```python
import numpy as np

kernel_name = "gpu_gemm_rs_m4096_k4096_n16384_bf16_gelu"


ABSMAX_RTOL = 0.008

COMPUTE_DTYPE = "bfloat16"
EPILOGUE = "gelu"
IN_QUANT = None
OUT_QUANT = None
IN_DTYPES = {"x": "bfloat16", "w_mat": "bfloat16"}
WORLD_SIZE = 8

def _to_bf16(x):
    u = np.ascontiguousarray(x, dtype=np.float32).view(np.uint32)
    return ((u + 0x7FFF + ((u >> 16) & 1)) & 0xFFFF0000).view(np.float32)

def setup_inputs():

    x = _to_bf16(np.random.randn(4096, 4096).astype(np.float32))
    w_mat = _to_bf16((np.random.randn(4096, 16384) / np.sqrt(4096)).astype(np.float32))
    return {"x": x, "w_mat": w_mat}

def _epilogue(y):
    c = 0.7978845608028654
    return (0.5 * y * (1.0 + np.tanh(c * (y + 0.044715 * y ** 3)))).astype(np.float32)

def reference(x, w_mat):
    y = (x @ w_mat).astype(np.float32)
    return _epilogue(y)
```

```cpp
#include <hip/hip_runtime.h>
#include <rccl/rccl.h>
#include <cstdio>
#include <cstdint>

#define HIP_CHECK(expr) do { hipError_t _e = (expr); if (_e != hipSuccess) { \
    fprintf(stderr, "HIP error %s at %s:%d\n", hipGetErrorString(_e), __FILE__, __LINE__); \
    return; } } while (0)

#define NSC 112
#define NSW 2
#define GLDS_AUX 17

namespace {
#define LAS __attribute__((address_space(3)))
typedef short bf16x8 __attribute__((ext_vector_type(8)));
typedef float f32x4 __attribute__((ext_vector_type(4)));
typedef float f32x2 __attribute__((ext_vector_type(2)));
typedef unsigned u32x4 __attribute__((ext_vector_type(4)));

constexpr int NR = 8;
constexpr int MM = 4096, NN = 16384, KS = 512, NS = 2048;
constexpr int NQ = 128;
constexpr int NH = 16;
constexpr unsigned long long DEADLINE = 20000000ull;

constexpr size_t OFF_CNT = 0;
constexpr size_t OFF_TMO = 16384;
constexpr size_t OFF_FX = 65536;
constexpr size_t OFF_FW = 65536 + 16384;
constexpr size_t OFF_FO = 65536 + 32768;
constexpr size_t OFF_STAMP = 262144;
constexpr int FSTR = 160;
constexpr size_t XQ = 256 * 1024, WQ = 128 * 1024;
constexpr size_t OFF_XI = 1u << 20, XI_SZ = (size_t)NQ * XQ;
constexpr size_t OFF_WI = OFF_XI + 2 * XI_SZ, WI_SZ = (size_t)NQ * WQ;
constexpr size_t WS_NEED = OFF_WI + 2 * WI_SZ;

struct Params {
    const unsigned short* x;
    const unsigned short* w;
    char* ws[NR];
    char* out[NR];
    int rank;
    int pad;
};

__device__ __forceinline__ void st16_sys(void* p, u32x4 v) {
    asm volatile("global_store_dwordx4 %0, %1, off sc0 sc1\n\ts_nop 1" :: "v"(p), "v"(v) : "memory");
}
__device__ __forceinline__ unsigned ld4_sys(const void* p) {
    unsigned v;
    asm volatile("global_load_dword %0, %1, off sc0 sc1\n\ts_waitcnt vmcnt(0)" : "=&v"(v) : "v"(p) : "memory");
    return v;
}
__device__ __forceinline__ void wait_vm0() { asm volatile("s_waitcnt vmcnt(0)" ::: "memory"); }
__device__ __forceinline__ unsigned cvt_pk_bf16(float lo, float hi) {
    unsigned r; asm("v_cvt_pk_bf16_f32 %0, %1, %2" : "=v"(r) : "v"(lo), "v"(hi)); return r;
}
__device__ __forceinline__ f32x2 gelu2(f32x2 y) {
    const float k1 = -2.0f * 0.7978845608028654f * 1.4426950408889634f, k2 = k1 * 0.044715f;
    f32x2 z = y * (y * y * k2 + k1);
    f32x2 e; e.x = __builtin_amdgcn_exp2f(z.x); e.y = __builtin_amdgcn_exp2f(z.y);
    e = e + 1.0f;
    f32x2 r; r.x = __builtin_amdgcn_rcpf(e.x); r.y = __builtin_amdgcn_rcpf(e.y);
    return y * r;
}

#define STAMP(slot) do {} while (0)

__device__ __forceinline__ bool give_up(unsigned* tmo, unsigned call, unsigned long long t0) {
    if (__builtin_amdgcn_readfirstlane(__hip_atomic_load(tmo, __ATOMIC_RELAXED, __HIP_MEMORY_SCOPE_AGENT)) == call ||
        __builtin_amdgcn_s_memrealtime() - t0 > DEADLINE) {
        __hip_atomic_store(tmo, call, __ATOMIC_RELAXED, __HIP_MEMORY_SCOPE_AGENT);
        return true;
    }
    return false;
}

struct Task { int h, u, dest; };

__device__ __forceinline__ void task_load(const Params& p, const Task& t, int lane, u32x4 (&b)[16]) {
    if (t.u < 16) {
        const unsigned short* src = p.x + (size_t)(t.u * 256 + (lane & 15)) * KS + t.h * 32 + (lane >> 4) * 8;
#pragma unroll
        for (int f = 0; f < 16; ++f) b[f] = *(const u32x4*)(src + (size_t)f * 16 * KS);
    } else {
        const unsigned short* src = p.w + (size_t)(t.h * 32 + (lane >> 4) * 8) * NN + t.dest * NS + (t.u - 16) * 256 + (lane & 15) * 8;
#pragma unroll
        for (int g = 0; g < 2; ++g)
#pragma unroll
            for (int r = 0; r < 8; ++r) b[g * 8 + r] = *(const u32x4*)(src + (size_t)r * NN + g * 128);
    }
}

__device__ __forceinline__ void task_store(const Params& p, const Task& t, int lane, unsigned call, u32x4 (&b)[16]) {
    const unsigned dest = (unsigned)t.dest & 7u;
    char* ws = p.ws[dest];
    const int q = (t.dest == p.rank) ? t.h : NH + t.h * 7 + ((p.rank - t.dest - 1) & 7);
    const size_t par = call & 1u;
    char* dst;
    unsigned* flag;
    if (t.u < 16) {
        dst = ws + OFF_XI + par * XI_SZ + (size_t)q * XQ + (size_t)t.u * 16384 + lane * 16;
        flag = (unsigned*)(ws + OFF_FX) + t.u * FSTR + q;
#pragma unroll
        for (int f = 0; f < 16; ++f) st16_sys(dst + f * 1024, b[f]);
    } else {
        dst = ws + OFF_WI + par * WI_SZ + (size_t)q * WQ + (size_t)(t.u - 16) * 16384 + lane * 16;
        flag = (unsigned*)(ws + OFF_FW) + (t.u - 16) * FSTR + q;
#pragma unroll
        for (int g = 0; g < 2; ++g)
#pragma unroll
            for (int i = 0; i < 8; ++i) {
                u32x4 o;
#pragma unroll
                for (int a = 0; a < 4; ++a)
                    o[a] = __builtin_amdgcn_perm(b[g * 8 + 2 * a + 1][i >> 1], b[g * 8 + 2 * a][i >> 1], (i & 1) ? 0x07060302u : 0x05040100u);
                st16_sys(dst + (g * 8 + i) * 1024, o);
            }
    }
    wait_vm0();
    if (lane == 0) __hip_atomic_store(flag, call, __ATOMIC_RELAXED, __HIP_MEMORY_SCOPE_SYSTEM);
}

__device__ __forceinline__ Task remote_task(int idx, int rank) {
    Task t; t.h = idx / 168; const int r = idx - t.h * 168; t.u = r / 7; t.dest = (rank + 1 + (r - t.u * 7)) & 7; return t;
}

__device__ __forceinline__ void comm_role(const Params& p, int cw, int wid, int lane, unsigned call) {
    constexpr int S = NSC * NSW, NT = NH * 168;
    if (wid >= 4) {
        const int id = (wid - 4) * 128 + cw;
        if (id < NH * 24) {
            Task t; t.h = id / 24; t.u = id - t.h * 24; t.dest = p.rank;
            u32x4 b[16];
            task_load(p, t, lane, b);
            task_store(p, t, lane, call, b);
        }
        STAMP(1);
    } else if (wid < NSW && cw < NSC) {
        int idx = wid * NSC + cw;
        u32x4 b0[16], b1[16];
        Task t0 = remote_task(idx, p.rank), t1;
        task_load(p, t0, lane, b0);
        for (;;) {
            const bool m1 = idx + S < NT;
            if (m1) { t1 = remote_task(idx + S, p.rank); task_load(p, t1, lane, b1); }
            task_store(p, t0, lane, call, b0);
            if (!m1) break;
            const bool m0 = idx + 2 * S < NT;
            if (m0) { t0 = remote_task(idx + 2 * S, p.rank); task_load(p, t0, lane, b0); }
            task_store(p, t1, lane, call, b1);
            if (!m0) break;
            idx += 2 * S;
        }
        STAMP(1);
    }
    if (cw == 0 && wid == 7) {
        unsigned* tmo = (unsigned*)(p.ws[p.rank] + OFF_TMO);
        const unsigned* fo = (const unsigned*)(p.ws[p.rank] + OFF_FO);
        const unsigned long long t0 = __builtin_amdgcn_s_memrealtime();
        bool bad = false;
        for (;;) {
            const unsigned a = ld4_sys(fo + (lane >> 4) * 32 + (lane & 15));
            const unsigned b = ld4_sys(fo + (4 + (lane >> 4)) * 32 + (lane & 15));
            if (__all(a >= call && b >= call)) break;
            if (give_up(tmo, call, t0)) { bad = true; break; }
            __builtin_amdgcn_s_sleep(2);
        }
        if (bad && lane == 0) ((unsigned short*)p.out[p.rank])[p.rank * NS] = 0x7fc0;
        STAMP(2);
    }
}

__device__ __forceinline__ int poll_ready(const unsigned* fx, const unsigned* fw, int ready, int need, unsigned call, unsigned* tmo) {
    const int lane = __builtin_amdgcn_mbcnt_hi(~0u, __builtin_amdgcn_mbcnt_lo(~0u, 0u));
    const unsigned long long t0 = __builtin_amdgcn_s_memrealtime();
    for (;;) {
        const int q = ready + (lane & 31);
        const unsigned v = ld4_sys((lane < 32 ? fx : fw) + q);
        const unsigned long long bal = __ballot(q >= NQ || v >= call);
        const unsigned both = (unsigned)bal & (unsigned)(bal >> 32);
        ready += (both == ~0u) ? 32 : __builtin_ctz(~both);
        if (ready >= need) break;
        if (give_up(tmo, call, t0)) { ready = 1 << 20; break; }
        __builtin_amdgcn_s_sleep(1);
    }
    return ready;
}

constexpr int HTB = 16384;

__device__ __forceinline__ void compute_role(const Params& p, LAS unsigned char* lds, int cid, int wid, int lane, unsigned call) {
    const int xcd = cid & 7, idx = cid >> 3;
    const int rt = (xcd >> 1) * 4 + (idx >> 2), ct = (xcd & 1) * 4 + (idx & 3);
    const int wr = wid >> 2, wc = wid & 3;
    char* ws = p.ws[p.rank];
    unsigned* tmo = (unsigned*)(ws + OFF_TMO);
    const unsigned* fx = (const unsigned*)(ws + OFF_FX) + rt * FSTR;
    const unsigned* fw = (const unsigned*)(ws + OFF_FW) + ct * FSTR;
    const size_t par = call & 1u;
    const char* xb = ws + OFF_XI + par * XI_SZ + (size_t)(rt * 16 + wid) * 1024;
    const char* wb = ws + OFF_WI + par * WI_SZ + (size_t)(ct * 16 + wr * 8 + wc) * 1024;
    const unsigned voff = lane * 16;
    const unsigned ldsw = wid * 1024;
    const int aoff = wr * 4096 + lane * 16, boff = wc * 2048 + lane * 16;
#define SA(b, h) (((b) * 2 + (h)) * HTB)
#define SB(b, h) ((4 + (b) * 2 + (h)) * HTB)
#define STAGE(bufoff, gbase, kq) do { _Pragma("unroll") for (int _i = 0; _i < 2; ++_i) \
        __builtin_amdgcn_global_load_lds((const unsigned*)((gbase) + (size_t)_i * (kq) + voff), (LAS unsigned*)(lds + (bufoff) + ldsw + _i * 8192), 16, 0, GLDS_AUX); } while (0)
#define STAGE_A(b, h, kt) STAGE(SA(b, h), wb + (size_t)(kt) * (2 * WQ) + (h) * 4096, WQ)
#define STAGE_B(b, h, kt) STAGE(SB(b, h), xb + (size_t)(kt) * (2 * XQ) + (h) * 8192, XQ)
#define LDA(dst, b, h) do { _Pragma("unroll") for (int m = 0; m < 4; ++m) _Pragma("unroll") for (int k = 0; k < 2; ++k) dst[m][k] = *(const LAS bf16x8*)(lds + SA(b, h) + aoff + m * 1024 + k * 8192); } while (0)
#define LDB(dst, b, h) do { _Pragma("unroll") for (int n = 0; n < 2; ++n) _Pragma("unroll") for (int k = 0; k < 2; ++k) dst[n][k] = *(const LAS bf16x8*)(lds + SB(b, h) + boff + n * 1024 + k * 8192); } while (0)
#define MMA(ai, bj, At, Bt) do { __builtin_amdgcn_s_setprio(1); _Pragma("unroll") for (int m = 0; m < 4; ++m) _Pragma("unroll") for (int n = 0; n < 2; ++n) _Pragma("unroll") for (int k = 0; k < 2; ++k) \
        acc[ai][bj][m][n] = __builtin_amdgcn_mfma_f32_16x16x32_bf16(Bt[n][k], At[m][k], acc[ai][bj][m][n], 0, 0, 0); __builtin_amdgcn_s_setprio(0); } while (0)
#define WAIT_V(n) asm volatile("s_waitcnt vmcnt(" #n ")" ::: "memory")
#define WAIT_L(n) asm volatile("s_waitcnt lgkmcnt(" #n ")" ::: "memory")
#define BAR __builtin_amdgcn_s_barrier()
#define SCHED __builtin_amdgcn_sched_barrier(0)
#define ENSURE(qn) do { if (__builtin_expect(ready < (qn), 0)) ready = poll_ready(fx, fw, ready, (qn), call, tmo); } while (0)
    f32x4 acc[2][2][4][2];
#pragma unroll
    for (int a = 0; a < 2; ++a)
#pragma unroll
        for (int b = 0; b < 2; ++b)
#pragma unroll
            for (int m = 0; m < 4; ++m)
#pragma unroll
                for (int n = 0; n < 2; ++n) acc[a][b][m][n] = (f32x4){0.f, 0.f, 0.f, 0.f};
    bf16x8 At[4][2], B0[2][2], B1[2][2];
    constexpr int nt = NQ / 2;
    int ready = 0;
    ENSURE(4);
    STAMP(1);
    STAGE_B(0, 0, 0); STAGE_A(0, 0, 0); STAGE_B(0, 1, 0); STAGE_A(0, 1, 0);
    if (wr == 1) BAR;
    WAIT_V(4); BAR;
    STAGE_B(1, 0, 1); STAGE_A(1, 0, 1); STAGE_B(1, 1, 1);
    WAIT_V(6); BAR;
    for (int t = 0; t < nt; t += 2) {
        const int t2 = (t + 2) & (nt - 1), t3 = t2 + 1;
        LDB(B0, 0, 0); SCHED; LDA(At, 0, 0); STAGE_A(1, 1, t + 1);
        WAIT_L(8); BAR; WAIT_L(0); MMA(0, 0, At, B0); BAR; SCHED;
        ENSURE(2 * t2 + 2);
        LDB(B1, 0, 1); STAGE_B(0, 0, t2);
        BAR; WAIT_L(0); MMA(0, 1, At, B1); BAR;
        LDA(At, 0, 1); STAGE_A(0, 0, t2);
        BAR; WAIT_L(0); MMA(1, 0, At, B0); BAR; SCHED;
        STAGE_B(0, 1, t2);
        WAIT_V(6); BAR; MMA(1, 1, At, B1); BAR;
        LDB(B0, 1, 0); SCHED; LDA(At, 1, 0); STAGE_A(0, 1, t2);
        WAIT_L(8); BAR; WAIT_L(0); MMA(0, 0, At, B0); BAR; SCHED;
        ENSURE(2 * t3 + 2);
        LDB(B1, 1, 1); STAGE_B(1, 0, t3);
        BAR; WAIT_L(0); MMA(0, 1, At, B1); BAR;
        LDA(At, 1, 1); STAGE_A(1, 0, t3);
        BAR; WAIT_L(0); MMA(1, 0, At, B0); BAR; SCHED;
        STAGE_B(1, 1, t3);
        WAIT_V(6); BAR; MMA(1, 1, At, B1); BAR;
    }
    STAMP(2);
    int tz = threadIdx.x; asm volatile("" : "+v"(tz));
    const int fr = tz & 15, fq = (tz >> 4) & 3;
    const unsigned nanmask = (__builtin_amdgcn_readfirstlane(__hip_atomic_load(tmo, __ATOMIC_RELAXED, __HIP_MEMORY_SCOPE_AGENT)) == call) ? 0x7fc07fc0u : 0u;
    const unsigned owner = (unsigned)(rt >> 1) & 7u;
    char* ob = p.out[owner] + ((size_t)((rt & 1) * 256 + wc * 32 + 4 * fq) * NN + p.rank * NS + ct * 256 + wr * 128 + 8 * fr) * 2;
#pragma unroll
    for (int bj = 0; bj < 2; ++bj)
#pragma unroll
        for (int n = 0; n < 2; ++n)
#pragma unroll
            for (int r = 0; r < 4; ++r) {
                u32x4 o;
#pragma unroll
                for (int ai = 0; ai < 2; ++ai)
#pragma unroll
                    for (int m2 = 0; m2 < 2; ++m2) {
                        const f32x2 g = gelu2((f32x2){acc[ai][bj][2 * m2][n][r], acc[ai][bj][2 * m2 + 1][n][r]});
                        o[ai * 2 + m2] = cvt_pk_bf16(g.x, g.y) | nanmask;
                    }
                st16_sys(ob + (size_t)(bj * 128 + n * 16 + r) * (NN * 2), o);
            }
    wait_vm0();
    if (wr == 0) BAR;
    BAR;
    if (threadIdx.x == 0)
        __hip_atomic_store((unsigned*)(p.ws[owner] + OFF_FO) + p.rank * 32 + (rt & 1) * 8 + ct, call, __ATOMIC_RELAXED, __HIP_MEMORY_SCOPE_SYSTEM);
    STAMP(3);
}

__global__ __launch_bounds__(512, 2) void fused_kernel(const Params p) {
    __shared__ __attribute__((aligned(16))) unsigned char shm[8 * HTB];
    const int wid = __builtin_amdgcn_readfirstlane(threadIdx.x >> 6), lane = threadIdx.x & 63;
    STAMP(0);
    unsigned c = 0;
    if (lane == 0) c = __hip_atomic_fetch_add((unsigned*)(p.ws[p.rank] + OFF_CNT) + blockIdx.x * 8 + wid, 1u, __ATOMIC_RELAXED, __HIP_MEMORY_SCOPE_AGENT);
    const unsigned call = __builtin_amdgcn_readfirstlane(c) + 1u;
    if (blockIdx.x < 128) comm_role(p, blockIdx.x, wid, lane, call);
    else compute_role(p, (LAS unsigned char*)shm, blockIdx.x - 128, wid, lane, call);
}
}

extern "C" void kernel_launch(int rank, int nranks, ncclComm_t comm, hipStream_t stream,
                              void* const* d_in, const int* in_sizes, int n_in,
                              void* d_out, int out_size, const int* in_dtypes, int out_dtype,
                              void* const* d_out_all, void* const* d_ws_all, size_t ws_bytes) {
    (void)comm; (void)in_dtypes; (void)out_dtype; (void)d_out;
    if (nranks != NR || (unsigned)rank >= (unsigned)NR || n_in < 2 || in_sizes[0] != MM * KS || in_sizes[1] != KS * NN ||
        out_size != (MM / NR) * NN || ws_bytes < WS_NEED) {
        fprintf(stderr, "kernel_launch: unexpected problem geometry\n");
        return;
    }
    Params p;
    p.x = (const unsigned short*)d_in[0];
    p.w = (const unsigned short*)d_in[1];
    for (int i = 0; i < NR; ++i) { p.ws[i] = (char*)d_ws_all[i]; p.out[i] = (char*)d_out_all[i]; }
    p.rank = rank; p.pad = 0;
    hipLaunchKernelGGL(fused_kernel, dim3(256), dim3(512), 0, stream, p);
    HIP_CHECK(hipGetLastError());
}
```

```cpp
#include <hip/hip_runtime.h>
#include <rccl/rccl.h>
#include <cstdio>
#include <cstdint>

#define HIP_CHECK(expr) do { hipError_t _e = (expr); if (_e != hipSuccess) { \
    fprintf(stderr, "HIP error %s at %s:%d\n", hipGetErrorString(_e), __FILE__, __LINE__); \
    return; } } while (0)

#define LAG 4
#define GLDS_AUX 17
#define FORCE_RAW 0
#define NUS 5
#define SWIN 7
#define SPASS 4
#define GATE_EARLY 0

namespace {
#define LAS __attribute__((address_space(3)))
typedef short bf16x8 __attribute__((ext_vector_type(8)));
typedef float f32x4 __attribute__((ext_vector_type(4)));
typedef float f32x2 __attribute__((ext_vector_type(2)));
typedef unsigned u32x4 __attribute__((ext_vector_type(4)));
typedef unsigned u32x2 __attribute__((ext_vector_type(2)));
typedef unsigned short u16x2 __attribute__((ext_vector_type(2)));

constexpr int NR = 8;
constexpr int MM = 4096, NN = 16384, KS = 512, NS = 2048;
constexpr int NQ = 128;
constexpr int NH = 16;
constexpr int NU = NH * 22;
constexpr int NUNP = NUS * 64;
constexpr int NFMT = (9 - NUS) * 64;
constexpr unsigned long long DEADLINE = 20000000ull;
constexpr unsigned RAWCNT = 0x3ffu;

constexpr size_t OFF_CNT = 0;
constexpr size_t OFF_TMO = 8192;
constexpr size_t OFF_TK = 8192 + 1024;
constexpr size_t OFF_SD = 131072;
constexpr int FSTR = 160;
constexpr size_t OFF_FX = 16384;
constexpr size_t OFF_FW = 32768;
constexpr size_t OFF_FA = 81920;
constexpr size_t OFF_FP = 122880;
constexpr size_t OFF_STAMP = 262144;
constexpr size_t UB = 16384;
constexpr size_t XQ = 256 * 1024, WQ = 128 * 1024;
constexpr size_t OFF_XI = 1u << 20;
constexpr size_t OFF_WI = OFF_XI + (size_t)NQ * XQ;
constexpr size_t OFF_WD = OFF_WI + (size_t)NQ * WQ;
constexpr size_t OFF_PL = OFF_WD + (size_t)7 * NH * WQ;
constexpr size_t OFF_LZ = OFF_PL + (size_t)1024 * UB;
constexpr size_t LZ_PAR = (size_t)7 * NU * UB;
constexpr size_t WS_NEED = OFF_LZ + 2 * LZ_PAR;
static_assert(OFF_FW + 64 * FSTR * 4 <= OFF_FA && OFF_FA + 14 * NU * 8 <= OFF_FP && OFF_FP + 1024 * 8 <= OFF_SD && OFF_SD + 56 * 128 <= OFF_STAMP, "layout");
static_assert(WS_NEED <= (size_t)161 << 20, "workspace");

struct Params {
    const char* x;
    const char* w;
    char* ws[NR];
    char* out;
    int rank;
    int pad;
};

template <int OFF> __device__ __forceinline__ void st16(const char* sbase, unsigned voff, u32x4 v) {
    asm volatile("global_store_dwordx4 %0, %1, %2 offset:%3 sc0 sc1\n\ts_nop 1" :: "v"(voff), "v"(v), "s"(sbase), "i"(OFF) : "memory");
}
__device__ __forceinline__ unsigned ld4_sys(const void* p) {
    unsigned v;
    asm volatile("global_load_dword %0, %1, off sc0 sc1\n\ts_waitcnt vmcnt(0)" : "=&v"(v) : "v"(p) : "memory");
    return v;
}
__device__ __forceinline__ u32x2 ld8_sys(const void* p) {
    u32x2 v;
    asm volatile("global_load_dwordx2 %0, %1, off sc0 sc1\n\ts_waitcnt vmcnt(0)" : "=&v"(v) : "v"(p) : "memory");
    return v;
}
__device__ __forceinline__ void st8_flag(void* p, unsigned call, unsigned hdr, int lane) {
    if (lane == 0) __hip_atomic_store((unsigned long long*)p, ((unsigned long long)hdr << 32) | call, __ATOMIC_RELAXED, __HIP_MEMORY_SCOPE_SYSTEM);
}
#define WAIT_V_(n) asm volatile("s_waitcnt vmcnt(" #n ")" ::: "memory")
#define WAIT_V(n) WAIT_V_(n)
#define WAIT_L(n) asm volatile("s_waitcnt lgkmcnt(" #n ")" ::: "memory")
#define BAR __builtin_amdgcn_s_barrier()
#define SCHED __builtin_amdgcn_sched_barrier(0)

__device__ __forceinline__ unsigned cvt_pk_bf16(float lo, float hi) {
    unsigned r; asm("v_cvt_pk_bf16_f32 %0, %1, %2" : "=v"(r) : "v"(lo), "v"(hi)); return r;
}
__device__ __forceinline__ f32x2 gelu2(f32x2 y) {
    const float k1 = -2.0f * 0.7978845608028654f * 1.4426950408889634f, k2 = k1 * 0.044715f;
    f32x2 z = y * (y * y * k2 + k1);
    f32x2 e; e.x = __builtin_amdgcn_exp2f(z.x); e.y = __builtin_amdgcn_exp2f(z.y);
    e = e + 1.0f;
    f32x2 r; r.x = __builtin_amdgcn_rcpf(e.x); r.y = __builtin_amdgcn_rcpf(e.y);
    return y * r;
}
__device__ __forceinline__ unsigned pk_sub(unsigned a, unsigned b) { return __builtin_bit_cast(unsigned, __builtin_bit_cast(u16x2, a) - __builtin_bit_cast(u16x2, b)); }
__device__ __forceinline__ unsigned pk_add(unsigned a, unsigned b) { return __builtin_bit_cast(unsigned, __builtin_bit_cast(u16x2, a) + __builtin_bit_cast(u16x2, b)); }
__device__ __forceinline__ unsigned pk_max(unsigned a, unsigned b) { return __builtin_bit_cast(unsigned, __builtin_elementwise_max(__builtin_bit_cast(u16x2, a), __builtin_bit_cast(u16x2, b))); }
__device__ __forceinline__ unsigned bfi(unsigned m, unsigned a, unsigned b) { return (a & m) | (b & ~m); }
__device__ __forceinline__ unsigned rotl(unsigned a, int n) { return __builtin_amdgcn_alignbit(a, a, 32 - n); }
__device__ __forceinline__ unsigned rotr(unsigned a, int n) { return __builtin_amdgcn_alignbit(a, a, n); }
__device__ __forceinline__ unsigned umax(unsigned a, unsigned b) { return a > b ? a : b; }

#define STAMP(slot) do {} while (0)

__device__ __forceinline__ bool give_up(unsigned* tmo, unsigned call, unsigned long long t0) {
    if (__builtin_amdgcn_readfirstlane(__hip_atomic_load(tmo, __ATOMIC_RELAXED, __HIP_MEMORY_SCOPE_AGENT)) == call ||
        __builtin_amdgcn_s_memrealtime() - t0 > DEADLINE) {
        __hip_atomic_store(tmo, call, __ATOMIC_RELAXED, __HIP_MEMORY_SCOPE_AGENT);
        return true;
    }
    return false;
}
__device__ __forceinline__ unsigned wait_unit(const char* flag, unsigned call, unsigned st, unsigned* tmo) {
    const unsigned long long t0 = __builtin_amdgcn_s_memrealtime();
    for (;;) {
        const u32x2 v = ld8_sys(flag);
        const unsigned hdr = __builtin_amdgcn_readfirstlane(v.y);
        if ((unsigned)__builtin_amdgcn_readfirstlane(v.x) == call && (hdr >> 28) >= st) return hdr;
        __builtin_amdgcn_s_sleep(1);
        if (give_up(tmo, call, t0)) return (RAWCNT << 16) | (4u << 28);
    }
}
__device__ __forceinline__ unsigned hdr_cnt(unsigned hdr) { return (hdr >> 16) & 0x3ffu; }

__device__ __forceinline__ unsigned pack_unit(const u32x4 (&b)[16], u32x4 (&pk)[12], LAS unsigned* lst, int lane, const char* early) {
    unsigned mx = 0;
#pragma unroll
    for (int f = 0; f < 16; ++f) mx = pk_max(mx, b[f][f & 3] & 0x7fff7fffu);
    unsigned m = umax(mx & 0xffffu, mx >> 16);
    m = umax(m, (unsigned)__builtin_amdgcn_update_dpp(0, (int)m, 0x128, 0xf, 0xf, false));
    m = umax(m, (unsigned)__builtin_amdgcn_update_dpp(0, (int)m, 0x124, 0xf, 0xf, false));
    m = umax(m, (unsigned)__builtin_amdgcn_update_dpp(0, (int)m, 0x122, 0xf, 0xf, false));
    m = umax(m, (unsigned)__builtin_amdgcn_update_dpp(0, (int)m, 0x121, 0xf, 0xf, false));
    const unsigned ms = umax(umax(__builtin_amdgcn_readlane(m, 0), __builtin_amdgcn_readlane(m, 16)), umax(__builtin_amdgcn_readlane(m, 32), __builtin_amdgcn_readlane(m, 48)));
    const unsigned L = umax(ms, 1023u) - 1023u, LL = L | (L << 16);
    const unsigned lanebits = ((lane & 7) << 7) | ((lane >> 3) << 15);
    unsigned cnt = 0;
#pragma unroll
    for (int s = 0; s < 4; ++s) {
        unsigned hb[8];
#pragma unroll
        for (int jj = 0; jj < 8; ++jj) {
            const int j = 8 * s + jj;
            const unsigned d0 = pk_sub(b[j >> 1][2 * (j & 1)], LL), d1 = pk_sub(b[j >> 1][2 * (j & 1) + 1], LL);
            pk[j >> 2][j & 3] = __builtin_amdgcn_perm(d1, d0, 0x06040200u);
            hb[jj] = __builtin_amdgcn_perm(d1, d0, 0x07050301u);
            const unsigned t = hb[jj] & 0x7c7c7c7cu;
            const unsigned long long bal = __ballot(t != 0);
            const unsigned pos = __builtin_amdgcn_mbcnt_hi((unsigned)(bal >> 32), __builtin_amdgcn_mbcnt_lo((unsigned)bal, cnt < 448u ? cnt : 448u));
            if (t != 0) lst[pos] = t | lanebits | (unsigned)((j & 3) | ((j >> 2) << 23));
            cnt += __builtin_popcountll(bal);
            if (j == 3 && early) st16<-4096>(early, lane * 16, pk[0]);
            if (j == 17 && early) { WAIT_V(0); st16<-3072>(early, lane * 16, pk[1]); }
        }
        pk[8][s] = bfi(0x83838383u, hb[0], bfi(0x1c1c1c1cu, rotl(hb[1], 3), hb[6] << 5));
        pk[9][s] = bfi(0x83838383u, hb[2], bfi(0x1c1c1c1cu, rotl(hb[3], 3), hb[7] << 5));
        pk[10][s] = bfi(0x83838383u, hb[4], bfi(0x1c1c1c1cu, rotl(hb[5], 3), bfi(0x20202020u, hb[6] >> 2, hb[7] >> 1)));
    }
    pk[11] = *(LAS u32x4*)(lst + lane * 4);
    if (cnt > 256u || FORCE_RAW) cnt = RAWCNT;
    return L | (cnt << 16);
}
__device__ __forceinline__ void st_list(const char* sb8k  , unsigned voff, u32x4 v, unsigned hdr, int lane) {
    const unsigned nl = umax((hdr_cnt(hdr) + 15u) >> 4, 1u) << 2;
    if ((unsigned)lane < nl) st16<3072>(sb8k, voff, v);
}
__device__ __forceinline__ void st_raw(const char* slot, unsigned voff, const u32x4 (&b)[16]) {
    const char* s0 = slot + 4096; const char* s1 = slot + 12288;
    st16<-4096>(s0, voff, b[0]); st16<-3072>(s0, voff, b[1]); st16<-2048>(s0, voff, b[2]); st16<-1024>(s0, voff, b[3]);
    st16<0>(s0, voff, b[4]); st16<1024>(s0, voff, b[5]); st16<2048>(s0, voff, b[6]); st16<3072>(s0, voff, b[7]);
    st16<-4096>(s1, voff, b[8]); st16<-3072>(s1, voff, b[9]); st16<-2048>(s1, voff, b[10]); st16<-1024>(s1, voff, b[11]);
    st16<0>(s1, voff, b[12]); st16<1024>(s1, voff, b[13]); st16<2048>(s1, voff, b[14]); st16<3072>(s1, voff, b[15]);
}
__device__ __forceinline__ void ld_unit(u32x4 (&r)[16], const char* slot, unsigned voff, bool raw) {
    const char* s0 = slot + 4096; const char* s1 = slot + 12288;
    r[12] = r[13] = r[14] = r[15] = (u32x4){0u, 0u, 0u, 0u};
    if (raw) {
        asm volatile("global_load_dwordx4 %0, %4, %5 sc0 sc1\n\tglobal_load_dwordx4 %1, %4, %5 offset:1024 sc0 sc1\n\t"
                     "global_load_dwordx4 %2, %4, %5 offset:2048 sc0 sc1\n\tglobal_load_dwordx4 %3, %4, %5 offset:3072 sc0 sc1\n\ts_waitcnt vmcnt(0)"
                     : "=&v"(r[12]), "=&v"(r[13]), "=&v"(r[14]), "=&v"(r[15]) : "v"(voff), "s"(s1) : "memory");
    }
    asm volatile("global_load_dwordx4 %11, %12, %14 offset:-1024 sc0 sc1\n\t"
                 "global_load_dwordx4 %8, %12, %14 offset:-4096 sc0 sc1\n\tglobal_load_dwordx4 %9, %12, %14 offset:-3072 sc0 sc1\n\t"
                 "global_load_dwordx4 %10, %12, %14 offset:-2048 sc0 sc1\n\t"
                 "global_load_dwordx4 %0, %12, %13 offset:-4096 sc0 sc1\n\tglobal_load_dwordx4 %1, %12, %13 offset:-3072 sc0 sc1\n\t"
                 "global_load_dwordx4 %2, %12, %13 offset:-2048 sc0 sc1\n\tglobal_load_dwordx4 %3, %12, %13 offset:-1024 sc0 sc1\n\t"
                 "global_load_dwordx4 %4, %12, %13 sc0 sc1\n\tglobal_load_dwordx4 %5, %12, %13 offset:1024 sc0 sc1\n\t"
                 "global_load_dwordx4 %6, %12, %13 offset:2048 sc0 sc1\n\tglobal_load_dwordx4 %7, %12, %13 offset:3072 sc0 sc1\n\t"
                 "s_waitcnt vmcnt(0)"
                 : "=&v"(r[0]), "=&v"(r[1]), "=&v"(r[2]), "=&v"(r[3]), "=&v"(r[4]), "=&v"(r[5]), "=&v"(r[6]), "=&v"(r[7]),
                   "=&v"(r[8]), "=&v"(r[9]), "=&v"(r[10]), "=&v"(r[11])
                 : "v"(voff), "s"(s0), "s"(s1) : "memory");
}
__device__ __forceinline__ void ld4x16(u32x4& a, u32x4& b, u32x4& c, u32x4& d, const char* sb, unsigned voff) {
    asm volatile("global_load_dwordx4 %0, %4, %5 sc0 sc1\n\tglobal_load_dwordx4 %1, %4, %5 offset:1024 sc0 sc1\n\t"
                 "global_load_dwordx4 %2, %4, %5 offset:2048 sc0 sc1\n\tglobal_load_dwordx4 %3, %4, %5 offset:3072 sc0 sc1\n\ts_waitcnt vmcnt(0)"
                 : "=&v"(a), "=&v"(b), "=&v"(c), "=&v"(d) : "v"(voff), "s"(sb) : "memory");
}
__device__ __forceinline__ void ld2x16t(u32x4& a, u32x4& b, unsigned& t, const char* sb, unsigned voff, const unsigned* tp) {
    asm volatile("global_load_dwordx4 %0, %3, %4 sc0 sc1\n\tglobal_load_dwordx4 %1, %3, %4 offset:1024 sc0 sc1\n\t"
                 "global_load_dword %2, %5, off sc1\n\ts_waitcnt vmcnt(0)"
                 : "=&v"(a), "=&v"(b), "=&v"(t) : "v"(voff), "s"(sb), "v"(tp) : "memory");
}
__device__ __forceinline__ unsigned patch_pos(unsigned e) { return (e & 3u) | (((e >> 7) & 7u) << 2) | (((e >> 15) & 7u) << 5) | (((e >> 23) & 7u) << 8); }
__device__ __forceinline__ void patch_apply(LAS unsigned* pat, u32x4 list, unsigned cnt, int lane, bool zero) {
#pragma unroll
    for (int c = 0; c < 4; ++c)
        if ((unsigned)(lane * 4 + c) < cnt) pat[patch_pos(list[c])] = zero ? 0u : list[c];
}
__device__ __forceinline__ void unpack_sub(const u32x4 (&r)[16], int s, LAS unsigned* pat, int lane, unsigned LL, u32x4 (&o)[4]) {
    const u32x4 p0 = *(LAS u32x4*)(pat + (2 * s) * 256 + lane * 4), p1 = *(LAS u32x4*)(pat + (2 * s + 1) * 256 + lane * 4);
    const unsigned A = r[8][s], B = r[9][s], C = r[10][s];
    unsigned hb[8];
    hb[0] = bfi(0x83838383u, A, p0[0]); hb[1] = bfi(0x83838383u, rotr(A, 3), p0[1]);
    hb[2] = bfi(0x83838383u, B, p0[2]); hb[3] = bfi(0x83838383u, rotr(B, 3), p0[3]);
    hb[4] = bfi(0x83838383u, C, p1[0]); hb[5] = bfi(0x83838383u, rotr(C, 3), p1[1]);
    hb[6] = bfi(0x03030303u, A >> 5, bfi(0x80808080u, C << 2, p1[2]));
    hb[7] = bfi(0x03030303u, B >> 5, bfi(0x80808080u, C << 1, p1[3]));
#pragma unroll
    for (int jj = 0; jj < 8; ++jj) {
        const unsigned lo = r[2 * s + (jj >> 2)][jj & 3];
        o[jj >> 1][2 * (jj & 1)] = pk_add(__builtin_amdgcn_perm(hb[jj], lo, 0x05010400u), LL);
        o[jj >> 1][2 * (jj & 1) + 1] = pk_add(__builtin_amdgcn_perm(hb[jj], lo, 0x07030602u), LL);
    }
}

__device__ __forceinline__ void unit_fetch(const Params& p, int h, int u, int lane, unsigned ldsbuf) {
    const bool isx = u < 16;
    const char* base = isx ? p.x + ((size_t)u * 256 * KS + h * 32) * 2 : p.w + ((size_t)h * 32 * NN + (u - 16) * 256) * 2;
    unsigned voff = isx ? (lane & 15) * (KS * 2) + (lane >> 4) * 16 : (lane >> 4) * (8 * NN * 2) + (lane & 15) * 16;
    const unsigned s1 = isx ? 16 * KS * 2 : NN * 2, s2 = isx ? 16 * KS * 2 : 256 - 7 * NN * 2;
#pragma unroll
    for (int i = 0; i < 16; ++i) {
        unsigned keep;
        asm volatile("s_mov_b32 %0, m0\n\ts_mov_b32 m0, %3\n\ts_nop 0\n\tglobal_load_lds_dwordx4 %1, %2\n\ts_mov_b32 m0, %0"
                     : "=&s"(keep) : "v"(voff), "s"(base), "s"(ldsbuf + i * 1024) : "memory");
        voff += (i == 7) ? s2 : s1;
    }
}
__device__ __forceinline__ void unit_read(LAS unsigned char* buf, int lane, u32x4 (&r)[16]) {
#pragma unroll
    for (int i = 0; i < 16; ++i) r[i] = *(LAS u32x4*)(buf + i * 1024 + lane * 16);
    asm volatile("s_waitcnt lgkmcnt(0)" : "+v"(r[0]), "+v"(r[1]), "+v"(r[2]), "+v"(r[3]), "+v"(r[4]), "+v"(r[5]), "+v"(r[6]), "+v"(r[7]),
                 "+v"(r[8]), "+v"(r[9]), "+v"(r[10]), "+v"(r[11]), "+v"(r[12]), "+v"(r[13]), "+v"(r[14]), "+v"(r[15]) :: "memory");
}
__device__ __forceinline__ void unit_format(int u, u32x4 (&r)[16]) {
    if (u >= 16) {
#pragma unroll
        for (int g = 0; g < 2; ++g) {
            u32x4 o[8];
#pragma unroll
            for (int i = 0; i < 8; ++i)
#pragma unroll
                for (int a = 0; a < 4; ++a)
                    o[i][a] = __builtin_amdgcn_perm(r[g * 8 + 2 * a + 1][i >> 1], r[g * 8 + 2 * a][i >> 1], (i & 1) ? 0x07060302u : 0x05040100u);
#pragma unroll
            for (int i = 0; i < 8; ++i) r[g * 8 + i] = o[i];
        }
    }
}
__device__ __forceinline__ int order(int k, int src) {
    int h, v;
    if (k < LAG * 15) { h = k / 15; v = k - h * 15; }
    else if (k < LAG * 15 + (NH - LAG) * 22) { const int k2 = k - LAG * 15; const int rho = k2 / 22; v = k2 - rho * 22; h = v >= 15 ? rho : rho + LAG; }
    else { const int k2 = k - LAG * 15 - (NH - LAG) * 22; const int rho = k2 / 7; v = 15 + k2 - rho * 7; h = rho + NH - LAG; }
    const int odd = v >= 15, m = (src + 1 + v - (odd ? 15 : 8)) & 7;
    return (h << 8) | (v < 8 ? 16 + v : 2 * m + odd);
}

__device__ __forceinline__ void sender(const Params& p, LAS unsigned char* buf, LAS unsigned* lst, LAS unsigned* lock, int xcd, int di, int mem, int lane, unsigned* cntp) {
    const unsigned dest = (unsigned)(p.rank + 1 + di) & 7u;
    char* dws = p.ws[dest];
    const int li = 6 - di;
    const unsigned voff = lane * 16, lbuf = (unsigned)(size_t)buf;
    int k = mem * 8 + xcd;
    int o = order(k, p.rank), h = o >> 8, u = (o & 255) + ((o & 16) ? dest * 8 : 0);
    unit_fetch(p, h, u, lane, lbuf);
    const unsigned call = __builtin_amdgcn_readfirstlane(ld4_sys(cntp)) + 1u;
    const char* lz = dws + OFF_LZ + (call & 1u) * LZ_PAR + (size_t)li * NU * UB;
    char* fa = dws + OFF_FA + (size_t)((call & 1u) * 7 + li) * NU * 8;
    unsigned* tk = (unsigned*)(p.ws[p.rank] + OFF_TK) + di * 32;
    const unsigned tk0 = (call - 1u) * (NU - 32) - 64u;
    unsigned kn = k + 32;
#define TICK(acc) do {} while (0)
    for (;;) {
        const int hc = h, uc = u;
        const bool more = kn < (unsigned)NU;
        u32x4 r[16], pk[12];
        unit_read(buf, lane, r);
        if (more) { o = order(kn, p.rank); h = o >> 8; u = (o & 255) + ((o & 16) ? dest * 8 : 0); unit_fetch(p, h, u, lane, lbuf); }
        unit_format(uc, r);
        const unsigned hdr = pack_unit(r, pk, lst, lane, nullptr) | (4u << 28);
        asm volatile("" : "+v"(pk[0]), "+v"(pk[1]), "+v"(pk[2]), "+v"(pk[3]), "+v"(pk[4]), "+v"(pk[5]), "+v"(pk[6]), "+v"(pk[7]), "+v"(pk[8]), "+v"(pk[9]), "+v"(pk[10]), "+v"(pk[11]));
        const bool raw = hdr_cnt(hdr) == RAWCNT;
        const char* slot = lz + (size_t)k * UB;
        unsigned my = 0;
        if (lane == 0) my = __hip_atomic_fetch_add(lock, 1u, __ATOMIC_RELAXED, __HIP_MEMORY_SCOPE_WORKGROUP);
        my = __builtin_amdgcn_readfirstlane(my);
        TICK(a_prep);
        while ((unsigned)__builtin_amdgcn_readfirstlane(*(volatile LAS unsigned*)(lock + 1)) != my) __builtin_amdgcn_s_sleep(1);
        TICK(a_wait);
        if (k == NU - 1 - GATE_EARLY && lane < 8)
            __hip_atomic_store((unsigned*)(p.ws[p.rank] + OFF_SD) + (di * 8 + lane) * 32, call, __ATOMIC_RELAXED, __HIP_MEMORY_SCOPE_AGENT);
        if (__builtin_expect(raw, 0)) {
            WAIT_V(0);
            unit_fetch(p, hc, uc, lane, lbuf);
            WAIT_V(0);
            unit_read(buf, lane, r);
            unit_fetch(p, h, u, lane, lbuf);
            unit_format(uc, r);
            st_raw(slot, voff, r);
        } else {
            const char* s0 = slot + 4096; const char* s1 = slot + 8192;
            st_list(s1, voff, pk[11], hdr, lane);
            st16<-4096>(s0, voff, pk[0]); st16<-3072>(s0, voff, pk[1]); st16<-2048>(s0, voff, pk[2]); st16<-1024>(s0, voff, pk[3]);
            st16<0>(s0, voff, pk[4]); st16<1024>(s0, voff, pk[5]); st16<2048>(s0, voff, pk[6]);
            WAIT_V(SWIN); st16<3072>(s0, voff, pk[7]);
            WAIT_V(SWIN); st16<0>(s1, voff, pk[8]);
            WAIT_V(SWIN); st16<1024>(s1, voff, pk[9]);
            WAIT_V(SWIN); st16<2048>(s1, voff, pk[10]);
        }
        WAIT_V(SPASS);
        *(volatile LAS unsigned*)(lock + 1) = my + 1;
        TICK(a_send);
        unsigned tn = 1u;
        if (more && lane == 0) asm volatile("global_atomic_add %0, %1, %0, off sc0\n\ts_waitcnt vmcnt(0)" : "+v"(tn) : "v"(tk) : "memory");
        WAIT_V(0);
        st8_flag(fa + k * 8, call, hdr, lane);
        TICK(a_ack);
        if (!more) break;
        k = kn; kn = __builtin_amdgcn_readfirstlane(tn) - tk0;
    }
}

__device__ __forceinline__ void unpacker(const Params& p, LAS unsigned* pat, int uw, int lane, unsigned call) {
    char* ws = p.ws[p.rank];
    unsigned* tmo = (unsigned*)(ws + OFF_TMO);
    const unsigned voff = lane * 16;
#pragma unroll
    for (int i = 0; i < 8; ++i) *(LAS u32x4*)(pat + i * 256 + lane * 4) = (u32x4){0u, 0u, 0u, 0u};
    for (int n = uw; n < 7 * NU; n += NUNP) {
        const int k = n / 7, li = n - k * 7, src = (p.rank + 1 + li) & 7;
        const int o = order(k, src), t = o & 15; const bool isx = !(o & 16);
        const int q = NH + (o >> 8) * 7 + li;
        const unsigned hdr = wait_unit(ws + OFF_FA + (size_t)(((call & 1u) * 7 + li) * NU + k) * 8, call, 4u, tmo);
        const unsigned cnt = hdr_cnt(hdr); const bool raw = cnt == RAWCNT;
        u32x4 r[16];
        ld_unit(r, ws + OFF_LZ + (call & 1u) * LZ_PAR + (size_t)(li * NU + k) * UB, voff, raw);
        const char* dst = ws + (isx ? OFF_XI + (size_t)q * XQ : OFF_WI + (size_t)q * WQ) + (size_t)t * UB;
        unsigned* flag = (unsigned*)(ws + (isx ? OFF_FX : OFF_FW)) + (isx ? t : p.rank * 8 + t) * FSTR + q;
        if (__builtin_expect(raw, 0)) st_raw(dst, voff, r);
        else {
            const unsigned LL = (hdr & 0xffffu) * 0x10001u;
            patch_apply(pat, r[11], cnt, lane, false);
#pragma unroll
            for (int s = 0; s < 4; ++s) {
                u32x4 o[4]; unpack_sub(r, s, pat, lane, LL, o);
                const char* sb = dst + s * 4096;
                st16<0>(sb, voff, o[0]); st16<1024>(sb, voff, o[1]); st16<2048>(sb, voff, o[2]); st16<3072>(sb, voff, o[3]);
            }
            patch_apply(pat, r[11], cnt, lane, true);
        }
        WAIT_V(0);
        if (lane == 0) __hip_atomic_store(flag, call, __ATOMIC_RELAXED, __HIP_MEMORY_SCOPE_SYSTEM);
    }
}

__device__ __forceinline__ void formatter(const Params& p, LAS unsigned char* buf, int fw, int n2nd, int lane, unsigned call) {
    char* ws = p.ws[p.rank];
    const unsigned voff = lane * 16;
    for (int n = fw; n < 1280; n = (n == fw) ? n2nd : n + NFMT) {
        int h, u; const char* dst; unsigned* flag;
        if (n < 384) { h = n / 24; const int v = n - h * 24;
            if (v < 16) { u = v; dst = ws + OFF_XI + (size_t)h * XQ + (size_t)v * UB; flag = (unsigned*)(ws + OFF_FX) + v * FSTR + h; }
            else { u = p.rank * 8 + v; dst = ws + OFF_WI + (size_t)h * WQ + (size_t)(v - 16) * UB; flag = (unsigned*)(ws + OFF_FW) + (u - 16) * FSTR + h; } }
        else { const int n2 = n - 384; h = n2 / 56; const int v = n2 - h * 56, li = v >> 3, cn = ((p.rank + 1 + li) & 7) * 8 + (v & 7);
            u = 16 + cn; dst = ws + OFF_WD + (size_t)(li * NH + h) * WQ + (size_t)(v & 7) * UB; flag = (unsigned*)(ws + OFF_FW) + cn * FSTR + h; }
        u32x4 r[16];
        unit_fetch(p, h, u, lane, (unsigned)(size_t)buf);
        WAIT_V(0);
        unit_read(buf, lane, r);
        unit_format(u, r);
        st_raw(dst, voff, r);
        WAIT_V(0);
        if (lane == 0) __hip_atomic_store(flag, call, __ATOMIC_RELAXED, __HIP_MEMORY_SCOPE_SYSTEM);
    }
}

__device__ __forceinline__ int poll_ready(const unsigned* fx, const unsigned* fw, int ready, int need, int skip, unsigned call, unsigned* tmo) {
    const int lane = __builtin_amdgcn_mbcnt_hi(~0u, __builtin_amdgcn_mbcnt_lo(~0u, 0u));
    const unsigned long long t0 = __builtin_amdgcn_s_memrealtime();
    for (;;) {
        const int q = ready + (lane & 31);
        const unsigned v = ld4_sys((lane < 32 ? fx : fw) + q);
        const bool skipped = lane < 32 && q >= NH && (q - NH) % 7 == skip;
        const unsigned long long bal = __ballot(q >= NQ || v >= call || skipped);
        const unsigned both = (unsigned)bal & (unsigned)(bal >> 32);
        ready += (both == ~0u) ? 32 : __builtin_ctz(~both);
        if (ready >= need) break;
        if (give_up(tmo, call, t0)) { ready = 1 << 20; break; }
        __builtin_amdgcn_s_sleep(1);
    }
    return ready;
}

constexpr int HTB = 16384;

__global__ __launch_bounds__(512, 2) void fused_kernel(const Params p) {
    __shared__ __attribute__((aligned(16))) unsigned char shm[8 * HTB + 8 * 2048 + 64];
    LAS unsigned char* lds = (LAS unsigned char*)shm;
    const int wid = __builtin_amdgcn_readfirstlane(threadIdx.x >> 6), lane = threadIdx.x & 63;
    const int xcd = blockIdx.x & 7, slot = blockIdx.x >> 3;
    char* ws = p.ws[p.rank];
    unsigned* tmo = (unsigned*)(ws + OFF_TMO);
    unsigned* cntp = (unsigned*)(ws + OFF_CNT) + blockIdx.x * 8 + wid;
    LAS unsigned* lst = (LAS unsigned*)(lds + 8 * HTB + wid * 2048);
    STAMP(0);
    const bool main_tile = slot >= 16;
    if (slot < 7) {
        LAS unsigned* lock = (LAS unsigned*)(lds + 8 * HTB + 8 * 2048);
        *(volatile LAS u32x2*)lock = (u32x2){0u, 0u};
        WAIT_L(0); BAR;
        if (wid < 4) sender(p, lds + wid * HTB, lst, lock, xcd, slot, wid, lane, cntp);
    }
    const unsigned call = __builtin_amdgcn_readfirstlane(ld4_sys(cntp)) + 1u;
    if (lane == 0) __hip_atomic_store(cntp, call, __ATOMIC_RELAXED, __HIP_MEMORY_SCOPE_AGENT);

    if (!main_tile) {
        if (slot >= 7) {
            const bool unp = slot < 7 + NUS;
            formatter(p, lds + wid * HTB, wid * 72 + (blockIdx.x - 56), unp ? 1 << 20 : 576 + wid * (NFMT / 8) + (blockIdx.x - 56 - NUS * 8), lane, call);
            if (unp) unpacker(p, (LAS unsigned*)(lds + wid * HTB), wid * (NUNP / 8) + (blockIdx.x - 56), lane, call);
        }
        STAMP(1);
        asm volatile("s_waitcnt vmcnt(0) lgkmcnt(0)" ::: "memory");
        BAR;
    }

    int rt, cn, q0, nt, skip;
    const char* wimg;
    if (main_tile) {
        const int idx = slot - 16;
        rt = ((xcd >> 1) * 2 + ((idx >> 2) & 1)) * 2 + (idx >> 3);
        cn = p.rank * 8 + (xcd & 1) * 4 + (idx & 3);
        const int m = rt >> 1;
        q0 = (m == p.rank) ? NH : 0; skip = (m - p.rank - 1) & 7; nt = 56;
        wimg = ws + OFF_WI;
    } else {
        rt = p.rank * 2 + (slot < 8 ? 1 : 0);
        cn = xcd * 8 + (slot & 7);
        q0 = 0; skip = 7; nt = 8;
        wimg = (xcd == p.rank) ? ws + OFF_WI : ws + OFF_WD + (size_t)((xcd - p.rank - 1) & 7) * NH * WQ;
    }
    const int wr = wid >> 2, wc = wid & 3;
    const unsigned* fx = (const unsigned*)(ws + OFF_FX) + rt * FSTR;
    const unsigned* fw = (const unsigned*)(ws + OFF_FW) + cn * FSTR;
    const char* xb = ws + OFF_XI + (size_t)(rt * 16 + wid) * 1024;
    const char* wb = wimg + (size_t)((cn & 7) * 16 + wr * 8 + wc) * 1024;
    const unsigned voff = lane * 16;
    const unsigned ldsw = wid * 1024;
    const int aoff = wr * 4096 + lane * 16, boff = wc * 2048 + lane * 16;
#define SA(b, h) (((b) * 4 + (h)) * HTB)
#define SB(b, h) (((b) * 4 + 2 + (h)) * HTB)
#define GLDS(g, l) __builtin_amdgcn_global_load_lds((const unsigned*)((g) + voff), (LAS unsigned*)(lds + (l)), 16, 0, GLDS_AUX)
#define STAGE_A(b, h, qa, qb) do { GLDS(wb + (size_t)(qa) * WQ + (h) * 4096, SA(b, h) + ldsw); GLDS(wb + (size_t)(qb) * WQ + (h) * 4096, SA(b, h) + ldsw + 8192); } while (0)
#define STAGE_B(b, h, qa, qb) do { GLDS(xb + (size_t)(qa) * XQ + (h) * 8192, SB(b, h) + ldsw); GLDS(xb + (size_t)(qb) * XQ + (h) * 8192, SB(b, h) + ldsw + 8192); } while (0)
#define LDA(dst, b, h) do { _Pragma("unroll") for (int m = 0; m < 4; ++m) _Pragma("unroll") for (int k = 0; k < 2; ++k) dst[m][k] = *(const LAS bf16x8*)(lds + SA(b, h) + aoff + m * 1024 + k * 8192); } while (0)
#define LDB(dst, b, h) do { _Pragma("unroll") for (int n = 0; n < 2; ++n) _Pragma("unroll") for (int k = 0; k < 2; ++k) dst[n][k] = *(const LAS bf16x8*)(lds + SB(b, h) + boff + n * 1024 + k * 8192); } while (0)
#define MMA(ai, bj, At, Bt) do { __builtin_amdgcn_s_setprio(1); _Pragma("unroll") for (int m = 0; m < 4; ++m) _Pragma("unroll") for (int n = 0; n < 2; ++n) _Pragma("unroll") for (int k = 0; k < 2; ++k) \
        acc[ai][bj][m][n] = __builtin_amdgcn_mfma_f32_16x16x32_bf16(Bt[n][k], At[m][k], acc[ai][bj][m][n], 0, 0, 0); __builtin_amdgcn_s_setprio(0); } while (0)
#define ENSURE(qn) do { if (__builtin_expect(ready <= (qn), 0)) ready = poll_ready(fx, fw, ready, (qn) + 1, skip, call, tmo); } while (0)
#define NEXTQ(dst) do { dst = qn; ++qn; if (qn == qskip) { ++qn; qskip += 7; } } while (0)
    f32x4 acc[2][2][4][2];
#pragma unroll
    for (int a = 0; a < 2; ++a)
#pragma unroll
        for (int b = 0; b < 2; ++b)
#pragma unroll
            for (int m = 0; m < 4; ++m)
#pragma unroll
                for (int n = 0; n < 2; ++n) acc[a][b][m][n] = (f32x4){0.f, 0.f, 0.f, 0.f};
    bf16x8 At[4][2], B0[2][2], B1[2][2];
    int ready = q0, qn = q0, qskip = (skip == 7) ? (1 << 20) : NH + skip;
    int q0a, q0b, q1a, q1b;
    NEXTQ(q0a); NEXTQ(q0b); NEXTQ(q1a); NEXTQ(q1b);
    ENSURE(q1b);
    STAMP(4);
    STAGE_B(0, 0, q0a, q0b); STAGE_A(0, 0, q0a, q0b); STAGE_B(0, 1, q0a, q0b); STAGE_A(0, 1, q0a, q0b);
    if (wr == 1) BAR;
    WAIT_V(4); BAR;
    STAGE_B(1, 0, q1a, q1b); STAGE_A(1, 0, q1a, q1b); STAGE_B(1, 1, q1a, q1b);
    WAIT_V(6); BAR;
    for (int t = 0; t < nt; t += 2) {
        if (t == nt - 2) { qn = q0; qskip = 1 << 20; }
        int q2a, q2b, q3a, q3b;
        NEXTQ(q2a); NEXTQ(q2b); NEXTQ(q3a); NEXTQ(q3b);
        LDB(B0, 0, 0); SCHED; LDA(At, 0, 0); STAGE_A(1, 1, q1a, q1b);
        WAIT_L(8); BAR; WAIT_L(0); MMA(0, 0, At, B0); BAR; SCHED;
        ENSURE(q2b);
        LDB(B1, 0, 1); STAGE_B(0, 0, q2a, q2b);
        BAR; WAIT_L(0); MMA(0, 1, At, B1); BAR;
        LDA(At, 0, 1); STAGE_A(0, 0, q2a, q2b);
        BAR; WAIT_L(0); MMA(1, 0, At, B0); BAR; SCHED;
        STAGE_B(0, 1, q2a, q2b);
        WAIT_V(6); BAR; MMA(1, 1, At, B1); BAR;
        LDB(B0, 1, 0); SCHED; LDA(At, 1, 0); STAGE_A(0, 1, q2a, q2b);
        WAIT_L(8); BAR; WAIT_L(0); MMA(0, 0, At, B0); BAR; SCHED;
        ENSURE(q3b);
        LDB(B1, 1, 1); STAGE_B(1, 0, q3a, q3b);
        BAR; WAIT_L(0); MMA(0, 1, At, B1); BAR;
        LDA(At, 1, 1); STAGE_A(1, 0, q3a, q3b);
        BAR; WAIT_L(0); MMA(1, 0, At, B0); BAR; SCHED;
        STAGE_B(1, 1, q3a, q3b);
        WAIT_V(6); BAR; MMA(1, 1, At, B1); BAR;
        q1a = q3a; q1b = q3b;
    }
    STAMP(2);
    int tz = threadIdx.x; asm volatile("" : "+v"(tz));
    const int ln = tz & 63;
    const unsigned vo = ln * 16;
    const unsigned tile = (unsigned)((rt & 1) * 64 + cn) * 8 + wid;
    if (main_tile) {
        if (wr == 0) BAR;
        const unsigned* gate = (skip == 7) ? cntp : (const unsigned*)(ws + OFF_SD) + (skip * 8 + xcd) * 32;
        unsigned gv = ld4_sys(ln == 1 ? tmo : gate);
        bool bad = (unsigned)__builtin_amdgcn_readlane(gv, 1) == call;
        const bool open = (unsigned)__builtin_amdgcn_readlane(gv, 0) >= call && !bad;
        const unsigned owner = (unsigned)(rt >> 1) & 7u;
        char* ows = p.ws[owner];
        const char* sl = ows + OFF_PL + (size_t)tile * UB;
        char* fl = ows + OFF_FP + (size_t)tile * 8;
        const char* s0 = sl + 4096; const char* s1 = sl + 8192;
        u32x4 b[16], pk[12];
#pragma unroll
        for (int bj = 0; bj < 2; ++bj)
#pragma unroll
            for (int n = 0; n < 2; ++n)
#pragma unroll
                for (int r = 0; r < 4; ++r)
#pragma unroll
                    for (int ai = 0; ai < 2; ++ai)
#pragma unroll
                        for (int m2 = 0; m2 < 2; ++m2)
                            b[bj * 8 + n * 4 + r][ai * 2 + m2] = cvt_pk_bf16(acc[ai][bj][2 * m2][n][r], acc[ai][bj][2 * m2 + 1][n][r]);
        unsigned hdr = pack_unit(b, pk, lst, ln, open ? s0 : nullptr);
        if (!open) {
            const unsigned long long t0 = __builtin_amdgcn_s_memrealtime();
            while (!bad && (unsigned)__builtin_amdgcn_readfirstlane(ld4_sys(gate)) < call) {
                __builtin_amdgcn_s_sleep(2);
                bad = give_up(tmo, call, t0);
            }
        }
        STAMP(6);
        if (bad) {
            hdr = RAWCNT << 16;
#pragma unroll
            for (int f = 0; f < 16; ++f) b[f] = (u32x4){0x7fc07fc0u, 0x7fc07fc0u, 0x7fc07fc0u, 0x7fc07fc0u};
        }
        if (__builtin_expect(hdr_cnt(hdr) == RAWCNT, 0)) { WAIT_V(0); st_raw(sl, vo, b); }
        else {
            if (!open) { st16<-4096>(s0, vo, pk[0]); WAIT_V(0); st16<-3072>(s0, vo, pk[1]); }
            WAIT_V(0); st_list(s1, vo, pk[11], hdr, ln); st16<0>(s1, vo, pk[8]);
            WAIT_V(0); st16<1024>(s1, vo, pk[9]);
            WAIT_V(0); st16<2048>(s1, vo, pk[10]);
            WAIT_V(0); st8_flag(fl, call, hdr | (1u << 28), ln); st16<-2048>(s0, vo, pk[2]);
            WAIT_V(0); st16<-1024>(s0, vo, pk[3]);
            WAIT_V(0); st8_flag(fl, call, hdr | (2u << 28), ln); st16<0>(s0, vo, pk[4]);
            WAIT_V(0); st16<1024>(s0, vo, pk[5]);
            WAIT_V(0); st8_flag(fl, call, hdr | (3u << 28), ln); st16<2048>(s0, vo, pk[6]);
            WAIT_V(0); st16<3072>(s0, vo, pk[7]);
        }
        WAIT_V(0);
        st8_flag(fl, call, hdr | (4u << 28), ln);
    } else {
        WAIT_V(0);
        if (wr == 0) BAR;
        BAR;
        LAS unsigned* wl = (LAS unsigned*)(lds + wid * HTB);
#pragma unroll
        for (int i = 0; i < 8; ++i) *(LAS u32x4*)(wl + i * 256 + ln * 4) = (u32x4){0u, 0u, 0u, 0u};
        const char* fl = ws + OFF_FP + (size_t)tile * 8;
        const char* sl = ws + OFF_PL + (size_t)tile * UB;
        unsigned hdr = wait_unit(fl, call, 1u, tmo);
        const unsigned cnt = hdr_cnt(hdr); const bool raw = cnt == RAWCNT;
        const unsigned LL = (hdr & 0xffffu) * 0x10001u;
        u32x4 r[16];
        if (!raw) { ld4x16(r[8], r[9], r[10], r[11], sl + 8192, vo); patch_apply(wl, r[11], cnt, ln, false); }
        const char* ob = p.out + ((size_t)((rt & 1) * 256 + wc * 32) * NN + cn * 256 + wr * 128) * 2;
        const unsigned oo = (ln >> 4) * (4 * NN * 2) + (ln & 15) * 16;
#pragma unroll
        for (int s = 0; s < 4; ++s) {
            if ((hdr >> 28) <= (unsigned)s) hdr = wait_unit(fl, call, s + 1, tmo);
            u32x4 o[4];
            unsigned tv;
            if (raw) { ld4x16(o[0], o[1], o[2], o[3], sl + s * 4096, vo); tv = ld4_sys(tmo); }
            else { ld2x16t(r[2 * s], r[2 * s + 1], tv, sl + s * 2048, vo, tmo); unpack_sub(r, s, wl, ln, LL, o); }
            const unsigned nanmask = (tv == call) ? 0x7fc07fc0u : 0u;
#pragma unroll
            for (int rr = 0; rr < 4; ++rr) {
                u32x4 y;
#pragma unroll
                for (int cc = 0; cc < 4; ++cc) {
                    const unsigned pv = o[rr][cc];
                    const f32x2 v = (f32x2){acc[cc >> 1][s >> 1][2 * (cc & 1)][s & 1][rr] + __uint_as_float(pv << 16),
                                            acc[cc >> 1][s >> 1][2 * (cc & 1) + 1][s & 1][rr] + __uint_as_float(pv & 0xffff0000u)};
                    const f32x2 g = gelu2(v);
                    y[cc] = cvt_pk_bf16(g.x, g.y) | nanmask;
                }
                st16<0>(ob + (size_t)((s >> 1) * 128 + (s & 1) * 16 + rr) * (NN * 2), oo, y);
            }
        }
    }
    STAMP(3);
}
}

extern "C" void kernel_launch(int rank, int nranks, ncclComm_t comm, hipStream_t stream,
                              void* const* d_in, const int* in_sizes, int n_in,
                              void* d_out, int out_size, const int* in_dtypes, int out_dtype,
                              void* const* d_out_all, void* const* d_ws_all, size_t ws_bytes) {
    (void)comm; (void)in_dtypes; (void)out_dtype; (void)d_out_all;
    if (nranks != NR || (unsigned)rank >= (unsigned)NR || n_in < 2 || in_sizes[0] != MM * KS || in_sizes[1] != KS * NN ||
        out_size != (MM / NR) * NN || ws_bytes < WS_NEED) {
        fprintf(stderr, "kernel_launch: unexpected problem geometry\n");
        return;
    }
    Params p;
    p.x = (const char*)d_in[0];
    p.w = (const char*)d_in[1];
    for (int i = 0; i < NR; ++i) p.ws[i] = (char*)d_ws_all[i];
    p.out = (char*)d_out;
    p.rank = rank; p.pad = 0;
    hipLaunchKernelGGL(fused_kernel, dim3(256), dim3(512), 0, stream, p);
    HIP_CHECK(hipGetLastError());
}
```

```cpp
#include <hip/hip_runtime.h>
#include <rccl/rccl.h>
#include <cstdio>
#include <cstdint>

#define HIP_CHECK(expr) do { hipError_t _e = (expr); if (_e != hipSuccess) { \
    fprintf(stderr, "HIP error %s at %s:%d\n", hipGetErrorString(_e), __FILE__, __LINE__); \
    return; } } while (0)

#define LAG 3
#define GLDS_AUX 17
#define FORCE_RAW 0
#define NUS 5
#define SWIN 6
#define RAW0 1
#define LATE_TK 1
#define KBAL 1
#define ASTAGE 0
#define NPK 3
#define GATE_EARLY 0

namespace {
#define LAS __attribute__((address_space(3)))
typedef short bf16x8 __attribute__((ext_vector_type(8)));
typedef float f32x4 __attribute__((ext_vector_type(4)));
typedef float f32x2 __attribute__((ext_vector_type(2)));
typedef unsigned u32x4 __attribute__((ext_vector_type(4)));
typedef unsigned u32x2 __attribute__((ext_vector_type(2)));
typedef unsigned short u16x2 __attribute__((ext_vector_type(2)));

constexpr int NR = 8;
constexpr int MM = 4096, NN = 16384, KS = 512, NS = 2048;
constexpr int NQ = 128;
constexpr int NH = 16;
constexpr int NU = NH * 22;
#define UNP7 1
constexpr int NUNP = UNP7 ? NUS * 64 / 7 * 7 : NUS * 64;
constexpr int NFMT = (9 - NUS) * 64;
constexpr unsigned long long DEADLINE = 20000000ull;
constexpr unsigned RAWCNT = 0x3ffu;

constexpr size_t OFF_CNT = 0;
constexpr size_t OFF_TMO = 8192;
constexpr size_t OFF_TK = 8192 + 1024;
constexpr size_t OFF_SD = 131072;
constexpr int FSTR = 160;
constexpr size_t OFF_FX = 16384;
constexpr size_t OFF_FW = 32768;
constexpr size_t OFF_FA = 81920;
constexpr size_t OFF_FP = 122880;
constexpr size_t OFF_STAMP = 262144;
constexpr size_t UB = 16384;
constexpr size_t XQ = 256 * 1024, WQ = 128 * 1024;
constexpr size_t OFF_XI = 1u << 20;
constexpr size_t OFF_WI = OFF_XI + (size_t)NQ * XQ;
constexpr size_t OFF_WD = OFF_WI + (size_t)NQ * WQ;
constexpr size_t OFF_PL = OFF_WD + (size_t)7 * NH * WQ;
constexpr size_t OFF_LZ = OFF_PL + (size_t)1024 * UB;
constexpr size_t LZ_PAR = (size_t)7 * NU * UB;
constexpr size_t WS_NEED = OFF_LZ + 2 * LZ_PAR;
static_assert(OFF_FW + 64 * FSTR * 4 <= OFF_FA && OFF_FA + 14 * NU * 8 <= OFF_FP && OFF_FP + 1024 * 8 <= OFF_SD && OFF_SD + 56 * 128 <= OFF_STAMP, "layout");
static_assert(WS_NEED <= (size_t)161 << 20, "workspace");

struct Params {
    const char* x;
    const char* w;
    char* ws[NR];
    char* out;
    int rank;
    int pad;
};

template <int OFF> __device__ __forceinline__ void st16(const char* sbase, unsigned voff, u32x4 v) {
    asm volatile("global_store_dwordx4 %0, %1, %2 offset:%3 sc0 sc1\n\ts_nop 1" :: "v"(voff), "v"(v), "s"(sbase), "i"(OFF) : "memory");
}
__device__ __forceinline__ unsigned ld4_sys(const void* p) {
    unsigned v;
    asm volatile("global_load_dword %0, %1, off sc0 sc1\n\ts_waitcnt vmcnt(0)" : "=&v"(v) : "v"(p) : "memory");
    return v;
}
__device__ __forceinline__ u32x2 ld8_sys(const void* p) {
    u32x2 v;
    asm volatile("global_load_dwordx2 %0, %1, off sc0 sc1\n\ts_waitcnt vmcnt(0)" : "=&v"(v) : "v"(p) : "memory");
    return v;
}
__device__ __forceinline__ void st8_flag(void* p, unsigned call, unsigned hdr, int lane) {
    if (lane == 0) __hip_atomic_store((unsigned long long*)p, ((unsigned long long)hdr << 32) | call, __ATOMIC_RELAXED, __HIP_MEMORY_SCOPE_SYSTEM);
}
#define WAIT_V_(n) asm volatile("s_waitcnt vmcnt(" #n ")" ::: "memory")
#define WAIT_V(n) WAIT_V_(n)
#define WAIT_L(n) asm volatile("s_waitcnt lgkmcnt(" #n ")" ::: "memory")
#define BAR __builtin_amdgcn_s_barrier()
#define SCHED __builtin_amdgcn_sched_barrier(0)

__device__ __forceinline__ unsigned cvt_pk_bf16(float lo, float hi) {
    unsigned r; asm("v_cvt_pk_bf16_f32 %0, %1, %2" : "=v"(r) : "v"(lo), "v"(hi)); return r;
}
__device__ __forceinline__ f32x2 gelu2(f32x2 y) {
    const float k1 = -2.0f * 0.7978845608028654f * 1.4426950408889634f, k2 = k1 * 0.044715f;
    f32x2 z = y * (y * y * k2 + k1);
    f32x2 e; e.x = __builtin_amdgcn_exp2f(z.x); e.y = __builtin_amdgcn_exp2f(z.y);
    e = e + 1.0f;
    f32x2 r; r.x = __builtin_amdgcn_rcpf(e.x); r.y = __builtin_amdgcn_rcpf(e.y);
    return y * r;
}
__device__ __forceinline__ unsigned pk_sub(unsigned a, unsigned b) { return __builtin_bit_cast(unsigned, __builtin_bit_cast(u16x2, a) - __builtin_bit_cast(u16x2, b)); }
__device__ __forceinline__ unsigned pk_add(unsigned a, unsigned b) { return __builtin_bit_cast(unsigned, __builtin_bit_cast(u16x2, a) + __builtin_bit_cast(u16x2, b)); }
__device__ __forceinline__ unsigned pk_max(unsigned a, unsigned b) { return __builtin_bit_cast(unsigned, __builtin_elementwise_max(__builtin_bit_cast(u16x2, a), __builtin_bit_cast(u16x2, b))); }
__device__ __forceinline__ unsigned bfi(unsigned m, unsigned a, unsigned b) { unsigned r; asm("v_bfi_b32 %0, %1, %2, %3" : "=v"(r) : "s"(m), "v"(a), "v"(b)); return r; }
__device__ __forceinline__ unsigned rotl(unsigned a, int n) { return __builtin_amdgcn_alignbit(a, a, 32 - n); }
__device__ __forceinline__ unsigned rotr(unsigned a, int n) { return __builtin_amdgcn_alignbit(a, a, n); }
__device__ __forceinline__ unsigned umax(unsigned a, unsigned b) { return a > b ? a : b; }

#define STAMP(slot) do {} while (0)

__device__ __forceinline__ bool give_up(unsigned* tmo, unsigned call, unsigned long long t0) {
    if (__builtin_amdgcn_readfirstlane(__hip_atomic_load(tmo, __ATOMIC_RELAXED, __HIP_MEMORY_SCOPE_AGENT)) == call ||
        __builtin_amdgcn_s_memrealtime() - t0 > DEADLINE) {
        __hip_atomic_store(tmo, call, __ATOMIC_RELAXED, __HIP_MEMORY_SCOPE_AGENT);
        return true;
    }
    return false;
}
__device__ __forceinline__ unsigned wait_unit(const char* flag, unsigned call, unsigned st, unsigned* tmo) {
    const unsigned long long t0 = __builtin_amdgcn_s_memrealtime();
    for (;;) {
        const u32x2 v = ld8_sys(flag);
        const unsigned hdr = __builtin_amdgcn_readfirstlane(v.y);
        if ((unsigned)__builtin_amdgcn_readfirstlane(v.x) == call && (hdr >> 28) >= st) return hdr;
        __builtin_amdgcn_s_sleep(1);
        if (give_up(tmo, call, t0)) return (RAWCNT << 16) | (4u << 28);
    }
}
__device__ __forceinline__ unsigned hdr_cnt(unsigned hdr) { return (hdr >> 16) & 0x3ffu; }

constexpr int SCR_DENSE = 36 * 256, SCR_LIST = 0;
__device__ __forceinline__ unsigned pack_unit(const u32x4 (&b)[16], u32x4 (&pk)[12], LAS unsigned char* scr, int lane, const char* early) {
    unsigned mx = 0;
#pragma unroll
    for (int f = 0; f < 16; ++f) mx = pk_max(mx, b[f][f & 3] & 0x7fff7fffu);
    unsigned m = umax(mx & 0xffffu, mx >> 16);
    m = umax(m, (unsigned)__builtin_amdgcn_update_dpp(0, (int)m, 0x128, 0xf, 0xf, false));
    m = umax(m, (unsigned)__builtin_amdgcn_update_dpp(0, (int)m, 0x124, 0xf, 0xf, false));
    m = umax(m, (unsigned)__builtin_amdgcn_update_dpp(0, (int)m, 0x122, 0xf, 0xf, false));
    m = umax(m, (unsigned)__builtin_amdgcn_update_dpp(0, (int)m, 0x121, 0xf, 0xf, false));
    const unsigned ms = umax(umax(__builtin_amdgcn_readlane(m, 0), __builtin_amdgcn_readlane(m, 16)), umax(__builtin_amdgcn_readlane(m, 32), __builtin_amdgcn_readlane(m, 48)));
    const unsigned L = umax(ms, 1023u) - 1023u, LL = L | (L << 16);
    const unsigned wp0 = (unsigned)(size_t)scr + lane * 4;
    unsigned wp = wp0;
#pragma unroll
    for (int s = 0; s < 4; ++s) {
        unsigned hb[8];
#pragma unroll
        for (int jj = 0; jj < 8; ++jj) {
            const int j = 8 * s + jj;
            const unsigned d0 = pk_sub(b[j >> 1][2 * (j & 1)], LL), d1 = pk_sub(b[j >> 1][2 * (j & 1) + 1], LL);
            pk[j >> 2][j & 3] = __builtin_amdgcn_perm(d1, d0, 0x06040200u);
            hb[jj] = __builtin_amdgcn_perm(d1, d0, 0x07050301u);
            const unsigned t = hb[jj] & 0x7c7c7c7cu;
            *(LAS unsigned*)(size_t)wp = t | (unsigned)((j & 3) | ((j >> 2) << 7));
            unsigned one; asm("v_min_u32 %0, 1, %1" : "=v"(one) : "v"(t));
            wp += one << 8;
            if (j == 3 && early) st16<-4096>(early, lane * 16, pk[0]);
            if (j == 19 && early) { WAIT_V(0); st16<-3072>(early, lane * 16, pk[1]); }
        }
        pk[8][s] = bfi(0x83838383u, hb[0], bfi(0x1c1c1c1cu, rotl(hb[1], 3), hb[6] << 5));
        pk[9][s] = bfi(0x83838383u, hb[2], bfi(0x1c1c1c1cu, rotl(hb[3], 3), hb[7] << 5));
        pk[10][s] = bfi(0x83838383u, hb[4], bfi(0x1c1c1c1cu, rotl(hb[5], 3), bfi(0x20202020u, hb[6] >> 2, hb[7] >> 1)));
    }
    const unsigned nrow = (wp - wp0) >> 8;
    unsigned inc = nrow;
    inc += __builtin_amdgcn_update_dpp(0, (int)inc, 0x111, 0xf, 0xf, true);
    inc += __builtin_amdgcn_update_dpp(0, (int)inc, 0x112, 0xf, 0xf, true);
    inc += __builtin_amdgcn_update_dpp(0, (int)inc, 0x114, 0xf, 0xf, true);
    inc += __builtin_amdgcn_update_dpp(0, (int)inc, 0x118, 0xf, 0xf, true);
    inc += __builtin_amdgcn_update_dpp(0, (int)inc, 0x142, 0xa, 0xf, false);
    inc += __builtin_amdgcn_update_dpp(0, (int)inc, 0x143, 0xc, 0xf, false);
    const unsigned ngrp = __builtin_amdgcn_readlane(inc, 63);
    unsigned cnt = ngrp, bad = 0;
    if (ngrp <= 512u) {
        LAS unsigned* dense = (LAS unsigned*)(scr + SCR_DENSE);
        const unsigned lanebits = ((lane & 7) << 15) | ((lane >> 3) << 23), base = inc - nrow;
        for (unsigned r0 = 0; __any(nrow > r0); r0 += 4) {
            unsigned e[4];
#pragma unroll
            for (int i = 0; i < 4; ++i) e[i] = *(LAS unsigned*)(scr + (r0 + i) * 256 + lane * 4);
#pragma unroll
            for (int i = 0; i < 4; ++i) dense[r0 + i < nrow ? base + r0 + i : 512 + lane] = e[i] | lanebits;
        }
        LAS unsigned short* lst = (LAS unsigned short*)(scr + SCR_LIST);
        const unsigned hz = ((0u - L) >> 10) & 31u;
        for (unsigned i0 = 0; i0 < ngrp; i0 += 64) {
            const unsigned e = dense[i0 + lane];
            unsigned t = i0 + lane < ngrp ? e & 0x7c7c7c7cu : 0u;
            const unsigned pos = ((e & 0x380u) << 3) | ((e >> 16) & 0x380u) | ((e >> 11) & 0x70u) | ((e & 3u) << 2);
            unsigned at = i0 + lane < ngrp ? i0 + lane : 576u + lane;
            for (;;) {
                const unsigned b8 = __builtin_ctz(t | 0x80000000u) & 24u, hi5 = (t >> (b8 + 2)) & 31u;
                const unsigned code = hi5 == hz ? 5u : (hi5 + 5u) & 31u;
                bad = umax(bad, code);
                lst[at] = (unsigned short)((code << 13) | pos | (b8 >> 3));
                t &= ~(0xffu << b8);
                const unsigned long long bal = __ballot(t != 0);
                if (!bal) break;
                at = t != 0 ? __builtin_amdgcn_mbcnt_hi((unsigned)(bal >> 32), __builtin_amdgcn_mbcnt_lo((unsigned)bal, cnt < 512u ? cnt : 512u)) : 576u + lane;
                cnt += __builtin_popcountll(bal);
            }
        }
    }
    pk[11] = *(LAS u32x4*)(scr + SCR_LIST + lane * 16);
    if (cnt > 512u || __any(bad > 7u) || FORCE_RAW) cnt = RAWCNT;
    return L | (cnt << 16);
}
__device__ __forceinline__ void st_list(const char* sb8k  , unsigned voff, u32x4 v, unsigned hdr, int lane) {
    const unsigned nl = umax((hdr_cnt(hdr) + 31u) >> 5, 1u) << 2;
    if ((unsigned)lane < nl) st16<3072>(sb8k, voff, v);
}
__device__ __forceinline__ void st_raw(const char* slot, unsigned voff, const u32x4 (&b)[16]) {
    const char* s0 = slot + 4096; const char* s1 = slot + 12288;
    st16<-4096>(s0, voff, b[0]); st16<-3072>(s0, voff, b[1]); st16<-2048>(s0, voff, b[2]); st16<-1024>(s0, voff, b[3]);
    st16<0>(s0, voff, b[4]); st16<1024>(s0, voff, b[5]); st16<2048>(s0, voff, b[6]); st16<3072>(s0, voff, b[7]);
    st16<-4096>(s1, voff, b[8]); st16<-3072>(s1, voff, b[9]); st16<-2048>(s1, voff, b[10]); st16<-1024>(s1, voff, b[11]);
    st16<0>(s1, voff, b[12]); st16<1024>(s1, voff, b[13]); st16<2048>(s1, voff, b[14]); st16<3072>(s1, voff, b[15]);
}
__device__ __forceinline__ void ld_unit(u32x4 (&r)[16], const char* slot, unsigned voff, bool raw) {
    const char* s0 = slot + 4096; const char* s1 = slot + 12288;
    r[12] = r[13] = r[14] = r[15] = (u32x4){0u, 0u, 0u, 0u};
    if (raw) {
        asm volatile("global_load_dwordx4 %0, %4, %5 sc0 sc1\n\tglobal_load_dwordx4 %1, %4, %5 offset:1024 sc0 sc1\n\t"
                     "global_load_dwordx4 %2, %4, %5 offset:2048 sc0 sc1\n\tglobal_load_dwordx4 %3, %4, %5 offset:3072 sc0 sc1\n\ts_waitcnt vmcnt(0)"
                     : "=&v"(r[12]), "=&v"(r[13]), "=&v"(r[14]), "=&v"(r[15]) : "v"(voff), "s"(s1) : "memory");
    }
    asm volatile("global_load_dwordx4 %11, %12, %14 offset:-1024 sc0 sc1\n\t"
                 "global_load_dwordx4 %8, %12, %14 offset:-4096 sc0 sc1\n\tglobal_load_dwordx4 %9, %12, %14 offset:-3072 sc0 sc1\n\t"
                 "global_load_dwordx4 %10, %12, %14 offset:-2048 sc0 sc1\n\t"
                 "global_load_dwordx4 %0, %12, %13 offset:-4096 sc0 sc1\n\tglobal_load_dwordx4 %1, %12, %13 offset:-3072 sc0 sc1\n\t"
                 "global_load_dwordx4 %2, %12, %13 offset:-2048 sc0 sc1\n\tglobal_load_dwordx4 %3, %12, %13 offset:-1024 sc0 sc1\n\t"
                 "global_load_dwordx4 %4, %12, %13 sc0 sc1\n\tglobal_load_dwordx4 %5, %12, %13 offset:1024 sc0 sc1\n\t"
                 "global_load_dwordx4 %6, %12, %13 offset:2048 sc0 sc1\n\tglobal_load_dwordx4 %7, %12, %13 offset:3072 sc0 sc1\n\t"
                 "s_waitcnt vmcnt(0)"
                 : "=&v"(r[0]), "=&v"(r[1]), "=&v"(r[2]), "=&v"(r[3]), "=&v"(r[4]), "=&v"(r[5]), "=&v"(r[6]), "=&v"(r[7]),
                   "=&v"(r[8]), "=&v"(r[9]), "=&v"(r[10]), "=&v"(r[11])
                 : "v"(voff), "s"(s0), "s"(s1) : "memory");
}
__device__ __forceinline__ void ld4x16(u32x4& a, u32x4& b, u32x4& c, u32x4& d, const char* sb, unsigned voff) {
    asm volatile("global_load_dwordx4 %0, %4, %5 sc0 sc1\n\tglobal_load_dwordx4 %1, %4, %5 offset:1024 sc0 sc1\n\t"
                 "global_load_dwordx4 %2, %4, %5 offset:2048 sc0 sc1\n\tglobal_load_dwordx4 %3, %4, %5 offset:3072 sc0 sc1\n\ts_waitcnt vmcnt(0)"
                 : "=&v"(a), "=&v"(b), "=&v"(c), "=&v"(d) : "v"(voff), "s"(sb) : "memory");
}
__device__ __forceinline__ void ld2x16t(u32x4& a, u32x4& b, unsigned& t, const char* sb, unsigned voff, const unsigned* tp) {
    asm volatile("global_load_dwordx4 %0, %3, %4 sc0 sc1\n\tglobal_load_dwordx4 %1, %3, %4 offset:1024 sc0 sc1\n\t"
                 "global_load_dword %2, %5, off sc1\n\ts_waitcnt vmcnt(0)"
                 : "=&v"(a), "=&v"(b), "=&v"(t) : "v"(voff), "s"(sb), "v"(tp) : "memory");
}
__device__ __forceinline__ void patch_apply(LAS unsigned* pat, u32x4 list, unsigned hdr, int lane, bool zero) {
    const unsigned cnt = hdr_cnt(hdr), lut_hi = 0x0804007cu | ((((0u - (hdr & 0xffffu)) >> 10) & 31u) << 10);
    if ((unsigned)lane * 8 < cnt) {
#pragma unroll
        for (int c = 0; c < 8; ++c) {
            const unsigned e = (c & 1) ? list[c >> 1] >> 16 : list[c >> 1] & 0xffffu;
            if ((unsigned)(lane * 8 + c) < cnt)
                ((LAS unsigned char*)pat)[e & 0x1fffu] = zero ? 0 : (unsigned char)__builtin_amdgcn_perm(lut_hi, 0x7874706cu, e >> 13);
        }
    }
}
__device__ __forceinline__ void unpack_sub(const u32x4 (&r)[16], int s, LAS unsigned* pat, int lane, unsigned LL, u32x4 (&o)[4]) {
    const u32x4 p0 = *(LAS u32x4*)(pat + (2 * s) * 256 + lane * 4), p1 = *(LAS u32x4*)(pat + (2 * s + 1) * 256 + lane * 4);
    const unsigned A = r[8][s], B = r[9][s], C = r[10][s];
    unsigned hb[8];
    hb[0] = bfi(0x83838383u, A, p0[0]); hb[1] = bfi(0x83838383u, rotr(A, 3), p0[1]);
    hb[2] = bfi(0x83838383u, B, p0[2]); hb[3] = bfi(0x83838383u, rotr(B, 3), p0[3]);
    hb[4] = bfi(0x83838383u, C, p1[0]); hb[5] = bfi(0x83838383u, rotr(C, 3), p1[1]);
    hb[6] = bfi(0x03030303u, A >> 5, bfi(0x80808080u, C << 2, p1[2]));
    hb[7] = bfi(0x03030303u, B >> 5, bfi(0x80808080u, C << 1, p1[3]));
#pragma unroll
    for (int jj = 0; jj < 8; ++jj) {
        const unsigned lo = r[2 * s + (jj >> 2)][jj & 3];
        o[jj >> 1][2 * (jj & 1)] = pk_add(__builtin_amdgcn_perm(hb[jj], lo, 0x05010400u), LL);
        o[jj >> 1][2 * (jj & 1) + 1] = pk_add(__builtin_amdgcn_perm(hb[jj], lo, 0x07030602u), LL);
    }
}

__device__ __forceinline__ void unit_fetch(const Params& p, int h, int u, int lane, unsigned ldsbuf) {
    const bool isx = u < 16;
    const char* base = isx ? p.x + ((size_t)u * 256 * KS + h * 32) * 2 : p.w + ((size_t)h * 32 * NN + (u - 16) * 256) * 2;
    unsigned voff = isx ? (lane & 15) * (KS * 2) + (lane >> 4) * 16 : (lane >> 4) * (8 * NN * 2) + (lane & 15) * 16;
    const unsigned s1 = isx ? 16 * KS * 2 : NN * 2, s2 = isx ? 16 * KS * 2 : 256 - 7 * NN * 2;
#pragma unroll
    for (int i = 0; i < 16; ++i) {
        unsigned keep;
        asm volatile("s_mov_b32 %0, m0\n\ts_mov_b32 m0, %3\n\ts_nop 0\n\tglobal_load_lds_dwordx4 %1, %2\n\ts_mov_b32 m0, %0"
                     : "=&s"(keep) : "v"(voff), "s"(base), "s"(ldsbuf + i * 1024) : "memory");
        voff += (i == 7) ? s2 : s1;
    }
}
__device__ __forceinline__ void unit_read(LAS unsigned char* buf, int lane, u32x4 (&r)[16]) {
#pragma unroll
    for (int i = 0; i < 16; ++i) r[i] = *(LAS u32x4*)(buf + i * 1024 + lane * 16);
    asm volatile("s_waitcnt lgkmcnt(0)" : "+v"(r[0]), "+v"(r[1]), "+v"(r[2]), "+v"(r[3]), "+v"(r[4]), "+v"(r[5]), "+v"(r[6]), "+v"(r[7]),
                 "+v"(r[8]), "+v"(r[9]), "+v"(r[10]), "+v"(r[11]), "+v"(r[12]), "+v"(r[13]), "+v"(r[14]), "+v"(r[15]) :: "memory");
}
__device__ __forceinline__ void unit_format(int u, u32x4 (&r)[16]) {
    if (u >= 16) {
#pragma unroll
        for (int g = 0; g < 2; ++g) {
            u32x4 o[8];
#pragma unroll
            for (int i = 0; i < 8; ++i)
#pragma unroll
                for (int a = 0; a < 4; ++a)
                    o[i][a] = __builtin_amdgcn_perm(r[g * 8 + 2 * a + 1][i >> 1], r[g * 8 + 2 * a][i >> 1], (i & 1) ? 0x07060302u : 0x05040100u);
#pragma unroll
            for (int i = 0; i < 8; ++i) r[g * 8 + i] = o[i];
        }
    }
}
__device__ __forceinline__ int order(int k, int src) {
    int h, v;
    if (k < LAG * 15) { h = k / 15; v = k - h * 15; }
    else if (k < LAG * 15 + (NH - LAG) * 22) { const int k2 = k - LAG * 15; const int rho = k2 / 22; v = k2 - rho * 22; h = v >= 15 ? rho : rho + LAG; }
    else { const int k2 = k - LAG * 15 - (NH - LAG) * 22; const int rho = k2 / 7; v = 15 + k2 - rho * 7; h = rho + NH - LAG; }
    const int odd = v >= 15, m = (src + 1 + v - (odd ? 15 : 8)) & 7;
    return (h << 8) | (v < 8 ? 16 + v : 2 * m + odd);
}

constexpr int PK_LDS = 40 * 1024;
struct SendCtl { u32x4 desc[8]; unsigned tail; unsigned freed[3]; };
__device__ __forceinline__ void packer(const Params& p, LAS unsigned char* buf, LAS SendCtl* ctl, int xcd, int di, int mem, int lane, unsigned* cntp) {
    const unsigned dest = (unsigned)(p.rank + 1 + di) & 7u;
    const unsigned voff = lane * 16, lbuf = (unsigned)(size_t)buf;
    unsigned k = (RAW0 + mem) * 8 + xcd;
    int o = order(k, p.rank), h = o >> 8, u = (o & 255) + ((o & 16) ? dest * 8 : 0);
    unit_fetch(p, h, u, lane, lbuf);
    const unsigned call = __builtin_amdgcn_readfirstlane(ld4_sys(cntp)) + 1u;
    unsigned* tk = (unsigned*)(p.ws[p.rank] + OFF_TK) + di * 32;
    const unsigned tk0 = (call - 1u) * (NU - 8 * RAW0) - (RAW0 + NPK) * 8;
    unsigned nput = 0;
    for (;;) {
        WAIT_V(0);
        u32x4 r[16], pk[12];
        unit_read(buf, lane, r);
        unit_format(u, r);
        const unsigned hdr = pack_unit(r, pk, buf + 28672, lane, nullptr) | (4u << 28);
        asm volatile("" : "+v"(pk[0]), "+v"(pk[1]), "+v"(pk[2]), "+v"(pk[3]), "+v"(pk[4]), "+v"(pk[5]), "+v"(pk[6]), "+v"(pk[7]), "+v"(pk[8]), "+v"(pk[9]), "+v"(pk[10]), "+v"(pk[11]));
        while ((unsigned)__builtin_amdgcn_readfirstlane(*(volatile LAS unsigned*)&ctl->freed[mem]) != nput) __builtin_amdgcn_s_sleep(1);
        ++nput;
        if (__builtin_expect(hdr_cnt(hdr) == RAWCNT, 0)) {
            unit_read(buf, lane, r);
            unit_format(u, r);
            st_raw(p.ws[dest] + OFF_LZ + (call & 1u) * LZ_PAR + (size_t)((6 - di) * NU + k) * UB, voff, r);
            WAIT_V(0);
        } else {
#pragma unroll
            for (int i = 0; i < 12; ++i) *(LAS u32x4*)(buf + 16384 + i * 1024 + lane * 16) = pk[i];
        }
        asm volatile("" ::: "memory");
        unsigned tn = 1u;
        if (lane == 0) {
            const unsigned pos = __hip_atomic_fetch_add(&ctl->tail, 1u, __ATOMIC_RELAXED, __HIP_MEMORY_SCOPE_WORKGROUP);
            *(volatile LAS u32x4*)&ctl->desc[pos & 7] = (u32x4){pos + 1u, k, hdr, (unsigned)mem | (nput << 8)};
            asm volatile("global_atomic_add %0, %1, %0, off sc0\n\ts_waitcnt vmcnt(0)" : "+v"(tn) : "v"(tk) : "memory");
        }
        k = __builtin_amdgcn_readfirstlane(tn) - tk0;
        if (k >= (unsigned)NU) break;
        o = order(k, p.rank); h = o >> 8; u = (o & 255) + ((o & 16) ? dest * 8 : 0);
        unit_fetch(p, h, u, lane, lbuf);
    }
    if (lane == 0) {
        const unsigned pos = __hip_atomic_fetch_add(&ctl->tail, 1u, __ATOMIC_RELAXED, __HIP_MEMORY_SCOPE_WORKGROUP);
        *(volatile LAS u32x4*)&ctl->desc[pos & 7] = (u32x4){pos + 1u, 0u, 0u, 0x80u};
    }
}
__device__ __forceinline__ void storer(const Params& p, LAS unsigned char* lds, LAS SendCtl* ctl, int xcd, int di, int lane, unsigned* cntp) {
    const unsigned dest = (unsigned)(p.rank + 1 + di) & 7u;
    char* dws = p.ws[dest];
    const int li = 6 - di;
    const unsigned voff = lane * 16;
    if (RAW0) unit_fetch(p, 0, 16 + dest * 8 + xcd, lane, (unsigned)(size_t)(lds + 3 * PK_LDS));
    const unsigned call = __builtin_amdgcn_readfirstlane(ld4_sys(cntp)) + 1u;
    const char* lz = dws + OFF_LZ + (call & 1u) * LZ_PAR + (size_t)li * NU * UB;
    char* fa = dws + OFF_FA + (size_t)((call & 1u) * 7 + li) * NU * 8;
    unsigned pos = 0, done = 0, pend_hdr = 0; int pend = -1;
    if (RAW0) {
        u32x4 r[16];
        unit_read(lds + 3 * PK_LDS, lane, r);
        unit_format(16, r);
        STAMP(5);
        const char* s0 = lz + (size_t)xcd * UB + 4096; const char* s1 = s0 + 8192;
        st16<-4096>(s0, voff, r[0]); st16<-3072>(s0, voff, r[1]); st16<-2048>(s0, voff, r[2]); st16<-1024>(s0, voff, r[3]);
        st16<0>(s0, voff, r[4]); st16<1024>(s0, voff, r[5]); WAIT_V(SWIN); st16<2048>(s0, voff, r[6]); WAIT_V(SWIN); st16<3072>(s0, voff, r[7]); WAIT_V(SWIN);
        st16<-4096>(s1, voff, r[8]); WAIT_V(SWIN); st16<-3072>(s1, voff, r[9]); WAIT_V(SWIN); st16<-2048>(s1, voff, r[10]); WAIT_V(SWIN); st16<-1024>(s1, voff, r[11]); WAIT_V(SWIN);
        st16<0>(s1, voff, r[12]); WAIT_V(SWIN); st16<1024>(s1, voff, r[13]); WAIT_V(SWIN); st16<2048>(s1, voff, r[14]); WAIT_V(SWIN); st16<3072>(s1, voff, r[15]); WAIT_V(SWIN);
        pend = xcd; pend_hdr = (RAWCNT << 16) | (4u << 28);
    }
    while (done < NPK) {
        const u32x4 d = *(volatile LAS u32x4*)&ctl->desc[pos & 7];
        if ((unsigned)__builtin_amdgcn_readfirstlane(d.x) != pos + 1u) {
            if (pend >= 0) { WAIT_V(0); st8_flag(fa + pend * 8, call, pend_hdr, lane); pend = -1; }
            __builtin_amdgcn_s_sleep(1);
            continue;
        }
        const unsigned k = __builtin_amdgcn_readfirstlane(d.y), hdr = __builtin_amdgcn_readfirstlane(d.z), w = __builtin_amdgcn_readfirstlane(d.w);
        ++pos;
        if (w & 0x80u) { ++done; continue; }
        if (k == NU - 1 - GATE_EARLY && lane < 8)
            __hip_atomic_store((unsigned*)(p.ws[p.rank] + OFF_SD) + (di * 8 + lane) * 32, call, __ATOMIC_RELAXED, __HIP_MEMORY_SCOPE_AGENT);
        if (__builtin_expect(hdr_cnt(hdr) == RAWCNT, 0)) {
            st8_flag(fa + k * 8, call, hdr, lane);
            *(volatile LAS unsigned*)&ctl->freed[w & 3u] = w >> 8;
            continue;
        }
        u32x4 pk[12];
        LAS unsigned char* src = lds + (w & 3u) * PK_LDS + 16384 + lane * 16;
#pragma unroll
        for (int i = 0; i < 12; ++i) pk[i] = *(LAS u32x4*)(src + i * 1024);
        asm volatile("s_waitcnt lgkmcnt(0)" : "+v"(pk[0]), "+v"(pk[1]), "+v"(pk[2]), "+v"(pk[3]), "+v"(pk[4]), "+v"(pk[5]), "+v"(pk[6]), "+v"(pk[7]), "+v"(pk[8]), "+v"(pk[9]), "+v"(pk[10]), "+v"(pk[11]) :: "memory");
        *(volatile LAS unsigned*)&ctl->freed[w & 3u] = w >> 8;
        const char* s0 = lz + (size_t)k * UB + 4096; const char* s1 = s0 + 4096;
#define PEND_FLAG(i) do { if (SWIN == (i) && pend >= 0) st8_flag(fa + pend * 8, call, pend_hdr, lane); } while (0)
        st_list(s1, voff, pk[11], hdr, lane); WAIT_V(SWIN);
        st16<-4096>(s0, voff, pk[0]); WAIT_V(SWIN);
        st16<-3072>(s0, voff, pk[1]); WAIT_V(SWIN); PEND_FLAG(3);
        st16<-2048>(s0, voff, pk[2]); WAIT_V(SWIN); PEND_FLAG(4);
        st16<-1024>(s0, voff, pk[3]); WAIT_V(SWIN); PEND_FLAG(5);
        st16<0>(s0, voff, pk[4]); WAIT_V(SWIN); PEND_FLAG(6);
        st16<1024>(s0, voff, pk[5]); WAIT_V(SWIN); PEND_FLAG(7);
        st16<2048>(s0, voff, pk[6]); WAIT_V(SWIN); PEND_FLAG(8);
        st16<3072>(s0, voff, pk[7]); WAIT_V(SWIN); PEND_FLAG(9);
        st16<0>(s1, voff, pk[8]); WAIT_V(SWIN);
        st16<1024>(s1, voff, pk[9]); WAIT_V(SWIN);
        st16<2048>(s1, voff, pk[10]); WAIT_V(SWIN);
        pend = k; pend_hdr = hdr;
    }
    WAIT_V(0);
    if (pend >= 0) st8_flag(fa + pend * 8, call, pend_hdr, lane);
}

__device__ __forceinline__ void unpacker(const Params& p, LAS unsigned* pat, int uw, int lane, unsigned call) {
    char* ws = p.ws[p.rank];
    unsigned* tmo = (unsigned*)(ws + OFF_TMO);
    const unsigned voff = lane * 16;
#pragma unroll
    for (int i = 0; i < 8; ++i) *(LAS u32x4*)(pat + i * 256 + lane * 4) = (u32x4){0u, 0u, 0u, 0u};
    for (int n = uw; n < 7 * NU; n += NUNP) {
        const int k = n / 7, li = n - k * 7, src = (p.rank + 1 + li) & 7;
        const int o = order(k, src), t = o & 15; const bool isx = !(o & 16);
        const int q = NH + (o >> 8) * 7 + li;
        const unsigned hdr = wait_unit(ws + OFF_FA + (size_t)(((call & 1u) * 7 + li) * NU + k) * 8, call, 4u, tmo);
        const unsigned cnt = hdr_cnt(hdr); const bool raw = cnt == RAWCNT;
        u32x4 r[16];
        ld_unit(r, ws + OFF_LZ + (call & 1u) * LZ_PAR + (size_t)(li * NU + k) * UB, voff, raw);
        const char* dst = ws + (isx ? OFF_XI + (size_t)q * XQ : OFF_WI + (size_t)q * WQ) + (size_t)t * UB;
        unsigned* flag = (unsigned*)(ws + (isx ? OFF_FX : OFF_FW)) + (isx ? t : p.rank * 8 + t) * FSTR + q;
        if (__builtin_expect(raw, 0)) st_raw(dst, voff, r);
        else {
            const unsigned LL = (hdr & 0xffffu) * 0x10001u;
            patch_apply(pat, r[11], hdr, lane, false);
#pragma unroll
            for (int s = 0; s < 4; ++s) {
                u32x4 o[4]; unpack_sub(r, s, pat, lane, LL, o);
                const char* sb = dst + s * 4096;
                st16<0>(sb, voff, o[0]); st16<1024>(sb, voff, o[1]); st16<2048>(sb, voff, o[2]); st16<3072>(sb, voff, o[3]);
            }
            patch_apply(pat, r[11], hdr, lane, true);
        }
        WAIT_V(0);
        if (lane == 0) __hip_atomic_store(flag, call, __ATOMIC_RELAXED, __HIP_MEMORY_SCOPE_SYSTEM);
    }
}

__device__ __forceinline__ void formatter(const Params& p, LAS unsigned char* buf, int fw, int n2nd, int lane, unsigned call) {
    char* ws = p.ws[p.rank];
    const unsigned voff = lane * 16;
    for (int n = fw; n < 1280; n = (n == fw) ? n2nd : n + NFMT) {
        int h, u; const char* dst; unsigned* flag;
        if (n < 384) { h = n / 24; const int v = n - h * 24;
            if (v < 16) { u = v; dst = ws + OFF_XI + (size_t)h * XQ + (size_t)v * UB; flag = (unsigned*)(ws + OFF_FX) + v * FSTR + h; }
            else { u = p.rank * 8 + v; dst = ws + OFF_WI + (size_t)h * WQ + (size_t)(v - 16) * UB; flag = (unsigned*)(ws + OFF_FW) + (u - 16) * FSTR + h; } }
        else { const int n2 = n - 384; h = n2 / 56; const int v = n2 - h * 56, li = v >> 3, cn = ((p.rank + 1 + li) & 7) * 8 + (v & 7);
            u = 16 + cn; dst = ws + OFF_WD + (size_t)(li * NH + h) * WQ + (size_t)(v & 7) * UB; flag = (unsigned*)(ws + OFF_FW) + cn * FSTR + h; }
        u32x4 r[16];
        unit_fetch(p, h, u, lane, (unsigned)(size_t)buf);
        WAIT_V(0);
        unit_read(buf, lane, r);
        unit_format(u, r);
        st_raw(dst, voff, r);
        WAIT_V(0);
        if (lane == 0) __hip_atomic_store(flag, call, __ATOMIC_RELAXED, __HIP_MEMORY_SCOPE_SYSTEM);
    }
}

__device__ __forceinline__ int poll_ready(const unsigned* fx, const unsigned* fw, int ready, int need, int skip, unsigned call, unsigned* tmo) {
    const int lane = __builtin_amdgcn_mbcnt_hi(~0u, __builtin_amdgcn_mbcnt_lo(~0u, 0u));
    const unsigned long long t0 = __builtin_amdgcn_s_memrealtime();
    for (;;) {
        const int q = ready + (lane & 31);
        const unsigned v = ld4_sys((lane < 32 ? fx : fw) + q);
        const bool skipped = lane < 32 && q >= NH && (q - NH) % 7 == skip;
        const unsigned long long bal = __ballot(q >= NQ || v >= call || skipped);
        const unsigned both = (unsigned)bal & (unsigned)(bal >> 32);
        ready += (both == ~0u) ? 32 : __builtin_ctz(~both);
        if (ready >= need) break;
        if (give_up(tmo, call, t0)) { ready = 1 << 20; break; }
        __builtin_amdgcn_s_sleep(1);
    }
    return ready;
}

constexpr int HTB = 16384;

__global__ __launch_bounds__(512, 2) void fused_kernel(const Params p) {
    __shared__ __attribute__((aligned(16))) unsigned char shm[8 * HTB + 8192 + 256];
    LAS unsigned char* lds = (LAS unsigned char*)shm;
    const int wid = __builtin_amdgcn_readfirstlane(threadIdx.x >> 6), lane = threadIdx.x & 63;
    const int xcd = blockIdx.x & 7, slot = blockIdx.x >> 3;
    char* ws = p.ws[p.rank];
    unsigned* tmo = (unsigned*)(ws + OFF_TMO);
    unsigned* cntp = (unsigned*)(ws + OFF_CNT) + blockIdx.x * 8 + wid;
    STAMP(0);
    const bool main_tile = slot >= 16;
    if (slot < 7) {
        LAS SendCtl* ctl = (LAS SendCtl*)(lds + 8 * HTB + 8192);
        if (threadIdx.x < 64) ((volatile LAS unsigned*)ctl)[lane] = 0u;
        WAIT_L(0); BAR;
        if (wid < NPK) packer(p, lds + wid * PK_LDS, ctl, xcd, slot, wid, lane, cntp);
        else if (wid == 3) storer(p, lds, ctl, xcd, slot, lane, cntp);
    }
    const unsigned call = __builtin_amdgcn_readfirstlane(ld4_sys(cntp)) + 1u;
    if (lane == 0) __hip_atomic_store(cntp, call, __ATOMIC_RELAXED, __HIP_MEMORY_SCOPE_AGENT);

    if (!main_tile) {
        if (slot >= 7) {
            const bool unp = slot < 7 + NUS;
            formatter(p, lds + wid * HTB, wid * 72 + (blockIdx.x - 56), unp ? 1 << 20 : 576 + wid * (NFMT / 8) + (blockIdx.x - 56 - NUS * 8), lane, call);
            const int uw = wid * (NUS * 8) + (blockIdx.x - 56);
            if (unp && uw < NUNP) unpacker(p, (LAS unsigned*)(lds + wid * HTB), uw, lane, call);
        }
        STAMP(1);
        asm volatile("s_waitcnt vmcnt(0) lgkmcnt(0)" ::: "memory");
        BAR;
    }

    int rt, cn, q0, nt, skip;
    const char* wimg;
    if (main_tile) {
        const int idx = slot - 16;
        rt = ((xcd >> 1) * 2 + ((idx >> 2) & 1)) * 2 + (idx >> 3);
        cn = p.rank * 8 + (xcd & 1) * 4 + (idx & 3);
        const int m = rt >> 1;
        q0 = (m == p.rank) ? NH : 0; skip = (m - p.rank - 1) & 7; nt = 56;
        wimg = ws + OFF_WI;
    } else {
        rt = p.rank * 2 + (slot < 8 ? 1 : 0);
        cn = xcd * 8 + (slot & 7);
        q0 = 0; skip = 7; nt = 8;
        wimg = (xcd == p.rank) ? ws + OFF_WI : ws + OFF_WD + (size_t)((xcd - p.rank - 1) & 7) * NH * WQ;
    }
    const int wr = wid >> 2, wc = wid & 3;
    const unsigned* fx = (const unsigned*)(ws + OFF_FX) + rt * FSTR;
    const unsigned* fw = (const unsigned*)(ws + OFF_FW) + cn * FSTR;
    const char* xb = ws + OFF_XI + (size_t)(rt * 16 + wid) * 1024;
    const char* wb = wimg + (size_t)((cn & 7) * 16 + wr * 8 + wc) * 1024;
    const unsigned voff = lane * 16;
    const unsigned ldsw = wid * 1024;
    const int aoff = wr * 4096 + lane * 16, boff = wc * 2048 + lane * 16;
#define SA(b, h) (((b) * 4 + (h)) * HTB)
#define SB(b, h) (((b) * 4 + 2 + (h)) * HTB)
#define GLDS(g, l) __builtin_amdgcn_global_load_lds((const unsigned*)((g) + voff), (LAS unsigned*)(lds + (l)), 16, 0, GLDS_AUX)
#define STAGE_A(b, h, qa, qb) do { GLDS(wb + (size_t)(qa) * WQ + (h) * 4096, SA(b, h) + ldsw); GLDS(wb + (size_t)(qb) * WQ + (h) * 4096, SA(b, h) + ldsw + 8192); } while (0)
#define STAGE_B(b, h, qa, qb) do { GLDS(xb + (size_t)(qa) * XQ + (h) * 8192, SB(b, h) + ldsw); GLDS(xb + (size_t)(qb) * XQ + (h) * 8192, SB(b, h) + ldsw + 8192); } while (0)
#define LDA(dst, b, h) do { _Pragma("unroll") for (int m = 0; m < 4; ++m) _Pragma("unroll") for (int k = 0; k < 2; ++k) dst[m][k] = *(const LAS bf16x8*)(lds + SA(b, h) + aoff + m * 1024 + k * 8192); } while (0)
#define LDB(dst, b, h) do { _Pragma("unroll") for (int n = 0; n < 2; ++n) _Pragma("unroll") for (int k = 0; k < 2; ++k) dst[n][k] = *(const LAS bf16x8*)(lds + SB(b, h) + boff + n * 1024 + k * 8192); } while (0)
#define MMA(ai, bj, At, Bt) do { __builtin_amdgcn_s_setprio(1); _Pragma("unroll") for (int m = 0; m < 4; ++m) _Pragma("unroll") for (int n = 0; n < 2; ++n) _Pragma("unroll") for (int k = 0; k < 2; ++k) \
        acc[ai][bj][m][n] = __builtin_amdgcn_mfma_f32_16x16x32_bf16(Bt[n][k], At[m][k], acc[ai][bj][m][n], 0, 0, 0); __builtin_amdgcn_s_setprio(0); } while (0)
#define ENSURE(qn) do { if (__builtin_expect(ready <= (qn), 0)) ready = poll_ready(fx, fw, ready, (qn) + 1, skip, call, tmo); } while (0)
#define NEXTQ(dst) do { dst = qn; ++qn; if (qn == qskip) { ++qn; qskip += 7; } } while (0)
    f32x4 acc[2][2][4][2];
#pragma unroll
    for (int a = 0; a < 2; ++a)
#pragma unroll
        for (int b = 0; b < 2; ++b)
#pragma unroll
            for (int m = 0; m < 4; ++m)
#pragma unroll
                for (int n = 0; n < 2; ++n) acc[a][b][m][n] = (f32x4){0.f, 0.f, 0.f, 0.f};
    bf16x8 At[4][2], B0[2][2], B1[2][2];
    int ready = q0, qn = q0, qskip = (skip == 7) ? (1 << 20) : NH + skip;
    int q0a, q0b, q1a, q1b;
    NEXTQ(q0a); NEXTQ(q0b); NEXTQ(q1a); NEXTQ(q1b);
    ENSURE(q1b);
    STAMP(4);
    STAGE_B(0, 0, q0a, q0b); STAGE_A(0, 0, q0a, q0b); STAGE_B(0, 1, q0a, q0b); STAGE_A(0, 1, q0a, q0b);
    if (wr == 1) BAR;
    WAIT_V(4); BAR;
    STAGE_B(1, 0, q1a, q1b); STAGE_A(1, 0, q1a, q1b); STAGE_B(1, 1, q1a, q1b);
    WAIT_V(6); BAR;
    LDB(B0, 0, 0);
    for (int t = 0; t < nt; t += 2) {
        if (t == nt - 2) { qn = q0; qskip = 1 << 20; }
        int q2a, q2b, q3a, q3b;
        NEXTQ(q2a); NEXTQ(q2b); NEXTQ(q3a); NEXTQ(q3b);
        LDA(At, 0, 0); STAGE_A(1, 1, q1a, q1b);
        BAR; WAIT_L(0); MMA(0, 0, At, B0); BAR; SCHED;
        ENSURE(q2b);
        LDB(B1, 0, 1); STAGE_B(0, 0, q2a, q2b);
        BAR; WAIT_L(0); MMA(0, 1, At, B1); BAR;
        LDA(At, 0, 1); STAGE_A(0, 0, q2a, q2b);
        WAIT_V(10); BAR; WAIT_L(0); MMA(1, 0, At, B0); BAR; SCHED;
        LDB(B0, 1, 0); STAGE_B(0, 1, q2a, q2b);
        WAIT_V(6); BAR; MMA(1, 1, At, B1); WAIT_L(0); BAR; SCHED;
        LDA(At, 1, 0); STAGE_A(0, 1, q2a, q2b);
        BAR; WAIT_L(0); MMA(0, 0, At, B0); BAR; SCHED;
        ENSURE(q3b);
        LDB(B1, 1, 1); STAGE_B(1, 0, q3a, q3b);
        BAR; WAIT_L(0); MMA(0, 1, At, B1); BAR;
        LDA(At, 1, 1); STAGE_A(1, 0, q3a, q3b);
        WAIT_V(10); BAR; WAIT_L(0); MMA(1, 0, At, B0); BAR; SCHED;
        LDB(B0, 0, 0); STAGE_B(1, 1, q3a, q3b);
        WAIT_V(6); BAR; MMA(1, 1, At, B1); WAIT_L(0); BAR; SCHED;
        q1a = q3a; q1b = q3b;
    }
    STAMP(2);
    int tz = threadIdx.x; asm volatile("" : "+v"(tz));
    const int ln = tz & 63;
    const unsigned vo = ln * 16;
    const unsigned tile = (unsigned)((rt & 1) * 64 + cn) * 8 + wid;
    if (main_tile) {
        const unsigned* gate = (skip == 7) ? cntp : (const unsigned*)(ws + OFF_SD) + (skip * 8 + xcd) * 32;
        unsigned gv = ld4_sys(ln == 1 ? tmo : gate);
        if (wr == 0) BAR;
        bool bad = (unsigned)__builtin_amdgcn_readlane(gv, 1) == call;
        const bool open = (unsigned)__builtin_amdgcn_readlane(gv, 0) >= call && !bad;
        const unsigned owner = (unsigned)(rt >> 1) & 7u;
        char* ows = p.ws[owner];
        const char* sl = ows + OFF_PL + (size_t)tile * UB;
        char* fl = ows + OFF_FP + (size_t)tile * 8;
        const char* s0 = sl + 4096; const char* s1 = sl + 8192;
        const bool staged = ASTAGE || (rt & 1);
        u32x4 b[16], pk[12];
#pragma unroll
        for (int bj = 0; bj < 2; ++bj)
#pragma unroll
            for (int n = 0; n < 2; ++n)
#pragma unroll
                for (int r = 0; r < 4; ++r)
#pragma unroll
                    for (int ai = 0; ai < 2; ++ai)
#pragma unroll
                        for (int m2 = 0; m2 < 2; ++m2)
                            b[bj * 8 + n * 4 + r][ai * 2 + m2] = cvt_pk_bf16(acc[ai][bj][2 * m2][n][r], acc[ai][bj][2 * m2 + 1][n][r]);
        asm volatile("" ::: "memory"); BAR; asm volatile("" ::: "memory");
        unsigned hdr = pack_unit(b, pk, lds + wid * HTB, ln, open ? s0 : nullptr);
        if (!open) {
            const unsigned long long t0 = __builtin_amdgcn_s_memrealtime();
            while (!bad && (unsigned)__builtin_amdgcn_readfirstlane(ld4_sys(gate)) < call) {
                __builtin_amdgcn_s_sleep(2);
                bad = give_up(tmo, call, t0);
            }
        }
        STAMP(6);
        if (bad) {
            hdr = RAWCNT << 16;
#pragma unroll
            for (int f = 0; f < 16; ++f) b[f] = (u32x4){0x7fc07fc0u, 0x7fc07fc0u, 0x7fc07fc0u, 0x7fc07fc0u};
        }
        if (__builtin_expect(hdr_cnt(hdr) == RAWCNT, 0)) { WAIT_V(0); st_raw(sl, vo, b); }
        else {
            if (!open) { st16<-4096>(s0, vo, pk[0]); WAIT_V(0); st16<-3072>(s0, vo, pk[1]); }
            WAIT_V(0); st_list(s1, vo, pk[11], hdr, ln); st16<0>(s1, vo, pk[8]);
            WAIT_V(0); st16<1024>(s1, vo, pk[9]);
            WAIT_V(0); st16<2048>(s1, vo, pk[10]);
            WAIT_V(0); if (staged) st8_flag(fl, call, hdr | (1u << 28), ln); st16<-2048>(s0, vo, pk[2]);
            WAIT_V(0); st16<-1024>(s0, vo, pk[3]);
            WAIT_V(0); if (staged) st8_flag(fl, call, hdr | (2u << 28), ln); st16<0>(s0, vo, pk[4]);
            WAIT_V(0); st16<1024>(s0, vo, pk[5]);
            WAIT_V(0); if (staged) st8_flag(fl, call, hdr | (3u << 28), ln); st16<2048>(s0, vo, pk[6]);
            WAIT_V(0); st16<3072>(s0, vo, pk[7]);
        }
        WAIT_V(0);
        st8_flag(fl, call, hdr | (4u << 28), ln);
    } else {
        WAIT_V(0);
        if (wr == 0) BAR;
        BAR;
        LAS unsigned* wl = (LAS unsigned*)(lds + wid * HTB);
#pragma unroll
        for (int i = 0; i < 8; ++i) *(LAS u32x4*)(wl + i * 256 + ln * 4) = (u32x4){0u, 0u, 0u, 0u};
        const char* fl = ws + OFF_FP + (size_t)tile * 8;
        const char* sl = ws + OFF_PL + (size_t)tile * UB;
        unsigned hdr = wait_unit(fl, call, 1u, tmo);
        const unsigned cnt = hdr_cnt(hdr); const bool raw = cnt == RAWCNT;
        const unsigned LL = (hdr & 0xffffu) * 0x10001u;
        u32x4 r[16];
        if (!raw) { ld4x16(r[8], r[9], r[10], r[11], sl + 8192, vo); patch_apply(wl, r[11], hdr, ln, false); }
        const char* ob = p.out + ((size_t)((rt & 1) * 256 + wc * 32) * NN + cn * 256 + wr * 128) * 2;
        const unsigned oo = (ln >> 4) * (4 * NN * 2) + (ln & 15) * 16;
#pragma unroll
        for (int s = 0; s < 4; ++s) {
            if ((hdr >> 28) <= (unsigned)s) hdr = wait_unit(fl, call, s + 1, tmo);
            u32x4 o[4];
            unsigned tv;
            if (raw) { ld4x16(o[0], o[1], o[2], o[3], sl + s * 4096, vo); tv = ld4_sys(tmo); }
            else { ld2x16t(r[2 * s], r[2 * s + 1], tv, sl + s * 2048, vo, tmo); unpack_sub(r, s, wl, ln, LL, o); }
            const unsigned nanmask = (tv == call) ? 0x7fc07fc0u : 0u;
#pragma unroll
            for (int rr = 0; rr < 4; ++rr) {
                u32x4 y;
#pragma unroll
                for (int cc = 0; cc < 4; ++cc) {
                    const unsigned pv = o[rr][cc];
                    const f32x2 v = (f32x2){acc[cc >> 1][s >> 1][2 * (cc & 1)][s & 1][rr] + __uint_as_float(pv << 16),
                                            acc[cc >> 1][s >> 1][2 * (cc & 1) + 1][s & 1][rr] + __uint_as_float(pv & 0xffff0000u)};
                    const f32x2 g = gelu2(v);
                    y[cc] = cvt_pk_bf16(g.x, g.y) | nanmask;
                }
                st16<0>(ob + (size_t)((s >> 1) * 128 + (s & 1) * 16 + rr) * (NN * 2), oo, y);
            }
        }
    }
    STAMP(3);
}
}

extern "C" void kernel_launch(int rank, int nranks, ncclComm_t comm, hipStream_t stream,
                              void* const* d_in, const int* in_sizes, int n_in,
                              void* d_out, int out_size, const int* in_dtypes, int out_dtype,
                              void* const* d_out_all, void* const* d_ws_all, size_t ws_bytes) {
    (void)comm; (void)in_dtypes; (void)out_dtype; (void)d_out_all;
    if (nranks != NR || (unsigned)rank >= (unsigned)NR || n_in < 2 || in_sizes[0] != MM * KS || in_sizes[1] != KS * NN ||
        out_size != (MM / NR) * NN || ws_bytes < WS_NEED) {
        fprintf(stderr, "kernel_launch: unexpected problem geometry\n");
        return;
    }
    Params p;
    p.x = (const char*)d_in[0];
    p.w = (const char*)d_in[1];
    for (int i = 0; i < NR; ++i) p.ws[i] = (char*)d_ws_all[i];
    p.out = (char*)d_out;
    p.rank = rank; p.pad = 0;
    hipLaunchKernelGGL(fused_kernel, dim3(256), dim3(512), 0, stream, p);
    HIP_CHECK(hipGetLastError());
}
```

```cpp
#include <hip/hip_runtime.h>
#include <rccl/rccl.h>
#include <cstdio>
#include <cstdint>

#define HIP_CHECK(expr) do { hipError_t _e = (expr); if (_e != hipSuccess) { \
    fprintf(stderr, "HIP error %s at %s:%d\n", hipGetErrorString(_e), __FILE__, __LINE__); \
    return; } } while (0)

#define NW0 4
#define LAGXB 36
#define LAGW0 0
#define LAGWS 0
#define LAGWH 4
#define LAGXA 1
#define GLDS_AUX 17
#define FORCE_RAW 0
#define NUS 5
#define SWIN 6
#define SHALLOW 1
#define RAW0 1
#define LATE_TK 1
#define PD0 100
#define PDT 24
#define FD0 1000
#define FDT 72
#define KTAIL 0
#define KPRIO 0
#define KBAL 2
#define EARLY3 1
#define EARLYJ 23
#define ENDW 0
#define BSTAGE 7
#define ASTAGE 0
#define FLS 128
#define NPK 4
#define GATE_EARLY 0

namespace {
#define LAS __attribute__((address_space(3)))
typedef short bf16x8 __attribute__((ext_vector_type(8)));
typedef float f32x4 __attribute__((ext_vector_type(4)));
typedef float f32x2 __attribute__((ext_vector_type(2)));
typedef unsigned u32x4 __attribute__((ext_vector_type(4)));
typedef unsigned u32x2 __attribute__((ext_vector_type(2)));
typedef unsigned short u16x2 __attribute__((ext_vector_type(2)));

constexpr int NR = 8;
constexpr int MM = 4096, NN = 16384, KS = 512, NS = 2048;
constexpr int NQ = 128;
constexpr int NH = 16;
constexpr int NU = NH * 22;
#define UNP7 1
constexpr int NUNP = UNP7 ? NUS * 64 / 7 * 7 : NUS * 64;
constexpr int NFMT = (9 - NUS) * 64;
constexpr unsigned long long DEADLINE = 20000000ull;
constexpr unsigned RAWCNT = 0x3ffu;

constexpr size_t OFF_CNT = 0;
constexpr size_t OFF_TMO = 8192;
constexpr size_t OFF_TK = 8192 + 1024;
constexpr size_t OFF_SD = 131072;
constexpr int FSTR = 160;
constexpr size_t OFF_FX = 16384;
constexpr size_t OFF_FW = 32768;
constexpr size_t OFF_STAMP = 262144;
constexpr size_t UB = 16384;
constexpr size_t XQ = 256 * 1024, WQ = 128 * 1024;
constexpr size_t OFF_XI = 1u << 20;
constexpr size_t OFF_WI = OFF_XI + (size_t)NQ * XQ;
constexpr size_t OFF_WD = OFF_WI + (size_t)NQ * WQ;
constexpr size_t OFF_PL = OFF_WD + (size_t)7 * NH * WQ;
constexpr size_t OFF_LZ = OFF_PL + (size_t)1024 * UB;
constexpr size_t LZ_PAR = (size_t)7 * NU * UB;
constexpr size_t OFF_FA = OFF_LZ + 2 * LZ_PAR;
constexpr size_t OFF_FP = OFF_FA + (size_t)14 * NU * 128;
constexpr size_t WS_NEED = OFF_FP + 1024 * 128;
static_assert(OFF_FW + 64 * FSTR * 4 <= OFF_SD && OFF_SD + 56 * 128 <= OFF_STAMP, "layout");
static_assert(WS_NEED <= (size_t)161 << 20, "workspace");

struct Params {
    const char* x;
    const char* w;
    char* ws[NR];
    char* out;
    int rank;
    int pad;
};

template <int OFF> __device__ __forceinline__ void st16(const char* sbase, unsigned voff, u32x4 v) {
    asm volatile("global_store_dwordx4 %0, %1, %2 offset:%3 sc0 sc1\n\ts_nop 1" :: "v"(voff), "v"(v), "s"(sbase), "i"(OFF) : "memory");
}
__device__ __forceinline__ unsigned ld4_sys(const void* p) {
    unsigned v;
    asm volatile("global_load_dword %0, %1, off sc0 sc1\n\ts_waitcnt vmcnt(0)" : "=&v"(v) : "v"(p) : "memory");
    return v;
}
__device__ __forceinline__ u32x2 ld8_sys(const void* p) {
    u32x2 v;
    asm volatile("global_load_dwordx2 %0, %1, off sc0 sc1\n\ts_waitcnt vmcnt(0)" : "=&v"(v) : "v"(p) : "memory");
    return v;
}
__device__ __forceinline__ void st8_flag(void* p, unsigned call, unsigned hdr, int lane) {
    if (lane == 0) __hip_atomic_store((unsigned long long*)p, ((unsigned long long)hdr << 32) | call, __ATOMIC_RELAXED, __HIP_MEMORY_SCOPE_SYSTEM);
}
#define WAIT_V_(n) asm volatile("s_waitcnt vmcnt(" #n ")" ::: "memory")
#define WAIT_V(n) WAIT_V_(n)
#define WAIT_L(n) asm volatile("s_waitcnt lgkmcnt(" #n ")" ::: "memory")
#define BAR __builtin_amdgcn_s_barrier()
#define SCHED __builtin_amdgcn_sched_barrier(0)

__device__ __forceinline__ unsigned cvt_pk_bf16(float lo, float hi) {
    unsigned r; asm("v_cvt_pk_bf16_f32 %0, %1, %2" : "=v"(r) : "v"(lo), "v"(hi)); return r;
}
__device__ __forceinline__ f32x2 gelu2(f32x2 y) {
    const float k1 = -2.0f * 0.7978845608028654f * 1.4426950408889634f, k2 = k1 * 0.044715f;
    f32x2 z = y * (y * y * k2 + k1);
    f32x2 e; e.x = __builtin_amdgcn_exp2f(z.x); e.y = __builtin_amdgcn_exp2f(z.y);
    e = e + 1.0f;
    f32x2 r; r.x = __builtin_amdgcn_rcpf(e.x); r.y = __builtin_amdgcn_rcpf(e.y);
    return y * r;
}
__device__ __forceinline__ unsigned pk_sub(unsigned a, unsigned b) { return __builtin_bit_cast(unsigned, __builtin_bit_cast(u16x2, a) - __builtin_bit_cast(u16x2, b)); }
__device__ __forceinline__ unsigned pk_add(unsigned a, unsigned b) { return __builtin_bit_cast(unsigned, __builtin_bit_cast(u16x2, a) + __builtin_bit_cast(u16x2, b)); }
__device__ __forceinline__ unsigned pk_max(unsigned a, unsigned b) { return __builtin_bit_cast(unsigned, __builtin_elementwise_max(__builtin_bit_cast(u16x2, a), __builtin_bit_cast(u16x2, b))); }
__device__ __forceinline__ unsigned bfi(unsigned m, unsigned a, unsigned b) { return __builtin_amdgcn_bitop3_b32(m, a, b, 0xca); }
__device__ __forceinline__ unsigned rotl(unsigned a, int n) { return __builtin_amdgcn_alignbit(a, a, 32 - n); }
__device__ __forceinline__ unsigned rotr(unsigned a, int n) { return __builtin_amdgcn_alignbit(a, a, n); }
__device__ __forceinline__ unsigned umax(unsigned a, unsigned b) { return a > b ? a : b; }

#define STAMP(slot) do {} while (0)

__device__ __forceinline__ bool give_up(unsigned* tmo, unsigned call, unsigned long long t0, unsigned& polls) {
    if ((++polls & 15u) != 0u) return false;
    if (__builtin_amdgcn_readfirstlane(__hip_atomic_load(tmo, __ATOMIC_RELAXED, __HIP_MEMORY_SCOPE_AGENT)) == call ||
        __builtin_amdgcn_s_memrealtime() - t0 > DEADLINE) {
        __hip_atomic_store(tmo, call, __ATOMIC_RELAXED, __HIP_MEMORY_SCOPE_AGENT);
        return true;
    }
    return false;
}
__device__ __forceinline__ unsigned wait_unit(const char* flag, unsigned call, unsigned st, unsigned* tmo) {
    const unsigned long long t0 = __builtin_amdgcn_s_memrealtime(); unsigned polls = 0;
    for (;;) {
        const u32x2 v = ld8_sys(flag);
        const unsigned hdr = __builtin_amdgcn_readfirstlane(v.y);
        if ((unsigned)__builtin_amdgcn_readfirstlane(v.x) == call && (hdr >> 28) >= st) return hdr;
        __builtin_amdgcn_s_sleep(1);
        if (give_up(tmo, call, t0, polls)) return (RAWCNT << 16) | (4u << 28);
    }
}
__device__ __forceinline__ unsigned hdr_cnt(unsigned hdr) { return (hdr >> 16) & 0x3ffu; }

constexpr int SCR_DENSE = 8192;
#define PT(i) do {} while (0)
__device__ __forceinline__ unsigned umin(unsigned a, unsigned b) { return a < b ? a : b; }
__device__ __forceinline__ unsigned wave_scan(unsigned inc) {
    inc += __builtin_amdgcn_update_dpp(0, (int)inc, 0x111, 0xf, 0xf, true);
    inc += __builtin_amdgcn_update_dpp(0, (int)inc, 0x112, 0xf, 0xf, true);
    inc += __builtin_amdgcn_update_dpp(0, (int)inc, 0x114, 0xf, 0xf, true);
    inc += __builtin_amdgcn_update_dpp(0, (int)inc, 0x118, 0xf, 0xf, true);
    inc += __builtin_amdgcn_update_dpp(0, (int)inc, 0x142, 0xa, 0xf, false);
    inc += __builtin_amdgcn_update_dpp(0, (int)inc, 0x143, 0xc, 0xf, false);
    return inc;
}
__device__ __forceinline__ unsigned pack_base(const u32x4 (&b)[16]) {
    unsigned mx = 0;
#pragma unroll
    for (int f = 0; f < 16; ++f) mx = pk_max(mx, b[f][f & 3] & 0x7fff7fffu);
    unsigned m = umax(mx & 0xffffu, mx >> 16);
    m = umax(m, (unsigned)__builtin_amdgcn_update_dpp(0, (int)m, 0x128, 0xf, 0xf, false));
    m = umax(m, (unsigned)__builtin_amdgcn_update_dpp(0, (int)m, 0x124, 0xf, 0xf, false));
    m = umax(m, (unsigned)__builtin_amdgcn_update_dpp(0, (int)m, 0x122, 0xf, 0xf, false));
    m = umax(m, (unsigned)__builtin_amdgcn_update_dpp(0, (int)m, 0x121, 0xf, 0xf, false));
    const unsigned m0 = __builtin_amdgcn_readlane(m, 0), m1 = __builtin_amdgcn_readlane(m, 16), m2 = __builtin_amdgcn_readlane(m, 32), m3 = __builtin_amdgcn_readlane(m, 48);
    const unsigned ms = umin(umax(umin(m0, m1), umin(m2, m3)), umin(umax(m0, m1), umax(m2, m3)));
    return umax(ms, 1023u) - 1023u;
}
__device__ __forceinline__ u32x4 pack_low(const u32x4 (&b)[16], int i, unsigned L) {
    const unsigned LL = L | (L << 16);
    u32x4 r;
#pragma unroll
    for (int c = 0; c < 4; ++c) r[c] = __builtin_amdgcn_perm(pk_sub(b[2 * i + (c >> 1)][2 * (c & 1) + 1], LL), pk_sub(b[2 * i + (c >> 1)][2 * (c & 1)], LL), 0x06040200u);
    return r;
}
__device__ __forceinline__ unsigned pack_unit(const u32x4 (&b)[16], u32x4 (&pk)[12], unsigned& tl, unsigned L, LAS unsigned char* scr, int lane, const char* early) {
    const unsigned LL = L | (L << 16);
    const unsigned wp0 = (unsigned)(size_t)scr + lane * 4;
    unsigned wp = wp0;
    unsigned z[4], seg[4], len[4], F = 0;
    PT(0);
#pragma unroll
    for (int s = 0; s < 4; ++s) {
        unsigned hb[8];
#pragma unroll
        for (int jj = 0; jj < 8; ++jj) {
            const int j = 8 * s + jj;
            const unsigned d0 = pk_sub(b[j >> 1][2 * (j & 1)], LL), d1 = pk_sub(b[j >> 1][2 * (j & 1) + 1], LL);
            pk[j >> 2][j & 3] = __builtin_amdgcn_perm(d1, d0, 0x06040200u);
            hb[jj] = __builtin_amdgcn_perm(d1, d0, 0x07050301u);
            const unsigned t = hb[jj] & 0x7c7c7c7cu; unsigned tmp;
            *(LAS unsigned*)(size_t)wp = t | (unsigned)((j & 3) | ((j >> 2) << 7));
            asm("v_min_u32 %0, 1, %2\n\tv_lshl_add_u32 %1, %0, 8, %1" : "=&v"(tmp), "+v"(wp) : "v"(t));
            if (jj & 3) hb[jj] = rotl(hb[jj], 2 * (jj & 3));
            if (j == EARLYJ && early) { WAIT_V(0); st16<-3072>(early, lane * 16, pk[1]); }
        }
        pk[8][s] = bfi(0x81818181u, hb[0], bfi(0x06060606u, hb[1], bfi(0x18181818u, hb[2], hb[3])));
        pk[9][s] = bfi(0x81818181u, hb[4], bfi(0x06060606u, hb[5], bfi(0x18181818u, hb[6], hb[7])));
        const unsigned ga = bfi(0x02020202u, hb[0], bfi(0x08080808u, hb[1], bfi(0x20202020u, hb[2], hb[3])));
        const unsigned gb = bfi(0x02020202u, hb[4], bfi(0x08080808u, hb[5], bfi(0x20202020u, hb[6], hb[7])));
        z[s] = ~bfi(0xaaaaaaaau, ga, gb >> 1);
    }
    PT(1);
    unsigned e8[8], nz4[4];
#pragma unroll
    for (int i = 0; i < 8; ++i) e8[i] = *(LAS unsigned*)(scr + i * 256 + lane * 4);
#pragma unroll
    for (int s = 0; s < 4; ++s) {
        unsigned t = z[s] | (z[s] >> 1); t |= t << 2;
        nz4[s] = t & 0x44444444u; F |= s < 2 ? nz4[s] >> (2 - s) : nz4[s] << (s - 2);
    }
    const unsigned nrow = (wp - wp0) >> 8, mine = (__builtin_popcount(F) << 14) | nrow;
    const unsigned inc = wave_scan(mine), tot = __builtin_amdgcn_readlane(inc, 63), ngrp = tot & 0xfffu;
    const unsigned lo = 256u + ((((tot >> 12) + 31u) >> 5) << 2);
    unsigned cnt = ngrp, bad = 0;
    PT(2);
    if (ngrp <= 512u) {
        LAS unsigned* dense = (LAS unsigned*)(scr + SCR_DENSE);
        const unsigned lanebits = ((lane & 7) << 15) | ((lane >> 3) << 23), base = (inc - mine) & 0xfffu;
#pragma unroll
        for (int i = 0; i < 8; ++i) dense[(unsigned)i < nrow ? base + i : 512 + lane] = e8[i] | lanebits;
        for (unsigned r = 8; __builtin_expect(__any(nrow > r), 0); ++r)
            dense[r < nrow ? base + r : 512 + lane] = *(LAS unsigned*)(scr + r * 256 + lane * 4) | lanebits;
        unsigned e2[2] = {dense[lane], dense[64 + lane]};
        *(LAS u32x4*)(scr + 256 + lane * 16) = (u32x4){0u, 0u, 0u, 0u};
        *(LAS unsigned*)(scr + lane * 4) = F;
        PT(3);
        if (EARLY3 && early) { WAIT_V(0); st16<0>(early + 4096, lane * 16, pk[8]); }
#pragma unroll
        for (int s = 0; s < 4; ++s) { seg[s] = z[s] & 15u; len[s] = nz4[s] & 7u; }
#pragma unroll
        for (int i = 1; i < 8; ++i)
#pragma unroll
            for (int s = 0; s < 4; ++s) { seg[s] |= ((z[s] >> (4 * i)) & 15u) << len[s]; len[s] += (nz4[s] >> (4 * i)) & 7u; }
        PT(4);
        unsigned bit = (inc - mine) >> 12;
#pragma unroll
        for (int s = 0; s < 4; ++s) {
            const unsigned long long v = (unsigned long long)seg[s] << (bit & 31u);
            LAS unsigned* w = (LAS unsigned*)(scr + 256) + (bit >> 5);
            __hip_atomic_fetch_or(w, (unsigned)v, __ATOMIC_RELAXED, __HIP_MEMORY_SCOPE_WAVEFRONT);
            __hip_atomic_fetch_or(w + 1, (unsigned)(v >> 32), __ATOMIC_RELAXED, __HIP_MEMORY_SCOPE_WAVEFRONT);
            bit += len[s];
        }
        PT(5);
        LAS unsigned short* lst = (LAS unsigned short*)(scr + lo);
        const unsigned hz = ((0u - L) >> 10) & 31u;
        for (unsigned i0 = 0;;) {
            unsigned t[2], pos[2], at[2];
#pragma unroll
            for (int c = 0; c < 2; ++c) {
                const unsigned e = e2[c], ix = i0 + c * 64 + lane;
                t[c] = ix < ngrp ? e & 0x7c7c7c7cu : 0u;
                pos[c] = ((e & 0x380u) << 3) | ((e >> 16) & 0x380u) | ((e >> 11) & 0x70u) | ((e & 3u) << 2);
                at[c] = ix < ngrp ? ix : 1024u + lane;
            }
            for (;;) {
#pragma unroll
                for (int c = 0; c < 2; ++c) {
                    const unsigned b8 = __builtin_ctz(t[c] | 0x80000000u) & 24u, hi5 = (t[c] >> (b8 + 2)) & 31u;
                    const unsigned code = hi5 == hz ? 5u : (hi5 + 5u) & 31u;
                    bad = umax(bad, code);
                    lst[at[c]] = (unsigned short)((code << 13) | pos[c] | (b8 >> 3));
                    t[c] &= ~(0xffu << b8);
                }
                const unsigned long long b0 = __ballot(t[0] != 0), b1 = __ballot(t[1] != 0);
                if (!(b0 | b1)) break;
                at[0] = t[0] != 0 ? __builtin_amdgcn_mbcnt_hi((unsigned)(b0 >> 32), __builtin_amdgcn_mbcnt_lo((unsigned)b0, umin(cnt, 512u))) : 1024u + lane;
                cnt += __builtin_popcountll(b0);
                at[1] = t[1] != 0 ? __builtin_amdgcn_mbcnt_hi((unsigned)(b1 >> 32), __builtin_amdgcn_mbcnt_lo((unsigned)b1, umin(cnt, 512u))) : 1024u + lane;
                cnt += __builtin_popcountll(b1);
            }
            i0 += 128;
            if (__builtin_expect(i0 >= ngrp, 1)) break;
            e2[0] = dense[i0 + lane]; e2[1] = dense[i0 + 64 + lane];
        }
    }
    PT(6);
    pk[10] = *(LAS u32x4*)(scr + lane * 16);
    pk[11] = *(LAS u32x4*)(scr + 1024 + lane * 16);
    tl = (lo + 2u * cnt + 63u) >> 6;
    if (cnt > 512u || tl > 32u || __any(bad > 7u) || FORCE_RAW) cnt = RAWCNT;
    return L | (cnt << 16) | (tl > 16u ? 1u << 26 : 0u);
}
__device__ __forceinline__ void st_tail(const char* sb8k  , unsigned voff, u32x4 t0, u32x4 t1, unsigned tl, int lane) {
    if (__builtin_expect(tl > 16u, 0)) { if ((unsigned)lane < (tl - 16u) * 4u) st16<3072>(sb8k, voff, t1); }
    if ((unsigned)lane < tl * 4u) st16<2048>(sb8k, voff, t0);
}
__device__ __forceinline__ void st_raw(const char* slot, unsigned voff, const u32x4 (&b)[16]) {
    const char* s0 = slot + 4096; const char* s1 = slot + 12288;
    st16<-4096>(s0, voff, b[0]); st16<-3072>(s0, voff, b[1]); st16<-2048>(s0, voff, b[2]); st16<-1024>(s0, voff, b[3]);
    st16<0>(s0, voff, b[4]); st16<1024>(s0, voff, b[5]); st16<2048>(s0, voff, b[6]); st16<3072>(s0, voff, b[7]);
    st16<-4096>(s1, voff, b[8]); st16<-3072>(s1, voff, b[9]); st16<-2048>(s1, voff, b[10]); st16<-1024>(s1, voff, b[11]);
    st16<0>(s1, voff, b[12]); st16<1024>(s1, voff, b[13]); st16<2048>(s1, voff, b[14]); st16<3072>(s1, voff, b[15]);
}
constexpr int DEC_TAIL = 8448;
__device__ __forceinline__ void ld_tail2(const char* slot, unsigned voff, unsigned ldst) {
    unsigned keep;
    asm volatile("s_mov_b32 %0, m0\n\ts_mov_b32 m0, %3\n\ts_nop 0\n\tglobal_load_lds_dwordx4 %1, %2 sc0 sc1\n\ts_mov_b32 m0, %0"
                 : "=&s"(keep) : "v"(voff), "s"(slot + 11264), "s"(ldst + 1024) : "memory");
}
__device__ __forceinline__ void ld_packed(u32x4 (&r)[10], unsigned& F, const char* slot, unsigned voff, unsigned ldst) {
    const char* s0 = slot + 4096; const char* s1 = slot + 10240;
    unsigned keep;
    asm volatile("s_mov_b32 %11, m0\n\ts_mov_b32 m0, %16\n\t"
                 "global_load_dword %10, %13, %15 sc0 sc1\n\tglobal_load_lds_dwordx4 %12, %15 sc0 sc1\n\t"
                 "global_load_dwordx4 %8, %12, %15 offset:-2048 sc0 sc1\n\tglobal_load_dwordx4 %9, %12, %15 offset:-1024 sc0 sc1\n\t"
                 "global_load_dwordx4 %0, %12, %14 offset:-4096 sc0 sc1\n\tglobal_load_dwordx4 %1, %12, %14 offset:-3072 sc0 sc1\n\t"
                 "global_load_dwordx4 %2, %12, %14 offset:-2048 sc0 sc1\n\tglobal_load_dwordx4 %3, %12, %14 offset:-1024 sc0 sc1\n\t"
                 "global_load_dwordx4 %4, %12, %14 sc0 sc1\n\tglobal_load_dwordx4 %5, %12, %14 offset:1024 sc0 sc1\n\t"
                 "global_load_dwordx4 %6, %12, %14 offset:2048 sc0 sc1\n\tglobal_load_dwordx4 %7, %12, %14 offset:3072 sc0 sc1\n\t"
                 "s_mov_b32 m0, %11\n\ts_waitcnt vmcnt(0)"
                 : "=&v"(r[0]), "=&v"(r[1]), "=&v"(r[2]), "=&v"(r[3]), "=&v"(r[4]), "=&v"(r[5]), "=&v"(r[6]), "=&v"(r[7]),
                   "=&v"(r[8]), "=&v"(r[9]), "=&v"(F), "=&s"(keep)
                 : "v"(voff), "v"(voff >> 2), "s"(s0), "s"(s1), "s"(ldst) : "memory");
}
__device__ __forceinline__ void ld_head(u32x4 (&r)[10], unsigned& F, unsigned& t, const char* slot, unsigned voff, unsigned ldst, const unsigned* tp) {
    const char* s1 = slot + 10240;
    unsigned keep;
    asm volatile("s_mov_b32 %6, m0\n\ts_mov_b32 m0, %11\n\t"
                 "global_load_dword %4, %8, %10 sc0 sc1\n\tglobal_load_lds_dwordx4 %7, %10 sc0 sc1\n\t"
                 "global_load_dwordx4 %2, %7, %10 offset:-2048 sc0 sc1\n\tglobal_load_dwordx4 %3, %7, %10 offset:-1024 sc0 sc1\n\t"
                 "global_load_dwordx4 %0, %7, %9 sc0 sc1\n\tglobal_load_dwordx4 %1, %7, %9 offset:1024 sc0 sc1\n\t"
                 "global_load_dword %5, %12, off sc1\n\ts_mov_b32 m0, %6\n\ts_waitcnt vmcnt(0)"
                 : "=&v"(r[0]), "=&v"(r[1]), "=&v"(r[8]), "=&v"(r[9]), "=&v"(F), "=&v"(t), "=&s"(keep)
                 : "v"(voff), "v"(voff >> 2), "s"(slot), "s"(s1), "s"(ldst), "v"(tp) : "memory");
}
__device__ __forceinline__ void ld4x16(u32x4& a, u32x4& b, u32x4& c, u32x4& d, const char* sb, unsigned voff) {
    asm volatile("global_load_dwordx4 %0, %4, %5 sc0 sc1\n\tglobal_load_dwordx4 %1, %4, %5 offset:1024 sc0 sc1\n\t"
                 "global_load_dwordx4 %2, %4, %5 offset:2048 sc0 sc1\n\tglobal_load_dwordx4 %3, %4, %5 offset:3072 sc0 sc1\n\ts_waitcnt vmcnt(0)"
                 : "=&v"(a), "=&v"(b), "=&v"(c), "=&v"(d) : "v"(voff), "s"(sb) : "memory");
}
__device__ __forceinline__ void ld2x16t(u32x4& a, u32x4& b, unsigned& t, const char* sb, unsigned voff, const unsigned* tp) {
    asm volatile("global_load_dwordx4 %0, %3, %4 sc0 sc1\n\tglobal_load_dwordx4 %1, %3, %4 offset:1024 sc0 sc1\n\t"
                 "global_load_dword %2, %5, off sc1\n\ts_waitcnt vmcnt(0)"
                 : "=&v"(a), "=&v"(b), "=&v"(t) : "v"(voff), "s"(sb), "v"(tp) : "memory");
}
__device__ __forceinline__ void patch_apply(LAS unsigned char* dl, unsigned lo, unsigned hdr, int lane, bool zero) {
    const unsigned cnt = hdr_cnt(hdr), lut_hi = 0x0804007cu | ((((0u - (hdr & 0x7fffu)) >> 10) & 31u) << 10);
    for (unsigned c = 0; c < cnt; c += 128) {
        const unsigned w = *(LAS unsigned*)(dl + DEC_TAIL + lo + c * 2 + lane * 4);
#pragma unroll
        for (int hf = 0; hf < 2; ++hf) {
            const unsigned e = hf ? w >> 16 : w & 0xffffu;
            const unsigned at = c + lane * 2 + hf < cnt ? e & 0x1fffu : 8192u + lane;
            dl[at] = zero ? 0 : (unsigned char)__builtin_amdgcn_perm(lut_hi, 0x7874706cu, e >> 13);
        }
    }
}
__device__ __forceinline__ unsigned unpack_head(unsigned F, unsigned hdr, LAS unsigned char* dl, int lane, u32x4& G) {
    PT(8);
    const unsigned mine = __builtin_popcount(F) << 2, inc = wave_scan(mine);
    const unsigned lo = 256u + ((((unsigned)__builtin_amdgcn_readlane(inc, 63) + 31u) >> 5) << 2);
    unsigned bit = inc - mine, sg[4];
#pragma unroll
    for (int s = 0; s < 4; ++s) {
        const LAS unsigned* w = (const LAS unsigned*)(dl + DEC_TAIL + 256) + (bit >> 5);
        sg[s] = __builtin_amdgcn_alignbit(w[1], w[0], bit);
        bit += __builtin_popcount(F & (0x11111111u << s)) << 2;
    }
    PT(9);
    patch_apply(dl, lo, hdr, lane, false);
    PT(10);
#pragma unroll
    for (int s = 0; s < 4; ++s) {
        const unsigned nz = (F >> s) & 0x11111111u, pre = nz * 0x11111110u;
        const unsigned xe = sg[s] & 0x0f0f0f0fu, xo = (sg[s] >> 4) & 0x0f0f0f0fu;
        const unsigned b0 = __builtin_amdgcn_perm(xo, xe, 0x05010400u), b1 = __builtin_amdgcn_perm(xo, xe, 0x07030602u);
        const unsigned ze = __builtin_amdgcn_perm(b1, b0, pre & 0x07070707u), zo = __builtin_amdgcn_perm(b1, b0, (pre >> 4) & 0x07070707u);
        G[s] = ~((ze | (zo << 4)) & ((nz << 4) - nz));
    }
    PT(11);
    return lo;
}
__device__ __forceinline__ void unpack_sub(const u32x4 (&r)[10], u32x4 G, int s, LAS unsigned char* dl, int lane, unsigned LL, u32x4 (&o)[4]) {
    const u32x4 p0 = *(LAS u32x4*)(dl + (2 * s) * 1024 + lane * 16), p1 = *(LAS u32x4*)(dl + (2 * s + 1) * 1024 + lane * 16);
    const unsigned A = r[8][s], B = r[9][s], ga = G[s], gb = G[s] << 1;
    unsigned hb[8];
    hb[0] = bfi(0x83838383u, bfi(0x02020202u, ga, A), p0[0]); hb[1] = bfi(0x83838383u, rotr(bfi(0x08080808u, ga, A), 2), p0[1]);
    hb[2] = bfi(0x83838383u, rotr(bfi(0x20202020u, ga, A), 4), p0[2]); hb[3] = bfi(0x83838383u, rotr(bfi(0x80808080u, ga, A), 6), p0[3]);
    hb[4] = bfi(0x83838383u, bfi(0x02020202u, gb, B), p1[0]); hb[5] = bfi(0x83838383u, rotr(bfi(0x08080808u, gb, B), 2), p1[1]);
    hb[6] = bfi(0x83838383u, rotr(bfi(0x20202020u, gb, B), 4), p1[2]); hb[7] = bfi(0x83838383u, rotr(bfi(0x80808080u, gb, B), 6), p1[3]);
#pragma unroll
    for (int jj = 0; jj < 8; ++jj) {
        const unsigned lo = r[2 * s + (jj >> 2)][jj & 3];
        o[jj >> 1][2 * (jj & 1)] = pk_add(__builtin_amdgcn_perm(hb[jj], lo, 0x05010400u), LL);
        o[jj >> 1][2 * (jj & 1) + 1] = pk_add(__builtin_amdgcn_perm(hb[jj], lo, 0x07030602u), LL);
    }
}

__device__ __forceinline__ void unit_fetch(const Params& p, int h, int u, int lane, unsigned ldsbuf) {
    const bool isx = u < 16;
    const char* base = isx ? p.x + ((size_t)u * 256 * KS + h * 32) * 2 : p.w + ((size_t)h * 32 * NN + (u - 16) * 256) * 2;
    unsigned voff = isx ? (lane & 15) * (KS * 2) + (lane >> 4) * 16 : (lane >> 4) * (8 * NN * 2) + (lane & 15) * 16;
    const unsigned s1 = isx ? 16 * KS * 2 : NN * 2, s2 = isx ? 16 * KS * 2 : 256 - 7 * NN * 2;
#pragma unroll
    for (int i = 0; i < 16; ++i) {
        unsigned keep;
        asm volatile("s_mov_b32 %0, m0\n\ts_mov_b32 m0, %3\n\ts_nop 0\n\tglobal_load_lds_dwordx4 %1, %2\n\ts_mov_b32 m0, %0"
                     : "=&s"(keep) : "v"(voff), "s"(base), "s"(ldsbuf + i * 1024) : "memory");
        voff += (i == 7) ? s2 : s1;
    }
}
__device__ __forceinline__ void unit_read(LAS unsigned char* buf, int lane, u32x4 (&r)[16]) {
#pragma unroll
    for (int i = 0; i < 16; ++i) r[i] = *(LAS u32x4*)(buf + i * 1024 + lane * 16);
    asm volatile("s_waitcnt lgkmcnt(0)" : "+v"(r[0]), "+v"(r[1]), "+v"(r[2]), "+v"(r[3]), "+v"(r[4]), "+v"(r[5]), "+v"(r[6]), "+v"(r[7]),
                 "+v"(r[8]), "+v"(r[9]), "+v"(r[10]), "+v"(r[11]), "+v"(r[12]), "+v"(r[13]), "+v"(r[14]), "+v"(r[15]) :: "memory");
}
__device__ __forceinline__ void unit_format(int u, u32x4 (&r)[16]) {
    if (u >= 16) {
#pragma unroll
        for (int g = 0; g < 2; ++g) {
            u32x4 o[8];
#pragma unroll
            for (int i = 0; i < 8; ++i)
#pragma unroll
                for (int a = 0; a < 4; ++a)
                    o[i][a] = __builtin_amdgcn_perm(r[g * 8 + 2 * a + 1][i >> 1], r[g * 8 + 2 * a][i >> 1], (i & 1) ? 0x07060302u : 0x05040100u);
#pragma unroll
            for (int i = 0; i < 8; ++i) r[g * 8 + i] = o[i];
        }
    }
}
struct OrderTab { unsigned e[NU]; };
constexpr OrderTab make_order() {
    OrderTab t{}; int n = 0;
    for (int v = 0; v < 8 * NW0; ++v) t.e[n++] = (v >> 3) << 8 | (v & 7);
    for (int key = 0; key < 8 * NH + 512; ++key)
        for (int v = 0; v < 22; ++v) {
            const int lag = v < 8 ? LAGW0 + (v & 3) * LAGWS + ((v & 2) ? LAGWH : 0) : v < 15 ? LAGXA : LAGXB;
            const int d = key - lag;
            if (d >= 0 && d % 8 == 0 && d / 8 < NH && !(v < 8 && d / 8 < NW0)) t.e[n++] = (unsigned)((d / 8) << 8 | v);
        }
    return t;
}
__constant__ const OrderTab g_order = make_order();
__device__ __forceinline__ int order(int k, int src) {
    const int e = g_order.e[k], v = e & 255;
    const int odd = v >= 15, m = (src + 1 + v - (odd ? 15 : 8)) & 7;
    return (e & ~255) | (v < 8 ? 16 + v : 2 * m + odd);
}

constexpr int PK_LDS = 28 * 1024;
struct SendCtl { u32x4 desc[8]; unsigned freed[4]; unsigned tail; };
__device__ __forceinline__ void packer(const Params& p, LAS unsigned char* buf, LAS SendCtl* ctl, int xcd, int di, int mem, int lane, unsigned* cntp) {
    const unsigned dest = (unsigned)(p.rank + 1 + di) & 7u;
    const unsigned voff = lane * 16, lbuf = (unsigned)(size_t)buf;
    unsigned k = (RAW0 + mem) * 8 + xcd;
    int o = order(k, p.rank), h = o >> 8, u = (o & 255) + ((o & 16) ? dest * 8 : 0);
    unit_fetch(p, h, u, lane, lbuf);
    const unsigned call = __builtin_amdgcn_readfirstlane(ld4_sys(cntp)) + 1u;
    unsigned* tk = (unsigned*)(p.ws[p.rank] + OFF_TK) + di * 32;
    const unsigned tk0 = (call - 1u) * (NU - 8 * RAW0) - (RAW0 + NPK) * 8;
    unsigned nput = 0;
    for (;;) {
        WAIT_V(0);
        u32x4 r[16], pk[12];
        unit_read(buf, lane, r);
        unit_format(u, r);
        unsigned tl;
        const unsigned hdr = pack_unit(r, pk, tl, pack_base(r), buf, lane, nullptr) | (4u << 28);
        asm volatile("" : "+v"(pk[0]), "+v"(pk[1]), "+v"(pk[2]), "+v"(pk[3]), "+v"(pk[4]), "+v"(pk[5]), "+v"(pk[6]), "+v"(pk[7]), "+v"(pk[8]), "+v"(pk[9]), "+v"(pk[10]), "+v"(pk[11]));
        while ((unsigned)__builtin_amdgcn_readfirstlane(*(volatile LAS unsigned*)&ctl->freed[mem]) != nput) __builtin_amdgcn_s_sleep(1);
        ++nput;
        if (__builtin_expect(hdr_cnt(hdr) == RAWCNT, 0)) {
            unit_fetch(p, h, u, lane, lbuf);
            WAIT_V(0);
            unit_read(buf, lane, r);
            unit_format(u, r);
            st_raw(p.ws[dest] + OFF_LZ + (call & 1u) * LZ_PAR + (size_t)((6 - di) * NU + k) * UB, voff, r);
            WAIT_V(0);
        } else {
#pragma unroll
            for (int i = 0; i < 12; ++i) *(LAS u32x4*)(buf + 16384 + i * 1024 + lane * 16) = pk[i];
        }
        asm volatile("" ::: "memory");
        unsigned tn = 1u;
        if (lane == 0) {
            const unsigned pos = __hip_atomic_fetch_add(&ctl->tail, 1u, __ATOMIC_RELAXED, __HIP_MEMORY_SCOPE_WORKGROUP);
            *(volatile LAS u32x4*)&ctl->desc[pos & 7] = (u32x4){pos + 1u, k, hdr, (unsigned)mem | (nput << 8) | (tl << 24)};
        }
        if (SHALLOW) while ((unsigned)__builtin_amdgcn_readfirstlane(*(volatile LAS unsigned*)&ctl->freed[mem]) != nput) __builtin_amdgcn_s_sleep(1);
        if (lane == 0) asm volatile("global_atomic_add %0, %1, %0, off sc0\n\ts_waitcnt vmcnt(0)" : "+v"(tn) : "v"(tk) : "memory");
        k = __builtin_amdgcn_readfirstlane(tn) - tk0;
        if (k >= (unsigned)NU) break;
        o = order(k, p.rank); h = o >> 8; u = (o & 255) + ((o & 16) ? dest * 8 : 0);
        unit_fetch(p, h, u, lane, lbuf);
    }
    if (lane == 0) {
        const unsigned pos = __hip_atomic_fetch_add(&ctl->tail, 1u, __ATOMIC_RELAXED, __HIP_MEMORY_SCOPE_WORKGROUP);
        *(volatile LAS u32x4*)&ctl->desc[pos & 7] = (u32x4){pos + 1u, 0u, 0u, 0x80u};
    }
}
__device__ __forceinline__ void storer(const Params& p, LAS unsigned char* lds, LAS SendCtl* ctl, int xcd, int di, int lane, unsigned* cntp) {
    const unsigned dest = (unsigned)(p.rank + 1 + di) & 7u;
    char* dws = p.ws[dest];
    const int li = 6 - di;
    const unsigned voff = lane * 16;
    if (RAW0) unit_fetch(p, 0, 16 + dest * 8 + xcd, lane, (unsigned)(size_t)(lds + 4 * PK_LDS));
    const unsigned call = __builtin_amdgcn_readfirstlane(ld4_sys(cntp)) + 1u;
    const char* lz = dws + OFF_LZ + (call & 1u) * LZ_PAR + (size_t)li * NU * UB;
    char* fa = dws + OFF_FA + (size_t)((call & 1u) * 7 + li) * NU * FLS;
    unsigned pos = 0, done = 0, pend_hdr = 0; int pend = -1;
#define POST(kk, hh) st8_flag(fa + (kk) * FLS, call, (hh), lane)
    if (RAW0) {
        u32x4 r[16];
        unit_read(lds + 4 * PK_LDS, lane, r);
        unit_format(16, r);
        STAMP(5);
        const char* s0 = lz + (size_t)xcd * UB + 4096; const char* s1 = s0 + 8192;
        st16<-4096>(s0, voff, r[0]); st16<-3072>(s0, voff, r[1]); st16<-2048>(s0, voff, r[2]); st16<-1024>(s0, voff, r[3]);
        st16<0>(s0, voff, r[4]); st16<1024>(s0, voff, r[5]); WAIT_V(SWIN); st16<2048>(s0, voff, r[6]); WAIT_V(SWIN); st16<3072>(s0, voff, r[7]); WAIT_V(SWIN);
        st16<-4096>(s1, voff, r[8]); WAIT_V(SWIN); st16<-3072>(s1, voff, r[9]); WAIT_V(SWIN); st16<-2048>(s1, voff, r[10]); WAIT_V(SWIN); st16<-1024>(s1, voff, r[11]); WAIT_V(SWIN);
        st16<0>(s1, voff, r[12]); WAIT_V(SWIN); st16<1024>(s1, voff, r[13]); WAIT_V(SWIN); st16<2048>(s1, voff, r[14]); WAIT_V(SWIN); st16<3072>(s1, voff, r[15]); WAIT_V(SWIN);
        pend = xcd; pend_hdr = (RAWCNT << 16) | (4u << 28);
    }
    while (done < NPK) {
        const u32x4 d = *(volatile LAS u32x4*)&ctl->desc[pos & 7];
        if ((unsigned)__builtin_amdgcn_readfirstlane(d.x) != pos + 1u) {
            if (pend >= 0) { WAIT_V(0); POST(pend, pend_hdr); pend = -1; }
            __builtin_amdgcn_s_sleep(1);
            continue;
        }
        const unsigned k = __builtin_amdgcn_readfirstlane(d.y), hdr = __builtin_amdgcn_readfirstlane(d.z), w = __builtin_amdgcn_readfirstlane(d.w);
        ++pos;
        if (w & 0x80u) { ++done; continue; }
        if (k == NU - 1 - GATE_EARLY && lane < 8)
            __hip_atomic_store((unsigned*)(p.ws[p.rank] + OFF_SD) + (di * 8 + lane) * 32, call, __ATOMIC_RELAXED, __HIP_MEMORY_SCOPE_AGENT);
        if (__builtin_expect(hdr_cnt(hdr) == RAWCNT, 0)) {
            POST(k, hdr);
            *(volatile LAS unsigned*)&ctl->freed[w & 3u] = (w >> 8) & 0xffffu;
            continue;
        }
        u32x4 pk[12];
        LAS unsigned char* src = lds + (w & 3u) * PK_LDS + 16384 + lane * 16;
#pragma unroll
        for (int i = 0; i < 12; ++i) pk[i] = *(LAS u32x4*)(src + i * 1024);
        asm volatile("s_waitcnt lgkmcnt(0)" : "+v"(pk[0]), "+v"(pk[1]), "+v"(pk[2]), "+v"(pk[3]), "+v"(pk[4]), "+v"(pk[5]), "+v"(pk[6]), "+v"(pk[7]), "+v"(pk[8]), "+v"(pk[9]), "+v"(pk[10]), "+v"(pk[11]) :: "memory");
        *(volatile LAS unsigned*)&ctl->freed[w & 3u] = (w >> 8) & 0xffffu;
        const char* s0 = lz + (size_t)k * UB + 4096; const char* s1 = s0 + 4096;
#define PEND_FLAG(i) do { if (SWIN == (i) && pend >= 0) POST(pend, pend_hdr); } while (0)
        st_tail(s1, voff, pk[10], pk[11], w >> 24, lane); WAIT_V(SWIN);
        st16<-4096>(s0, voff, pk[0]); WAIT_V(SWIN);
        st16<-3072>(s0, voff, pk[1]); WAIT_V(SWIN); PEND_FLAG(3);
        st16<-2048>(s0, voff, pk[2]); WAIT_V(SWIN); PEND_FLAG(4);
        st16<-1024>(s0, voff, pk[3]); WAIT_V(SWIN); PEND_FLAG(5);
        st16<0>(s0, voff, pk[4]); WAIT_V(SWIN); PEND_FLAG(6);
        st16<1024>(s0, voff, pk[5]); WAIT_V(SWIN); PEND_FLAG(7);
        st16<2048>(s0, voff, pk[6]); WAIT_V(SWIN); PEND_FLAG(8);
        st16<3072>(s0, voff, pk[7]); WAIT_V(SWIN); PEND_FLAG(9);
        st16<0>(s1, voff, pk[8]); WAIT_V(SWIN);
        st16<1024>(s1, voff, pk[9]); WAIT_V(SWIN);
        pend = k; pend_hdr = hdr;
    }
    WAIT_V(0);
    if (pend >= 0) st8_flag(fa + pend * FLS, call, pend_hdr, lane);
}

__device__ __forceinline__ void unpacker(const Params& p, LAS unsigned char* dl, int uw, int lane, unsigned call) {
    char* ws = p.ws[p.rank];
    unsigned* tmo = (unsigned*)(ws + OFF_TMO);
    const unsigned voff = lane * 16;
#pragma unroll
    for (int i = 0; i < 8; ++i) *(LAS u32x4*)(dl + i * 1024 + lane * 16) = (u32x4){0u, 0u, 0u, 0u};
    for (int n = uw; n < 7 * NU; n += NUNP) {
        const int k = n / 7, li = n - k * 7, src = (p.rank + 1 + li) & 7;
        const int o = order(k, src), t = o & 15; const bool isx = !(o & 16);
        const int q = NH + (o >> 8) * 7 + li;
        const unsigned hdr = wait_unit(ws + OFF_FA + (size_t)(((call & 1u) * 7 + li) * NU + k) * FLS, call, 4u, tmo);
        const unsigned cnt = hdr_cnt(hdr); const bool raw = cnt == RAWCNT;
        unsigned lo = 0;
        const char* sl = ws + OFF_LZ + (call & 1u) * LZ_PAR + (size_t)(li * NU + k) * UB;
        const char* dst = ws + (isx ? OFF_XI + (size_t)q * XQ : OFF_WI + (size_t)q * WQ) + (size_t)t * UB;
        unsigned* flag = (unsigned*)(ws + (isx ? OFF_FX : OFF_FW)) + (isx ? t : p.rank * 8 + t) * FSTR + q;
        if (__builtin_expect(raw, 0)) {
            for (int s = 0; s < 4; ++s) {
                u32x4 o[4]; ld4x16(o[0], o[1], o[2], o[3], sl + s * 4096, voff);
                const char* sb = dst + s * 4096;
                st16<0>(sb, voff, o[0]); st16<1024>(sb, voff, o[1]); st16<2048>(sb, voff, o[2]); st16<3072>(sb, voff, o[3]);
            }
        } else {
            u32x4 r[10], G; unsigned F;
            if (hdr & (1u << 26)) ld_tail2(sl, voff, (unsigned)(size_t)(dl + DEC_TAIL));
            ld_packed(r, F, sl, voff, (unsigned)(size_t)(dl + DEC_TAIL));
            const unsigned LL = (hdr & 0x7fffu) * 0x10001u;
            lo = unpack_head(F, hdr, dl, lane, G);
#pragma unroll
            for (int s = 0; s < 4; ++s) {
                u32x4 o[4]; unpack_sub(r, G, s, dl, lane, LL, o);
                const char* sb = dst + s * 4096;
                st16<0>(sb, voff, o[0]); st16<1024>(sb, voff, o[1]); st16<2048>(sb, voff, o[2]); st16<3072>(sb, voff, o[3]);
            }
        }
        WAIT_V(0);
        if (lane == 0) __hip_atomic_store(flag, call, __ATOMIC_RELAXED, __HIP_MEMORY_SCOPE_SYSTEM);
        if (!raw) { patch_apply(dl, lo, hdr, lane, true); WAIT_L(0); }
    }
}

__device__ __forceinline__ void formatter(const Params& p, LAS unsigned char* buf, int fw, int n2nd, int lane, unsigned call) {
    char* ws = p.ws[p.rank];
    const unsigned voff = lane * 16;
    const unsigned long long ts = __builtin_amdgcn_s_memrealtime();
    for (int n = fw < 384 ? fw : n2nd; n < 1280; n = (n < 384) ? n2nd : n + NFMT) {
        const unsigned at = n < 384 ? PD0 + n * PDT / 16 : FD0 + (n - 384) * FDT / 16;
        while (__builtin_amdgcn_s_memrealtime() - ts < at) __builtin_amdgcn_s_sleep(4);
        int h, u; const char* dst; unsigned* flag;
        if (n < 384) { h = n / 24; const int v = n - h * 24;
            if (v < 16) { u = v; dst = ws + OFF_XI + (size_t)h * XQ + (size_t)v * UB; flag = (unsigned*)(ws + OFF_FX) + v * FSTR + h; }
            else { u = p.rank * 8 + v; dst = ws + OFF_WI + (size_t)h * WQ + (size_t)(v - 16) * UB; flag = (unsigned*)(ws + OFF_FW) + (u - 16) * FSTR + h; } }
        else { const int n2 = n - 384; h = n2 / 56; const int v = n2 - h * 56, li = v >> 3, cn = ((p.rank + 1 + li) & 7) * 8 + (v & 7);
            u = 16 + cn; dst = ws + OFF_WD + (size_t)(li * NH + h) * WQ + (size_t)(v & 7) * UB; flag = (unsigned*)(ws + OFF_FW) + cn * FSTR + h; }
        u32x4 r[16];
        unit_fetch(p, h, u, lane, (unsigned)(size_t)buf);
        WAIT_V(0);
        unit_read(buf, lane, r);
        unit_format(u, r);
        st_raw(dst, voff, r);
        WAIT_V(0);
        if (lane == 0) __hip_atomic_store(flag, call, __ATOMIC_RELAXED, __HIP_MEMORY_SCOPE_SYSTEM);
    }
}

__device__ __forceinline__ int poll_ready(const unsigned* fx, const unsigned* fw, int ready, int need, int skip, unsigned call, unsigned* tmo) {
    const int lane = __builtin_amdgcn_mbcnt_hi(~0u, __builtin_amdgcn_mbcnt_lo(~0u, 0u));
    const unsigned long long t0 = __builtin_amdgcn_s_memrealtime(); unsigned polls = 0;
    for (;;) {
        const int q = ready + (lane & 31);
        const unsigned v = ld4_sys((lane < 32 ? fx : fw) + q);
        const bool skipped = lane < 32 && q >= NH && (q - NH) % 7 == skip;
        const unsigned long long bal = __ballot(q >= NQ || v >= call || skipped);
        const unsigned both = (unsigned)bal & (unsigned)(bal >> 32);
        ready += (both == ~0u) ? 32 : __builtin_ctz(~both);
        if (ready >= need) break;
        if (give_up(tmo, call, t0, polls)) { ready = 1 << 20; break; }
        __builtin_amdgcn_s_sleep(1);
    }
    return ready;
}

constexpr int HTB = 16384;

__global__ __launch_bounds__(512, 2) void fused_kernel(const Params p) {
    __shared__ __attribute__((aligned(16))) unsigned char shm[8 * HTB + 8192 + 256];
    LAS unsigned char* lds = (LAS unsigned char*)shm;
    const int wid = __builtin_amdgcn_readfirstlane(threadIdx.x >> 6), lane = threadIdx.x & 63;
    const int xcd = blockIdx.x & 7, slot = blockIdx.x >> 3;
    char* ws = p.ws[p.rank];
    unsigned* tmo = (unsigned*)(ws + OFF_TMO);
    unsigned* cntp = (unsigned*)(ws + OFF_CNT) + blockIdx.x * 8 + wid;
    STAMP(0);
    const bool main_tile = slot >= 16;
    if (slot < 7) {
        LAS SendCtl* ctl = (LAS SendCtl*)(lds + 8 * HTB + 8192);
        if (threadIdx.x < 64) ((volatile LAS unsigned*)ctl)[lane] = 0u;
        WAIT_L(0); BAR;
        if (wid < NPK) packer(p, lds + wid * PK_LDS, ctl, xcd, slot, wid, lane, cntp);
        else if (wid == 4) { __builtin_amdgcn_s_setprio(3); storer(p, lds, ctl, xcd, slot, lane, cntp); __builtin_amdgcn_s_setprio(0); }
    }
    const unsigned call = __builtin_amdgcn_readfirstlane(ld4_sys(cntp)) + 1u;
    if (lane == 0) __hip_atomic_store(cntp, call, __ATOMIC_RELAXED, __HIP_MEMORY_SCOPE_AGENT);

    if (!main_tile) {
        if (slot >= 7) {
            const bool unp = slot < 7 + NUS;
            formatter(p, lds + wid * HTB, wid * 72 + (blockIdx.x - 56), unp ? 1 << 20 : 384 + wid * (NFMT / 8) + (blockIdx.x - 56 - NUS * 8), lane, call);
            const int uw = wid * (NUS * 8) + (blockIdx.x - 56);
            if (unp && uw < NUNP) unpacker(p, lds + wid * HTB, uw, lane, call);
        }
        STAMP(1);
        asm volatile("s_waitcnt vmcnt(0) lgkmcnt(0)" ::: "memory");
        BAR;
    }

    int rt, cn, q0, nt, skip;
    const char* wimg;
    if (main_tile) {
        const int idx = slot - 16;
        rt = ((xcd >> 1) * 2 + ((idx >> 2) & 1)) * 2 + (idx >> 3);
        cn = p.rank * 8 + (xcd & 1) * 4 + (idx & 3);
        const int m = rt >> 1;
        q0 = (m == p.rank) ? NH : 0; skip = (m - p.rank - 1) & 7; nt = 56;
        wimg = ws + OFF_WI;
    } else {
        rt = p.rank * 2 + (slot < 8 ? 1 : 0);
        cn = xcd * 8 + (slot & 7);
        q0 = 0; skip = 7; nt = 8;
        wimg = (xcd == p.rank) ? ws + OFF_WI : ws + OFF_WD + (size_t)((xcd - p.rank - 1) & 7) * NH * WQ;
    }
    const int wr = wid >> 2, wc = wid & 3;
    const unsigned* gate = (skip == 7) ? cntp : (const unsigned*)(ws + OFF_SD) + (skip * 8 + xcd) * 32;
    const unsigned* fx = (const unsigned*)(ws + OFF_FX) + rt * FSTR;
    const unsigned* fw = (const unsigned*)(ws + OFF_FW) + cn * FSTR;
    const char* xb = ws + OFF_XI + (size_t)(rt * 16 + wid) * 1024;
    const char* wb = wimg + (size_t)((cn & 7) * 16 + wr * 8 + wc) * 1024;
    const unsigned voff = lane * 16;
    const unsigned ldsw = wid * 1024;
    const int aoff = wr * 4096 + lane * 16, boff = wc * 2048 + lane * 16;
#define SA(b, h) (((b) * 4 + (h)) * HTB)
#define SB(b, h) (((b) * 4 + 2 + (h)) * HTB)
#define GLDS(g, l) __builtin_amdgcn_global_load_lds((const unsigned*)((g) + voff), (LAS unsigned*)(lds + (l)), 16, 0, GLDS_AUX)
#define STAGE_A(b, h, qa, qb) do { GLDS(wb + (size_t)(qa) * WQ + (h) * 4096, SA(b, h) + ldsw); GLDS(wb + (size_t)(qb) * WQ + (h) * 4096, SA(b, h) + ldsw + 8192); } while (0)
#define STAGE_B(b, h, qa, qb) do { GLDS(xb + (size_t)(qa) * XQ + (h) * 8192, SB(b, h) + ldsw); GLDS(xb + (size_t)(qb) * XQ + (h) * 8192, SB(b, h) + ldsw + 8192); } while (0)
#define LDA(dst, b, h) do { _Pragma("unroll") for (int m = 0; m < 4; ++m) _Pragma("unroll") for (int k = 0; k < 2; ++k) dst[m][k] = *(const LAS bf16x8*)(lds + SA(b, h) + aoff + m * 1024 + k * 8192); } while (0)
#define LDB(dst, b, h) do { _Pragma("unroll") for (int n = 0; n < 2; ++n) _Pragma("unroll") for (int k = 0; k < 2; ++k) dst[n][k] = *(const LAS bf16x8*)(lds + SB(b, h) + boff + n * 1024 + k * 8192); } while (0)
#define MMA(ai, bj, At, Bt) do { __builtin_amdgcn_s_setprio(1); _Pragma("unroll") for (int m = 0; m < 4; ++m) _Pragma("unroll") for (int n = 0; n < 2; ++n) _Pragma("unroll") for (int k = 0; k < 2; ++k) \
        acc[ai][bj][m][n] = __builtin_amdgcn_mfma_f32_16x16x32_bf16(Bt[n][k], At[m][k], acc[ai][bj][m][n], 0, 0, 0); __builtin_amdgcn_s_setprio(0); } while (0)
#define MMA2(ai) do { __builtin_amdgcn_s_setprio(1); _Pragma("unroll") for (int k = 0; k < 2; ++k) _Pragma("unroll") for (int m = 0; m < 4; ++m) _Pragma("unroll") for (int n = 0; n < 2; ++n) { \
        if (k * 8 + m * 2 + n == 16 - KTAIL) { SCHED; if (KPRIO) __builtin_amdgcn_s_setprio(2); BAR; SCHED; } \
        acc[ai][0][m][n] = __builtin_amdgcn_mfma_f32_16x16x32_bf16(B0[n][k], At[m][k], acc[ai][0][m][n], 0, 0, 0); \
        acc[ai][1][m][n] = __builtin_amdgcn_mfma_f32_16x16x32_bf16(B1[n][k], At[m][k], acc[ai][1][m][n], 0, 0, 0); } __builtin_amdgcn_s_setprio(0); \
        if (KTAIL == 0) BAR; } while (0)
#define ENSURE(qn) do { if (__builtin_expect(ready <= (qn), 0)) ready = poll_ready(fx, fw, ready, (qn) + 1, skip, call, tmo); } while (0)
#define NEXTQ(dst) do { dst = qn; ++qn; if (qn == qskip) { ++qn; qskip += 7; } } while (0)
    f32x4 acc[2][2][4][2];
#pragma unroll
    for (int a = 0; a < 2; ++a)
#pragma unroll
        for (int b = 0; b < 2; ++b)
#pragma unroll
            for (int m = 0; m < 4; ++m)
#pragma unroll
                for (int n = 0; n < 2; ++n) acc[a][b][m][n] = (f32x4){0.f, 0.f, 0.f, 0.f};
    bf16x8 At[4][2], B0[2][2], B1[2][2];
    int ready = q0, qn = q0, qskip = (skip == 7) ? (1 << 20) : NH + skip;
    int q0a, q0b, q1a, q1b;
    NEXTQ(q0a); NEXTQ(q0b); NEXTQ(q1a); NEXTQ(q1b);
    ENSURE(q1b);
    STAMP(4);
    STAGE_B(0, 0, q0a, q0b); STAGE_A(0, 0, q0a, q0b); STAGE_B(0, 1, q0a, q0b); STAGE_A(0, 1, q0a, q0b);
    STAGE_B(1, 0, q1a, q1b); STAGE_A(1, 0, q1a, q1b); STAGE_B(1, 1, q1a, q1b);
    if (wr == 1) BAR;
    WAIT_V(8); BAR; BAR;
    for (int t = 0; t < nt; t += 2) {
        if (t == nt - 2) { qn = q0; qskip = 1 << 20; }
        int q2a, q2b, q3a, q3b;
        NEXTQ(q2a); NEXTQ(q2b); NEXTQ(q3a); NEXTQ(q3b);
        LDB(B0, 0, 0); LDA(At, 0, 0); LDB(B1, 0, 1); STAGE_A(1, 1, q1a, q1b);
        WAIT_V(8); WAIT_L(0); BAR; SCHED; MMA2(0); SCHED;
        ENSURE(q2b);
        LDA(At, 0, 1); STAGE_B(0, 0, q2a, q2b); STAGE_A(0, 0, q2a, q2b); STAGE_B(0, 1, q2a, q2b);
        WAIT_V(8); WAIT_L(0); BAR; SCHED; MMA2(1); SCHED;
        if (t == nt - 2)
            __builtin_amdgcn_global_load_lds(lane == 1 ? tmo : gate, (LAS unsigned*)(lds + 8 * HTB + wid * 256), 4, 0, 17);
        LDB(B0, 1, 0); LDA(At, 1, 0); LDB(B1, 1, 1); STAGE_A(0, 1, q2a, q2b);
        WAIT_V(8); WAIT_L(0); BAR; SCHED; MMA2(0); SCHED;
        ENSURE(q3b);
        LDA(At, 1, 1); STAGE_B(1, 0, q3a, q3b); STAGE_A(1, 0, q3a, q3b); STAGE_B(1, 1, q3a, q3b);
        WAIT_V(8); WAIT_L(0); BAR; SCHED; MMA2(1); SCHED;
        q1a = q3a; q1b = q3b;
    }
    STAMP(2);
    int tz = threadIdx.x; asm volatile("" : "+v"(tz));
    const int ln = tz & 63;
    const unsigned vo = ln * 16;
    const unsigned tile = (unsigned)((rt & 1) * 64 + cn) * 8 + wid;
    if (main_tile) {
        const u32x2 gv = *(volatile LAS u32x2*)(lds + 8 * HTB + wid * 256);
        bool bad = (unsigned)__builtin_amdgcn_readfirstlane(gv.y) == call;
        const bool open = (unsigned)__builtin_amdgcn_readfirstlane(gv.x) >= call && !bad;
        const unsigned owner = (unsigned)(rt >> 1) & 7u;
        char* ows = p.ws[owner];
        const char* sl = ows + OFF_PL + (size_t)tile * UB;
        char* fl = ows + OFF_FP + (size_t)tile * FLS;
        const char* s0 = sl + 4096; const char* s1 = sl + 8192;
        const int staged = (rt & 1) ? BSTAGE : ASTAGE;
        u32x4 b[16], pk[12];
#pragma unroll
        for (int bj = 0; bj < 2; ++bj)
#pragma unroll
            for (int n = 0; n < 2; ++n)
#pragma unroll
                for (int r = 0; r < 4; ++r)
#pragma unroll
                    for (int ai = 0; ai < 2; ++ai)
#pragma unroll
                        for (int m2 = 0; m2 < 2; ++m2)
                            b[bj * 8 + n * 4 + r][ai * 2 + m2] = cvt_pk_bf16(acc[ai][bj][2 * m2][n][r], acc[ai][bj][2 * m2 + 1][n][r]);
        asm volatile("" : "+v"(b[0]), "+v"(b[1]), "+v"(b[2]), "+v"(b[3]), "+v"(b[4]), "+v"(b[5]), "+v"(b[6]), "+v"(b[7]), "+v"(b[8]), "+v"(b[9]), "+v"(b[10]), "+v"(b[11]), "+v"(b[12]), "+v"(b[13]), "+v"(b[14]), "+v"(b[15]) :: "memory");
        if (wr == 0) BAR;
        const unsigned L = pack_base(b);
        if (open) { st16<-4096>(s0, vo, pack_low(b, 0, L)); WAIT_V(1); } else WAIT_V(0);
        asm volatile("" ::: "memory"); BAR; asm volatile("" ::: "memory");
        unsigned tl;
        unsigned hdr = pack_unit(b, pk, tl, L, lds + wid * HTB, ln, open ? s0 : nullptr);
        if (!open) {
            const unsigned long long t0 = __builtin_amdgcn_s_memrealtime(); unsigned polls = 0;
            while (!bad && (unsigned)__builtin_amdgcn_readfirstlane(ld4_sys(gate)) < call) {
                __builtin_amdgcn_s_sleep(2);
                bad = give_up(tmo, call, t0, polls);
            }
        }
        STAMP(6);
        if (bad) {
            hdr = RAWCNT << 16;
#pragma unroll
            for (int f = 0; f < 16; ++f) b[f] = (u32x4){0x7fc07fc0u, 0x7fc07fc0u, 0x7fc07fc0u, 0x7fc07fc0u};
        }
        if (__builtin_expect(hdr_cnt(hdr) == RAWCNT, 0)) { WAIT_V(0); st_raw(sl, vo, b); }
        else {
            if (!open) { st16<-4096>(s0, vo, pk[0]); WAIT_V(0); st16<-3072>(s0, vo, pk[1]); }
            if (!open || !EARLY3) { WAIT_V(0); st16<0>(s1, vo, pk[8]); }
            WAIT_V(0); st16<1024>(s1, vo, pk[9]);
            WAIT_V(0); st_tail(s1, vo, pk[10], pk[11], tl, ln);
            WAIT_V(0); STAMP(7); if (staged & 1) st8_flag(fl, call, hdr | (1u << 28), ln); st16<-2048>(s0, vo, pk[2]);
            WAIT_V(0); st16<-1024>(s0, vo, pk[3]);
            WAIT_V(0); if (staged & 2) st8_flag(fl, call, hdr | (2u << 28), ln); st16<0>(s0, vo, pk[4]);
            WAIT_V(0); st16<1024>(s0, vo, pk[5]);
            if (ENDW) { WAIT_V(0); if (staged & 4) st8_flag(fl, call, hdr | (3u << 28), ln); st16<2048>(s0, vo, pk[6]); st16<3072>(s0, vo, pk[7]); }
            else {
            WAIT_V(0); if (staged & 4) st8_flag(fl, call, hdr | (3u << 28), ln); st16<2048>(s0, vo, pk[6]);
            WAIT_V(0); st16<3072>(s0, vo, pk[7]); }
        }
        WAIT_V(0);
        st8_flag(fl, call, hdr | (4u << 28), ln);
    } else {
        WAIT_V(0);
        if (wr == 0) BAR;
        BAR;
        LAS unsigned char* dl = lds + wid * HTB;
#pragma unroll
        for (int i = 0; i < 8; ++i) *(LAS u32x4*)(dl + i * 1024 + ln * 16) = (u32x4){0u, 0u, 0u, 0u};
        const char* fl = ws + OFF_FP + (size_t)tile * FLS;
        const char* sl = ws + OFF_PL + (size_t)tile * UB;
        unsigned hdr = wait_unit(fl, call, 1u, tmo);
        const unsigned cnt = hdr_cnt(hdr); const bool raw = cnt == RAWCNT;
        const unsigned LL = (hdr & 0x7fffu) * 0x10001u;
        u32x4 r[10], G;
        unsigned tv = 0;
        if (!raw) {
            unsigned F;
            if (hdr & (1u << 26)) ld_tail2(sl, vo, (unsigned)(size_t)(dl + DEC_TAIL));
            ld_head(r, F, tv, sl, vo, (unsigned)(size_t)(dl + DEC_TAIL), tmo);
            unpack_head(F, hdr, dl, ln, G);
        }
        const char* ob = p.out + ((size_t)((rt & 1) * 256 + wc * 32) * NN + cn * 256 + wr * 128) * 2;
        const unsigned oo = (ln >> 4) * (4 * NN * 2) + (ln & 15) * 16;
#pragma unroll
        for (int s = 0; s < 4; ++s) {
            if ((hdr >> 28) <= (unsigned)s) hdr = wait_unit(fl, call, s + 1, tmo);
            u32x4 o[4];
            if (raw) { ld4x16(o[0], o[1], o[2], o[3], sl + s * 4096, vo); tv = ld4_sys(tmo); }
            else { if (s) ld2x16t(r[2 * s], r[2 * s + 1], tv, sl + s * 2048, vo, tmo); unpack_sub(r, G, s, dl, ln, LL, o); }
            const unsigned nanmask = (tv == call) ? 0x7fc07fc0u : 0u;
#pragma unroll
            for (int rr = 0; rr < 4; ++rr) {
                u32x4 y;
#pragma unroll
                for (int cc = 0; cc < 4; ++cc) {
                    const unsigned pv = o[rr][cc];
                    const f32x2 v = (f32x2){acc[cc >> 1][s >> 1][2 * (cc & 1)][s & 1][rr] + __uint_as_float(pv << 16),
                                            acc[cc >> 1][s >> 1][2 * (cc & 1) + 1][s & 1][rr] + __uint_as_float(pv & 0xffff0000u)};
                    const f32x2 g = gelu2(v);
                    y[cc] = cvt_pk_bf16(g.x, g.y) | nanmask;
                }
                st16<0>(ob + (size_t)((s >> 1) * 128 + (s & 1) * 16 + rr) * (NN * 2), oo, y);
            }
        }
    }
    STAMP(3);
}
}

extern "C" void kernel_launch(int rank, int nranks, ncclComm_t comm, hipStream_t stream,
                              void* const* d_in, const int* in_sizes, int n_in,
                              void* d_out, int out_size, const int* in_dtypes, int out_dtype,
                              void* const* d_out_all, void* const* d_ws_all, size_t ws_bytes) {
    (void)comm; (void)in_dtypes; (void)out_dtype; (void)d_out_all;
    if (nranks != NR || (unsigned)rank >= (unsigned)NR || n_in < 2 || in_sizes[0] != MM * KS || in_sizes[1] != KS * NN ||
        out_size != (MM / NR) * NN || ws_bytes < WS_NEED) {
        fprintf(stderr, "kernel_launch: unexpected problem geometry\n");
        return;
    }
    Params p;
    p.x = (const char*)d_in[0];
    p.w = (const char*)d_in[1];
    for (int i = 0; i < NR; ++i) p.ws[i] = (char*)d_ws_all[i];
    p.out = (char*)d_out;
    p.rank = rank; p.pad = 0;
    hipLaunchKernelGGL(fused_kernel, dim3(256), dim3(512), 0, stream, p);
    HIP_CHECK(hipGetLastError());
}
```

```cpp
#include <hip/hip_runtime.h>
#include <rccl/rccl.h>
#include <cstdio>
#include <cstdint>

#define HIP_CHECK(expr) do { hipError_t _e = (expr); if (_e != hipSuccess) { \
    fprintf(stderr, "HIP error %s at %s:%d\n", hipGetErrorString(_e), __FILE__, __LINE__); \
    return; } } while (0)

#define NW0 5
#define LAGXB 36
#define LAGW0 0
#define LAGWS 0
#define LAGWH 4
#define LAGXA 1
#define GLDS_AUX 17
#define FORCE_RAW 0
#define NUS 5
#define SWIN 6
#define SHALLOW 1
#define PKDELAY 4
#define RAW0 1
#define PD0 100
#define PDT 24
#define FD0 1000
#define FDT 72
#define EARLY3 2
#define EARLYJ 19
#define ENDW 0
#define BSTAGE 7
#define ASTAGE 0
#define FLS 128
#define NPK 4
#define GATE_EARLY 12

namespace {
#define LAS __attribute__((address_space(3)))
typedef short bf16x8 __attribute__((ext_vector_type(8)));
typedef float f32x4 __attribute__((ext_vector_type(4)));
typedef float f32x2 __attribute__((ext_vector_type(2)));
typedef unsigned u32x4 __attribute__((ext_vector_type(4)));
typedef unsigned u32x2 __attribute__((ext_vector_type(2)));
typedef unsigned short u16x2 __attribute__((ext_vector_type(2)));

constexpr int NR = 8;
constexpr int MM = 4096, NN = 16384, KS = 512, NS = 2048;
constexpr int NQ = 128;
constexpr int NH = 16;
constexpr int NU = NH * 22;
#define UNP7 1
constexpr int NUNP = UNP7 ? NUS * 64 / 7 * 7 : NUS * 64;
constexpr int NFMT = (9 - NUS) * 64;
constexpr unsigned long long DEADLINE = 20000000ull;
constexpr unsigned RAWCNT = 0x3ffu;

constexpr size_t OFF_CNT = 0;
constexpr size_t OFF_TMO = 8192;
constexpr size_t OFF_TK = 8192 + 1024;
constexpr size_t OFF_SD = 131072;
constexpr int FSTR = 160;
constexpr size_t OFF_FX = 16384;
constexpr size_t OFF_FW = 32768;
constexpr size_t OFF_STAMP = 262144;
constexpr size_t UB = 16384;
constexpr size_t XQ = 256 * 1024, WQ = 128 * 1024;
constexpr size_t OFF_XI = 1u << 20;
constexpr size_t OFF_WI = OFF_XI + (size_t)NQ * XQ;
constexpr size_t OFF_WD = OFF_WI + (size_t)NQ * WQ;
constexpr size_t OFF_PL = OFF_WD + (size_t)7 * NH * WQ;
constexpr size_t OFF_LZ = OFF_PL + (size_t)1024 * UB;
constexpr size_t LZ_PAR = (size_t)7 * NU * UB;
constexpr size_t OFF_FA = OFF_LZ + 2 * LZ_PAR;
constexpr size_t OFF_FP = OFF_FA + (size_t)14 * NU * 128;
constexpr size_t WS_NEED = OFF_FP + 1024 * 128;
static_assert(OFF_FW + 64 * FSTR * 4 <= OFF_SD && OFF_SD + 56 * 128 <= OFF_STAMP, "layout");
static_assert(WS_NEED <= (size_t)161 << 20, "workspace");

struct Params {
    const char* x;
    const char* w;
    char* ws[NR];
    char* out;
    int rank;
    int pad;
};

template <int OFF> __device__ __forceinline__ void st16(const char* sbase, unsigned voff, u32x4 v) {
    asm volatile("global_store_dwordx4 %0, %1, %2 offset:%3 sc0 sc1\n\ts_nop 1" :: "v"(voff), "v"(v), "s"(sbase), "i"(OFF) : "memory");
}
__device__ __forceinline__ unsigned ld4_sys(const void* p) {
    unsigned v;
    asm volatile("global_load_dword %0, %1, off sc0 sc1\n\ts_waitcnt vmcnt(0)" : "=&v"(v) : "v"(p) : "memory");
    return v;
}
__device__ __forceinline__ u32x2 ld8_sys(const void* p) {
    u32x2 v;
    asm volatile("global_load_dwordx2 %0, %1, off sc0 sc1\n\ts_waitcnt vmcnt(0)" : "=&v"(v) : "v"(p) : "memory");
    return v;
}
__device__ __forceinline__ void st8_flag(void* p, unsigned call, unsigned hdr, int lane) {
    if (lane == 0) __hip_atomic_store((unsigned long long*)p, ((unsigned long long)hdr << 32) | call, __ATOMIC_RELAXED, __HIP_MEMORY_SCOPE_SYSTEM);
}
#define WAIT_V_(n) asm volatile("s_waitcnt vmcnt(" #n ")" ::: "memory")
#define WAIT_V(n) WAIT_V_(n)
#define WAIT_L(n) asm volatile("s_waitcnt lgkmcnt(" #n ")" ::: "memory")
#define BAR __builtin_amdgcn_s_barrier()
#define SCHED __builtin_amdgcn_sched_barrier(0)

__device__ __forceinline__ unsigned cvt_pk_bf16(float lo, float hi) {
    unsigned r; asm("v_cvt_pk_bf16_f32 %0, %1, %2" : "=v"(r) : "v"(lo), "v"(hi)); return r;
}
__device__ __forceinline__ f32x2 gelu2(f32x2 y) {
    const float k1 = -2.0f * 0.7978845608028654f * 1.4426950408889634f, k2 = k1 * 0.044715f;
    f32x2 z = y * (y * y * k2 + k1);
    f32x2 e; e.x = __builtin_amdgcn_exp2f(z.x); e.y = __builtin_amdgcn_exp2f(z.y);
    e = e + 1.0f;
    f32x2 r; r.x = __builtin_amdgcn_rcpf(e.x); r.y = __builtin_amdgcn_rcpf(e.y);
    return y * r;
}
__device__ __forceinline__ unsigned pk_sub(unsigned a, unsigned b) { return __builtin_bit_cast(unsigned, __builtin_bit_cast(u16x2, a) - __builtin_bit_cast(u16x2, b)); }
__device__ __forceinline__ unsigned pk_add(unsigned a, unsigned b) { return __builtin_bit_cast(unsigned, __builtin_bit_cast(u16x2, a) + __builtin_bit_cast(u16x2, b)); }
__device__ __forceinline__ unsigned pk_max(unsigned a, unsigned b) { return __builtin_bit_cast(unsigned, __builtin_elementwise_max(__builtin_bit_cast(u16x2, a), __builtin_bit_cast(u16x2, b))); }
__device__ __forceinline__ unsigned bfi(unsigned m, unsigned a, unsigned b) { return __builtin_amdgcn_bitop3_b32(m, a, b, 0xca); }
__device__ __forceinline__ unsigned rotl(unsigned a, int n) { return __builtin_amdgcn_alignbit(a, a, 32 - n); }
__device__ __forceinline__ unsigned rotr(unsigned a, int n) { return __builtin_amdgcn_alignbit(a, a, n); }
__device__ __forceinline__ unsigned umax(unsigned a, unsigned b) { return a > b ? a : b; }

#define STAMP(slot) do {} while (0)

__device__ __forceinline__ bool give_up(unsigned* tmo, unsigned call, unsigned long long t0, unsigned& polls) {
    if ((++polls & 15u) != 0u) return false;
    if (__builtin_amdgcn_readfirstlane(__hip_atomic_load(tmo, __ATOMIC_RELAXED, __HIP_MEMORY_SCOPE_AGENT)) == call ||
        __builtin_amdgcn_s_memrealtime() - t0 > DEADLINE) {
        __hip_atomic_store(tmo, call, __ATOMIC_RELAXED, __HIP_MEMORY_SCOPE_AGENT);
        return true;
    }
    return false;
}
__device__ __forceinline__ unsigned wait_unit(const char* flag, unsigned call, unsigned st, unsigned* tmo) {
    const unsigned long long t0 = __builtin_amdgcn_s_memrealtime(); unsigned polls = 0;
    for (;;) {
        const u32x2 v = ld8_sys(flag);
        const unsigned hdr = __builtin_amdgcn_readfirstlane(v.y);
        if ((unsigned)__builtin_amdgcn_readfirstlane(v.x) == call && (hdr >> 28) >= st) return hdr;
        __builtin_amdgcn_s_sleep(1);
        if (give_up(tmo, call, t0, polls)) return (RAWCNT << 16) | (4u << 28);
    }
}
__device__ __forceinline__ unsigned hdr_cnt(unsigned hdr) { return (hdr >> 16) & 0x3ffu; }

constexpr int SCR_DENSE = 8192;
#define PT(i) do {} while (0)
__device__ __forceinline__ unsigned umin(unsigned a, unsigned b) { return a < b ? a : b; }
__device__ __forceinline__ unsigned wave_scan(unsigned inc) {
    inc += __builtin_amdgcn_update_dpp(0, (int)inc, 0x111, 0xf, 0xf, true);
    inc += __builtin_amdgcn_update_dpp(0, (int)inc, 0x112, 0xf, 0xf, true);
    inc += __builtin_amdgcn_update_dpp(0, (int)inc, 0x114, 0xf, 0xf, true);
    inc += __builtin_amdgcn_update_dpp(0, (int)inc, 0x118, 0xf, 0xf, true);
    inc += __builtin_amdgcn_update_dpp(0, (int)inc, 0x142, 0xa, 0xf, false);
    inc += __builtin_amdgcn_update_dpp(0, (int)inc, 0x143, 0xc, 0xf, false);
    return inc;
}
__device__ __forceinline__ unsigned pack_base(const u32x4 (&b)[16]) {
    unsigned mx = 0;
#pragma unroll
    for (int f = 0; f < 16; ++f) mx = pk_max(mx, b[f][f & 3] & 0x7fff7fffu);
    unsigned m = umax(mx & 0xffffu, mx >> 16);
    m = umax(m, (unsigned)__builtin_amdgcn_update_dpp(0, (int)m, 0x128, 0xf, 0xf, false));
    m = umax(m, (unsigned)__builtin_amdgcn_update_dpp(0, (int)m, 0x124, 0xf, 0xf, false));
    m = umax(m, (unsigned)__builtin_amdgcn_update_dpp(0, (int)m, 0x122, 0xf, 0xf, false));
    m = umax(m, (unsigned)__builtin_amdgcn_update_dpp(0, (int)m, 0x121, 0xf, 0xf, false));
    const unsigned m0 = __builtin_amdgcn_readlane(m, 0), m1 = __builtin_amdgcn_readlane(m, 16), m2 = __builtin_amdgcn_readlane(m, 32), m3 = __builtin_amdgcn_readlane(m, 48);
    const unsigned ms = umin(umax(umin(m0, m1), umin(m2, m3)), umin(umax(m0, m1), umax(m2, m3)));
    return umax(ms, 1023u) - 1023u;
}
__device__ __forceinline__ u32x4 pack_low(const u32x4 (&b)[16], int i, unsigned L) {
    const unsigned LL = L | (L << 16);
    u32x4 r;
#pragma unroll
    for (int c = 0; c < 4; ++c) r[c] = __builtin_amdgcn_perm(pk_sub(b[2 * i + (c >> 1)][2 * (c & 1) + 1], LL), pk_sub(b[2 * i + (c >> 1)][2 * (c & 1)], LL), 0x06040200u);
    return r;
}
__device__ __forceinline__ unsigned pack_unit(const u32x4 (&b)[16], u32x4 (&pk)[12], unsigned& tl, unsigned L, LAS unsigned char* scr, int lane, const char* early) {
    const unsigned LL = L | (L << 16);
    const unsigned wp0 = (unsigned)(size_t)scr + lane * 4;
    unsigned wp = wp0;
    unsigned z[4], seg[4], len[4], F = 0;
    PT(0);
#pragma unroll
    for (int s = 0; s < 4; ++s) {
        unsigned hb[8];
#pragma unroll
        for (int jj = 0; jj < 8; ++jj) {
            const int j = 8 * s + jj;
            const unsigned d0 = pk_sub(b[j >> 1][2 * (j & 1)], LL), d1 = pk_sub(b[j >> 1][2 * (j & 1) + 1], LL);
            pk[j >> 2][j & 3] = __builtin_amdgcn_perm(d1, d0, 0x06040200u);
            hb[jj] = __builtin_amdgcn_perm(d1, d0, 0x07050301u);
            const unsigned t = hb[jj] & 0x7c7c7c7cu; unsigned tmp;
            *(LAS unsigned*)(size_t)wp = t | (unsigned)((j & 3) | ((j >> 2) << 7));
            asm("v_min_u32 %0, 1, %2\n\tv_lshl_add_u32 %1, %0, 8, %1" : "=&v"(tmp), "+v"(wp) : "v"(t));
            if (jj & 3) hb[jj] = rotl(hb[jj], 2 * (jj & 3));
            if (j == EARLYJ && early) { WAIT_V(0); st16<-3072>(early, lane * 16, pk[1]); }
        }
        pk[8][s] = bfi(0x81818181u, hb[0], bfi(0x06060606u, hb[1], bfi(0x18181818u, hb[2], hb[3])));
        pk[9][s] = bfi(0x81818181u, hb[4], bfi(0x06060606u, hb[5], bfi(0x18181818u, hb[6], hb[7])));
        const unsigned ga = bfi(0x02020202u, hb[0], bfi(0x08080808u, hb[1], bfi(0x20202020u, hb[2], hb[3])));
        const unsigned gb = bfi(0x02020202u, hb[4], bfi(0x08080808u, hb[5], bfi(0x20202020u, hb[6], hb[7])));
        z[s] = ~bfi(0xaaaaaaaau, ga, gb >> 1);
    }
    PT(1);
    unsigned e8[8], nz4[4];
#pragma unroll
    for (int i = 0; i < 8; ++i) e8[i] = *(LAS unsigned*)(scr + i * 256 + lane * 4);
#pragma unroll
    for (int s = 0; s < 4; ++s) {
        unsigned t = z[s] | (z[s] >> 1); t |= t << 2;
        nz4[s] = t & 0x44444444u; F |= s < 2 ? nz4[s] >> (2 - s) : nz4[s] << (s - 2);
    }
    const unsigned nrow = (wp - wp0) >> 8, mine = (__builtin_popcount(F) << 14) | nrow;
    const unsigned inc = wave_scan(mine), tot = __builtin_amdgcn_readlane(inc, 63), ngrp = tot & 0xfffu;
    const unsigned lo = 256u + ((((tot >> 12) + 31u) >> 5) << 2);
    unsigned cnt = ngrp, bad = 0;
    PT(2);
    if (ngrp <= 512u) {
        LAS unsigned* dense = (LAS unsigned*)(scr + SCR_DENSE);
        const unsigned lanebits = ((lane & 7) << 15) | ((lane >> 3) << 23), base = (inc - mine) & 0xfffu;
#pragma unroll
        for (int i = 0; i < 8; ++i) dense[(unsigned)i < nrow ? base + i : 512 + lane] = e8[i] | lanebits;
        for (unsigned r = 8; __builtin_expect(__any(nrow > r), 0); ++r)
            dense[r < nrow ? base + r : 512 + lane] = *(LAS unsigned*)(scr + r * 256 + lane * 4) | lanebits;
        unsigned e2[2] = {dense[lane], dense[64 + lane]};
        *(LAS u32x4*)(scr + 256 + lane * 16) = (u32x4){0u, 0u, 0u, 0u};
        *(LAS unsigned*)(scr + lane * 4) = F;
        PT(3);
        if (EARLY3 && early) { WAIT_V(0); st16<0>(early + 4096, lane * 16, pk[8]); }
#pragma unroll
        for (int s = 0; s < 4; ++s) { seg[s] = z[s] & 15u; len[s] = nz4[s] & 7u; }
#pragma unroll
        for (int i = 1; i < 8; ++i)
#pragma unroll
            for (int s = 0; s < 4; ++s) { seg[s] |= ((z[s] >> (4 * i)) & 15u) << len[s]; len[s] += (nz4[s] >> (4 * i)) & 7u; }
        PT(4);
        unsigned bit = (inc - mine) >> 12;
#pragma unroll
        for (int s = 0; s < 4; ++s) {
            const unsigned long long v = (unsigned long long)seg[s] << (bit & 31u);
            LAS unsigned* w = (LAS unsigned*)(scr + 256) + (bit >> 5);
            __hip_atomic_fetch_or(w, (unsigned)v, __ATOMIC_RELAXED, __HIP_MEMORY_SCOPE_WAVEFRONT);
            __hip_atomic_fetch_or(w + 1, (unsigned)(v >> 32), __ATOMIC_RELAXED, __HIP_MEMORY_SCOPE_WAVEFRONT);
            bit += len[s];
        }
        PT(5);
        if (EARLY3 > 1 && early) { WAIT_V(0); st16<1024>(early + 4096, lane * 16, pk[9]); }
        LAS unsigned short* lst = (LAS unsigned short*)(scr + lo);
        const unsigned hz = ((0u - L) >> 10) & 31u;
        for (unsigned i0 = 0;;) {
            unsigned t[2], pos[2], at[2];
#pragma unroll
            for (int c = 0; c < 2; ++c) {
                const unsigned e = e2[c], ix = i0 + c * 64 + lane;
                t[c] = ix < ngrp ? e & 0x7c7c7c7cu : 0u;
                pos[c] = ((e & 0x380u) << 3) | ((e >> 16) & 0x380u) | ((e >> 11) & 0x70u) | ((e & 3u) << 2);
                at[c] = ix < ngrp ? ix : 1024u + lane;
            }
            for (;;) {
#pragma unroll
                for (int c = 0; c < 2; ++c) {
                    const unsigned b8 = __builtin_ctz(t[c] | 0x80000000u) & 24u, hi5 = (t[c] >> (b8 + 2)) & 31u;
                    const unsigned code = hi5 == hz ? 5u : (hi5 + 5u) & 31u;
                    bad = umax(bad, code);
                    lst[at[c]] = (unsigned short)((code << 13) | pos[c] | (b8 >> 3));
                    t[c] &= ~(0xffu << b8);
                }
                const unsigned long long b0 = __ballot(t[0] != 0), b1 = __ballot(t[1] != 0);
                if (!(b0 | b1)) break;
                at[0] = t[0] != 0 ? __builtin_amdgcn_mbcnt_hi((unsigned)(b0 >> 32), __builtin_amdgcn_mbcnt_lo((unsigned)b0, umin(cnt, 512u))) : 1024u + lane;
                cnt += __builtin_popcountll(b0);
                at[1] = t[1] != 0 ? __builtin_amdgcn_mbcnt_hi((unsigned)(b1 >> 32), __builtin_amdgcn_mbcnt_lo((unsigned)b1, umin(cnt, 512u))) : 1024u + lane;
                cnt += __builtin_popcountll(b1);
            }
            i0 += 128;
            if (__builtin_expect(i0 >= ngrp, 1)) break;
            e2[0] = dense[i0 + lane]; e2[1] = dense[i0 + 64 + lane];
        }
    }
    PT(6);
    pk[10] = *(LAS u32x4*)(scr + lane * 16);
    pk[11] = *(LAS u32x4*)(scr + 1024 + lane * 16);
    tl = (lo + 2u * cnt + 63u) >> 6;
    if (cnt > 512u || tl > 32u || __any(bad > 7u) || FORCE_RAW) cnt = RAWCNT;
    return L | (cnt << 16) | (tl > 16u ? 1u << 26 : 0u);
}
__device__ __forceinline__ void st_tail(const char* sb8k  , unsigned voff, u32x4 t0, u32x4 t1, unsigned tl, int lane) {
    if (__builtin_expect(tl > 16u, 0)) { if ((unsigned)lane < (tl - 16u) * 4u) st16<3072>(sb8k, voff, t1); }
    if ((unsigned)lane < tl * 4u) st16<2048>(sb8k, voff, t0);
}
__device__ __forceinline__ void st_raw(const char* slot, unsigned voff, const u32x4 (&b)[16]) {
    const char* s0 = slot + 4096; const char* s1 = slot + 12288;
    st16<-4096>(s0, voff, b[0]); st16<-3072>(s0, voff, b[1]); st16<-2048>(s0, voff, b[2]); st16<-1024>(s0, voff, b[3]);
    st16<0>(s0, voff, b[4]); st16<1024>(s0, voff, b[5]); st16<2048>(s0, voff, b[6]); st16<3072>(s0, voff, b[7]);
    st16<-4096>(s1, voff, b[8]); st16<-3072>(s1, voff, b[9]); st16<-2048>(s1, voff, b[10]); st16<-1024>(s1, voff, b[11]);
    st16<0>(s1, voff, b[12]); st16<1024>(s1, voff, b[13]); st16<2048>(s1, voff, b[14]); st16<3072>(s1, voff, b[15]);
}
constexpr int DEC_TAIL = 8448;
__device__ __forceinline__ void ld_tail2(const char* slot, unsigned voff, unsigned ldst) {
    unsigned keep;
    asm volatile("s_mov_b32 %0, m0\n\ts_mov_b32 m0, %3\n\ts_nop 0\n\tglobal_load_lds_dwordx4 %1, %2 sc0 sc1\n\ts_mov_b32 m0, %0"
                 : "=&s"(keep) : "v"(voff), "s"(slot + 11264), "s"(ldst + 1024) : "memory");
}
__device__ __forceinline__ void ld_packed(u32x4 (&r)[10], unsigned& F, const char* slot, unsigned voff, unsigned ldst) {
    const char* s0 = slot + 4096; const char* s1 = slot + 10240;
    unsigned keep;
    asm volatile("s_mov_b32 %11, m0\n\ts_mov_b32 m0, %16\n\t"
                 "global_load_dword %10, %13, %15 sc0 sc1\n\tglobal_load_lds_dwordx4 %12, %15 sc0 sc1\n\t"
                 "global_load_dwordx4 %8, %12, %15 offset:-2048 sc0 sc1\n\tglobal_load_dwordx4 %9, %12, %15 offset:-1024 sc0 sc1\n\t"
                 "global_load_dwordx4 %0, %12, %14 offset:-4096 sc0 sc1\n\tglobal_load_dwordx4 %1, %12, %14 offset:-3072 sc0 sc1\n\t"
                 "global_load_dwordx4 %2, %12, %14 offset:-2048 sc0 sc1\n\tglobal_load_dwordx4 %3, %12, %14 offset:-1024 sc0 sc1\n\t"
                 "global_load_dwordx4 %4, %12, %14 sc0 sc1\n\tglobal_load_dwordx4 %5, %12, %14 offset:1024 sc0 sc1\n\t"
                 "global_load_dwordx4 %6, %12, %14 offset:2048 sc0 sc1\n\tglobal_load_dwordx4 %7, %12, %14 offset:3072 sc0 sc1\n\t"
                 "s_mov_b32 m0, %11\n\ts_waitcnt vmcnt(0)"
                 : "=&v"(r[0]), "=&v"(r[1]), "=&v"(r[2]), "=&v"(r[3]), "=&v"(r[4]), "=&v"(r[5]), "=&v"(r[6]), "=&v"(r[7]),
                   "=&v"(r[8]), "=&v"(r[9]), "=&v"(F), "=&s"(keep)
                 : "v"(voff), "v"(voff >> 2), "s"(s0), "s"(s1), "s"(ldst) : "memory");
}
__device__ __forceinline__ void ld_head(u32x4 (&r)[10], unsigned& F, unsigned& t, const char* slot, unsigned voff, unsigned ldst, const unsigned* tp) {
    const char* s1 = slot + 10240;
    unsigned keep;
    asm volatile("s_mov_b32 %6, m0\n\ts_mov_b32 m0, %11\n\t"
                 "global_load_dword %4, %8, %10 sc0 sc1\n\tglobal_load_lds_dwordx4 %7, %10 sc0 sc1\n\t"
                 "global_load_dwordx4 %2, %7, %10 offset:-2048 sc0 sc1\n\tglobal_load_dwordx4 %3, %7, %10 offset:-1024 sc0 sc1\n\t"
                 "global_load_dwordx4 %0, %7, %9 sc0 sc1\n\tglobal_load_dwordx4 %1, %7, %9 offset:1024 sc0 sc1\n\t"
                 "global_load_dword %5, %12, off sc1\n\ts_mov_b32 m0, %6\n\ts_waitcnt vmcnt(0)"
                 : "=&v"(r[0]), "=&v"(r[1]), "=&v"(r[8]), "=&v"(r[9]), "=&v"(F), "=&v"(t), "=&s"(keep)
                 : "v"(voff), "v"(voff >> 2), "s"(slot), "s"(s1), "s"(ldst), "v"(tp) : "memory");
}
__device__ __forceinline__ void ld4x16(u32x4& a, u32x4& b, u32x4& c, u32x4& d, const char* sb, unsigned voff) {
    asm volatile("global_load_dwordx4 %0, %4, %5 sc0 sc1\n\tglobal_load_dwordx4 %1, %4, %5 offset:1024 sc0 sc1\n\t"
                 "global_load_dwordx4 %2, %4, %5 offset:2048 sc0 sc1\n\tglobal_load_dwordx4 %3, %4, %5 offset:3072 sc0 sc1\n\ts_waitcnt vmcnt(0)"
                 : "=&v"(a), "=&v"(b), "=&v"(c), "=&v"(d) : "v"(voff), "s"(sb) : "memory");
}
__device__ __forceinline__ void ld2x16t(u32x4& a, u32x4& b, unsigned& t, const char* sb, unsigned voff, const unsigned* tp) {
    asm volatile("global_load_dwordx4 %0, %3, %4 sc0 sc1\n\tglobal_load_dwordx4 %1, %3, %4 offset:1024 sc0 sc1\n\t"
                 "global_load_dword %2, %5, off sc1\n\ts_waitcnt vmcnt(0)"
                 : "=&v"(a), "=&v"(b), "=&v"(t) : "v"(voff), "s"(sb), "v"(tp) : "memory");
}
__device__ __forceinline__ void patch_apply(LAS unsigned char* dl, unsigned lo, unsigned hdr, int lane, bool zero) {
    const unsigned cnt = hdr_cnt(hdr), lut_hi = 0x0804007cu | ((((0u - (hdr & 0x7fffu)) >> 10) & 31u) << 10);
    for (unsigned c = 0; c < cnt; c += 128) {
        const unsigned w = *(LAS unsigned*)(dl + DEC_TAIL + lo + c * 2 + lane * 4);
#pragma unroll
        for (int hf = 0; hf < 2; ++hf) {
            const unsigned e = hf ? w >> 16 : w & 0xffffu;
            const unsigned at = c + lane * 2 + hf < cnt ? e & 0x1fffu : 8192u + lane;
            dl[at] = zero ? 0 : (unsigned char)__builtin_amdgcn_perm(lut_hi, 0x7874706cu, e >> 13);
        }
    }
}
__device__ __forceinline__ unsigned unpack_head(unsigned F, unsigned hdr, LAS unsigned char* dl, int lane, u32x4& G) {
    PT(8);
    const unsigned mine = __builtin_popcount(F) << 2, inc = wave_scan(mine);
    const unsigned lo = 256u + ((((unsigned)__builtin_amdgcn_readlane(inc, 63) + 31u) >> 5) << 2);
    unsigned bit = inc - mine, sg[4];
#pragma unroll
    for (int s = 0; s < 4; ++s) {
        const LAS unsigned* w = (const LAS unsigned*)(dl + DEC_TAIL + 256) + (bit >> 5);
        sg[s] = __builtin_amdgcn_alignbit(w[1], w[0], bit);
        bit += __builtin_popcount(F & (0x11111111u << s)) << 2;
    }
    PT(9);
    patch_apply(dl, lo, hdr, lane, false);
    PT(10);
#pragma unroll
    for (int s = 0; s < 4; ++s) {
        const unsigned nz = (F >> s) & 0x11111111u, pre = nz * 0x11111110u;
        const unsigned xe = sg[s] & 0x0f0f0f0fu, xo = (sg[s] >> 4) & 0x0f0f0f0fu;
        const unsigned b0 = __builtin_amdgcn_perm(xo, xe, 0x05010400u), b1 = __builtin_amdgcn_perm(xo, xe, 0x07030602u);
        const unsigned ze = __builtin_amdgcn_perm(b1, b0, pre & 0x07070707u), zo = __builtin_amdgcn_perm(b1, b0, (pre >> 4) & 0x07070707u);
        G[s] = ~((ze | (zo << 4)) & ((nz << 4) - nz));
    }
    PT(11);
    return lo;
}
__device__ __forceinline__ void unpack_sub(const u32x4 (&r)[10], u32x4 G, int s, LAS unsigned char* dl, int lane, unsigned LL, u32x4 (&o)[4]) {
    const u32x4 p0 = *(LAS u32x4*)(dl + (2 * s) * 1024 + lane * 16), p1 = *(LAS u32x4*)(dl + (2 * s + 1) * 1024 + lane * 16);
    const unsigned A = r[8][s], B = r[9][s], ga = G[s], gb = G[s] << 1;
    unsigned hb[8];
    hb[0] = bfi(0x83838383u, bfi(0x02020202u, ga, A), p0[0]); hb[1] = bfi(0x83838383u, rotr(bfi(0x08080808u, ga, A), 2), p0[1]);
    hb[2] = bfi(0x83838383u, rotr(bfi(0x20202020u, ga, A), 4), p0[2]); hb[3] = bfi(0x83838383u, rotr(bfi(0x80808080u, ga, A), 6), p0[3]);
    hb[4] = bfi(0x83838383u, bfi(0x02020202u, gb, B), p1[0]); hb[5] = bfi(0x83838383u, rotr(bfi(0x08080808u, gb, B), 2), p1[1]);
    hb[6] = bfi(0x83838383u, rotr(bfi(0x20202020u, gb, B), 4), p1[2]); hb[7] = bfi(0x83838383u, rotr(bfi(0x80808080u, gb, B), 6), p1[3]);
#pragma unroll
    for (int jj = 0; jj < 8; ++jj) {
        const unsigned lo = r[2 * s + (jj >> 2)][jj & 3];
        o[jj >> 1][2 * (jj & 1)] = pk_add(__builtin_amdgcn_perm(hb[jj], lo, 0x05010400u), LL);
        o[jj >> 1][2 * (jj & 1) + 1] = pk_add(__builtin_amdgcn_perm(hb[jj], lo, 0x07030602u), LL);
    }
}

__device__ __forceinline__ void unit_fetch(const Params& p, int h, int u, int lane, unsigned ldsbuf) {
    const bool isx = u < 16;
    const char* base = isx ? p.x + ((size_t)u * 256 * KS + h * 32) * 2 : p.w + ((size_t)h * 32 * NN + (u - 16) * 256) * 2;
    unsigned voff = isx ? (lane & 15) * (KS * 2) + (lane >> 4) * 16 : (lane >> 4) * (8 * NN * 2) + (lane & 15) * 16;
    const unsigned s1 = isx ? 16 * KS * 2 : NN * 2, s2 = isx ? 16 * KS * 2 : 256 - 7 * NN * 2;
#pragma unroll
    for (int i = 0; i < 16; ++i) {
        unsigned keep;
        asm volatile("s_mov_b32 %0, m0\n\ts_mov_b32 m0, %3\n\ts_nop 0\n\tglobal_load_lds_dwordx4 %1, %2\n\ts_mov_b32 m0, %0"
                     : "=&s"(keep) : "v"(voff), "s"(base), "s"(ldsbuf + i * 1024) : "memory");
        voff += (i == 7) ? s2 : s1;
    }
}
__device__ __forceinline__ void unit_read(LAS unsigned char* buf, int lane, u32x4 (&r)[16]) {
#pragma unroll
    for (int i = 0; i < 16; ++i) r[i] = *(LAS u32x4*)(buf + i * 1024 + lane * 16);
    asm volatile("s_waitcnt lgkmcnt(0)" : "+v"(r[0]), "+v"(r[1]), "+v"(r[2]), "+v"(r[3]), "+v"(r[4]), "+v"(r[5]), "+v"(r[6]), "+v"(r[7]),
                 "+v"(r[8]), "+v"(r[9]), "+v"(r[10]), "+v"(r[11]), "+v"(r[12]), "+v"(r[13]), "+v"(r[14]), "+v"(r[15]) :: "memory");
}
__device__ __forceinline__ void unit_format(int u, u32x4 (&r)[16]) {
    if (u >= 16) {
#pragma unroll
        for (int g = 0; g < 2; ++g) {
            u32x4 o[8];
#pragma unroll
            for (int i = 0; i < 8; ++i)
#pragma unroll
                for (int a = 0; a < 4; ++a)
                    o[i][a] = __builtin_amdgcn_perm(r[g * 8 + 2 * a + 1][i >> 1], r[g * 8 + 2 * a][i >> 1], (i & 1) ? 0x07060302u : 0x05040100u);
#pragma unroll
            for (int i = 0; i < 8; ++i) r[g * 8 + i] = o[i];
        }
    }
}
struct OrderTab { unsigned e[NU]; };
constexpr OrderTab make_order() {
    OrderTab t{}; int n = 0;
    for (int v = 0; v < 8 * NW0; ++v) t.e[n++] = (v >> 3) << 8 | (v & 7);
    for (int key = 0; key < 8 * NH + 512; ++key)
        for (int v = 0; v < 22; ++v) {
            const int lag = v < 8 ? LAGW0 + (v & 3) * LAGWS + ((v & 2) ? LAGWH : 0) : v < 15 ? LAGXA : LAGXB;
            const int d = key - lag;
            if (d >= 0 && d % 8 == 0 && d / 8 < NH && !(v < 8 && d / 8 < NW0)) t.e[n++] = (unsigned)((d / 8) << 8 | v);
        }
    return t;
}
__constant__ const OrderTab g_order = make_order();
__device__ __forceinline__ int order(int k, int src) {
    const int e = g_order.e[k], v = e & 255;
    const int odd = v >= 15, m = (src + 1 + v - (odd ? 15 : 8)) & 7;
    return (e & ~255) | (v < 8 ? 16 + v : 2 * m + odd);
}

constexpr int PK_LDS = 28 * 1024;
struct SendCtl { u32x4 desc[8]; unsigned freed[4]; unsigned tail; };
__device__ __forceinline__ void packer(const Params& p, LAS unsigned char* buf, LAS SendCtl* ctl, int xcd, int di, int mem, int lane, unsigned* cntp) {
    const unsigned dest = (unsigned)(p.rank + 1 + di) & 7u;
    const unsigned voff = lane * 16, lbuf = (unsigned)(size_t)buf;
    unsigned k = (RAW0 + mem) * 8 + xcd;
    int o, h = RAW0 + mem, u = 16 + dest * 8 + xcd;
    const unsigned call = __builtin_amdgcn_readfirstlane(ld4_sys(cntp)) + 1u;
    unsigned* tk = (unsigned*)(p.ws[p.rank] + OFF_TK) + di * 32;
    const unsigned tk0 = (call - 1u) * (NU - 8 * RAW0) - (RAW0 + NPK) * 8;
    unsigned nput = 0;
    for (;;) {
        WAIT_V(0);
        u32x4 r[16], pk[12];
        unit_read(buf, lane, r);
        unit_format(u, r);
        unsigned tl;
        const unsigned hdr = pack_unit(r, pk, tl, pack_base(r), buf, lane, nullptr) | (4u << 28);
        asm volatile("" : "+v"(pk[0]), "+v"(pk[1]), "+v"(pk[2]), "+v"(pk[3]), "+v"(pk[4]), "+v"(pk[5]), "+v"(pk[6]), "+v"(pk[7]), "+v"(pk[8]), "+v"(pk[9]), "+v"(pk[10]), "+v"(pk[11]));
        while ((unsigned)__builtin_amdgcn_readfirstlane(*(volatile LAS unsigned*)&ctl->freed[mem]) != nput) __builtin_amdgcn_s_sleep(1);
        ++nput;
        if (__builtin_expect(hdr_cnt(hdr) == RAWCNT, 0)) {
            unit_fetch(p, h, u, lane, lbuf);
            WAIT_V(0);
            unit_read(buf, lane, r);
            unit_format(u, r);
            st_raw(p.ws[dest] + OFF_LZ + (call & 1u) * LZ_PAR + (size_t)((6 - di) * NU + k) * UB, voff, r);
            WAIT_V(0);
        } else {
#pragma unroll
            for (int i = 0; i < 12; ++i) *(LAS u32x4*)(buf + 16384 + i * 1024 + lane * 16) = pk[i];
        }
        asm volatile("" ::: "memory");
        unsigned tn = 1u;
        if (lane == 0) {
            const unsigned pos = __hip_atomic_fetch_add(&ctl->tail, 1u, __ATOMIC_RELAXED, __HIP_MEMORY_SCOPE_WORKGROUP);
            *(volatile LAS u32x4*)&ctl->desc[pos & 7] = (u32x4){pos + 1u, k, hdr, (unsigned)mem | (nput << 8) | (tl << 24)};
        }
        if (SHALLOW) while ((unsigned)__builtin_amdgcn_readfirstlane(*(volatile LAS unsigned*)&ctl->freed[mem]) != nput) __builtin_amdgcn_s_sleep(1);
        if (lane == 0) asm volatile("global_atomic_add %0, %1, %0, off sc0\n\ts_waitcnt vmcnt(0)" : "+v"(tn) : "v"(tk) : "memory");
        k = __builtin_amdgcn_readfirstlane(tn) - tk0;
        if (k >= (unsigned)NU) break;
        o = order(k, p.rank); h = o >> 8; u = (o & 255) + ((o & 16) ? dest * 8 : 0);
        unit_fetch(p, h, u, lane, lbuf);
    }
    if (lane == 0) {
        const unsigned pos = __hip_atomic_fetch_add(&ctl->tail, 1u, __ATOMIC_RELAXED, __HIP_MEMORY_SCOPE_WORKGROUP);
        *(volatile LAS u32x4*)&ctl->desc[pos & 7] = (u32x4){pos + 1u, 0u, 0u, 0x80u};
    }
}
__device__ __forceinline__ void storer(const Params& p, LAS unsigned char* lds, LAS SendCtl* ctl, int xcd, int di, int lane, unsigned* cntp) {
    const unsigned dest = (unsigned)(p.rank + 1 + di) & 7u;
    char* dws = p.ws[dest];
    const int li = 6 - di;
    const unsigned voff = lane * 16;
    const unsigned call = __builtin_amdgcn_readfirstlane(ld4_sys(cntp)) + 1u;
    const char* lz = dws + OFF_LZ + (call & 1u) * LZ_PAR + (size_t)li * NU * UB;
    char* fa = dws + OFF_FA + (size_t)((call & 1u) * 7 + li) * NU * FLS;
    unsigned pos = 0, done = 0, pend_hdr = 0; int pend = -1;
#define POST(kk, hh) st8_flag(fa + (kk) * FLS, call, (hh), lane)
    if (RAW0) {
        u32x4 r[16];
        unit_read(lds + 4 * PK_LDS, lane, r);
        unit_format(16, r);
        STAMP(5);
        const char* s0 = lz + (size_t)xcd * UB + 4096; const char* s1 = s0 + 8192;
        st16<-4096>(s0, voff, r[0]); st16<-3072>(s0, voff, r[1]); st16<-2048>(s0, voff, r[2]); st16<-1024>(s0, voff, r[3]);
        st16<0>(s0, voff, r[4]); st16<1024>(s0, voff, r[5]); WAIT_V(SWIN); st16<2048>(s0, voff, r[6]); WAIT_V(SWIN); st16<3072>(s0, voff, r[7]); WAIT_V(SWIN);
        st16<-4096>(s1, voff, r[8]); WAIT_V(SWIN); st16<-3072>(s1, voff, r[9]); WAIT_V(SWIN); st16<-2048>(s1, voff, r[10]); WAIT_V(SWIN); st16<-1024>(s1, voff, r[11]); WAIT_V(SWIN);
        st16<0>(s1, voff, r[12]); WAIT_V(SWIN); st16<1024>(s1, voff, r[13]); WAIT_V(SWIN); st16<2048>(s1, voff, r[14]); WAIT_V(SWIN); st16<3072>(s1, voff, r[15]); WAIT_V(SWIN);
        pend = xcd; pend_hdr = (RAWCNT << 16) | (4u << 28);
    }
    while (done < NPK) {
        const u32x4 d = *(volatile LAS u32x4*)&ctl->desc[pos & 7];
        if ((unsigned)__builtin_amdgcn_readfirstlane(d.x) != pos + 1u) {
            if (pend >= 0) { WAIT_V(0); POST(pend, pend_hdr); pend = -1; }
            __builtin_amdgcn_s_sleep(1);
            continue;
        }
        const unsigned k = __builtin_amdgcn_readfirstlane(d.y), hdr = __builtin_amdgcn_readfirstlane(d.z), w = __builtin_amdgcn_readfirstlane(d.w);
        ++pos;
        if (w & 0x80u) { ++done; continue; }
        if (k == NU - 1 - GATE_EARLY && lane < 8)
            __hip_atomic_store((unsigned*)(p.ws[p.rank] + OFF_SD) + (di * 8 + lane) * 32, call, __ATOMIC_RELAXED, __HIP_MEMORY_SCOPE_AGENT);
        if (__builtin_expect(hdr_cnt(hdr) == RAWCNT, 0)) {
            POST(k, hdr);
            *(volatile LAS unsigned*)&ctl->freed[w & 3u] = (w >> 8) & 0xffffu;
            continue;
        }
        u32x4 pk[12];
        LAS unsigned char* src = lds + (w & 3u) * PK_LDS + 16384 + lane * 16;
#pragma unroll
        for (int i = 0; i < 12; ++i) pk[i] = *(LAS u32x4*)(src + i * 1024);
        asm volatile("s_waitcnt lgkmcnt(0)" : "+v"(pk[0]), "+v"(pk[1]), "+v"(pk[2]), "+v"(pk[3]), "+v"(pk[4]), "+v"(pk[5]), "+v"(pk[6]), "+v"(pk[7]), "+v"(pk[8]), "+v"(pk[9]), "+v"(pk[10]), "+v"(pk[11]) :: "memory");
        *(volatile LAS unsigned*)&ctl->freed[w & 3u] = (w >> 8) & 0xffffu;
        const char* s0 = lz + (size_t)k * UB + 4096; const char* s1 = s0 + 4096;
#define PEND_FLAG(i) do { if (SWIN == (i) && pend >= 0) POST(pend, pend_hdr); } while (0)
        st_tail(s1, voff, pk[10], pk[11], w >> 24, lane); WAIT_V(SWIN);
        st16<-4096>(s0, voff, pk[0]); WAIT_V(SWIN);
        st16<-3072>(s0, voff, pk[1]); WAIT_V(SWIN); PEND_FLAG(3);
        st16<-2048>(s0, voff, pk[2]); WAIT_V(SWIN); PEND_FLAG(4);
        st16<-1024>(s0, voff, pk[3]); WAIT_V(SWIN); PEND_FLAG(5);
        st16<0>(s0, voff, pk[4]); WAIT_V(SWIN); PEND_FLAG(6);
        st16<1024>(s0, voff, pk[5]); WAIT_V(SWIN); PEND_FLAG(7);
        st16<2048>(s0, voff, pk[6]); WAIT_V(SWIN); PEND_FLAG(8);
        st16<3072>(s0, voff, pk[7]); WAIT_V(SWIN); PEND_FLAG(9);
        st16<0>(s1, voff, pk[8]); WAIT_V(SWIN);
        st16<1024>(s1, voff, pk[9]); WAIT_V(SWIN);
        pend = k; pend_hdr = hdr;
    }
    WAIT_V(0);
    if (pend >= 0) st8_flag(fa + pend * FLS, call, pend_hdr, lane);
}

__device__ __forceinline__ void unpacker(const Params& p, LAS unsigned char* dl, int uw, int lane, unsigned call) {
    char* ws = p.ws[p.rank];
    unsigned* tmo = (unsigned*)(ws + OFF_TMO);
    const unsigned voff = lane * 16;
#pragma unroll
    for (int i = 0; i < 8; ++i) *(LAS u32x4*)(dl + i * 1024 + lane * 16) = (u32x4){0u, 0u, 0u, 0u};
    for (int n = uw; n < 7 * NU; n += NUNP) {
        const int k = n / 7, li = n - k * 7, src = (p.rank + 1 + li) & 7;
        const int o = order(k, src), t = o & 15; const bool isx = !(o & 16);
        const int q = NH + (o >> 8) * 7 + li;
        const unsigned hdr = wait_unit(ws + OFF_FA + (size_t)(((call & 1u) * 7 + li) * NU + k) * FLS, call, 4u, tmo);
        const unsigned cnt = hdr_cnt(hdr); const bool raw = cnt == RAWCNT;
        unsigned lo = 0;
        const char* sl = ws + OFF_LZ + (call & 1u) * LZ_PAR + (size_t)(li * NU + k) * UB;
        const char* dst = ws + (isx ? OFF_XI + (size_t)q * XQ : OFF_WI + (size_t)q * WQ) + (size_t)t * UB;
        unsigned* flag = (unsigned*)(ws + (isx ? OFF_FX : OFF_FW)) + (isx ? t : p.rank * 8 + t) * FSTR + q;
        if (__builtin_expect(raw, 0)) {
            for (int s = 0; s < 4; ++s) {
                u32x4 o[4]; ld4x16(o[0], o[1], o[2], o[3], sl + s * 4096, voff);
                const char* sb = dst + s * 4096;
                st16<0>(sb, voff, o[0]); st16<1024>(sb, voff, o[1]); st16<2048>(sb, voff, o[2]); st16<3072>(sb, voff, o[3]);
            }
        } else {
            u32x4 r[10], G; unsigned F;
            if (hdr & (1u << 26)) ld_tail2(sl, voff, (unsigned)(size_t)(dl + DEC_TAIL));
            ld_packed(r, F, sl, voff, (unsigned)(size_t)(dl + DEC_TAIL));
            const unsigned LL = (hdr & 0x7fffu) * 0x10001u;
            lo = unpack_head(F, hdr, dl, lane, G);
#pragma unroll
            for (int s = 0; s < 4; ++s) {
                u32x4 o[4]; unpack_sub(r, G, s, dl, lane, LL, o);
                const char* sb = dst + s * 4096;
                st16<0>(sb, voff, o[0]); st16<1024>(sb, voff, o[1]); st16<2048>(sb, voff, o[2]); st16<3072>(sb, voff, o[3]);
            }
        }
        WAIT_V(0);
        if (lane == 0) __hip_atomic_store(flag, call, __ATOMIC_RELAXED, __HIP_MEMORY_SCOPE_SYSTEM);
        if (!raw) { patch_apply(dl, lo, hdr, lane, true); WAIT_L(0); }
    }
}

__device__ __forceinline__ void formatter(const Params& p, LAS unsigned char* buf, int fw, int n2nd, int lane, unsigned call) {
    char* ws = p.ws[p.rank];
    const unsigned voff = lane * 16;
    const unsigned long long ts = __builtin_amdgcn_s_memrealtime();
    for (int n = fw < 384 ? fw : n2nd; n < 1280; n = (n < 384) ? n2nd : n + NFMT) {
        const unsigned at = n < 384 ? PD0 + n * PDT / 16 : FD0 + (n - 384) * FDT / 16;
        while (__builtin_amdgcn_s_memrealtime() - ts < at) __builtin_amdgcn_s_sleep(4);
        int h, u; const char* dst; unsigned* flag;
        if (n < 384) { h = n / 24; const int v = n - h * 24;
            if (v < 16) { u = v; dst = ws + OFF_XI + (size_t)h * XQ + (size_t)v * UB; flag = (unsigned*)(ws + OFF_FX) + v * FSTR + h; }
            else { u = p.rank * 8 + v; dst = ws + OFF_WI + (size_t)h * WQ + (size_t)(v - 16) * UB; flag = (unsigned*)(ws + OFF_FW) + (u - 16) * FSTR + h; } }
        else { const int n2 = n - 384; h = n2 / 56; const int v = n2 - h * 56, li = v >> 3, cn = ((p.rank + 1 + li) & 7) * 8 + (v & 7);
            u = 16 + cn; dst = ws + OFF_WD + (size_t)(li * NH + h) * WQ + (size_t)(v & 7) * UB; flag = (unsigned*)(ws + OFF_FW) + cn * FSTR + h; }
        u32x4 r[16];
        unit_fetch(p, h, u, lane, (unsigned)(size_t)buf);
        WAIT_V(0);
        unit_read(buf, lane, r);
        unit_format(u, r);
        st_raw(dst, voff, r);
        WAIT_V(0);
        if (lane == 0) __hip_atomic_store(flag, call, __ATOMIC_RELAXED, __HIP_MEMORY_SCOPE_SYSTEM);
    }
}

__device__ __forceinline__ void poll_ready(const unsigned* fx, const unsigned* fw, unsigned long long& td0, unsigned long long& td1,
                                           LAS unsigned* queue, int& nq, int need, unsigned call, unsigned* tmo) {
    const unsigned lane = __builtin_amdgcn_mbcnt_hi(~0u, __builtin_amdgcn_mbcnt_lo(~0u, 0u));
    const unsigned long long t0 = __builtin_amdgcn_s_memrealtime(); unsigned polls = 0;
    bool all = false;
    for (;;) {
        unsigned x0, x1, w0, w1;
        asm volatile("global_load_dword %0, %4, %5 sc0 sc1\n\tglobal_load_dword %1, %4, %5 offset:256 sc0 sc1\n\t"
                     "global_load_dword %2, %4, %6 sc0 sc1\n\tglobal_load_dword %3, %4, %6 offset:256 sc0 sc1\n\ts_waitcnt vmcnt(0)"
                     : "=&v"(x0), "=&v"(x1), "=&v"(w0), "=&v"(w1) : "v"(lane * 4), "s"(fx), "s"(fw) : "memory");
        const unsigned long long a0 = __ballot(all || (x0 >= call && w0 >= call)) & td0, a1 = __ballot(all || (x1 >= call && w1 >= call)) & td1;
        if ((a0 >> lane) & 1) ((volatile LAS unsigned*)queue)[__builtin_amdgcn_mbcnt_hi((unsigned)(a0 >> 32), __builtin_amdgcn_mbcnt_lo((unsigned)a0, nq))] = lane | 0x10000u;
        nq += __builtin_popcountll(a0);
        if ((a1 >> lane) & 1) ((volatile LAS unsigned*)queue)[__builtin_amdgcn_mbcnt_hi((unsigned)(a1 >> 32), __builtin_amdgcn_mbcnt_lo((unsigned)a1, nq))] = lane | 0x10040u;
        nq += __builtin_popcountll(a1);
        td0 &= ~a0; td1 &= ~a1;
        if (nq >= need) break;
        all = give_up(tmo, call, t0, polls);
        __builtin_amdgcn_s_sleep(1);
    }
    WAIT_L(0);
}

constexpr int HTB = 16384;

__global__ __launch_bounds__(512, 2) void fused_kernel(const Params p) {
    __shared__ __attribute__((aligned(16))) unsigned char shm[8 * HTB + 8192 + 256];
    LAS unsigned char* lds = (LAS unsigned char*)shm;
    const int wid = __builtin_amdgcn_readfirstlane(threadIdx.x >> 6), lane = threadIdx.x & 63;
    const int xcd = blockIdx.x & 7, slot = blockIdx.x >> 3;
    char* ws = p.ws[p.rank];
    unsigned* tmo = (unsigned*)(ws + OFF_TMO);
    unsigned* cntp = (unsigned*)(ws + OFF_CNT) + blockIdx.x * 8 + wid;
    STAMP(0);
    const bool main_tile = slot >= 16;
    if (slot < 7) {
        LAS SendCtl* ctl = (LAS SendCtl*)(lds + 8 * HTB + 8192);
        static_assert(RAW0 == 1 && NPK == 4 && NW0 >= RAW0 + NPK, "opening units");
        if (wid == 7) ((volatile LAS unsigned*)ctl)[lane] = 0u;
        WAIT_L(0); BAR;
        if (wid == 0) __builtin_amdgcn_s_setprio(3); else __builtin_amdgcn_s_sleep(PKDELAY);
        if (wid <= 4) unit_fetch(p, wid, 16 + ((p.rank + 1 + slot) & 7) * 8 + xcd, lane, (unsigned)(size_t)(lds + ((wid + 4) % 5) * PK_LDS));
        if (wid == 0) { storer(p, lds, ctl, xcd, slot, lane, cntp); __builtin_amdgcn_s_setprio(0); }
        else if (wid <= NPK) packer(p, lds + (wid - 1) * PK_LDS, ctl, xcd, slot, wid - 1, lane, cntp);
    }
    const unsigned call = __builtin_amdgcn_readfirstlane(ld4_sys(cntp)) + 1u;
    if (lane == 0) __hip_atomic_store(cntp, call, __ATOMIC_RELAXED, __HIP_MEMORY_SCOPE_AGENT);

    if (!main_tile) {
        if (slot >= 7) {
            const bool unp = slot < 7 + NUS;
            formatter(p, lds + wid * HTB, wid * 72 + (blockIdx.x - 56), unp ? 1 << 20 : 384 + wid * (NFMT / 8) + (blockIdx.x - 56 - NUS * 8), lane, call);
            const int uw = wid * (NUS * 8) + (blockIdx.x - 56);
            if (unp && uw < NUNP) unpacker(p, lds + wid * HTB, uw, lane, call);
        }
        STAMP(1);
        asm volatile("s_waitcnt vmcnt(0) lgkmcnt(0)" ::: "memory");
        BAR;
    }

    int rt, cn, q0, nt, skip;
    const char* wimg;
    if (main_tile) {
        const int idx = slot - 16;
        rt = ((xcd >> 1) * 2 + ((idx >> 2) & 1)) * 2 + (idx >> 3);
        cn = p.rank * 8 + (xcd & 1) * 4 + (idx & 3);
        const int m = rt >> 1;
        q0 = (m == p.rank) ? NH : 0; skip = (m - p.rank - 1) & 7; nt = 56;
        wimg = ws + OFF_WI;
    } else {
        rt = p.rank * 2 + (slot < 8 ? 1 : 0);
        cn = xcd * 8 + (slot & 7);
        q0 = 0; skip = 7; nt = 8;
        wimg = (xcd == p.rank) ? ws + OFF_WI : ws + OFF_WD + (size_t)((xcd - p.rank - 1) & 7) * NH * WQ;
    }
    const int wr = wid >> 2, wc = wid & 3;
    const unsigned* gate = (skip == 7) ? cntp : (const unsigned*)(ws + OFF_SD) + (skip * 8 + xcd) * 32;
    const unsigned* fx = (const unsigned*)(ws + OFF_FX) + rt * FSTR;
    const unsigned* fw = (const unsigned*)(ws + OFF_FW) + cn * FSTR;
    const unsigned* gate2 = (skip == 7) ? cntp : fw + NH + skip;
    const char* xb = ws + OFF_XI + (size_t)(rt * 16 + wid) * 1024;
    const char* wb = wimg + (size_t)((cn & 7) * 16 + wr * 8 + wc) * 1024;
    const unsigned voff = lane * 16;
    const unsigned ldsw = wid * 1024;
    const int aoff = wr * 4096 + lane * 16, boff = wc * 2048 + lane * 16;
#define SA(b, h) (((b) * 4 + (h)) * HTB)
#define SB(b, h) (((b) * 4 + 2 + (h)) * HTB)
#define GLDS(g, o, l) __builtin_amdgcn_global_load_lds((const unsigned*)((g) + (size_t)(unsigned)(o)), (LAS unsigned*)(lds + (l)), 16, 0, GLDS_AUX)
#define STAGE_A(b, h, qq) do { GLDS(wb + (h) * 4096, (qq.x << 17) + voff, SA(b, h) + ldsw); GLDS(wb + (h) * 4096, (qq.y << 17) + voff, SA(b, h) + ldsw + 8192); } while (0)
#define STAGE_B(b, h, qq) do { GLDS(xb + (h) * 8192, (qq.x << 18) + voff, SB(b, h) + ldsw); GLDS(xb + (h) * 8192, (qq.y << 18) + voff, SB(b, h) + ldsw + 8192); } while (0)
#define LDA(dst, b, h) do { _Pragma("unroll") for (int m = 0; m < 4; ++m) _Pragma("unroll") for (int k = 0; k < 2; ++k) dst[m][k] = *(const LAS bf16x8*)(lds + SA(b, h) + aoff + m * 1024 + k * 8192); } while (0)
#define LDB(dst, b, h) do { _Pragma("unroll") for (int n = 0; n < 2; ++n) _Pragma("unroll") for (int k = 0; k < 2; ++k) dst[n][k] = *(const LAS bf16x8*)(lds + SB(b, h) + boff + n * 1024 + k * 8192); } while (0)
#define MMA2(ai) do { __builtin_amdgcn_s_setprio(1); _Pragma("unroll") for (int k = 0; k < 2; ++k) _Pragma("unroll") for (int m = 0; m < 4; ++m) _Pragma("unroll") for (int n = 0; n < 2; ++n) { \
        acc[ai][0][m][n] = __builtin_amdgcn_mfma_f32_16x16x32_bf16(B0[n][k], At[m][k], acc[ai][0][m][n], 0, 0, 0); \
        acc[ai][1][m][n] = __builtin_amdgcn_mfma_f32_16x16x32_bf16(B1[n][k], At[m][k], acc[ai][1][m][n], 0, 0, 0); } __builtin_amdgcn_s_setprio(0); } while (0)
#define MUST_PICK(kt) do { if (__builtin_expect(nq < 2 * (kt) + 2, 0)) poll_ready(fx, fw, td0, td1, queue, nq, 2 * (kt) + 2, call, tmo); } while (0)
#define GETQ(kt) (*(volatile LAS u32x2*)(queue + 2 * (kt)))
#define FIXQ(qq, kt) do { if (__builtin_expect(__builtin_amdgcn_readfirstlane(qq.x & qq.y) < 0x10000, 0)) qq = GETQ(kt); } while (0)
    f32x4 acc[2][2][4][2];
#pragma unroll
    for (int a = 0; a < 2; ++a)
#pragma unroll
        for (int b = 0; b < 2; ++b)
#pragma unroll
            for (int m = 0; m < 4; ++m)
#pragma unroll
                for (int n = 0; n < 2; ++n) acc[a][b][m][n] = (f32x4){0.f, 0.f, 0.f, 0.f};
    bf16x8 At[4][2], B0[2][2], B1[2][2];
    LAS unsigned* queue = (LAS unsigned*)(lds + 8 * HTB + 4096);
    unsigned long long td0 = __ballot(lane >= q0 && lane < 2 * nt + 64 * main_tile && !(lane >= NH && (lane - NH) % 7 == skip));
    unsigned long long td1 = __ballot(main_tile && (lane + 64 - NH) % 7 != skip);
    int nq = 1 << 20;
    if (wid == 0) { nq = 0; *(volatile LAS u32x2*)(queue + 2 * lane) = (u32x2){0u, 0u}; poll_ready(fx, fw, td0, td1, queue, nq, 4, call, tmo); }
    asm volatile("" ::: "memory"); BAR; asm volatile("" ::: "memory");
    u32x2 qe = GETQ(0), qo = GETQ(1);
    STAMP(4);
    STAGE_B(0, 0, qe); STAGE_A(0, 0, qe); STAGE_B(0, 1, qe); STAGE_A(0, 1, qe);
    STAGE_B(1, 0, qo); STAGE_A(1, 0, qo); STAGE_B(1, 1, qo);
    if (wr == 1) BAR;
    WAIT_V(8); BAR; BAR;
    for (int t = 0; t < nt; t += 2) {
        if (t == nt - 2) nq = 1 << 20;
        qe = GETQ(t + 2);
        LDB(B0, 0, 0); LDA(At, 0, 0); LDB(B1, 0, 1); STAGE_A(1, 1, qo);
        WAIT_V(8); WAIT_L(0); BAR; SCHED; MMA2(0); SCHED; MUST_PICK(t + 2); BAR; SCHED;
        FIXQ(qe, t + 2);
        LDA(At, 0, 1); STAGE_B(0, 0, qe); STAGE_A(0, 0, qe); STAGE_B(0, 1, qe);
        WAIT_V(8); WAIT_L(0); BAR; SCHED; MMA2(1); BAR; SCHED;
        if (t == nt - 2)
            __builtin_amdgcn_global_load_lds(lane == 1 ? tmo : lane == 2 ? gate2 : gate, (LAS unsigned*)(lds + 8 * HTB + wid * 256), 4, 0, 17);
        qo = GETQ(t + 3);
        LDB(B0, 1, 0); LDA(At, 1, 0); LDB(B1, 1, 1); STAGE_A(0, 1, qe);
        WAIT_V(8); WAIT_L(0); BAR; SCHED; MMA2(0); SCHED; MUST_PICK(t + 3); BAR; SCHED;
        FIXQ(qo, t + 3);
        LDA(At, 1, 1); STAGE_B(1, 0, qo); STAGE_A(1, 0, qo); STAGE_B(1, 1, qo);
        WAIT_V(8); WAIT_L(0); BAR; SCHED; MMA2(1); BAR; SCHED;
    }
    STAMP(2);
    int tz = threadIdx.x; asm volatile("" : "+v"(tz));
    const int ln = tz & 63;
    const unsigned vo = ln * 16;
    const unsigned tile = (unsigned)((rt & 1) * 64 + cn) * 8 + wid;
    if (main_tile) {
        const u32x4 gv = *(volatile LAS u32x4*)(lds + 8 * HTB + wid * 256);
        bool bad = (unsigned)__builtin_amdgcn_readfirstlane(gv.y) == call;
        const bool open = (unsigned)__builtin_amdgcn_readfirstlane(gv.x) >= call && (unsigned)__builtin_amdgcn_readfirstlane(gv.z) >= call && !bad;
        const unsigned owner = (unsigned)(rt >> 1) & 7u;
        char* ows = p.ws[owner];
        const char* sl = ows + OFF_PL + (size_t)tile * UB;
        char* fl = ows + OFF_FP + (size_t)tile * FLS;
        const char* s0 = sl + 4096; const char* s1 = sl + 8192;
        const int staged = (rt & 1) ? BSTAGE : ASTAGE;
        u32x4 b[16], pk[12];
#pragma unroll
        for (int bj = 0; bj < 2; ++bj)
#pragma unroll
            for (int n = 0; n < 2; ++n)
#pragma unroll
                for (int r = 0; r < 4; ++r)
#pragma unroll
                    for (int ai = 0; ai < 2; ++ai)
#pragma unroll
                        for (int m2 = 0; m2 < 2; ++m2)
                            b[bj * 8 + n * 4 + r][ai * 2 + m2] = cvt_pk_bf16(acc[ai][bj][2 * m2][n][r], acc[ai][bj][2 * m2 + 1][n][r]);
        asm volatile("" : "+v"(b[0]), "+v"(b[1]), "+v"(b[2]), "+v"(b[3]), "+v"(b[4]), "+v"(b[5]), "+v"(b[6]), "+v"(b[7]), "+v"(b[8]), "+v"(b[9]), "+v"(b[10]), "+v"(b[11]), "+v"(b[12]), "+v"(b[13]), "+v"(b[14]), "+v"(b[15]) :: "memory");
        if (wr == 0) BAR;
        const unsigned L = pack_base(b);
        if (open) { st16<-4096>(s0, vo, pack_low(b, 0, L)); WAIT_V(1); } else WAIT_V(0);
        asm volatile("" ::: "memory"); BAR; asm volatile("" ::: "memory");
        unsigned tl;
        unsigned hdr = pack_unit(b, pk, tl, L, lds + wid * HTB, ln, open ? s0 : nullptr);
        if (!open) {
            const unsigned long long t0 = __builtin_amdgcn_s_memrealtime(); unsigned polls = 0;
            while (!bad && !__all(ld4_sys(ln & 1 ? gate2 : gate) >= call)) {
                __builtin_amdgcn_s_sleep(2);
                bad = give_up(tmo, call, t0, polls);
            }
        }
        STAMP(6);
        if (bad) {
            hdr = RAWCNT << 16;
#pragma unroll
            for (int f = 0; f < 16; ++f) b[f] = (u32x4){0x7fc07fc0u, 0x7fc07fc0u, 0x7fc07fc0u, 0x7fc07fc0u};
        }
        if (__builtin_expect(hdr_cnt(hdr) == RAWCNT, 0)) { WAIT_V(0); st_raw(sl, vo, b); }
        else {
            if (!open) { st16<-4096>(s0, vo, pk[0]); WAIT_V(0); st16<-3072>(s0, vo, pk[1]); }
            if (!open || !EARLY3) { WAIT_V(0); st16<0>(s1, vo, pk[8]); }
            if (!open || EARLY3 < 2) { WAIT_V(0); st16<1024>(s1, vo, pk[9]); }
            WAIT_V(0); st_tail(s1, vo, pk[10], pk[11], tl, ln);
            WAIT_V(0); STAMP(7); if (staged & 1) st8_flag(fl, call, hdr | (1u << 28), ln); st16<-2048>(s0, vo, pk[2]);
            WAIT_V(0); st16<-1024>(s0, vo, pk[3]);
            WAIT_V(0); if (staged & 2) st8_flag(fl, call, hdr | (2u << 28), ln); st16<0>(s0, vo, pk[4]);
            WAIT_V(0); st16<1024>(s0, vo, pk[5]);
            if (ENDW) { WAIT_V(0); if (staged & 4) st8_flag(fl, call, hdr | (3u << 28), ln); st16<2048>(s0, vo, pk[6]); st16<3072>(s0, vo, pk[7]); }
            else {
            WAIT_V(0); if (staged & 4) st8_flag(fl, call, hdr | (3u << 28), ln); st16<2048>(s0, vo, pk[6]);
            WAIT_V(0); st16<3072>(s0, vo, pk[7]); }
        }
        WAIT_V(0);
        st8_flag(fl, call, hdr | (4u << 28), ln);
    } else {
        WAIT_V(0);
        if (wr == 0) BAR;
        BAR;
        LAS unsigned char* dl = lds + wid * HTB;
#pragma unroll
        for (int i = 0; i < 8; ++i) *(LAS u32x4*)(dl + i * 1024 + ln * 16) = (u32x4){0u, 0u, 0u, 0u};
        const char* fl = ws + OFF_FP + (size_t)tile * FLS;
        const char* sl = ws + OFF_PL + (size_t)tile * UB;
        unsigned hdr = wait_unit(fl, call, 1u, tmo);
        const unsigned cnt = hdr_cnt(hdr); const bool raw = cnt == RAWCNT;
        const unsigned LL = (hdr & 0x7fffu) * 0x10001u;
        u32x4 r[10], G;
        unsigned tv = 0;
        if (!raw) {
            unsigned F;
            if (hdr & (1u << 26)) ld_tail2(sl, vo, (unsigned)(size_t)(dl + DEC_TAIL));
            ld_head(r, F, tv, sl, vo, (unsigned)(size_t)(dl + DEC_TAIL), tmo);
            unpack_head(F, hdr, dl, ln, G);
        }
        const char* ob = p.out + ((size_t)((rt & 1) * 256 + wc * 32) * NN + cn * 256 + wr * 128) * 2;
        const unsigned oo = (ln >> 4) * (4 * NN * 2) + (ln & 15) * 16;
#pragma unroll
        for (int s = 0; s < 4; ++s) {
            if ((hdr >> 28) <= (unsigned)s) hdr = wait_unit(fl, call, s + 1, tmo);
            u32x4 o[4];
            if (raw) { ld4x16(o[0], o[1], o[2], o[3], sl + s * 4096, vo); tv = ld4_sys(tmo); }
            else { if (s) ld2x16t(r[2 * s], r[2 * s + 1], tv, sl + s * 2048, vo, tmo); unpack_sub(r, G, s, dl, ln, LL, o); }
            const unsigned nanmask = (tv == call) ? 0x7fc07fc0u : 0u;
#pragma unroll
            for (int rr = 0; rr < 4; ++rr) {
                u32x4 y;
#pragma unroll
                for (int cc = 0; cc < 4; ++cc) {
                    const unsigned pv = o[rr][cc];
                    const f32x2 v = (f32x2){acc[cc >> 1][s >> 1][2 * (cc & 1)][s & 1][rr] + __uint_as_float(pv << 16),
                                            acc[cc >> 1][s >> 1][2 * (cc & 1) + 1][s & 1][rr] + __uint_as_float(pv & 0xffff0000u)};
                    const f32x2 g = gelu2(v);
                    y[cc] = cvt_pk_bf16(g.x, g.y) | nanmask;
                }
                st16<0>(ob + (size_t)((s >> 1) * 128 + (s & 1) * 16 + rr) * (NN * 2), oo, y);
            }
        }
    }
    STAMP(3);
}
}

extern "C" void kernel_launch(int rank, int nranks, ncclComm_t comm, hipStream_t stream,
                              void* const* d_in, const int* in_sizes, int n_in,
                              void* d_out, int out_size, const int* in_dtypes, int out_dtype,
                              void* const* d_out_all, void* const* d_ws_all, size_t ws_bytes) {
    (void)comm; (void)in_dtypes; (void)out_dtype; (void)d_out_all;
    if (nranks != NR || (unsigned)rank >= (unsigned)NR || n_in < 2 || in_sizes[0] != MM * KS || in_sizes[1] != KS * NN ||
        out_size != (MM / NR) * NN || ws_bytes < WS_NEED) {
        fprintf(stderr, "kernel_launch: unexpected problem geometry\n");
        return;
    }
    Params p;
    p.x = (const char*)d_in[0];
    p.w = (const char*)d_in[1];
    for (int i = 0; i < NR; ++i) p.ws[i] = (char*)d_ws_all[i];
    p.out = (char*)d_out;
    p.rank = rank; p.pad = 0;
    hipLaunchKernelGGL(fused_kernel, dim3(256), dim3(512), 0, stream, p);
    HIP_CHECK(hipGetLastError());
}
```

```cpp
#include <hip/hip_runtime.h>
#include <rccl/rccl.h>
#include <cstdio>
#include <cstdint>

#define HIP_CHECK(expr) do { hipError_t _e = (expr); if (_e != hipSuccess) { \
    fprintf(stderr, "HIP error %s at %s:%d\n", hipGetErrorString(_e), __FILE__, __LINE__); \
    return; } } while (0)

#define NW0 (NPK + 1)
#define LAGXB 36
#define LAGW0 0
#define LAGWS 0
#define LAGWH 4
#define LAGXA 1
#define GLDS_AUX 17
#define FORCE_RAW 0
#define NUS 5
#define SWIN 6
#define SHALLOW 1
#define PKDELAY 4
#define DIAGD 750
#define DIAGD2 DIAGD
#define RAW0 1
#define PD0 100
#define PDT 24
#define FD0 1000
#define FDT 72
#define EARLYJ 19
#define BSTAGE 7
#define ASTAGE 0
#define FLS 128
#define NPK 4
#define GATE_EARLY 12

namespace {
#define LAS __attribute__((address_space(3)))
typedef short bf16x8 __attribute__((ext_vector_type(8)));
typedef float f32x4 __attribute__((ext_vector_type(4)));
typedef float f32x2 __attribute__((ext_vector_type(2)));
typedef unsigned u32x4 __attribute__((ext_vector_type(4)));
typedef unsigned u32x2 __attribute__((ext_vector_type(2)));
typedef unsigned short u16x2 __attribute__((ext_vector_type(2)));

constexpr int NR = 8;
constexpr int MM = 4096, NN = 16384, KS = 512, NS = 2048;
constexpr int NQ = 128;
constexpr int NH = 16;
constexpr int NU = NH * 22;
#define UNP7 1
constexpr int NUNP = UNP7 ? NUS * 64 / 7 * 7 : NUS * 64;
constexpr int NFMT = (9 - NUS) * 64;
constexpr unsigned long long DEADLINE = 20000000ull;
constexpr unsigned RAWCNT = 0x3ffu;

constexpr size_t OFF_CNT = 0;
constexpr size_t OFF_TMO = 8192;
constexpr size_t OFF_TK = 8192 + 1024;
constexpr size_t OFF_SD = 131072;
constexpr int FSTR = 160;
constexpr size_t OFF_FX = 16384;
constexpr size_t OFF_FW = 32768;
constexpr size_t OFF_STAMP = 262144;
constexpr size_t UB = 16384;
constexpr size_t XQ = 256 * 1024, WQ = 128 * 1024;
constexpr size_t OFF_XI = 1u << 20;
constexpr size_t OFF_WI = OFF_XI + (size_t)NQ * XQ;
constexpr size_t OFF_WD = OFF_WI + (size_t)NQ * WQ;
constexpr size_t OFF_PL = OFF_WD + (size_t)7 * NH * WQ;
constexpr size_t OFF_LZ = OFF_PL + (size_t)1024 * UB;
constexpr size_t LZ_PAR = (size_t)7 * NU * UB;
constexpr size_t OFF_FA = OFF_LZ + 2 * LZ_PAR;
constexpr size_t OFF_FP = OFF_FA + (size_t)14 * NU * 128;
constexpr size_t WS_NEED = OFF_FP + 1024 * 128;
static_assert(OFF_FW + 64 * FSTR * 4 <= OFF_SD && OFF_SD + 56 * 128 <= OFF_STAMP, "layout");
static_assert(WS_NEED <= (size_t)161 << 20, "workspace");

struct Params {
    const char* x;
    const char* w;
    char* ws[NR];
    char* out;
    int rank;
    int pad;
};

template <int OFF> __device__ __forceinline__ void st16(const char* sbase, unsigned voff, u32x4 v) {
    asm volatile("global_store_dwordx4 %0, %1, %2 offset:%3 sc0 sc1\n\ts_nop 1" :: "v"(voff), "v"(v), "s"(sbase), "i"(OFF) : "memory");
}
__device__ __forceinline__ unsigned ld4_sys(const void* p) {
    unsigned v;
    asm volatile("global_load_dword %0, %1, off sc0 sc1\n\ts_waitcnt vmcnt(0)" : "=&v"(v) : "v"(p) : "memory");
    return v;
}
__device__ __forceinline__ u32x2 ld8_sys(const void* p) {
    u32x2 v;
    asm volatile("global_load_dwordx2 %0, %1, off sc0 sc1\n\ts_waitcnt vmcnt(0)" : "=&v"(v) : "v"(p) : "memory");
    return v;
}
__device__ __forceinline__ void st8_flag(void* p, unsigned call, unsigned hdr, int lane) {
    if (lane == 0) __hip_atomic_store((unsigned long long*)p, ((unsigned long long)hdr << 32) | call, __ATOMIC_RELAXED, __HIP_MEMORY_SCOPE_SYSTEM);
}
#define WAIT_V_(n) asm volatile("s_waitcnt vmcnt(" #n ")" ::: "memory")
#define WAIT_V(n) WAIT_V_(n)
#define WAIT_L(n) asm volatile("s_waitcnt lgkmcnt(" #n ")" ::: "memory")
#define BAR __builtin_amdgcn_s_barrier()
#define SCHED __builtin_amdgcn_sched_barrier(0)

__device__ __forceinline__ unsigned cvt_pk_bf16(float lo, float hi) {
    unsigned r; asm("v_cvt_pk_bf16_f32 %0, %1, %2" : "=v"(r) : "v"(lo), "v"(hi)); return r;
}
__device__ __forceinline__ f32x2 gelu2(f32x2 y) {
    const float k1 = -2.0f * 0.7978845608028654f * 1.4426950408889634f, k2 = k1 * 0.044715f;
    f32x2 z = y * (y * y * k2 + k1);
    f32x2 e; e.x = __builtin_amdgcn_exp2f(z.x); e.y = __builtin_amdgcn_exp2f(z.y);
    e = e + 1.0f;
    f32x2 r; r.x = __builtin_amdgcn_rcpf(e.x); r.y = __builtin_amdgcn_rcpf(e.y);
    return y * r;
}
__device__ __forceinline__ unsigned pk_sub(unsigned a, unsigned b) { return __builtin_bit_cast(unsigned, __builtin_bit_cast(u16x2, a) - __builtin_bit_cast(u16x2, b)); }
__device__ __forceinline__ unsigned pk_add(unsigned a, unsigned b) { return __builtin_bit_cast(unsigned, __builtin_bit_cast(u16x2, a) + __builtin_bit_cast(u16x2, b)); }
__device__ __forceinline__ unsigned pk_max(unsigned a, unsigned b) { return __builtin_bit_cast(unsigned, __builtin_elementwise_max(__builtin_bit_cast(u16x2, a), __builtin_bit_cast(u16x2, b))); }
__device__ __forceinline__ unsigned bfi(unsigned m, unsigned a, unsigned b) { return __builtin_amdgcn_bitop3_b32(m, a, b, 0xca); }
__device__ __forceinline__ unsigned rotl(unsigned a, int n) { return __builtin_amdgcn_alignbit(a, a, 32 - n); }
__device__ __forceinline__ unsigned rotr(unsigned a, int n) { return __builtin_amdgcn_alignbit(a, a, n); }
__device__ __forceinline__ unsigned umax(unsigned a, unsigned b) { return a > b ? a : b; }

#define STAMP(slot) do {} while (0)

__device__ __forceinline__ bool give_up(unsigned* tmo, unsigned call, unsigned long long t0, unsigned& polls) {
    if ((++polls & 15u) != 0u) return false;
    if (__builtin_amdgcn_readfirstlane(__hip_atomic_load(tmo, __ATOMIC_RELAXED, __HIP_MEMORY_SCOPE_AGENT)) == call ||
        __builtin_amdgcn_s_memrealtime() - t0 > DEADLINE) {
        __hip_atomic_store(tmo, call, __ATOMIC_RELAXED, __HIP_MEMORY_SCOPE_AGENT);
        return true;
    }
    return false;
}
__device__ __forceinline__ unsigned wait_unit(const char* flag, unsigned call, unsigned st, unsigned* tmo) {
    const unsigned long long t0 = __builtin_amdgcn_s_memrealtime(); unsigned polls = 0;
    for (;;) {
        const u32x2 v = ld8_sys(flag);
        const unsigned hdr = __builtin_amdgcn_readfirstlane(v.y);
        if ((unsigned)__builtin_amdgcn_readfirstlane(v.x) == call && (hdr >> 28) >= st) return hdr;
        __builtin_amdgcn_s_sleep(1);
        if (give_up(tmo, call, t0, polls)) return (RAWCNT << 16) | (4u << 28);
    }
}
__device__ __forceinline__ unsigned hdr_cnt(unsigned hdr) { return (hdr >> 16) & 0x3ffu; }

constexpr int SCR_DENSE = 8192;
#define PT(i) do {} while (0)
__device__ __forceinline__ unsigned umin(unsigned a, unsigned b) { return a < b ? a : b; }
__device__ __forceinline__ unsigned wave_scan(unsigned inc) {
    inc += __builtin_amdgcn_update_dpp(0, (int)inc, 0x111, 0xf, 0xf, true);
    inc += __builtin_amdgcn_update_dpp(0, (int)inc, 0x112, 0xf, 0xf, true);
    inc += __builtin_amdgcn_update_dpp(0, (int)inc, 0x114, 0xf, 0xf, true);
    inc += __builtin_amdgcn_update_dpp(0, (int)inc, 0x118, 0xf, 0xf, true);
    inc += __builtin_amdgcn_update_dpp(0, (int)inc, 0x142, 0xa, 0xf, false);
    inc += __builtin_amdgcn_update_dpp(0, (int)inc, 0x143, 0xc, 0xf, false);
    return inc;
}
__device__ __forceinline__ unsigned pack_base(const u32x4 (&b)[16]) {
    unsigned mx = 0;
#pragma unroll
    for (int f = 0; f < 16; ++f) mx = pk_max(mx, b[f][f & 3] & 0x7fff7fffu);
    unsigned m = umax(mx & 0xffffu, mx >> 16);
    m = umax(m, (unsigned)__builtin_amdgcn_update_dpp(0, (int)m, 0x128, 0xf, 0xf, false));
    m = umax(m, (unsigned)__builtin_amdgcn_update_dpp(0, (int)m, 0x124, 0xf, 0xf, false));
    m = umax(m, (unsigned)__builtin_amdgcn_update_dpp(0, (int)m, 0x122, 0xf, 0xf, false));
    m = umax(m, (unsigned)__builtin_amdgcn_update_dpp(0, (int)m, 0x121, 0xf, 0xf, false));
    const unsigned m0 = __builtin_amdgcn_readlane(m, 0), m1 = __builtin_amdgcn_readlane(m, 16), m2 = __builtin_amdgcn_readlane(m, 32), m3 = __builtin_amdgcn_readlane(m, 48);
    const unsigned ms = umin(umax(umin(m0, m1), umin(m2, m3)), umin(umax(m0, m1), umax(m2, m3)));
    return umax(ms, 1023u) - 1023u;
}
__device__ __forceinline__ u32x4 pack_low(const u32x4 (&b)[16], int i, unsigned L) {
    const unsigned LL = L | (L << 16);
    u32x4 r;
#pragma unroll
    for (int c = 0; c < 4; ++c) r[c] = __builtin_amdgcn_perm(pk_sub(b[2 * i + (c >> 1)][2 * (c & 1) + 1], LL), pk_sub(b[2 * i + (c >> 1)][2 * (c & 1)], LL), 0x06040200u);
    return r;
}
__device__ __forceinline__ unsigned pack_unit(const u32x4 (&b)[16], u32x4 (&pk)[12], unsigned& tl, unsigned L, LAS unsigned char* scr, int lane, const char* early) {
    const unsigned LL = L | (L << 16);
    const unsigned wp0 = (unsigned)(size_t)scr + lane * 4;
    unsigned wp = wp0;
    unsigned z[4], seg[4], len[4], F = 0;
    PT(0);
#pragma unroll
    for (int s = 0; s < 4; ++s) {
        unsigned hb[8];
#pragma unroll
        for (int jj = 0; jj < 8; ++jj) {
            const int j = 8 * s + jj;
            const unsigned d0 = pk_sub(b[j >> 1][2 * (j & 1)], LL), d1 = pk_sub(b[j >> 1][2 * (j & 1) + 1], LL);
            pk[j >> 2][j & 3] = __builtin_amdgcn_perm(d1, d0, 0x06040200u);
            hb[jj] = __builtin_amdgcn_perm(d1, d0, 0x07050301u);
            const unsigned t = hb[jj] & 0x7c7c7c7cu; unsigned tmp;
            *(LAS unsigned*)(size_t)wp = t | (unsigned)((j & 3) | ((j >> 2) << 7));
            asm("v_min_u32 %0, 1, %2\n\tv_lshl_add_u32 %1, %0, 8, %1" : "=&v"(tmp), "+v"(wp) : "v"(t));
            if (jj & 3) hb[jj] = rotl(hb[jj], 2 * (jj & 3));
            if (j == EARLYJ && early) { WAIT_V(0); st16<-3072>(early, lane * 16, pk[1]); }
        }
        pk[8][s] = bfi(0x81818181u, hb[0], bfi(0x06060606u, hb[1], bfi(0x18181818u, hb[2], hb[3])));
        pk[9][s] = bfi(0x81818181u, hb[4], bfi(0x06060606u, hb[5], bfi(0x18181818u, hb[6], hb[7])));
        const unsigned ga = bfi(0x02020202u, hb[0], bfi(0x08080808u, hb[1], bfi(0x20202020u, hb[2], hb[3])));
        const unsigned gb = bfi(0x02020202u, hb[4], bfi(0x08080808u, hb[5], bfi(0x20202020u, hb[6], hb[7])));
        z[s] = ~bfi(0xaaaaaaaau, ga, gb >> 1);
    }
    PT(1);
    unsigned e8[8], nz4[4];
#pragma unroll
    for (int i = 0; i < 8; ++i) e8[i] = *(LAS unsigned*)(scr + i * 256 + lane * 4);
#pragma unroll
    for (int s = 0; s < 4; ++s) {
        unsigned t = z[s] | (z[s] >> 1); t |= t << 2;
        nz4[s] = t & 0x44444444u; F |= s < 2 ? nz4[s] >> (2 - s) : nz4[s] << (s - 2);
    }
    const unsigned nrow = (wp - wp0) >> 8, mine = (__builtin_popcount(F) << 14) | nrow;
    const unsigned inc = wave_scan(mine), tot = __builtin_amdgcn_readlane(inc, 63), ngrp = tot & 0xfffu;
    const unsigned lo = 256u + ((((tot >> 12) + 31u) >> 5) << 2);
    unsigned cnt = ngrp, bad = 0;
    PT(2);
    if (ngrp <= 512u) {
        LAS unsigned* dense = (LAS unsigned*)(scr + SCR_DENSE);
        const unsigned lanebits = ((lane & 7) << 15) | ((lane >> 3) << 23), base = (inc - mine) & 0xfffu;
#pragma unroll
        for (int i = 0; i < 8; ++i) dense[(unsigned)i < nrow ? base + i : 512 + lane] = e8[i] | lanebits;
        for (unsigned r = 8; __builtin_expect(__any(nrow > r), 0); ++r)
            dense[r < nrow ? base + r : 512 + lane] = *(LAS unsigned*)(scr + r * 256 + lane * 4) | lanebits;
        unsigned e2[2] = {dense[lane], dense[64 + lane]};
        *(LAS u32x4*)(scr + 256 + lane * 16) = (u32x4){0u, 0u, 0u, 0u};
        *(LAS unsigned*)(scr + lane * 4) = F;
        PT(3);
        if (early) { WAIT_V(0); st16<0>(early + 4096, lane * 16, pk[8]); }
#pragma unroll
        for (int s = 0; s < 4; ++s) { seg[s] = z[s] & 15u; len[s] = nz4[s] & 7u; }
#pragma unroll
        for (int i = 1; i < 8; ++i)
#pragma unroll
            for (int s = 0; s < 4; ++s) { seg[s] |= ((z[s] >> (4 * i)) & 15u) << len[s]; len[s] += (nz4[s] >> (4 * i)) & 7u; }
        PT(4);
        unsigned bit = (inc - mine) >> 12;
#pragma unroll
        for (int s = 0; s < 4; ++s) {
            const unsigned long long v = (unsigned long long)seg[s] << (bit & 31u);
            LAS unsigned* w = (LAS unsigned*)(scr + 256) + (bit >> 5);
            __hip_atomic_fetch_or(w, (unsigned)v, __ATOMIC_RELAXED, __HIP_MEMORY_SCOPE_WAVEFRONT);
            __hip_atomic_fetch_or(w + 1, (unsigned)(v >> 32), __ATOMIC_RELAXED, __HIP_MEMORY_SCOPE_WAVEFRONT);
            bit += len[s];
        }
        PT(5);
        if (early) { WAIT_V(0); st16<1024>(early + 4096, lane * 16, pk[9]); }
        LAS unsigned short* lst = (LAS unsigned short*)(scr + lo);
        const unsigned hz = ((0u - L) >> 10) & 31u;
        for (unsigned i0 = 0;;) {
            unsigned t[2], pos[2], at[2];
#pragma unroll
            for (int c = 0; c < 2; ++c) {
                const unsigned e = e2[c], ix = i0 + c * 64 + lane;
                t[c] = ix < ngrp ? e & 0x7c7c7c7cu : 0u;
                pos[c] = ((e & 0x380u) << 3) | ((e >> 16) & 0x380u) | ((e >> 11) & 0x70u) | ((e & 3u) << 2);
                at[c] = ix < ngrp ? ix : 1024u + lane;
            }
            for (;;) {
#pragma unroll
                for (int c = 0; c < 2; ++c) {
                    const unsigned b8 = __builtin_ctz(t[c] | 0x80000000u) & 24u, hi5 = (t[c] >> (b8 + 2)) & 31u;
                    const unsigned code = hi5 == hz ? 5u : (hi5 + 5u) & 31u;
                    bad = umax(bad, code);
                    lst[at[c]] = (unsigned short)((code << 13) | pos[c] | (b8 >> 3));
                    t[c] &= ~(0xffu << b8);
                }
                const unsigned long long b0 = __ballot(t[0] != 0), b1 = __ballot(t[1] != 0);
                if (!(b0 | b1)) break;
                at[0] = t[0] != 0 ? __builtin_amdgcn_mbcnt_hi((unsigned)(b0 >> 32), __builtin_amdgcn_mbcnt_lo((unsigned)b0, umin(cnt, 512u))) : 1024u + lane;
                cnt += __builtin_popcountll(b0);
                at[1] = t[1] != 0 ? __builtin_amdgcn_mbcnt_hi((unsigned)(b1 >> 32), __builtin_amdgcn_mbcnt_lo((unsigned)b1, umin(cnt, 512u))) : 1024u + lane;
                cnt += __builtin_popcountll(b1);
            }
            i0 += 128;
            if (__builtin_expect(i0 >= ngrp, 1)) break;
            e2[0] = dense[i0 + lane]; e2[1] = dense[i0 + 64 + lane];
        }
    }
    PT(6);
    pk[10] = *(LAS u32x4*)(scr + lane * 16);
    pk[11] = *(LAS u32x4*)(scr + 1024 + lane * 16);
    tl = (lo + 2u * cnt + 63u) >> 6;
    if (cnt > 512u || tl > 32u || __any(bad > 7u) || FORCE_RAW) cnt = RAWCNT;
    return L | (cnt << 16) | (tl > 16u ? 1u << 26 : 0u);
}
__device__ __forceinline__ void st_tail(const char* sb8k  , unsigned voff, u32x4 t0, u32x4 t1, unsigned tl, int lane) {
    if (__builtin_expect(tl > 16u, 0)) { if ((unsigned)lane < (tl - 16u) * 4u) st16<3072>(sb8k, voff, t1); }
    if ((unsigned)lane < tl * 4u) st16<2048>(sb8k, voff, t0);
}
__device__ __forceinline__ void st_raw(const char* slot, unsigned voff, const u32x4 (&b)[16]) {
    const char* s0 = slot + 4096; const char* s1 = slot + 12288;
    st16<-4096>(s0, voff, b[0]); st16<-3072>(s0, voff, b[1]); st16<-2048>(s0, voff, b[2]); st16<-1024>(s0, voff, b[3]);
    st16<0>(s0, voff, b[4]); st16<1024>(s0, voff, b[5]); st16<2048>(s0, voff, b[6]); st16<3072>(s0, voff, b[7]);
    st16<-4096>(s1, voff, b[8]); st16<-3072>(s1, voff, b[9]); st16<-2048>(s1, voff, b[10]); st16<-1024>(s1, voff, b[11]);
    st16<0>(s1, voff, b[12]); st16<1024>(s1, voff, b[13]); st16<2048>(s1, voff, b[14]); st16<3072>(s1, voff, b[15]);
}
constexpr int DEC_TAIL = 8448;
__device__ __forceinline__ void ld_tail2(const char* slot, unsigned voff, unsigned ldst) {
    unsigned keep;
    asm volatile("s_mov_b32 %0, m0\n\ts_mov_b32 m0, %3\n\ts_nop 0\n\tglobal_load_lds_dwordx4 %1, %2 sc0 sc1\n\ts_mov_b32 m0, %0"
                 : "=&s"(keep) : "v"(voff), "s"(slot + 11264), "s"(ldst + 1024) : "memory");
}
__device__ __forceinline__ void ld_packed(u32x4 (&r)[10], unsigned& F, const char* slot, unsigned voff, unsigned ldst) {
    const char* s0 = slot + 4096; const char* s1 = slot + 10240;
    unsigned keep;
    asm volatile("s_mov_b32 %11, m0\n\ts_mov_b32 m0, %16\n\t"
                 "global_load_dword %10, %13, %15 sc0 sc1\n\tglobal_load_lds_dwordx4 %12, %15 sc0 sc1\n\t"
                 "global_load_dwordx4 %8, %12, %15 offset:-2048 sc0 sc1\n\tglobal_load_dwordx4 %9, %12, %15 offset:-1024 sc0 sc1\n\t"
                 "global_load_dwordx4 %0, %12, %14 offset:-4096 sc0 sc1\n\tglobal_load_dwordx4 %1, %12, %14 offset:-3072 sc0 sc1\n\t"
                 "global_load_dwordx4 %2, %12, %14 offset:-2048 sc0 sc1\n\tglobal_load_dwordx4 %3, %12, %14 offset:-1024 sc0 sc1\n\t"
                 "global_load_dwordx4 %4, %12, %14 sc0 sc1\n\tglobal_load_dwordx4 %5, %12, %14 offset:1024 sc0 sc1\n\t"
                 "global_load_dwordx4 %6, %12, %14 offset:2048 sc0 sc1\n\tglobal_load_dwordx4 %7, %12, %14 offset:3072 sc0 sc1\n\t"
                 "s_mov_b32 m0, %11\n\ts_waitcnt vmcnt(0)"
                 : "=&v"(r[0]), "=&v"(r[1]), "=&v"(r[2]), "=&v"(r[3]), "=&v"(r[4]), "=&v"(r[5]), "=&v"(r[6]), "=&v"(r[7]),
                   "=&v"(r[8]), "=&v"(r[9]), "=&v"(F), "=&s"(keep)
                 : "v"(voff), "v"(voff >> 2), "s"(s0), "s"(s1), "s"(ldst) : "memory");
}
__device__ __forceinline__ void ld_head(u32x4 (&r)[10], unsigned& F, unsigned& t, const char* slot, unsigned voff, unsigned ldst, const unsigned* tp) {
    const char* s1 = slot + 10240;
    unsigned keep;
    asm volatile("s_mov_b32 %6, m0\n\ts_mov_b32 m0, %11\n\t"
                 "global_load_dword %4, %8, %10 sc0 sc1\n\tglobal_load_lds_dwordx4 %7, %10 sc0 sc1\n\t"
                 "global_load_dwordx4 %2, %7, %10 offset:-2048 sc0 sc1\n\tglobal_load_dwordx4 %3, %7, %10 offset:-1024 sc0 sc1\n\t"
                 "global_load_dwordx4 %0, %7, %9 sc0 sc1\n\tglobal_load_dwordx4 %1, %7, %9 offset:1024 sc0 sc1\n\t"
                 "global_load_dword %5, %12, off sc1\n\ts_mov_b32 m0, %6\n\ts_waitcnt vmcnt(0)"
                 : "=&v"(r[0]), "=&v"(r[1]), "=&v"(r[8]), "=&v"(r[9]), "=&v"(F), "=&v"(t), "=&s"(keep)
                 : "v"(voff), "v"(voff >> 2), "s"(slot), "s"(s1), "s"(ldst), "v"(tp) : "memory");
}
__device__ __forceinline__ void ld4x16(u32x4& a, u32x4& b, u32x4& c, u32x4& d, const char* sb, unsigned voff) {
    asm volatile("global_load_dwordx4 %0, %4, %5 sc0 sc1\n\tglobal_load_dwordx4 %1, %4, %5 offset:1024 sc0 sc1\n\t"
                 "global_load_dwordx4 %2, %4, %5 offset:2048 sc0 sc1\n\tglobal_load_dwordx4 %3, %4, %5 offset:3072 sc0 sc1\n\ts_waitcnt vmcnt(0)"
                 : "=&v"(a), "=&v"(b), "=&v"(c), "=&v"(d) : "v"(voff), "s"(sb) : "memory");
}
__device__ __forceinline__ void ld2x16t(u32x4& a, u32x4& b, unsigned& t, const char* sb, unsigned voff, const unsigned* tp) {
    asm volatile("global_load_dwordx4 %0, %3, %4 sc0 sc1\n\tglobal_load_dwordx4 %1, %3, %4 offset:1024 sc0 sc1\n\t"
                 "global_load_dword %2, %5, off sc1\n\ts_waitcnt vmcnt(0)"
                 : "=&v"(a), "=&v"(b), "=&v"(t) : "v"(voff), "s"(sb), "v"(tp) : "memory");
}
__device__ __forceinline__ void patch_apply(LAS unsigned char* dl, unsigned lo, unsigned hdr, int lane, bool zero) {
    const unsigned cnt = hdr_cnt(hdr), lut_hi = 0x0804007cu | ((((0u - (hdr & 0x7fffu)) >> 10) & 31u) << 10);
    for (unsigned c = 0; c < cnt; c += 128) {
        const unsigned w = *(LAS unsigned*)(dl + DEC_TAIL + lo + c * 2 + lane * 4);
#pragma unroll
        for (int hf = 0; hf < 2; ++hf) {
            const unsigned e = hf ? w >> 16 : w & 0xffffu;
            const unsigned at = c + lane * 2 + hf < cnt ? e & 0x1fffu : 8192u + lane;
            dl[at] = zero ? 0 : (unsigned char)__builtin_amdgcn_perm(lut_hi, 0x7874706cu, e >> 13);
        }
    }
}
__device__ __forceinline__ unsigned unpack_head(unsigned F, unsigned hdr, LAS unsigned char* dl, int lane, u32x4& G) {
    PT(8);
    const unsigned mine = __builtin_popcount(F) << 2, inc = wave_scan(mine);
    const unsigned lo = 256u + ((((unsigned)__builtin_amdgcn_readlane(inc, 63) + 31u) >> 5) << 2);
    unsigned bit = inc - mine, sg[4];
#pragma unroll
    for (int s = 0; s < 4; ++s) {
        const LAS unsigned* w = (const LAS unsigned*)(dl + DEC_TAIL + 256) + (bit >> 5);
        sg[s] = __builtin_amdgcn_alignbit(w[1], w[0], bit);
        bit += __builtin_popcount(F & (0x11111111u << s)) << 2;
    }
    PT(9);
    patch_apply(dl, lo, hdr, lane, false);
    PT(10);
#pragma unroll
    for (int s = 0; s < 4; ++s) {
        const unsigned nz = (F >> s) & 0x11111111u, pre = nz * 0x11111110u;
        const unsigned xe = sg[s] & 0x0f0f0f0fu, xo = (sg[s] >> 4) & 0x0f0f0f0fu;
        const unsigned b0 = __builtin_amdgcn_perm(xo, xe, 0x05010400u), b1 = __builtin_amdgcn_perm(xo, xe, 0x07030602u);
        const unsigned ze = __builtin_amdgcn_perm(b1, b0, pre & 0x07070707u), zo = __builtin_amdgcn_perm(b1, b0, (pre >> 4) & 0x07070707u);
        G[s] = ~((ze | (zo << 4)) & ((nz << 4) - nz));
    }
    PT(11);
    return lo;
}
__device__ __forceinline__ void unpack_sub(const u32x4 (&r)[10], u32x4 G, int s, LAS unsigned char* dl, int lane, unsigned LL, u32x4 (&o)[4]) {
    const u32x4 p0 = *(LAS u32x4*)(dl + (2 * s) * 1024 + lane * 16), p1 = *(LAS u32x4*)(dl + (2 * s + 1) * 1024 + lane * 16);
    const unsigned A = r[8][s], B = r[9][s], ga = G[s], gb = G[s] << 1;
    unsigned hb[8];
    hb[0] = bfi(0x83838383u, bfi(0x02020202u, ga, A), p0[0]); hb[1] = bfi(0x83838383u, rotr(bfi(0x08080808u, ga, A), 2), p0[1]);
    hb[2] = bfi(0x83838383u, rotr(bfi(0x20202020u, ga, A), 4), p0[2]); hb[3] = bfi(0x83838383u, rotr(bfi(0x80808080u, ga, A), 6), p0[3]);
    hb[4] = bfi(0x83838383u, bfi(0x02020202u, gb, B), p1[0]); hb[5] = bfi(0x83838383u, rotr(bfi(0x08080808u, gb, B), 2), p1[1]);
    hb[6] = bfi(0x83838383u, rotr(bfi(0x20202020u, gb, B), 4), p1[2]); hb[7] = bfi(0x83838383u, rotr(bfi(0x80808080u, gb, B), 6), p1[3]);
#pragma unroll
    for (int jj = 0; jj < 8; ++jj) {
        const unsigned lo = r[2 * s + (jj >> 2)][jj & 3];
        o[jj >> 1][2 * (jj & 1)] = pk_add(__builtin_amdgcn_perm(hb[jj], lo, 0x05010400u), LL);
        o[jj >> 1][2 * (jj & 1) + 1] = pk_add(__builtin_amdgcn_perm(hb[jj], lo, 0x07030602u), LL);
    }
}

__device__ __forceinline__ void unit_fetch(const Params& p, int h, int u, int lane, unsigned ldsbuf) {
    const bool isx = u < 16;
    const char* base = isx ? p.x + ((size_t)u * 256 * KS + h * 32) * 2 : p.w + ((size_t)h * 32 * NN + (u - 16) * 256) * 2;
    unsigned voff = isx ? (lane & 15) * (KS * 2) + (lane >> 4) * 16 : (lane >> 4) * (8 * NN * 2) + (lane & 15) * 16;
    const unsigned s1 = isx ? 16 * KS * 2 : NN * 2, s2 = isx ? 16 * KS * 2 : 256 - 7 * NN * 2;
#pragma unroll
    for (int i = 0; i < 16; ++i) {
        unsigned keep;
        asm volatile("s_mov_b32 %0, m0\n\ts_mov_b32 m0, %3\n\ts_nop 0\n\tglobal_load_lds_dwordx4 %1, %2\n\ts_mov_b32 m0, %0"
                     : "=&s"(keep) : "v"(voff), "s"(base), "s"(ldsbuf + i * 1024) : "memory");
        voff += (i == 7) ? s2 : s1;
    }
}
__device__ __forceinline__ void unit_read(LAS unsigned char* buf, int lane, u32x4 (&r)[16]) {
#pragma unroll
    for (int i = 0; i < 16; ++i) r[i] = *(LAS u32x4*)(buf + i * 1024 + lane * 16);
    asm volatile("s_waitcnt lgkmcnt(0)" : "+v"(r[0]), "+v"(r[1]), "+v"(r[2]), "+v"(r[3]), "+v"(r[4]), "+v"(r[5]), "+v"(r[6]), "+v"(r[7]),
                 "+v"(r[8]), "+v"(r[9]), "+v"(r[10]), "+v"(r[11]), "+v"(r[12]), "+v"(r[13]), "+v"(r[14]), "+v"(r[15]) :: "memory");
}
__device__ __forceinline__ void unit_format(int u, u32x4 (&r)[16]) {
    if (u >= 16) {
#pragma unroll
        for (int g = 0; g < 2; ++g) {
            u32x4 o[8];
#pragma unroll
            for (int i = 0; i < 8; ++i)
#pragma unroll
                for (int a = 0; a < 4; ++a)
                    o[i][a] = __builtin_amdgcn_perm(r[g * 8 + 2 * a + 1][i >> 1], r[g * 8 + 2 * a][i >> 1], (i & 1) ? 0x07060302u : 0x05040100u);
#pragma unroll
            for (int i = 0; i < 8; ++i) r[g * 8 + i] = o[i];
        }
    }
}
struct OrderTab { unsigned e[NU]; };
constexpr OrderTab make_order() {
    OrderTab t{}; int n = 0;
    for (int v = 0; v < 8 * NW0; ++v) t.e[n++] = (v >> 3) << 8 | (v & 7);
    for (int key = 0; key < 8 * NH + 512; ++key)
        for (int v = 0; v < 22; ++v) {
            const int lag = v < 8 ? LAGW0 + (v & 3) * LAGWS + ((v & 2) ? LAGWH : 0) : v < 15 ? LAGXA : LAGXB;
            const int d = key - lag;
            if (d >= 0 && d % 8 == 0 && d / 8 < NH && !(v < 8 && d / 8 < NW0)) t.e[n++] = (unsigned)((d / 8) << 8 | v);
        }
    return t;
}
__constant__ const OrderTab g_order = make_order();
__device__ __forceinline__ int order(int k, int src) {
    const int e = g_order.e[k], v = e & 255;
    const int odd = v >= 15, m = (src + 1 + v - (odd ? 15 : 8)) & 7;
    return (e & ~255) | (v < 8 ? 16 + v : 2 * m + odd);
}

constexpr int PK_LDS = 28 * 1024;
struct SendCtl { u32x4 desc[16]; unsigned freed[7]; unsigned tail; };
__device__ __forceinline__ void packer(const Params& p, LAS unsigned char* buf, LAS SendCtl* ctl, int xcd, int di, int mem, int lane, unsigned* cntp) {
    const unsigned dest = (unsigned)(p.rank + 1 + di) & 7u;
    const unsigned voff = lane * 16, lbuf = (unsigned)(size_t)buf;
    unsigned k = (RAW0 + mem) * 8 + xcd;
    int o, h = RAW0 + mem, u = 16 + dest * 8 + xcd;
    const unsigned call = __builtin_amdgcn_readfirstlane(ld4_sys(cntp)) + 1u;
    unsigned* tk = (unsigned*)(p.ws[p.rank] + OFF_TK) + di * 32;
    const unsigned tk0 = (call - 1u) * (NU - 8 * RAW0) - (RAW0 + NPK) * 8;
    unsigned nput = 0;
    for (;;) {
        WAIT_V(0);
        u32x4 r[16], pk[12];
        unit_read(buf, lane, r);
        unit_format(u, r);
        unsigned tl;
        const unsigned hdr = pack_unit(r, pk, tl, pack_base(r), buf, lane, nullptr) | (4u << 28);
        asm volatile("" : "+v"(pk[0]), "+v"(pk[1]), "+v"(pk[2]), "+v"(pk[3]), "+v"(pk[4]), "+v"(pk[5]), "+v"(pk[6]), "+v"(pk[7]), "+v"(pk[8]), "+v"(pk[9]), "+v"(pk[10]), "+v"(pk[11]));
        while ((unsigned)__builtin_amdgcn_readfirstlane(*(volatile LAS unsigned*)&ctl->freed[mem]) != nput) __builtin_amdgcn_s_sleep(1);
        ++nput;
        if (__builtin_expect(hdr_cnt(hdr) == RAWCNT, 0)) {
            unit_fetch(p, h, u, lane, lbuf);
            WAIT_V(0);
            unit_read(buf, lane, r);
            unit_format(u, r);
            st_raw(p.ws[dest] + OFF_LZ + (call & 1u) * LZ_PAR + (size_t)((6 - di) * NU + k) * UB, voff, r);
            WAIT_V(0);
        } else {
#pragma unroll
            for (int i = 0; i < 12; ++i) *(LAS u32x4*)(buf + 16384 + i * 1024 + lane * 16) = pk[i];
        }
        asm volatile("" ::: "memory");
        unsigned tn = 1u;
        if (lane == 0) {
            const unsigned pos = __hip_atomic_fetch_add(&ctl->tail, 1u, __ATOMIC_RELAXED, __HIP_MEMORY_SCOPE_WORKGROUP);
            *(volatile LAS u32x4*)&ctl->desc[pos & 15] = (u32x4){pos + 1u, k, hdr, (unsigned)mem | (nput << 8) | (tl << 24)};
        }
        if (SHALLOW) while ((unsigned)__builtin_amdgcn_readfirstlane(*(volatile LAS unsigned*)&ctl->freed[mem]) != nput) __builtin_amdgcn_s_sleep(1);
        if (lane == 0) asm volatile("global_atomic_add %0, %1, %0, off sc0\n\ts_waitcnt vmcnt(0)" : "+v"(tn) : "v"(tk) : "memory");
        k = __builtin_amdgcn_readfirstlane(tn) - tk0;
        if (k >= (unsigned)NU) break;
        o = order(k, p.rank); h = o >> 8; u = (o & 255) + ((o & 16) ? dest * 8 : 0);
        unit_fetch(p, h, u, lane, lbuf);
    }
    if (lane == 0) {
        const unsigned pos = __hip_atomic_fetch_add(&ctl->tail, 1u, __ATOMIC_RELAXED, __HIP_MEMORY_SCOPE_WORKGROUP);
        *(volatile LAS u32x4*)&ctl->desc[pos & 15] = (u32x4){pos + 1u, 0u, 0u, 0x80u};
    }
}
__device__ __forceinline__ void storer(const Params& p, LAS unsigned char* lds, LAS SendCtl* ctl, int xcd, int di, int lane, unsigned* cntp) {
    const unsigned dest = (unsigned)(p.rank + 1 + di) & 7u;
    char* dws = p.ws[dest];
    const int li = 6 - di;
    const unsigned voff = lane * 16;
    const unsigned call = __builtin_amdgcn_readfirstlane(ld4_sys(cntp)) + 1u;
    const char* lz = dws + OFF_LZ + (call & 1u) * LZ_PAR + (size_t)li * NU * UB;
    char* fa = dws + OFF_FA + (size_t)((call & 1u) * 7 + li) * NU * FLS;
    unsigned pos = 0, done = 0, pend_hdr = 0; int pend = -1;
#define POST(kk, hh) st8_flag(fa + (kk) * FLS, call, (hh), lane)
    if (RAW0) {
        u32x4 r[16];
        unit_read(lds + NPK * PK_LDS, lane, r);
        unit_format(16, r);
        STAMP(5);
        const char* s0 = lz + (size_t)xcd * UB + 4096; const char* s1 = s0 + 8192;
        st16<-4096>(s0, voff, r[0]); st16<-3072>(s0, voff, r[1]); st16<-2048>(s0, voff, r[2]); st16<-1024>(s0, voff, r[3]);
        st16<0>(s0, voff, r[4]); st16<1024>(s0, voff, r[5]); WAIT_V(SWIN); st16<2048>(s0, voff, r[6]); WAIT_V(SWIN); st16<3072>(s0, voff, r[7]); WAIT_V(SWIN);
        st16<-4096>(s1, voff, r[8]); WAIT_V(SWIN); st16<-3072>(s1, voff, r[9]); WAIT_V(SWIN); st16<-2048>(s1, voff, r[10]); WAIT_V(SWIN); st16<-1024>(s1, voff, r[11]); WAIT_V(SWIN);
        st16<0>(s1, voff, r[12]); WAIT_V(SWIN); st16<1024>(s1, voff, r[13]); WAIT_V(SWIN); st16<2048>(s1, voff, r[14]); WAIT_V(SWIN); st16<3072>(s1, voff, r[15]); WAIT_V(SWIN);
        pend = xcd; pend_hdr = (RAWCNT << 16) | (4u << 28);
    }
    while (done < NPK) {
        const u32x4 d = *(volatile LAS u32x4*)&ctl->desc[pos & 15];
        if ((unsigned)__builtin_amdgcn_readfirstlane(d.x) != pos + 1u) {
            if (pend >= 0) { WAIT_V(0); POST(pend, pend_hdr); pend = -1; }
            __builtin_amdgcn_s_sleep(1);
            continue;
        }
        const unsigned k = __builtin_amdgcn_readfirstlane(d.y), hdr = __builtin_amdgcn_readfirstlane(d.z), w = __builtin_amdgcn_readfirstlane(d.w);
        ++pos;
        if (w & 0x80u) { ++done; continue; }
        if (k == NU - 1 - GATE_EARLY && lane < 8)
            __hip_atomic_store((unsigned*)(p.ws[p.rank] + OFF_SD) + (di * 8 + lane) * 32, call, __ATOMIC_RELAXED, __HIP_MEMORY_SCOPE_AGENT);
        if (__builtin_expect(hdr_cnt(hdr) == RAWCNT, 0)) {
            POST(k, hdr);
            *(volatile LAS unsigned*)&ctl->freed[w & 7u] = (w >> 8) & 0xffffu;
            continue;
        }
        u32x4 pk[12];
        LAS unsigned char* src = lds + (w & 7u) * PK_LDS + 16384 + lane * 16;
#pragma unroll
        for (int i = 0; i < 12; ++i) pk[i] = *(LAS u32x4*)(src + i * 1024);
        asm volatile("s_waitcnt lgkmcnt(0)" : "+v"(pk[0]), "+v"(pk[1]), "+v"(pk[2]), "+v"(pk[3]), "+v"(pk[4]), "+v"(pk[5]), "+v"(pk[6]), "+v"(pk[7]), "+v"(pk[8]), "+v"(pk[9]), "+v"(pk[10]), "+v"(pk[11]) :: "memory");
        *(volatile LAS unsigned*)&ctl->freed[w & 7u] = (w >> 8) & 0xffffu;
        const char* s0 = lz + (size_t)k * UB + 4096; const char* s1 = s0 + 4096;
#define PEND_FLAG(i) do { if (SWIN == (i) && pend >= 0) POST(pend, pend_hdr); } while (0)
        st16<-4096>(s0, voff, pk[0]); WAIT_V(SWIN);
        st16<-3072>(s0, voff, pk[1]); WAIT_V(SWIN);
        st16<-2048>(s0, voff, pk[2]); WAIT_V(SWIN); PEND_FLAG(3);
        st16<-1024>(s0, voff, pk[3]); WAIT_V(SWIN); PEND_FLAG(4);
        st16<0>(s0, voff, pk[4]); WAIT_V(SWIN); PEND_FLAG(5);
        st16<1024>(s0, voff, pk[5]); WAIT_V(SWIN); PEND_FLAG(6);
        st16<2048>(s0, voff, pk[6]); WAIT_V(SWIN); PEND_FLAG(7);
        st16<3072>(s0, voff, pk[7]); WAIT_V(SWIN); PEND_FLAG(8);
        st16<0>(s1, voff, pk[8]); WAIT_V(SWIN); PEND_FLAG(9);
        st16<1024>(s1, voff, pk[9]); WAIT_V(SWIN);
        st_tail(s1, voff, pk[10], pk[11], w >> 24, lane); WAIT_V(SWIN);
        pend = k; pend_hdr = hdr;
    }
    WAIT_V(0);
    if (pend >= 0) st8_flag(fa + pend * FLS, call, pend_hdr, lane);
}

__device__ __forceinline__ void unpacker(const Params& p, LAS unsigned char* dl, int uw, int lane, unsigned call) {
    char* ws = p.ws[p.rank];
    unsigned* tmo = (unsigned*)(ws + OFF_TMO);
    const unsigned voff = lane * 16;
#pragma unroll
    for (int i = 0; i < 8; ++i) *(LAS u32x4*)(dl + i * 1024 + lane * 16) = (u32x4){0u, 0u, 0u, 0u};
    for (int n = uw; n < 7 * NU; n += NUNP) {
        const int k = n / 7, li = n - k * 7, src = (p.rank + 1 + li) & 7;
        const int o = order(k, src), t = o & 15; const bool isx = !(o & 16);
        const int q = NH + (o >> 8) * 7 + li;
        const unsigned hdr = wait_unit(ws + OFF_FA + (size_t)(((call & 1u) * 7 + li) * NU + k) * FLS, call, 4u, tmo);
        const unsigned cnt = hdr_cnt(hdr); const bool raw = cnt == RAWCNT;
        unsigned lo = 0;
        const char* sl = ws + OFF_LZ + (call & 1u) * LZ_PAR + (size_t)(li * NU + k) * UB;
        const char* dst = ws + (isx ? OFF_XI + (size_t)q * XQ : OFF_WI + (size_t)q * WQ) + (size_t)t * UB;
        unsigned* flag = (unsigned*)(ws + (isx ? OFF_FX : OFF_FW)) + (isx ? t : p.rank * 8 + t) * FSTR + q;
        if (__builtin_expect(raw, 0)) {
            for (int s = 0; s < 4; ++s) {
                u32x4 o[4]; ld4x16(o[0], o[1], o[2], o[3], sl + s * 4096, voff);
                const char* sb = dst + s * 4096;
                st16<0>(sb, voff, o[0]); st16<1024>(sb, voff, o[1]); st16<2048>(sb, voff, o[2]); st16<3072>(sb, voff, o[3]);
            }
        } else {
            u32x4 r[10], G; unsigned F;
            if (hdr & (1u << 26)) ld_tail2(sl, voff, (unsigned)(size_t)(dl + DEC_TAIL));
            ld_packed(r, F, sl, voff, (unsigned)(size_t)(dl + DEC_TAIL));
            const unsigned LL = (hdr & 0x7fffu) * 0x10001u;
            lo = unpack_head(F, hdr, dl, lane, G);
#pragma unroll
            for (int s = 0; s < 4; ++s) {
                u32x4 o[4]; unpack_sub(r, G, s, dl, lane, LL, o);
                const char* sb = dst + s * 4096;
                st16<0>(sb, voff, o[0]); st16<1024>(sb, voff, o[1]); st16<2048>(sb, voff, o[2]); st16<3072>(sb, voff, o[3]);
            }
        }
        WAIT_V(0);
        if (lane == 0) __hip_atomic_store(flag, call, __ATOMIC_RELAXED, __HIP_MEMORY_SCOPE_SYSTEM);
        if (!raw) { patch_apply(dl, lo, hdr, lane, true); WAIT_L(0); }
    }
}

__device__ __forceinline__ void formatter(const Params& p, LAS unsigned char* buf, int fw, int n2nd, int lane, unsigned call) {
    char* ws = p.ws[p.rank];
    const unsigned voff = lane * 16;
    const unsigned long long ts = __builtin_amdgcn_s_memrealtime();
    for (int n = fw < 384 ? fw : n2nd; n < 1280; n = (n < 384) ? n2nd : n + NFMT) {
        const unsigned at = n < 384 ? PD0 + n * PDT / 16 : FD0 + (n - 384) * FDT / 16;
        while (__builtin_amdgcn_s_memrealtime() - ts < at) __builtin_amdgcn_s_sleep(4);
        int h, u; const char* dst; unsigned* flag;
        if (n < 384) { h = n / 24; const int v = n - h * 24;
            if (v < 16) { u = v; dst = ws + OFF_XI + (size_t)h * XQ + (size_t)v * UB; flag = (unsigned*)(ws + OFF_FX) + v * FSTR + h; }
            else { u = p.rank * 8 + v; dst = ws + OFF_WI + (size_t)h * WQ + (size_t)(v - 16) * UB; flag = (unsigned*)(ws + OFF_FW) + (u - 16) * FSTR + h; } }
        else { const int n2 = n - 384; h = n2 / 56; const int v = n2 - h * 56, li = v >> 3, cn = ((p.rank + 1 + li) & 7) * 8 + (v & 7);
            u = 16 + cn; dst = ws + OFF_WD + (size_t)(li * NH + h) * WQ + (size_t)(v & 7) * UB; flag = (unsigned*)(ws + OFF_FW) + cn * FSTR + h; }
        u32x4 r[16];
        unit_fetch(p, h, u, lane, (unsigned)(size_t)buf);
        WAIT_V(0);
        unit_read(buf, lane, r);
        unit_format(u, r);
        st_raw(dst, voff, r);
        WAIT_V(0);
        if (lane == 0) __hip_atomic_store(flag, call, __ATOMIC_RELAXED, __HIP_MEMORY_SCOPE_SYSTEM);
    }
}

__device__ __forceinline__ void poll_ready(const unsigned* fx, const unsigned* fw, unsigned long long& td0, unsigned long long& td1,
                                           LAS unsigned* queue, int& nq, int need, unsigned call, unsigned* tmo) {
    const unsigned lane = __builtin_amdgcn_mbcnt_hi(~0u, __builtin_amdgcn_mbcnt_lo(~0u, 0u));
    const unsigned long long t0 = __builtin_amdgcn_s_memrealtime(); unsigned polls = 0;
    bool all = false;
    for (;;) {
        unsigned x0, x1, w0, w1;
        asm volatile("global_load_dword %0, %4, %5 sc0 sc1\n\tglobal_load_dword %1, %4, %5 offset:256 sc0 sc1\n\t"
                     "global_load_dword %2, %4, %6 sc0 sc1\n\tglobal_load_dword %3, %4, %6 offset:256 sc0 sc1\n\ts_waitcnt vmcnt(0)"
                     : "=&v"(x0), "=&v"(x1), "=&v"(w0), "=&v"(w1) : "v"(lane * 4), "s"(fx), "s"(fw) : "memory");
        const unsigned long long a0 = __ballot(all || (x0 >= call && w0 >= call)) & td0, a1 = __ballot(all || (x1 >= call && w1 >= call)) & td1;
        if ((a0 >> lane) & 1) ((volatile LAS unsigned*)queue)[__builtin_amdgcn_mbcnt_hi((unsigned)(a0 >> 32), __builtin_amdgcn_mbcnt_lo((unsigned)a0, nq))] = lane | 0x10000u;
        nq += __builtin_popcountll(a0);
        if ((a1 >> lane) & 1) ((volatile LAS unsigned*)queue)[__builtin_amdgcn_mbcnt_hi((unsigned)(a1 >> 32), __builtin_amdgcn_mbcnt_lo((unsigned)a1, nq))] = lane | 0x10040u;
        nq += __builtin_popcountll(a1);
        td0 &= ~a0; td1 &= ~a1;
        if (nq >= need) break;
        all = give_up(tmo, call, t0, polls);
        __builtin_amdgcn_s_sleep(1);
    }
    WAIT_L(0);
}

constexpr int HTB = 16384;
constexpr int LDS_CTL = (NPK * PK_LDS + 16384 > 8 * HTB + 8192) ? NPK * PK_LDS + 16384 : 8 * HTB + 8192;

__global__ __launch_bounds__(512, 2) void fused_kernel(const Params p) {
    __shared__ __attribute__((aligned(16))) unsigned char shm[LDS_CTL + 512];
    LAS unsigned char* lds = (LAS unsigned char*)shm;
    const int wid = __builtin_amdgcn_readfirstlane(threadIdx.x >> 6), lane = threadIdx.x & 63;
    const int xcd = blockIdx.x & 7, slot = blockIdx.x >> 3;
    char* ws = p.ws[p.rank];
    unsigned* tmo = (unsigned*)(ws + OFF_TMO);
    unsigned* cntp = (unsigned*)(ws + OFF_CNT) + blockIdx.x * 8 + wid;
    STAMP(0);
    const bool main_tile = slot >= 16;
    if (slot < 7) {
        LAS SendCtl* ctl = (LAS SendCtl*)(lds + LDS_CTL);
        static_assert(RAW0 == 1 && NPK <= 5 && NW0 >= RAW0 + NPK, "opening units");
        if (wid >= 6) ((volatile LAS unsigned*)ctl)[(wid - 6) * 64 + lane] = 0u;
        WAIT_L(0); BAR;
        if (wid == 0) __builtin_amdgcn_s_setprio(3); else __builtin_amdgcn_s_sleep(PKDELAY);
        if (wid <= NPK) unit_fetch(p, wid, 16 + ((p.rank + 1 + slot) & 7) * 8 + xcd, lane, (unsigned)(size_t)(lds + ((wid + NPK) % (NPK + 1)) * PK_LDS));
        if (wid == 0) { storer(p, lds, ctl, xcd, slot, lane, cntp); __builtin_amdgcn_s_setprio(0); }
        else if (wid <= NPK) packer(p, lds + (wid - 1) * PK_LDS, ctl, xcd, slot, wid - 1, lane, cntp);
    }
    const unsigned call = __builtin_amdgcn_readfirstlane(ld4_sys(cntp)) + 1u;
    if (lane == 0) __hip_atomic_store(cntp, call, __ATOMIC_RELAXED, __HIP_MEMORY_SCOPE_AGENT);

    if (!main_tile) {
        if (slot >= 7) {
            const bool unp = slot < 7 + NUS;
            formatter(p, lds + wid * HTB, wid * 72 + (blockIdx.x - 56), unp ? 1 << 20 : 384 + wid * (NFMT / 8) + (blockIdx.x - 56 - NUS * 8), lane, call);
            const int uw = wid * (NUS * 8) + (blockIdx.x - 56);
            if (unp && uw < NUNP) unpacker(p, lds + wid * HTB, uw, lane, call);
        }
        STAMP(1);
        asm volatile("s_waitcnt vmcnt(0) lgkmcnt(0)" ::: "memory");
        BAR;
        if (DIAGD && slot < 7 + NUS) { const unsigned long long ts = __builtin_amdgcn_s_memrealtime(); while (__builtin_amdgcn_s_memrealtime() - ts < (slot < 8 ? DIAGD : DIAGD2)) __builtin_amdgcn_s_sleep(8); }
    }

    int rt, cn, q0, nt, skip;
    const char* wimg;
    if (main_tile) {
        const int idx = slot - 16;
        rt = ((xcd >> 1) * 2 + ((idx >> 2) & 1)) * 2 + (idx >> 3);
        cn = p.rank * 8 + (xcd & 1) * 4 + (idx & 3);
        const int m = rt >> 1;
        q0 = (m == p.rank) ? NH : 0; skip = (m - p.rank - 1) & 7; nt = 56;
        wimg = ws + OFF_WI;
    } else {
        rt = p.rank * 2 + (slot < 8 ? 1 : 0);
        cn = xcd * 8 + (slot & 7);
        q0 = 0; skip = 7; nt = 8;
        wimg = (xcd == p.rank) ? ws + OFF_WI : ws + OFF_WD + (size_t)((xcd - p.rank - 1) & 7) * NH * WQ;
    }
    const int wr = wid >> 2, wc = wid & 3;
    const unsigned* gate = (skip == 7) ? cntp : (const unsigned*)(ws + OFF_SD) + (skip * 8 + xcd) * 32;
    const unsigned* fx = (const unsigned*)(ws + OFF_FX) + rt * FSTR;
    const unsigned* fw = (const unsigned*)(ws + OFF_FW) + cn * FSTR;
    const unsigned* gate2 = (skip == 7) ? cntp : fw + NH + skip;
    const char* xb = ws + OFF_XI + (size_t)(rt * 16 + wid) * 1024;
    const char* wb = wimg + (size_t)((cn & 7) * 16 + wr * 8 + wc) * 1024;
    const unsigned voff = lane * 16;
    const unsigned ldsw = wid * 1024;
    const int aoff = wr * 4096 + lane * 16, boff = wc * 2048 + lane * 16;
#define SA(b, h) (((b) * 4 + (h)) * HTB)
#define SB(b, h) (((b) * 4 + 2 + (h)) * HTB)
#define GLDS(g, o, l) __builtin_amdgcn_global_load_lds((const unsigned*)((g) + (size_t)(unsigned)(o)), (LAS unsigned*)(lds + (l)), 16, 0, GLDS_AUX)
#define STAGE_A(b, h, qq) do { GLDS(wb + (h) * 4096, (qq.x << 17) + voff, SA(b, h) + ldsw); GLDS(wb + (h) * 4096, (qq.y << 17) + voff, SA(b, h) + ldsw + 8192); } while (0)
#define STAGE_B(b, h, qq) do { GLDS(xb + (h) * 8192, (qq.x << 18) + voff, SB(b, h) + ldsw); GLDS(xb + (h) * 8192, (qq.y << 18) + voff, SB(b, h) + ldsw + 8192); } while (0)
#define LDA(dst, b, h) do { _Pragma("unroll") for (int m = 0; m < 4; ++m) _Pragma("unroll") for (int k = 0; k < 2; ++k) dst[m][k] = *(const LAS bf16x8*)(lds + SA(b, h) + aoff + m * 1024 + k * 8192); } while (0)
#define LDB(dst, b, h) do { _Pragma("unroll") for (int n = 0; n < 2; ++n) _Pragma("unroll") for (int k = 0; k < 2; ++k) dst[n][k] = *(const LAS bf16x8*)(lds + SB(b, h) + boff + n * 1024 + k * 8192); } while (0)
#define MMA2(ai) do { __builtin_amdgcn_s_setprio(1); _Pragma("unroll") for (int k = 0; k < 2; ++k) _Pragma("unroll") for (int m = 0; m < 4; ++m) _Pragma("unroll") for (int n = 0; n < 2; ++n) { \
        acc[ai][0][m][n] = __builtin_amdgcn_mfma_f32_16x16x32_bf16(B0[n][k], At[m][k], acc[ai][0][m][n], 0, 0, 0); \
        acc[ai][1][m][n] = __builtin_amdgcn_mfma_f32_16x16x32_bf16(B1[n][k], At[m][k], acc[ai][1][m][n], 0, 0, 0); } __builtin_amdgcn_s_setprio(0); } while (0)
#define MUST_PICK(kt) do { if (__builtin_expect(nq < 2 * (kt) + 2, 0)) poll_ready(fx, fw, td0, td1, queue, nq, 2 * (kt) + 2, call, tmo); } while (0)
#define GETQ(kt) (*(volatile LAS u32x2*)(queue + 2 * (kt)))
#define FIXQ(qq, kt) do { if (__builtin_expect(__builtin_amdgcn_readfirstlane(qq.x & qq.y) < 0x10000, 0)) qq = GETQ(kt); } while (0)
    f32x4 acc[2][2][4][2];
#pragma unroll
    for (int a = 0; a < 2; ++a)
#pragma unroll
        for (int b = 0; b < 2; ++b)
#pragma unroll
            for (int m = 0; m < 4; ++m)
#pragma unroll
                for (int n = 0; n < 2; ++n) acc[a][b][m][n] = (f32x4){0.f, 0.f, 0.f, 0.f};
    bf16x8 At[4][2], B0[2][2], B1[2][2];
    LAS unsigned* queue = (LAS unsigned*)(lds + 8 * HTB + 4096);
    unsigned long long td0 = __ballot(lane >= q0 && lane < 2 * nt + 64 * main_tile && !(lane >= NH && (lane - NH) % 7 == skip));
    unsigned long long td1 = __ballot(main_tile && (lane + 64 - NH) % 7 != skip);
    int nq = 1 << 20;
    if (wid == 0) { nq = 0; *(volatile LAS u32x2*)(queue + 2 * lane) = (u32x2){0u, 0u}; poll_ready(fx, fw, td0, td1, queue, nq, 4, call, tmo); }
    asm volatile("" ::: "memory"); BAR; asm volatile("" ::: "memory");
    u32x2 qe = GETQ(0), qo = GETQ(1);
    STAMP(4);
    STAGE_B(0, 0, qe); STAGE_A(0, 0, qe); STAGE_B(0, 1, qe); STAGE_A(0, 1, qe);
    STAGE_B(1, 0, qo); STAGE_A(1, 0, qo); STAGE_B(1, 1, qo);
    if (wr == 1) BAR;
    WAIT_V(8); BAR; BAR;
    for (int t = 0; t < nt; t += 2) {
        if (t == nt - 2) nq = 1 << 20;
        qe = GETQ(t + 2);
        LDB(B0, 0, 0); LDA(At, 0, 0); LDB(B1, 0, 1); STAGE_A(1, 1, qo);
        WAIT_V(8); WAIT_L(0); BAR; SCHED; MMA2(0); SCHED; MUST_PICK(t + 2); BAR; SCHED;
        FIXQ(qe, t + 2);
        LDA(At, 0, 1); STAGE_B(0, 0, qe); STAGE_A(0, 0, qe); STAGE_B(0, 1, qe);
        WAIT_V(8); WAIT_L(0); BAR; SCHED; MMA2(1); BAR; SCHED;
        if (t == nt - 2)
            __builtin_amdgcn_global_load_lds(lane == 1 ? tmo : lane == 2 ? gate2 : gate, (LAS unsigned*)(lds + 8 * HTB + wid * 256), 4, 0, 17);
        qo = GETQ(t + 3);
        LDB(B0, 1, 0); LDA(At, 1, 0); LDB(B1, 1, 1); STAGE_A(0, 1, qe);
        WAIT_V(8); WAIT_L(0); BAR; SCHED; MMA2(0); SCHED; MUST_PICK(t + 3); BAR; SCHED;
        FIXQ(qo, t + 3);
        LDA(At, 1, 1); STAGE_B(1, 0, qo); STAGE_A(1, 0, qo); STAGE_B(1, 1, qo);
        WAIT_V(8); WAIT_L(0); BAR; SCHED; MMA2(1); BAR; SCHED;
    }
    STAMP(2);
    int tz = threadIdx.x; asm volatile("" : "+v"(tz));
    const int ln = tz & 63;
    const unsigned vo = ln * 16;
    const unsigned tile = (unsigned)((rt & 1) * 64 + cn) * 8 + wid;
    if (main_tile) {
        const u32x4 gv = *(volatile LAS u32x4*)(lds + 8 * HTB + wid * 256);
        bool bad = (unsigned)__builtin_amdgcn_readfirstlane(gv.y) == call;
        const bool open = (unsigned)__builtin_amdgcn_readfirstlane(gv.x) >= call && (unsigned)__builtin_amdgcn_readfirstlane(gv.z) >= call && !bad;
        const unsigned owner = (unsigned)(rt >> 1) & 7u;
        char* ows = p.ws[owner];
        const char* sl = ows + OFF_PL + (size_t)tile * UB;
        char* fl = ows + OFF_FP + (size_t)tile * FLS;
        const char* s0 = sl + 4096; const char* s1 = sl + 8192;
        const int staged = (rt & 1) ? BSTAGE : ASTAGE;
        u32x4 b[16], pk[12];
#pragma unroll
        for (int bj = 0; bj < 2; ++bj)
#pragma unroll
            for (int n = 0; n < 2; ++n)
#pragma unroll
                for (int r = 0; r < 4; ++r)
#pragma unroll
                    for (int ai = 0; ai < 2; ++ai)
#pragma unroll
                        for (int m2 = 0; m2 < 2; ++m2)
                            b[bj * 8 + n * 4 + r][ai * 2 + m2] = cvt_pk_bf16(acc[ai][bj][2 * m2][n][r], acc[ai][bj][2 * m2 + 1][n][r]);
        asm volatile("" : "+v"(b[0]), "+v"(b[1]), "+v"(b[2]), "+v"(b[3]), "+v"(b[4]), "+v"(b[5]), "+v"(b[6]), "+v"(b[7]), "+v"(b[8]), "+v"(b[9]), "+v"(b[10]), "+v"(b[11]), "+v"(b[12]), "+v"(b[13]), "+v"(b[14]), "+v"(b[15]) :: "memory");
        if (wr == 0) BAR;
        const unsigned L = pack_base(b);
        if (open) { st16<-4096>(s0, vo, pack_low(b, 0, L)); WAIT_V(1); } else WAIT_V(0);
        asm volatile("" ::: "memory"); BAR; asm volatile("" ::: "memory");
        unsigned tl;
        unsigned hdr = pack_unit(b, pk, tl, L, lds + wid * HTB, ln, open ? s0 : nullptr);
        if (!open) {
            const unsigned long long t0 = __builtin_amdgcn_s_memrealtime(); unsigned polls = 0;
            while (!bad && !__all(ld4_sys(ln & 1 ? gate2 : gate) >= call)) {
                __builtin_amdgcn_s_sleep(2);
                bad = give_up(tmo, call, t0, polls);
            }
        }
        STAMP(6);
        if (bad) {
            hdr = RAWCNT << 16;
#pragma unroll
            for (int f = 0; f < 16; ++f) b[f] = (u32x4){0x7fc07fc0u, 0x7fc07fc0u, 0x7fc07fc0u, 0x7fc07fc0u};
        }
        if (__builtin_expect(hdr_cnt(hdr) == RAWCNT, 0)) { WAIT_V(0); st_raw(sl, vo, b); }
        else {
            {
                if (!open) { st16<-4096>(s0, vo, pk[0]); WAIT_V(0); st16<-3072>(s0, vo, pk[1]); WAIT_V(0); st16<0>(s1, vo, pk[8]); WAIT_V(0); st16<1024>(s1, vo, pk[9]); }
                WAIT_V(0); st_tail(s1, vo, pk[10], pk[11], tl, ln);
                WAIT_V(0); STAMP(7); if (staged & 1) st8_flag(fl, call, hdr | (1u << 28), ln); st16<-2048>(s0, vo, pk[2]);
                WAIT_V(0); st16<-1024>(s0, vo, pk[3]);
                WAIT_V(0); if (staged & 2) st8_flag(fl, call, hdr | (2u << 28), ln); st16<0>(s0, vo, pk[4]);
                WAIT_V(0); st16<1024>(s0, vo, pk[5]);
                WAIT_V(0); if (staged & 4) st8_flag(fl, call, hdr | (3u << 28), ln); st16<2048>(s0, vo, pk[6]);
                WAIT_V(0); st16<3072>(s0, vo, pk[7]);
            }
        }
        WAIT_V(0);
        st8_flag(fl, call, hdr | (4u << 28), ln);
    } else {
        WAIT_V(0);
        if (wr == 0) BAR;
        BAR;
        LAS unsigned char* dl = lds + wid * HTB;
#pragma unroll
        for (int i = 0; i < 8; ++i) *(LAS u32x4*)(dl + i * 1024 + ln * 16) = (u32x4){0u, 0u, 0u, 0u};
        const char* fl = ws + OFF_FP + (size_t)tile * FLS;
        const char* sl = ws + OFF_PL + (size_t)tile * UB;
        unsigned hdr = wait_unit(fl, call, 1u, tmo);
        const unsigned cnt = hdr_cnt(hdr); const bool raw = cnt == RAWCNT;
        const unsigned LL = (hdr & 0x7fffu) * 0x10001u;
        u32x4 r[10], G;
        unsigned tv = 0;
        if (!raw) {
            unsigned F;
            if (hdr & (1u << 26)) ld_tail2(sl, vo, (unsigned)(size_t)(dl + DEC_TAIL));
            ld_head(r, F, tv, sl, vo, (unsigned)(size_t)(dl + DEC_TAIL), tmo);
            unpack_head(F, hdr, dl, ln, G);
        }
        const char* ob = p.out + ((size_t)((rt & 1) * 256 + wc * 32) * NN + cn * 256 + wr * 128) * 2;
        const unsigned oo = (ln >> 4) * (4 * NN * 2) + (ln & 15) * 16;
#pragma unroll
        for (int s = 0; s < 4; ++s) {
            if ((hdr >> 28) <= (unsigned)s) hdr = wait_unit(fl, call, s + 1, tmo);
            u32x4 o[4];
            if (raw) { ld4x16(o[0], o[1], o[2], o[3], sl + s * 4096, vo); tv = ld4_sys(tmo); }
            else { if (s) ld2x16t(r[2 * s], r[2 * s + 1], tv, sl + s * 2048, vo, tmo); unpack_sub(r, G, s, dl, ln, LL, o); }
            const unsigned nanmask = (tv == call) ? 0x7fc07fc0u : 0u;
#pragma unroll
            for (int rr = 0; rr < 4; ++rr) {
                u32x4 y;
#pragma unroll
                for (int cc = 0; cc < 4; ++cc) {
                    const unsigned pv = o[rr][cc];
                    const f32x2 v = (f32x2){acc[cc >> 1][s >> 1][2 * (cc & 1)][s & 1][rr] + __uint_as_float(pv << 16),
                                            acc[cc >> 1][s >> 1][2 * (cc & 1) + 1][s & 1][rr] + __uint_as_float(pv & 0xffff0000u)};
                    const f32x2 g = gelu2(v);
                    y[cc] = cvt_pk_bf16(g.x, g.y) | nanmask;
                }
                st16<0>(ob + (size_t)((s >> 1) * 128 + (s & 1) * 16 + rr) * (NN * 2), oo, y);
            }
        }
    }
    STAMP(3);
}
}

extern "C" void kernel_launch(int rank, int nranks, ncclComm_t comm, hipStream_t stream,
                              void* const* d_in, const int* in_sizes, int n_in,
                              void* d_out, int out_size, const int* in_dtypes, int out_dtype,
                              void* const* d_out_all, void* const* d_ws_all, size_t ws_bytes) {
    (void)comm; (void)in_dtypes; (void)out_dtype; (void)d_out_all;
    if (nranks != NR || (unsigned)rank >= (unsigned)NR || n_in < 2 || in_sizes[0] != MM * KS || in_sizes[1] != KS * NN ||
        out_size != (MM / NR) * NN || ws_bytes < WS_NEED) {
        fprintf(stderr, "kernel_launch: unexpected problem geometry\n");
        return;
    }
    Params p;
    p.x = (const char*)d_in[0];
    p.w = (const char*)d_in[1];
    for (int i = 0; i < NR; ++i) p.ws[i] = (char*)d_ws_all[i];
    p.out = (char*)d_out;
    p.rank = rank; p.pad = 0;
    hipLaunchKernelGGL(fused_kernel, dim3(256), dim3(512), 0, stream, p);
    HIP_CHECK(hipGetLastError());
}
```

```cpp
#include <hip/hip_runtime.h>
#include <rccl/rccl.h>
#include <cstdio>
#include <cstdint>

#define HIP_CHECK(expr) do { hipError_t _e = (expr); if (_e != hipSuccess) { \
    fprintf(stderr, "HIP error %s at %s:%d\n", hipGetErrorString(_e), __FILE__, __LINE__); \
    return; } } while (0)

#define NW0 (NPK + RAW0)
#define LAGXB 36
#define LAGW0 0
#define LAGWS 0
#define LAGWH 4
#define LAGXA 1
#define GLDS_AUX 17
#define FORCE_RAW 0
#define NUS 5
#define SWIN 6
#define SHALLOW 1
#define DIAGD 750
#define DIAGD2 DIAGD
#define ASYNCP 1
#define WHOLE 1
#define PKST 24
#define RAW0 2
#define PD0 100
#define PDT 24
#define FD0 1000
#define FDT 72
#define EARLYJ 19
#define BSTAGE 7
#define ASTAGE 0
#define FLS 128
#define SWIN1 SWIN
#define SWIN2 7
#define NPK 5
#define GATE_EARLY 12

namespace {
#define LAS __attribute__((address_space(3)))
typedef short bf16x8 __attribute__((ext_vector_type(8)));
typedef float f32x4 __attribute__((ext_vector_type(4)));
typedef float f32x2 __attribute__((ext_vector_type(2)));
typedef unsigned u32x4 __attribute__((ext_vector_type(4)));
typedef unsigned u32x2 __attribute__((ext_vector_type(2)));
typedef unsigned short u16x2 __attribute__((ext_vector_type(2)));

constexpr int NR = 8;
constexpr int MM = 4096, NN = 16384, KS = 512, NS = 2048;
constexpr int NQ = 128;
constexpr int NH = 16;
constexpr int NU = NH * 22;
#define UNP7 1
constexpr int NUNP = UNP7 ? NUS * 64 / 7 * 7 : NUS * 64;
constexpr int NFMT = (9 - NUS) * 64;
constexpr unsigned long long DEADLINE = 20000000ull;
constexpr unsigned RAWCNT = 0x3ffu;

constexpr size_t OFF_CNT = 0;
constexpr size_t OFF_TMO = 8192;
constexpr size_t OFF_TK = 8192 + 1024;
constexpr size_t OFF_SD = 131072;
constexpr int FSTR = 160;
constexpr size_t OFF_FX = 16384;
constexpr size_t OFF_FW = 32768;
constexpr size_t OFF_STAMP = 262144;
constexpr size_t UB = 16384;
constexpr size_t XQ = 256 * 1024, WQ = 128 * 1024;
constexpr size_t OFF_XI = 1u << 20;
constexpr size_t OFF_WI = OFF_XI + (size_t)NQ * XQ;
constexpr size_t OFF_WD = OFF_WI + (size_t)NQ * WQ;
constexpr size_t OFF_PL = OFF_WD + (size_t)7 * NH * WQ;
constexpr size_t OFF_LZ = OFF_PL + (size_t)1024 * UB;
constexpr size_t LZ_PAR = (size_t)7 * NU * UB;
constexpr size_t OFF_FA = OFF_LZ + 2 * LZ_PAR;
constexpr size_t OFF_FP = OFF_FA + (size_t)14 * NU * 128;
constexpr size_t WS_NEED = OFF_FP + 1024 * 128;
static_assert(OFF_FW + 64 * FSTR * 4 <= OFF_SD && OFF_SD + 56 * 128 <= OFF_STAMP, "layout");
static_assert(WS_NEED <= (size_t)161 << 20, "workspace");

struct Params {
    const char* x;
    const char* w;
    char* ws[NR];
    char* out;
    int rank;
    int pad;
};

template <int OFF> __device__ __forceinline__ void st16(const char* sbase, unsigned voff, u32x4 v) {
    asm volatile("global_store_dwordx4 %0, %1, %2 offset:%3 sc0 sc1\n\ts_nop 1" :: "v"(voff), "v"(v), "s"(sbase), "i"(OFF) : "memory");
}
__device__ __forceinline__ unsigned ld4_sys(const void* p) {
    unsigned v;
    asm volatile("global_load_dword %0, %1, off sc0 sc1\n\ts_waitcnt vmcnt(0)" : "=&v"(v) : "v"(p) : "memory");
    return v;
}
__device__ __forceinline__ u32x2 ld8_sys(const void* p) {
    u32x2 v;
    asm volatile("global_load_dwordx2 %0, %1, off sc0 sc1\n\ts_waitcnt vmcnt(0)" : "=&v"(v) : "v"(p) : "memory");
    return v;
}
__device__ __forceinline__ void st8_flag(void* p, unsigned call, unsigned hdr, int lane) {
    if (lane == 0) __hip_atomic_store((unsigned long long*)p, ((unsigned long long)hdr << 32) | call, __ATOMIC_RELAXED, __HIP_MEMORY_SCOPE_SYSTEM);
}
#define WAIT_V_(n) asm volatile("s_waitcnt vmcnt(" #n ")" ::: "memory")
#define WAIT_V(n) WAIT_V_(n)
#define WAIT_L(n) asm volatile("s_waitcnt lgkmcnt(" #n ")" ::: "memory")
#define BAR __builtin_amdgcn_s_barrier()
#define SCHED __builtin_amdgcn_sched_barrier(0)

__device__ __forceinline__ unsigned cvt_pk_bf16(float lo, float hi) {
    unsigned r; asm("v_cvt_pk_bf16_f32 %0, %1, %2" : "=v"(r) : "v"(lo), "v"(hi)); return r;
}
__device__ __forceinline__ f32x2 gelu2(f32x2 y) {
    const float k1 = -2.0f * 0.7978845608028654f * 1.4426950408889634f, k2 = k1 * 0.044715f;
    f32x2 z = y * (y * y * k2 + k1);
    f32x2 e; e.x = __builtin_amdgcn_exp2f(z.x); e.y = __builtin_amdgcn_exp2f(z.y);
    e = e + 1.0f;
    f32x2 r; r.x = __builtin_amdgcn_rcpf(e.x); r.y = __builtin_amdgcn_rcpf(e.y);
    return y * r;
}
__device__ __forceinline__ unsigned pk_sub(unsigned a, unsigned b) { return __builtin_bit_cast(unsigned, __builtin_bit_cast(u16x2, a) - __builtin_bit_cast(u16x2, b)); }
__device__ __forceinline__ unsigned pk_add(unsigned a, unsigned b) { return __builtin_bit_cast(unsigned, __builtin_bit_cast(u16x2, a) + __builtin_bit_cast(u16x2, b)); }
__device__ __forceinline__ unsigned pk_max(unsigned a, unsigned b) { return __builtin_bit_cast(unsigned, __builtin_elementwise_max(__builtin_bit_cast(u16x2, a), __builtin_bit_cast(u16x2, b))); }
__device__ __forceinline__ unsigned bfi(unsigned m, unsigned a, unsigned b) { return __builtin_amdgcn_bitop3_b32(m, a, b, 0xca); }
__device__ __forceinline__ unsigned rotl(unsigned a, int n) { return __builtin_amdgcn_alignbit(a, a, 32 - n); }
__device__ __forceinline__ unsigned rotr(unsigned a, int n) { return __builtin_amdgcn_alignbit(a, a, n); }
__device__ __forceinline__ unsigned umax(unsigned a, unsigned b) { return a > b ? a : b; }

#define STAMP(slot) do {} while (0)

__device__ __forceinline__ bool give_up(unsigned* tmo, unsigned call, unsigned long long t0, unsigned& polls) {
    if ((++polls & 15u) != 0u) return false;
    if (__builtin_amdgcn_readfirstlane(__hip_atomic_load(tmo, __ATOMIC_RELAXED, __HIP_MEMORY_SCOPE_AGENT)) == call ||
        __builtin_amdgcn_s_memrealtime() - t0 > DEADLINE) {
        __hip_atomic_store(tmo, call, __ATOMIC_RELAXED, __HIP_MEMORY_SCOPE_AGENT);
        return true;
    }
    return false;
}
__device__ __forceinline__ unsigned wait_unit(const char* flag, unsigned call, unsigned st, unsigned* tmo) {
    const unsigned long long t0 = __builtin_amdgcn_s_memrealtime(); unsigned polls = 0;
    for (;;) {
        const u32x2 v = ld8_sys(flag);
        const unsigned hdr = __builtin_amdgcn_readfirstlane(v.y);
        if ((unsigned)__builtin_amdgcn_readfirstlane(v.x) == call && (hdr >> 28) >= st) return hdr;
        __builtin_amdgcn_s_sleep(1);
        if (give_up(tmo, call, t0, polls)) return (RAWCNT << 16) | (4u << 28);
    }
}
__device__ __forceinline__ unsigned hdr_cnt(unsigned hdr) { return (hdr >> 16) & 0x3ffu; }

constexpr int SCR_DENSE = 8192;
#define PT(i) do {} while (0)
__device__ __forceinline__ unsigned umin(unsigned a, unsigned b) { return a < b ? a : b; }
__device__ __forceinline__ unsigned wave_scan(unsigned inc) {
    inc += __builtin_amdgcn_update_dpp(0, (int)inc, 0x111, 0xf, 0xf, true);
    inc += __builtin_amdgcn_update_dpp(0, (int)inc, 0x112, 0xf, 0xf, true);
    inc += __builtin_amdgcn_update_dpp(0, (int)inc, 0x114, 0xf, 0xf, true);
    inc += __builtin_amdgcn_update_dpp(0, (int)inc, 0x118, 0xf, 0xf, true);
    inc += __builtin_amdgcn_update_dpp(0, (int)inc, 0x142, 0xa, 0xf, false);
    inc += __builtin_amdgcn_update_dpp(0, (int)inc, 0x143, 0xc, 0xf, false);
    return inc;
}
__device__ __forceinline__ unsigned pack_base(const u32x4 (&b)[16]) {
    unsigned mx = 0;
#pragma unroll
    for (int f = 0; f < 16; ++f) mx = pk_max(mx, b[f][f & 3] & 0x7fff7fffu);
    unsigned m = umax(mx & 0xffffu, mx >> 16);
    m = umax(m, (unsigned)__builtin_amdgcn_update_dpp(0, (int)m, 0x128, 0xf, 0xf, false));
    m = umax(m, (unsigned)__builtin_amdgcn_update_dpp(0, (int)m, 0x124, 0xf, 0xf, false));
    m = umax(m, (unsigned)__builtin_amdgcn_update_dpp(0, (int)m, 0x122, 0xf, 0xf, false));
    m = umax(m, (unsigned)__builtin_amdgcn_update_dpp(0, (int)m, 0x121, 0xf, 0xf, false));
    const unsigned m0 = __builtin_amdgcn_readlane(m, 0), m1 = __builtin_amdgcn_readlane(m, 16), m2 = __builtin_amdgcn_readlane(m, 32), m3 = __builtin_amdgcn_readlane(m, 48);
    const unsigned ms = umin(umax(umin(m0, m1), umin(m2, m3)), umin(umax(m0, m1), umax(m2, m3)));
    return umax(ms, 1023u) - 1023u;
}
__device__ __forceinline__ u32x4 pack_low(const u32x4 (&b)[16], int i, unsigned L) {
    const unsigned LL = L | (L << 16);
    u32x4 r;
#pragma unroll
    for (int c = 0; c < 4; ++c) r[c] = __builtin_amdgcn_perm(pk_sub(b[2 * i + (c >> 1)][2 * (c & 1) + 1], LL), pk_sub(b[2 * i + (c >> 1)][2 * (c & 1)], LL), 0x06040200u);
    return r;
}
__device__ __forceinline__ unsigned pack_unit(const u32x4 (&b)[16], u32x4 (&pk)[12], unsigned& tl, unsigned L, LAS unsigned char* scr, int lane, const char* early) {
    const unsigned LL = L | (L << 16);
    const unsigned wp0 = (unsigned)(size_t)scr + lane * 4;
    unsigned wp = wp0;
    unsigned z[4], seg[4], len[4], F = 0;
    PT(0);
#pragma unroll
    for (int s = 0; s < 4; ++s) {
        unsigned hb[8];
#pragma unroll
        for (int jj = 0; jj < 8; ++jj) {
            const int j = 8 * s + jj;
            const unsigned d0 = pk_sub(b[j >> 1][2 * (j & 1)], LL), d1 = pk_sub(b[j >> 1][2 * (j & 1) + 1], LL);
            pk[j >> 2][j & 3] = __builtin_amdgcn_perm(d1, d0, 0x06040200u);
            hb[jj] = __builtin_amdgcn_perm(d1, d0, 0x07050301u);
            const unsigned t = hb[jj] & 0x7c7c7c7cu; unsigned tmp;
            *(LAS unsigned*)(size_t)wp = t | (unsigned)((j & 3) | ((j >> 2) << 7));
            asm("v_min_u32 %0, 1, %2\n\tv_lshl_add_u32 %1, %0, 8, %1" : "=&v"(tmp), "+v"(wp) : "v"(t));
            if (jj & 3) hb[jj] = rotl(hb[jj], 2 * (jj & 3));
            if (j == EARLYJ && early) { WAIT_V(0); st16<-3072>(early, lane * 16, pk[1]); }
        }
        pk[8][s] = bfi(0x81818181u, hb[0], bfi(0x06060606u, hb[1], bfi(0x18181818u, hb[2], hb[3])));
        pk[9][s] = bfi(0x81818181u, hb[4], bfi(0x06060606u, hb[5], bfi(0x18181818u, hb[6], hb[7])));
        const unsigned ga = bfi(0x02020202u, hb[0], bfi(0x08080808u, hb[1], bfi(0x20202020u, hb[2], hb[3])));
        const unsigned gb = bfi(0x02020202u, hb[4], bfi(0x08080808u, hb[5], bfi(0x20202020u, hb[6], hb[7])));
        z[s] = ~bfi(0xaaaaaaaau, ga, gb >> 1);
    }
    PT(1);
    unsigned e8[8], nz4[4];
#pragma unroll
    for (int i = 0; i < 8; ++i) e8[i] = *(LAS unsigned*)(scr + i * 256 + lane * 4);
#pragma unroll
    for (int s = 0; s < 4; ++s) {
        unsigned t = z[s] | (z[s] >> 1); t |= t << 2;
        nz4[s] = t & 0x44444444u; F |= s < 2 ? nz4[s] >> (2 - s) : nz4[s] << (s - 2);
    }
    const unsigned nrow = (wp - wp0) >> 8, mine = (__builtin_popcount(F) << 14) | nrow;
    const unsigned inc = wave_scan(mine), tot = __builtin_amdgcn_readlane(inc, 63), ngrp = tot & 0xfffu;
    const unsigned lo = 256u + ((((tot >> 12) + 31u) >> 5) << 2);
    unsigned cnt = ngrp, bad = 0;
    PT(2);
    if (ngrp <= 512u) {
        LAS unsigned* dense = (LAS unsigned*)(scr + SCR_DENSE);
        const unsigned lanebits = ((lane & 7) << 15) | ((lane >> 3) << 23), base = (inc - mine) & 0xfffu;
#pragma unroll
        for (int i = 0; i < 8; ++i) dense[(unsigned)i < nrow ? base + i : 512 + lane] = e8[i] | lanebits;
        for (unsigned r = 8; __builtin_expect(__any(nrow > r), 0); ++r)
            dense[r < nrow ? base + r : 512 + lane] = *(LAS unsigned*)(scr + r * 256 + lane * 4) | lanebits;
        unsigned e2[2] = {dense[lane], dense[64 + lane]};
        *(LAS u32x4*)(scr + 256 + lane * 16) = (u32x4){0u, 0u, 0u, 0u};
        *(LAS unsigned*)(scr + lane * 4) = F;
        PT(3);
        if (early) { WAIT_V(0); st16<0>(early + 4096, lane * 16, pk[8]); }
#pragma unroll
        for (int s = 0; s < 4; ++s) { seg[s] = z[s] & 15u; len[s] = nz4[s] & 7u; }
#pragma unroll
        for (int i = 1; i < 8; ++i)
#pragma unroll
            for (int s = 0; s < 4; ++s) { seg[s] |= ((z[s] >> (4 * i)) & 15u) << len[s]; len[s] += (nz4[s] >> (4 * i)) & 7u; }
        PT(4);
        unsigned bit = (inc - mine) >> 12;
#pragma unroll
        for (int s = 0; s < 4; ++s) {
            const unsigned long long v = (unsigned long long)seg[s] << (bit & 31u);
            LAS unsigned* w = (LAS unsigned*)(scr + 256) + (bit >> 5);
            __hip_atomic_fetch_or(w, (unsigned)v, __ATOMIC_RELAXED, __HIP_MEMORY_SCOPE_WAVEFRONT);
            __hip_atomic_fetch_or(w + 1, (unsigned)(v >> 32), __ATOMIC_RELAXED, __HIP_MEMORY_SCOPE_WAVEFRONT);
            bit += len[s];
        }
        PT(5);
        if (early) { WAIT_V(0); st16<1024>(early + 4096, lane * 16, pk[9]); }
        LAS unsigned short* lst = (LAS unsigned short*)(scr + lo);
        const unsigned hz = ((0u - L) >> 10) & 31u;
        for (unsigned i0 = 0;;) {
            unsigned t[2], pos[2], at[2];
#pragma unroll
            for (int c = 0; c < 2; ++c) {
                const unsigned e = e2[c], ix = i0 + c * 64 + lane;
                t[c] = ix < ngrp ? e & 0x7c7c7c7cu : 0u;
                pos[c] = ((e & 0x380u) << 3) | ((e >> 16) & 0x380u) | ((e >> 11) & 0x70u) | ((e & 3u) << 2);
                at[c] = ix < ngrp ? ix : 1024u + lane;
            }
            for (;;) {
#pragma unroll
                for (int c = 0; c < 2; ++c) {
                    const unsigned b8 = __builtin_ctz(t[c] | 0x80000000u) & 24u, hi5 = (t[c] >> (b8 + 2)) & 31u;
                    const unsigned code = hi5 == hz ? 5u : (hi5 + 5u) & 31u;
                    bad = umax(bad, code);
                    lst[at[c]] = (unsigned short)((code << 13) | pos[c] | (b8 >> 3));
                    t[c] &= ~(0xffu << b8);
                }
                const unsigned long long b0 = __ballot(t[0] != 0), b1 = __ballot(t[1] != 0);
                if (!(b0 | b1)) break;
                at[0] = t[0] != 0 ? __builtin_amdgcn_mbcnt_hi((unsigned)(b0 >> 32), __builtin_amdgcn_mbcnt_lo((unsigned)b0, umin(cnt, 512u))) : 1024u + lane;
                cnt += __builtin_popcountll(b0);
                at[1] = t[1] != 0 ? __builtin_amdgcn_mbcnt_hi((unsigned)(b1 >> 32), __builtin_amdgcn_mbcnt_lo((unsigned)b1, umin(cnt, 512u))) : 1024u + lane;
                cnt += __builtin_popcountll(b1);
            }
            i0 += 128;
            if (__builtin_expect(i0 >= ngrp, 1)) break;
            e2[0] = dense[i0 + lane]; e2[1] = dense[i0 + 64 + lane];
        }
    }
    PT(6);
    pk[10] = *(LAS u32x4*)(scr + lane * 16);
    pk[11] = *(LAS u32x4*)(scr + 1024 + lane * 16);
    tl = (lo + 2u * cnt + 31u) >> 5;
    if (cnt > 512u || tl > 64u || __any(bad > 7u) || FORCE_RAW) cnt = RAWCNT;
    return L | (cnt << 16) | (tl > 32u ? 1u << 26 : 0u);
}
__device__ __forceinline__ void st_tail(const char* sb8k  , unsigned voff, u32x4 t0, u32x4 t1, unsigned tl, int lane) {
    if (__builtin_expect(tl > 32u, 0)) { if ((unsigned)lane < (tl - 32u) * 2u) st16<3072>(sb8k, voff, t1); }
    if ((unsigned)lane < tl * 2u) st16<2048>(sb8k, voff, t0);
}
__device__ __forceinline__ void st_raw(const char* slot, unsigned voff, const u32x4 (&b)[16]) {
    const char* s0 = slot + 4096; const char* s1 = slot + 12288;
    st16<-4096>(s0, voff, b[0]); st16<-3072>(s0, voff, b[1]); st16<-2048>(s0, voff, b[2]); st16<-1024>(s0, voff, b[3]);
    st16<0>(s0, voff, b[4]); st16<1024>(s0, voff, b[5]); st16<2048>(s0, voff, b[6]); st16<3072>(s0, voff, b[7]);
    st16<-4096>(s1, voff, b[8]); st16<-3072>(s1, voff, b[9]); st16<-2048>(s1, voff, b[10]); st16<-1024>(s1, voff, b[11]);
    st16<0>(s1, voff, b[12]); st16<1024>(s1, voff, b[13]); st16<2048>(s1, voff, b[14]); st16<3072>(s1, voff, b[15]);
}
constexpr int DEC_TAIL = 8448;
__device__ __forceinline__ void ld_tail2(const char* slot, unsigned voff, unsigned ldst) {
    unsigned keep;
    asm volatile("s_mov_b32 %0, m0\n\ts_mov_b32 m0, %3\n\ts_nop 0\n\tglobal_load_lds_dwordx4 %1, %2 sc0 sc1\n\ts_mov_b32 m0, %0"
                 : "=&s"(keep) : "v"(voff), "s"(slot + 11264), "s"(ldst + 1024) : "memory");
}
__device__ __forceinline__ void ld_packed(u32x4 (&r)[10], unsigned& F, const char* slot, unsigned voff, unsigned ldst) {
    const char* s0 = slot + 4096; const char* s1 = slot + 10240;
    unsigned keep;
    asm volatile("s_mov_b32 %11, m0\n\ts_mov_b32 m0, %16\n\t"
                 "global_load_dword %10, %13, %15 sc0 sc1\n\tglobal_load_lds_dwordx4 %12, %15 sc0 sc1\n\t"
                 "global_load_dwordx4 %8, %12, %15 offset:-2048 sc0 sc1\n\tglobal_load_dwordx4 %9, %12, %15 offset:-1024 sc0 sc1\n\t"
                 "global_load_dwordx4 %0, %12, %14 offset:-4096 sc0 sc1\n\tglobal_load_dwordx4 %1, %12, %14 offset:-3072 sc0 sc1\n\t"
                 "global_load_dwordx4 %2, %12, %14 offset:-2048 sc0 sc1\n\tglobal_load_dwordx4 %3, %12, %14 offset:-1024 sc0 sc1\n\t"
                 "global_load_dwordx4 %4, %12, %14 sc0 sc1\n\tglobal_load_dwordx4 %5, %12, %14 offset:1024 sc0 sc1\n\t"
                 "global_load_dwordx4 %6, %12, %14 offset:2048 sc0 sc1\n\tglobal_load_dwordx4 %7, %12, %14 offset:3072 sc0 sc1\n\t"
                 "s_mov_b32 m0, %11\n\ts_waitcnt vmcnt(0)"
                 : "=&v"(r[0]), "=&v"(r[1]), "=&v"(r[2]), "=&v"(r[3]), "=&v"(r[4]), "=&v"(r[5]), "=&v"(r[6]), "=&v"(r[7]),
                   "=&v"(r[8]), "=&v"(r[9]), "=&v"(F), "=&s"(keep)
                 : "v"(voff), "v"(voff >> 2), "s"(s0), "s"(s1), "s"(ldst) : "memory");
}
__device__ __forceinline__ void ld_head(u32x4 (&r)[10], unsigned& F, unsigned& t, const char* slot, unsigned voff, unsigned ldst, const unsigned* tp) {
    const char* s1 = slot + 10240;
    unsigned keep;
    asm volatile("s_mov_b32 %6, m0\n\ts_mov_b32 m0, %11\n\t"
                 "global_load_dword %4, %8, %10 sc0 sc1\n\tglobal_load_lds_dwordx4 %7, %10 sc0 sc1\n\t"
                 "global_load_dwordx4 %2, %7, %10 offset:-2048 sc0 sc1\n\tglobal_load_dwordx4 %3, %7, %10 offset:-1024 sc0 sc1\n\t"
                 "global_load_dwordx4 %0, %7, %9 sc0 sc1\n\tglobal_load_dwordx4 %1, %7, %9 offset:1024 sc0 sc1\n\t"
                 "global_load_dword %5, %12, off sc1\n\ts_mov_b32 m0, %6\n\ts_waitcnt vmcnt(0)"
                 : "=&v"(r[0]), "=&v"(r[1]), "=&v"(r[8]), "=&v"(r[9]), "=&v"(F), "=&v"(t), "=&s"(keep)
                 : "v"(voff), "v"(voff >> 2), "s"(slot), "s"(s1), "s"(ldst), "v"(tp) : "memory");
}
__device__ __forceinline__ void ld4x16(u32x4& a, u32x4& b, u32x4& c, u32x4& d, const char* sb, unsigned voff) {
    asm volatile("global_load_dwordx4 %0, %4, %5 sc0 sc1\n\tglobal_load_dwordx4 %1, %4, %5 offset:1024 sc0 sc1\n\t"
                 "global_load_dwordx4 %2, %4, %5 offset:2048 sc0 sc1\n\tglobal_load_dwordx4 %3, %4, %5 offset:3072 sc0 sc1\n\ts_waitcnt vmcnt(0)"
                 : "=&v"(a), "=&v"(b), "=&v"(c), "=&v"(d) : "v"(voff), "s"(sb) : "memory");
}
__device__ __forceinline__ void ld2x16t(u32x4& a, u32x4& b, unsigned& t, const char* sb, unsigned voff, const unsigned* tp) {
    asm volatile("global_load_dwordx4 %0, %3, %4 sc0 sc1\n\tglobal_load_dwordx4 %1, %3, %4 offset:1024 sc0 sc1\n\t"
                 "global_load_dword %2, %5, off sc1\n\ts_waitcnt vmcnt(0)"
                 : "=&v"(a), "=&v"(b), "=&v"(t) : "v"(voff), "s"(sb), "v"(tp) : "memory");
}
__device__ __forceinline__ void patch_apply(LAS unsigned char* dl, unsigned lo, unsigned hdr, int lane, bool zero) {
    const unsigned cnt = hdr_cnt(hdr), lut_hi = 0x0804007cu | ((((0u - (hdr & 0x7fffu)) >> 10) & 31u) << 10);
    for (unsigned c = 0; c < cnt; c += 128) {
        const unsigned w = *(LAS unsigned*)(dl + DEC_TAIL + lo + c * 2 + lane * 4);
#pragma unroll
        for (int hf = 0; hf < 2; ++hf) {
            const unsigned e = hf ? w >> 16 : w & 0xffffu;
            const unsigned at = c + lane * 2 + hf < cnt ? e & 0x1fffu : 8192u + lane;
            dl[at] = zero ? 0 : (unsigned char)__builtin_amdgcn_perm(lut_hi, 0x7874706cu, e >> 13);
        }
    }
}
__device__ __forceinline__ unsigned unpack_head(unsigned F, unsigned hdr, LAS unsigned char* dl, int lane, u32x4& G) {
    PT(8);
    const unsigned mine = __builtin_popcount(F) << 2, inc = wave_scan(mine);
    const unsigned lo = 256u + ((((unsigned)__builtin_amdgcn_readlane(inc, 63) + 31u) >> 5) << 2);
    unsigned bit = inc - mine, sg[4];
#pragma unroll
    for (int s = 0; s < 4; ++s) {
        const LAS unsigned* w = (const LAS unsigned*)(dl + DEC_TAIL + 256) + (bit >> 5);
        sg[s] = __builtin_amdgcn_alignbit(w[1], w[0], bit);
        bit += __builtin_popcount(F & (0x11111111u << s)) << 2;
    }
    PT(9);
    patch_apply(dl, lo, hdr, lane, false);
    PT(10);
#pragma unroll
    for (int s = 0; s < 4; ++s) {
        const unsigned nz = (F >> s) & 0x11111111u, pre = nz * 0x11111110u;
        const unsigned xe = sg[s] & 0x0f0f0f0fu, xo = (sg[s] >> 4) & 0x0f0f0f0fu;
        const unsigned b0 = __builtin_amdgcn_perm(xo, xe, 0x05010400u), b1 = __builtin_amdgcn_perm(xo, xe, 0x07030602u);
        const unsigned ze = __builtin_amdgcn_perm(b1, b0, pre & 0x07070707u), zo = __builtin_amdgcn_perm(b1, b0, (pre >> 4) & 0x07070707u);
        G[s] = ~((ze | (zo << 4)) & ((nz << 4) - nz));
    }
    PT(11);
    return lo;
}
__device__ __forceinline__ void unpack_sub(const u32x4 (&r)[10], u32x4 G, int s, LAS unsigned char* dl, int lane, unsigned LL, u32x4 (&o)[4]) {
    const u32x4 p0 = *(LAS u32x4*)(dl + (2 * s) * 1024 + lane * 16), p1 = *(LAS u32x4*)(dl + (2 * s + 1) * 1024 + lane * 16);
    const unsigned A = r[8][s], B = r[9][s], ga = G[s], gb = G[s] << 1;
    unsigned hb[8];
    hb[0] = bfi(0x83838383u, bfi(0x02020202u, ga, A), p0[0]); hb[1] = bfi(0x83838383u, rotr(bfi(0x08080808u, ga, A), 2), p0[1]);
    hb[2] = bfi(0x83838383u, rotr(bfi(0x20202020u, ga, A), 4), p0[2]); hb[3] = bfi(0x83838383u, rotr(bfi(0x80808080u, ga, A), 6), p0[3]);
    hb[4] = bfi(0x83838383u, bfi(0x02020202u, gb, B), p1[0]); hb[5] = bfi(0x83838383u, rotr(bfi(0x08080808u, gb, B), 2), p1[1]);
    hb[6] = bfi(0x83838383u, rotr(bfi(0x20202020u, gb, B), 4), p1[2]); hb[7] = bfi(0x83838383u, rotr(bfi(0x80808080u, gb, B), 6), p1[3]);
#pragma unroll
    for (int jj = 0; jj < 8; ++jj) {
        const unsigned lo = r[2 * s + (jj >> 2)][jj & 3];
        o[jj >> 1][2 * (jj & 1)] = pk_add(__builtin_amdgcn_perm(hb[jj], lo, 0x05010400u), LL);
        o[jj >> 1][2 * (jj & 1) + 1] = pk_add(__builtin_amdgcn_perm(hb[jj], lo, 0x07030602u), LL);
    }
}

__device__ __forceinline__ void unit_fetch(const Params& p, int h, int u, int lane, unsigned ldsbuf) {
    const bool isx = u < 16;
    const char* base = isx ? p.x + ((size_t)u * 256 * KS + h * 32) * 2 : p.w + ((size_t)h * 32 * NN + (u - 16) * 256) * 2;
    unsigned voff = isx ? (lane & 15) * (KS * 2) + (lane >> 4) * 16 : (lane >> 4) * (8 * NN * 2) + (lane & 15) * 16;
    const unsigned s1 = isx ? 16 * KS * 2 : NN * 2, s2 = isx ? 16 * KS * 2 : 256 - 7 * NN * 2;
#pragma unroll
    for (int i = 0; i < 16; ++i) {
        unsigned keep;
        asm volatile("s_mov_b32 %0, m0\n\ts_mov_b32 m0, %3\n\ts_nop 0\n\tglobal_load_lds_dwordx4 %1, %2\n\ts_mov_b32 m0, %0"
                     : "=&s"(keep) : "v"(voff), "s"(base), "s"(ldsbuf + i * 1024) : "memory");
        voff += (i == 7) ? s2 : s1;
    }
}
__device__ __forceinline__ void unit_read(LAS unsigned char* buf, int lane, u32x4 (&r)[16]) {
#pragma unroll
    for (int i = 0; i < 16; ++i) r[i] = *(LAS u32x4*)(buf + i * 1024 + lane * 16);
    asm volatile("s_waitcnt lgkmcnt(0)" : "+v"(r[0]), "+v"(r[1]), "+v"(r[2]), "+v"(r[3]), "+v"(r[4]), "+v"(r[5]), "+v"(r[6]), "+v"(r[7]),
                 "+v"(r[8]), "+v"(r[9]), "+v"(r[10]), "+v"(r[11]), "+v"(r[12]), "+v"(r[13]), "+v"(r[14]), "+v"(r[15]) :: "memory");
}
__device__ __forceinline__ void unit_format(int u, u32x4 (&r)[16]) {
    if (u >= 16) {
#pragma unroll
        for (int g = 0; g < 2; ++g) {
            u32x4 o[8];
#pragma unroll
            for (int i = 0; i < 8; ++i)
#pragma unroll
                for (int a = 0; a < 4; ++a)
                    o[i][a] = __builtin_amdgcn_perm(r[g * 8 + 2 * a + 1][i >> 1], r[g * 8 + 2 * a][i >> 1], (i & 1) ? 0x07060302u : 0x05040100u);
#pragma unroll
            for (int i = 0; i < 8; ++i) r[g * 8 + i] = o[i];
        }
    }
}
struct OrderTab { unsigned e[NU]; };
constexpr OrderTab make_order() {
    OrderTab t{}; int n = 0;
    for (int v = 0; v < 8 * NW0; ++v) t.e[n++] = (v >> 3) << 8 | (v & 7);
    for (int key = 0; key < 8 * NH + 512; ++key)
        for (int v = 0; v < 22; ++v) {
            const int lag = v < 8 ? LAGW0 + (v & 3) * LAGWS + ((v & 2) ? LAGWH : 0) : v < 15 ? LAGXA : LAGXB;
            const int d = key - lag;
            if (d >= 0 && d % 8 == 0 && d / 8 < NH && !(v < 8 && d / 8 < NW0)) t.e[n++] = (unsigned)((d / 8) << 8 | v);
        }
    return t;
}
__constant__ const OrderTab g_order = make_order();
__device__ __forceinline__ int order(int k, int src) {
    const int e = g_order.e[k], v = e & 255;
    const int odd = v >= 15, m = (src + 1 + v - (odd ? 15 : 8)) & 7;
    return (e & ~255) | (v < 8 ? 16 + v : 2 * m + odd);
}

constexpr int PK_LDS = 28 * 1024;
struct SendCtl { u32x4 desc[16]; unsigned freed[6]; unsigned go; unsigned tail; };
__device__ __forceinline__ void packer(const Params& p, LAS unsigned char* buf, LAS SendCtl* ctl, int xcd, int di, int mem, int lane, unsigned* cntp) {
    const unsigned dest = (unsigned)(p.rank + 1 + di) & 7u;
    const unsigned voff = lane * 16, lbuf = (unsigned)(size_t)buf;
    unsigned k = (RAW0 + mem) * 8 + xcd;
    int o, h = RAW0 + mem, u = 16 + dest * 8 + xcd;
    const unsigned call = __builtin_amdgcn_readfirstlane(ld4_sys(cntp)) + 1u;
    unsigned* tk = (unsigned*)(p.ws[p.rank] + OFF_TK) + di * 32;
    const unsigned tk0 = (call - 1u) * (NU - 8 * RAW0) - (RAW0 + NPK) * 8;
    unsigned nput = 0;
    for (;;) {
        WAIT_V(0);
        u32x4 r[16], pk[12];
        unit_read(buf, lane, r);
        unit_format(u, r);
        unsigned tl;
        const unsigned hdr = pack_unit(r, pk, tl, pack_base(r), buf, lane, nullptr) | (4u << 28);
        asm volatile("" : "+v"(pk[0]), "+v"(pk[1]), "+v"(pk[2]), "+v"(pk[3]), "+v"(pk[4]), "+v"(pk[5]), "+v"(pk[6]), "+v"(pk[7]), "+v"(pk[8]), "+v"(pk[9]), "+v"(pk[10]), "+v"(pk[11]));
        while ((unsigned)__builtin_amdgcn_readfirstlane(*(volatile LAS unsigned*)&ctl->freed[mem]) != nput) __builtin_amdgcn_s_sleep(1);
        ++nput;
        if (__builtin_expect(hdr_cnt(hdr) == RAWCNT, 0)) {
            unit_fetch(p, h, u, lane, lbuf);
            WAIT_V(0);
            unit_read(buf, lane, r);
            unit_format(u, r);
            st_raw(p.ws[dest] + OFF_LZ + (call & 1u) * LZ_PAR + (size_t)((6 - di) * NU + k) * UB, voff, r);
            WAIT_V(0);
        } else {
#pragma unroll
            for (int i = 0; i < 12; ++i) *(LAS u32x4*)(buf + 16384 + i * 1024 + lane * 16) = pk[i];
        }
        asm volatile("" ::: "memory");
        unsigned tn = 1u;
        if (lane == 0) {
            const unsigned pos = __hip_atomic_fetch_add(&ctl->tail, 1u, __ATOMIC_RELAXED, __HIP_MEMORY_SCOPE_WORKGROUP);
            *(volatile LAS u32x4*)&ctl->desc[pos & 15] = (u32x4){pos + 1u, k, hdr, (unsigned)mem | (nput << 8) | (tl << 24)};
        }
        if (SHALLOW) while ((unsigned)__builtin_amdgcn_readfirstlane(*(volatile LAS unsigned*)&ctl->freed[mem]) != nput) __builtin_amdgcn_s_sleep(1);
        if (lane == 0) asm volatile("global_atomic_add %0, %1, %0, off sc0\n\ts_waitcnt vmcnt(0)" : "+v"(tn) : "v"(tk) : "memory");
        k = __builtin_amdgcn_readfirstlane(tn) - tk0;
        if (k >= (unsigned)NU) break;
        o = order(k, p.rank); h = o >> 8; u = (o & 255) + ((o & 16) ? dest * 8 : 0);
        unit_fetch(p, h, u, lane, lbuf);
    }
    if (lane == 0) {
        const unsigned pos = __hip_atomic_fetch_add(&ctl->tail, 1u, __ATOMIC_RELAXED, __HIP_MEMORY_SCOPE_WORKGROUP);
        *(volatile LAS u32x4*)&ctl->desc[pos & 15] = (u32x4){pos + 1u, 0u, 0u, 0x80u};
    }
}
__device__ __forceinline__ void storer(const Params& p, LAS unsigned char* lds, LAS SendCtl* ctl, int xcd, int di, int lane, unsigned* cntp) {
    const unsigned dest = (unsigned)(p.rank + 1 + di) & 7u;
    char* dws = p.ws[dest];
    const int li = 6 - di;
    const unsigned voff = lane * 16;
    WAIT_V(16);
    const unsigned call = __builtin_amdgcn_readfirstlane(*(volatile LAS unsigned*)((LAS unsigned char*)ctl + 512)) + 1u;
    const char* lz = dws + OFF_LZ + (call & 1u) * LZ_PAR + (size_t)li * NU * UB;
    char* fa = dws + OFF_FA + (size_t)((call & 1u) * 7 + li) * NU * FLS;
    unsigned pos = 0, done = 0, pend_hdr = 0, idle = 0; int pend = -1;
#define POST(kk, hh) st8_flag(fa + (kk) * FLS, call, (hh), lane)
#define PEND_FLAG(i) do { if (SWIN == (i) && pend >= 0) POST(pend, pend_hdr); } while (0)
    for (int i = 0; i < RAW0; ++i) {
        u32x4 r[16];
        unit_read(lds + (NPK - i) * PK_LDS, lane, r);
        if (i) *(volatile LAS unsigned*)&ctl->go = 1u;
        unit_format(16, r);
        const char* s0 = lz + (size_t)(i * 8 + xcd) * UB + 4096; const char* s1 = s0 + 8192;
        if (i == 0) { st16<-4096>(s0, voff, r[0]); st16<-3072>(s0, voff, r[1]); st16<-2048>(s0, voff, r[2]); st16<-1024>(s0, voff, r[3]); st16<0>(s0, voff, r[4]); st16<1024>(s0, voff, r[5]); }
        else { WAIT_V(SWIN); st16<-4096>(s0, voff, r[0]); WAIT_V(SWIN); st16<-3072>(s0, voff, r[1]); WAIT_V(SWIN); st16<-2048>(s0, voff, r[2]);
               WAIT_V(SWIN); st16<-1024>(s0, voff, r[3]); WAIT_V(SWIN); st16<0>(s0, voff, r[4]); WAIT_V(SWIN); st16<1024>(s0, voff, r[5]); }
        WAIT_V(SWIN); st16<2048>(s0, voff, r[6]); WAIT_V(SWIN); st16<3072>(s0, voff, r[7]);
        WAIT_V(SWIN); if (pend >= 0) POST(pend, pend_hdr); static_assert(SWIN <= 8, "window");
        st16<-4096>(s1, voff, r[8]); WAIT_V(SWIN); st16<-3072>(s1, voff, r[9]); WAIT_V(SWIN); st16<-2048>(s1, voff, r[10]); WAIT_V(SWIN); st16<-1024>(s1, voff, r[11]); WAIT_V(SWIN);
        st16<0>(s1, voff, r[12]); WAIT_V(SWIN); st16<1024>(s1, voff, r[13]); WAIT_V(SWIN); st16<2048>(s1, voff, r[14]); WAIT_V(SWIN); st16<3072>(s1, voff, r[15]);
        pend = i * 8 + xcd; pend_hdr = (RAWCNT << 16) | (4u << 28);
    }
    STAMP(24);
    while (done < NPK) {
        const u32x4 d = *(volatile LAS u32x4*)&ctl->desc[pos & 15];
        if ((unsigned)__builtin_amdgcn_readfirstlane(d.x) != pos + 1u) {
            if (pend < 0) __builtin_amdgcn_s_sleep(1);
            else if (idle == 0) WAIT_V(4);
            else if (idle == 1) WAIT_V(2);
            else { WAIT_V(0); POST(pend, pend_hdr); pend = -1; }
            ++idle;
            continue;
        }
        idle = 0;
        const unsigned k = __builtin_amdgcn_readfirstlane(d.y), hdr = __builtin_amdgcn_readfirstlane(d.z), w = __builtin_amdgcn_readfirstlane(d.w);
        ++pos;
        if (w & 0x80u) { ++done; continue; }
        if (k == NU - 1 - GATE_EARLY && lane < 8)
            __hip_atomic_store((unsigned*)(p.ws[p.rank] + OFF_SD) + (di * 8 + lane) * 32, call, __ATOMIC_RELAXED, __HIP_MEMORY_SCOPE_AGENT);
        if (__builtin_expect(hdr_cnt(hdr) == RAWCNT, 0)) {
            POST(k, hdr);
            *(volatile LAS unsigned*)&ctl->freed[w & 7u] = (w >> 8) & 0xffffu;
            continue;
        }
        u32x4 pk[12];
        LAS unsigned char* src = lds + (w & 7u) * PK_LDS + 16384 + lane * 16;
#pragma unroll
        for (int i = 0; i < 12; ++i) pk[i] = *(LAS u32x4*)(src + i * 1024);
        asm volatile("s_waitcnt lgkmcnt(0)" : "+v"(pk[0]), "+v"(pk[1]), "+v"(pk[2]), "+v"(pk[3]), "+v"(pk[4]), "+v"(pk[5]), "+v"(pk[6]), "+v"(pk[7]), "+v"(pk[8]), "+v"(pk[9]), "+v"(pk[10]), "+v"(pk[11]) :: "memory");
        *(volatile LAS unsigned*)&ctl->freed[w & 7u] = (w >> 8) & 0xffffu;
        const char* s0 = lz + (size_t)k * UB + 4096; const char* s1 = s0 + 4096;
        WAIT_V(SWIN); st16<-4096>(s0, voff, pk[0]);
        WAIT_V(SWIN); st16<-3072>(s0, voff, pk[1]);
        WAIT_V(SWIN); st16<-2048>(s0, voff, pk[2]);
        WAIT_V(SWIN); PEND_FLAG(3); st16<-1024>(s0, voff, pk[3]);
        WAIT_V(SWIN); PEND_FLAG(4); st16<0>(s0, voff, pk[4]);
        WAIT_V(SWIN); PEND_FLAG(5); st16<1024>(s0, voff, pk[5]);
        WAIT_V(SWIN); PEND_FLAG(6); st16<2048>(s0, voff, pk[6]);
        WAIT_V(SWIN); PEND_FLAG(7); st16<3072>(s0, voff, pk[7]);
        WAIT_V(SWIN); PEND_FLAG(8); st16<0>(s1, voff, pk[8]);
        WAIT_V(SWIN1); st16<1024>(s1, voff, pk[9]);
        WAIT_V(SWIN2); st_tail(s1, voff, pk[10], pk[11], w >> 24, lane);
        pend = k; pend_hdr = hdr;
    }
    WAIT_V(0);
    if (pend >= 0) st8_flag(fa + pend * FLS, call, pend_hdr, lane);
}

__device__ __forceinline__ void unpacker(const Params& p, LAS unsigned char* dl, int uw, int lane, unsigned call) {
    char* ws = p.ws[p.rank];
    unsigned* tmo = (unsigned*)(ws + OFF_TMO);
    const unsigned voff = lane * 16;
#pragma unroll
    for (int i = 0; i < 8; ++i) *(LAS u32x4*)(dl + i * 1024 + lane * 16) = (u32x4){0u, 0u, 0u, 0u};
    for (int n = uw; n < 7 * NU; n += NUNP) {
        const int k = n / 7, li = n - k * 7, src = (p.rank + 1 + li) & 7;
        const int o = order(k, src), t = o & 15; const bool isx = !(o & 16);
        const int q = NH + (o >> 8) * 7 + li;
        const unsigned hdr = wait_unit(ws + OFF_FA + (size_t)(((call & 1u) * 7 + li) * NU + k) * FLS, call, 4u, tmo);
        const unsigned cnt = hdr_cnt(hdr); const bool raw = cnt == RAWCNT;
        unsigned lo = 0;
        const char* sl = ws + OFF_LZ + (call & 1u) * LZ_PAR + (size_t)(li * NU + k) * UB;
        const char* dst = ws + (isx ? OFF_XI + (size_t)q * XQ : OFF_WI + (size_t)q * WQ) + (size_t)t * UB;
        unsigned* flag = (unsigned*)(ws + (isx ? OFF_FX : OFF_FW)) + (isx ? t : p.rank * 8 + t) * FSTR + q;
        if (__builtin_expect(raw, 0)) {
            for (int s = 0; s < 4; ++s) {
                u32x4 o[4]; ld4x16(o[0], o[1], o[2], o[3], sl + s * 4096, voff);
                const char* sb = dst + s * 4096;
                st16<0>(sb, voff, o[0]); st16<1024>(sb, voff, o[1]); st16<2048>(sb, voff, o[2]); st16<3072>(sb, voff, o[3]);
            }
        } else {
            u32x4 r[10], G; unsigned F;
            if (hdr & (1u << 26)) ld_tail2(sl, voff, (unsigned)(size_t)(dl + DEC_TAIL));
            ld_packed(r, F, sl, voff, (unsigned)(size_t)(dl + DEC_TAIL));
            const unsigned LL = (hdr & 0x7fffu) * 0x10001u;
            lo = unpack_head(F, hdr, dl, lane, G);
#pragma unroll
            for (int s = 0; s < 4; ++s) {
                u32x4 o[4]; unpack_sub(r, G, s, dl, lane, LL, o);
                const char* sb = dst + s * 4096;
                st16<0>(sb, voff, o[0]); st16<1024>(sb, voff, o[1]); st16<2048>(sb, voff, o[2]); st16<3072>(sb, voff, o[3]);
            }
        }
        WAIT_V(0);
        if (lane == 0) __hip_atomic_store(flag, call, __ATOMIC_RELAXED, __HIP_MEMORY_SCOPE_SYSTEM);
        if (!raw) { patch_apply(dl, lo, hdr, lane, true); WAIT_L(0); }
    }
}

__device__ __forceinline__ void formatter(const Params& p, LAS unsigned char* buf, int fw, int n2nd, int lane, unsigned call) {
    char* ws = p.ws[p.rank];
    const unsigned voff = lane * 16;
    const unsigned long long ts = __builtin_amdgcn_s_memrealtime();
    for (int n = fw < 384 ? fw : n2nd; n < 1280; n = (n < 384) ? n2nd : n + NFMT) {
        const unsigned at = n < 384 ? PD0 + n * PDT / 16 : FD0 + (n - 384) * FDT / 16;
        while (__builtin_amdgcn_s_memrealtime() - ts < at) __builtin_amdgcn_s_sleep(4);
        int h, u; const char* dst; unsigned* flag;
        if (n < 384) { h = n / 24; const int v = n - h * 24;
            if (v < 16) { u = v; dst = ws + OFF_XI + (size_t)h * XQ + (size_t)v * UB; flag = (unsigned*)(ws + OFF_FX) + v * FSTR + h; }
            else { u = p.rank * 8 + v; dst = ws + OFF_WI + (size_t)h * WQ + (size_t)(v - 16) * UB; flag = (unsigned*)(ws + OFF_FW) + (u - 16) * FSTR + h; } }
        else { const int n2 = n - 384; h = n2 / 56; const int v = n2 - h * 56, li = v >> 3, cn = ((p.rank + 1 + li) & 7) * 8 + (v & 7);
            u = 16 + cn; dst = ws + OFF_WD + (size_t)(li * NH + h) * WQ + (size_t)(v & 7) * UB; flag = (unsigned*)(ws + OFF_FW) + cn * FSTR + h; }
        u32x4 r[16];
        unit_fetch(p, h, u, lane, (unsigned)(size_t)buf);
        WAIT_V(0);
        unit_read(buf, lane, r);
        unit_format(u, r);
        st_raw(dst, voff, r);
        WAIT_V(0);
        if (lane == 0) __hip_atomic_store(flag, call, __ATOMIC_RELAXED, __HIP_MEMORY_SCOPE_SYSTEM);
    }
}

__device__ __forceinline__ void post_ready(bool r0, bool r1, unsigned long long& td0, unsigned long long& td1, LAS unsigned* queue, int& nq, unsigned lane) {
    const unsigned long long a0 = __ballot(r0) & td0, a1 = __ballot(r1) & td1;
    if ((a0 >> lane) & 1) ((volatile LAS unsigned*)queue)[__builtin_amdgcn_mbcnt_hi((unsigned)(a0 >> 32), __builtin_amdgcn_mbcnt_lo((unsigned)a0, nq))] = lane | 0x10000u;
    nq += __builtin_popcountll(a0);
    if ((a1 >> lane) & 1) ((volatile LAS unsigned*)queue)[__builtin_amdgcn_mbcnt_hi((unsigned)(a1 >> 32), __builtin_amdgcn_mbcnt_lo((unsigned)a1, nq))] = lane | 0x10040u;
    nq += __builtin_popcountll(a1);
    td0 &= ~a0; td1 &= ~a1;
}
__device__ __forceinline__ void poll_ready(const unsigned* fx, const unsigned* fw, unsigned long long& td0, unsigned long long& td1,
                                           LAS unsigned* queue, int& nq, int need, unsigned call, unsigned* tmo) {
    const unsigned lane = __builtin_amdgcn_mbcnt_hi(~0u, __builtin_amdgcn_mbcnt_lo(~0u, 0u));
    {
        const volatile LAS unsigned* sn = queue + 256 + lane;
        const unsigned x0 = sn[0], x1 = sn[64], w0 = sn[128], w1 = sn[192];
        post_ready(x0 == call && w0 == call, x1 == call && w1 == call, td0, td1, queue, nq, lane);
    }
    const unsigned long long t0 = __builtin_amdgcn_s_memrealtime(); unsigned polls = 0;
    bool all = false;
    while (nq < need) {
        unsigned x0, x1, w0, w1;
        asm volatile("global_load_dword %0, %4, %5 sc0 sc1\n\tglobal_load_dword %1, %4, %5 offset:256 sc0 sc1\n\t"
                     "global_load_dword %2, %4, %6 sc0 sc1\n\tglobal_load_dword %3, %4, %6 offset:256 sc0 sc1\n\ts_waitcnt vmcnt(0)"
                     : "=&v"(x0), "=&v"(x1), "=&v"(w0), "=&v"(w1) : "v"(lane * 4), "s"(fx), "s"(fw) : "memory");
        post_ready(all || (x0 >= call && w0 >= call), all || (x1 >= call && w1 >= call), td0, td1, queue, nq, lane);
        if (nq >= need) break;
        all = give_up(tmo, call, t0, polls);
        __builtin_amdgcn_s_sleep(1);
    }
    WAIT_L(0);
}

constexpr int HTB = 16384;
constexpr int LDS_CTL = (NPK * PK_LDS + 16384 > 8 * HTB + 8192) ? NPK * PK_LDS + 16384 : 8 * HTB + 8192;

__global__ __launch_bounds__(512, 2) void fused_kernel(const Params p) {
    __shared__ __attribute__((aligned(16))) unsigned char shm[LDS_CTL + 768];
    LAS unsigned char* lds = (LAS unsigned char*)shm;
    const int wid = __builtin_amdgcn_readfirstlane(threadIdx.x >> 6), lane = threadIdx.x & 63;
    const int xcd = blockIdx.x & 7, slot = blockIdx.x >> 3;
    char* ws = p.ws[p.rank];
    unsigned* tmo = (unsigned*)(ws + OFF_TMO);
    unsigned* cntp = (unsigned*)(ws + OFF_CNT) + blockIdx.x * 8 + wid;
    STAMP(0);
    const bool main_tile = slot >= 16;
    if (slot < 7) {
        LAS SendCtl* ctl = (LAS SendCtl*)(lds + LDS_CTL);
        static_assert(RAW0 == 2 && NPK <= 5 && NW0 >= RAW0 + NPK, "opening units");
        const int u0 = 16 + ((p.rank + 1 + slot) & 7) * 8 + xcd;
        if (wid >= 6) ((volatile LAS unsigned*)ctl)[(wid - 6) * 64 + lane] = 0u;
        if (wid == 0) {
            __builtin_amdgcn_s_setprio(3);
            unsigned keep;
            asm volatile("s_mov_b32 %0, m0\n\ts_mov_b32 m0, %2\n\ts_nop 0\n\tglobal_load_lds_dword %1, off sc0 sc1\n\ts_mov_b32 m0, %0"
                         : "=&s"(keep) : "v"(cntp), "s"((unsigned)(size_t)(lds + LDS_CTL + 512)) : "memory");
            unit_fetch(p, 0, u0, lane, (unsigned)(size_t)(lds + NPK * PK_LDS));
        } else { WAIT_L(0); BAR; }
        for (int i = 1; i < wid && wid < NPK; ++i) __builtin_amdgcn_s_sleep(PKST);
        if (wid == NPK) while (__builtin_amdgcn_readfirstlane(*(volatile LAS unsigned*)&ctl->go) == 0) __builtin_amdgcn_s_sleep(2);
        if (wid <= NPK) unit_fetch(p, wid + 1, u0, lane, (unsigned)(size_t)(lds + ((wid + NPK - 1) % NPK) * PK_LDS));
        if (wid == 0) BAR;
        if (wid == 0) { storer(p, lds, ctl, xcd, slot, lane, cntp); __builtin_amdgcn_s_setprio(0); }
        else if (wid <= NPK) packer(p, lds + (wid - 1) * PK_LDS, ctl, xcd, slot, wid - 1, lane, cntp);
    }
    const unsigned call = __builtin_amdgcn_readfirstlane(ld4_sys(cntp)) + 1u;
    if (lane == 0) __hip_atomic_store(cntp, call, __ATOMIC_RELAXED, __HIP_MEMORY_SCOPE_AGENT);

    if (!main_tile) {
        if (slot >= 7) {
            const bool unp = slot < 7 + NUS;
            formatter(p, lds + wid * HTB, wid * 72 + (blockIdx.x - 56), unp ? 1 << 20 : 384 + wid * (NFMT / 8) + (blockIdx.x - 56 - NUS * 8), lane, call);
            const int uw = wid * (NUS * 8) + (blockIdx.x - 56);
            if (unp && uw < NUNP) unpacker(p, lds + wid * HTB, uw, lane, call);
        }
        STAMP(1);
        asm volatile("s_waitcnt vmcnt(0) lgkmcnt(0)" ::: "memory");
        BAR;
        if (DIAGD && slot < 7 + NUS) { const unsigned long long ts = __builtin_amdgcn_s_memrealtime(); while (__builtin_amdgcn_s_memrealtime() - ts < (slot < 8 ? DIAGD : DIAGD2)) __builtin_amdgcn_s_sleep(8); }
    }

    int rt, cn, q0, nt, skip;
    const char* wimg;
    if (main_tile) {
        const int idx = slot - 16;
        rt = ((xcd >> 1) * 2 + ((idx >> 2) & 1)) * 2 + (idx >> 3);
        cn = p.rank * 8 + (xcd & 1) * 4 + (idx & 3);
        const int m = rt >> 1;
        q0 = (m == p.rank) ? NH : 0; skip = (m - p.rank - 1) & 7; nt = 56;
        wimg = ws + OFF_WI;
    } else {
        rt = p.rank * 2 + (slot < 8 ? 1 : 0);
        cn = xcd * 8 + (slot & 7);
        q0 = 0; skip = 7; nt = 8;
        wimg = (xcd == p.rank) ? ws + OFF_WI : ws + OFF_WD + (size_t)((xcd - p.rank - 1) & 7) * NH * WQ;
    }
    const int wr = wid >> 2, wc = wid & 3;
    const unsigned* gate = (skip == 7) ? cntp : (const unsigned*)(ws + OFF_SD) + (skip * 8 + xcd) * 32;
    const unsigned* fx = (const unsigned*)(ws + OFF_FX) + rt * FSTR;
    const unsigned* fw = (const unsigned*)(ws + OFF_FW) + cn * FSTR;
    const unsigned* gate2 = (skip == 7) ? cntp : fw + NH + skip;
    const char* xb = ws + OFF_XI + (size_t)(rt * 16 + wid) * 1024;
    const char* wb = wimg + (size_t)((cn & 7) * 16 + wr * 8 + wc) * 1024;
    const unsigned voff = lane * 16;
    const unsigned ldsw = wid * 1024;
    const int aoff = wr * 4096 + lane * 16, boff = wc * 2048 + lane * 16;
#define SA(b, h) (((b) * 4 + (h)) * HTB)
#define SB(b, h) (((b) * 4 + 2 + (h)) * HTB)
#define GLDS(g, o, l) __builtin_amdgcn_global_load_lds((const unsigned*)((g) + (size_t)(unsigned)(o)), (LAS unsigned*)(lds + (l)), 16, 0, GLDS_AUX)
#define STAGE_A(b, h, qq) do { GLDS(wb + (h) * 4096, (qq.x << 17) + voff, SA(b, h) + ldsw); GLDS(wb + (h) * 4096, (qq.y << 17) + voff, SA(b, h) + ldsw + 8192); } while (0)
#define STAGE_B(b, h, qq) do { GLDS(xb + (h) * 8192, (qq.x << 18) + voff, SB(b, h) + ldsw); GLDS(xb + (h) * 8192, (qq.y << 18) + voff, SB(b, h) + ldsw + 8192); } while (0)
#define LDA(dst, b, h) do { _Pragma("unroll") for (int m = 0; m < 4; ++m) _Pragma("unroll") for (int k = 0; k < 2; ++k) dst[m][k] = *(const LAS bf16x8*)(lds + SA(b, h) + aoff + m * 1024 + k * 8192); } while (0)
#define LDB(dst, b, h) do { _Pragma("unroll") for (int n = 0; n < 2; ++n) _Pragma("unroll") for (int k = 0; k < 2; ++k) dst[n][k] = *(const LAS bf16x8*)(lds + SB(b, h) + boff + n * 1024 + k * 8192); } while (0)
#define MMA2(ai) do { __builtin_amdgcn_s_setprio(1); _Pragma("unroll") for (int k = 0; k < 2; ++k) _Pragma("unroll") for (int m = 0; m < 4; ++m) _Pragma("unroll") for (int n = 0; n < 2; ++n) { \
        acc[ai][0][m][n] = __builtin_amdgcn_mfma_f32_16x16x32_bf16(B0[n][k], At[m][k], acc[ai][0][m][n], 0, 0, 0); \
        acc[ai][1][m][n] = __builtin_amdgcn_mfma_f32_16x16x32_bf16(B1[n][k], At[m][k], acc[ai][1][m][n], 0, 0, 0); } __builtin_amdgcn_s_setprio(0); } while (0)
#define MUST_PICK(kt) do { if (__builtin_expect(nq < 2 * (kt) + 2, 0)) poll_ready(fx, fw, td0, td1, queue, nq, 2 * (kt) + 2, call, tmo); } while (0)
#define SNAP(kt) do { if (ASYNCP && __builtin_expect(nq < 2 * (kt) + 2, 0)) \
        __builtin_amdgcn_global_load_lds((lane < 32 ? fx : fw - 128) + lane * 4, queue + 256, 16, 0, 17); } while (0)
#define GETQ(kt) (*(volatile LAS u32x2*)(queue + 2 * (kt)))
#define FIXQ(qq, kt) do { if (__builtin_expect(__builtin_amdgcn_readfirstlane(qq.x & qq.y) < 0x10000, 0)) qq = GETQ(kt); } while (0)
    f32x4 acc[2][2][4][2];
#pragma unroll
    for (int a = 0; a < 2; ++a)
#pragma unroll
        for (int b = 0; b < 2; ++b)
#pragma unroll
            for (int m = 0; m < 4; ++m)
#pragma unroll
                for (int n = 0; n < 2; ++n) acc[a][b][m][n] = (f32x4){0.f, 0.f, 0.f, 0.f};
    bf16x8 At[4][2], B0[2][2], B1[2][2];
    LAS unsigned* queue = (LAS unsigned*)(lds + 8 * HTB + 4096);
    unsigned long long td0 = __ballot(lane >= q0 && lane < 2 * nt + 64 * main_tile && !(lane >= NH && (lane - NH) % 7 == skip));
    unsigned long long td1 = __ballot(main_tile && (lane + 64 - NH) % 7 != skip);
    int nq = 1 << 20;
    if (wid == 0) { nq = 0; *(volatile LAS u32x2*)(queue + 2 * lane) = (u32x2){0u, 0u}; *(volatile LAS u32x4*)(queue + 256 + 4 * lane) = (u32x4){0u, 0u, 0u, 0u}; poll_ready(fx, fw, td0, td1, queue, nq, 4, call, tmo); }
    asm volatile("" ::: "memory"); BAR; asm volatile("" ::: "memory");
    u32x2 qe = GETQ(0), qo = GETQ(1);
    STAMP(4);
    STAGE_B(0, 0, qe); STAGE_A(0, 0, qe); STAGE_B(0, 1, qe); STAGE_A(0, 1, qe);
    STAGE_B(1, 0, qo); STAGE_A(1, 0, qo); STAGE_B(1, 1, qo);
    if (wr == 1) BAR;
    WAIT_V(8); BAR; BAR;
    for (int t = 0; t < nt; t += 2) {
        if (t == nt - 2) nq = 1 << 20;
        SNAP(t + 3);
        qe = GETQ(t + 2);
        LDB(B0, 0, 0); LDA(At, 0, 0); LDB(B1, 0, 1); STAGE_A(1, 1, qo);
        WAIT_V(8); WAIT_L(0); BAR; SCHED; MMA2(0); SCHED; MUST_PICK(t + 2); BAR; SCHED;
        FIXQ(qe, t + 2);
        LDA(At, 0, 1); STAGE_B(0, 0, qe); STAGE_A(0, 0, qe); STAGE_B(0, 1, qe);
        WAIT_V(8); WAIT_L(0); BAR; SCHED; MMA2(1); BAR; SCHED;
        if (t == nt - 2)
            __builtin_amdgcn_global_load_lds(lane == 1 ? tmo : lane == 2 ? gate2 : gate, (LAS unsigned*)(lds + 8 * HTB + wid * 256), 4, 0, 17);
        SNAP(t + 4);
        qo = GETQ(t + 3);
        LDB(B0, 1, 0); LDA(At, 1, 0); LDB(B1, 1, 1); STAGE_A(0, 1, qe);
        WAIT_V(8); WAIT_L(0); BAR; SCHED; MMA2(0); SCHED; MUST_PICK(t + 3); BAR; SCHED;
        FIXQ(qo, t + 3);
        LDA(At, 1, 1); STAGE_B(1, 0, qo); STAGE_A(1, 0, qo); STAGE_B(1, 1, qo);
        WAIT_V(8); WAIT_L(0); BAR; SCHED; MMA2(1); BAR; SCHED;
    }
    STAMP(2);
    int tz = threadIdx.x; asm volatile("" : "+v"(tz));
    const int ln = tz & 63;
    const unsigned vo = ln * 16;
    const unsigned tile = (unsigned)((rt & 1) * 64 + cn) * 8 + wid;
    if (main_tile) {
        const u32x4 gv = *(volatile LAS u32x4*)(lds + 8 * HTB + wid * 256);
        bool bad = (unsigned)__builtin_amdgcn_readfirstlane(gv.y) == call;
        const bool open = (unsigned)__builtin_amdgcn_readfirstlane(gv.x) >= call && (unsigned)__builtin_amdgcn_readfirstlane(gv.z) >= call && !bad;
        const unsigned owner = (unsigned)(rt >> 1) & 7u;
        char* ows = p.ws[owner];
        const char* sl = ows + OFF_PL + (size_t)tile * UB;
        char* fl = ows + OFF_FP + (size_t)tile * FLS;
        const char* s0 = sl + 4096; const char* s1 = sl + 8192;
        const int staged = (rt & 1) ? BSTAGE : ASTAGE;
        u32x4 b[16], pk[12];
#pragma unroll
        for (int bj = 0; bj < 2; ++bj)
#pragma unroll
            for (int n = 0; n < 2; ++n)
#pragma unroll
                for (int r = 0; r < 4; ++r)
#pragma unroll
                    for (int ai = 0; ai < 2; ++ai)
#pragma unroll
                        for (int m2 = 0; m2 < 2; ++m2)
                            b[bj * 8 + n * 4 + r][ai * 2 + m2] = cvt_pk_bf16(acc[ai][bj][2 * m2][n][r], acc[ai][bj][2 * m2 + 1][n][r]);
        asm volatile("" : "+v"(b[0]), "+v"(b[1]), "+v"(b[2]), "+v"(b[3]), "+v"(b[4]), "+v"(b[5]), "+v"(b[6]), "+v"(b[7]), "+v"(b[8]), "+v"(b[9]), "+v"(b[10]), "+v"(b[11]), "+v"(b[12]), "+v"(b[13]), "+v"(b[14]), "+v"(b[15]) :: "memory");
        if (wr == 0) BAR;
        const unsigned L = pack_base(b);
        if (open) { st16<-4096>(s0, vo, pack_low(b, 0, L)); WAIT_V(1); } else WAIT_V(0);
        asm volatile("" ::: "memory"); BAR; asm volatile("" ::: "memory");
        unsigned tl;
        unsigned hdr = pack_unit(b, pk, tl, L, lds + wid * HTB, ln, open ? s0 : nullptr);
        if (!open) {
            const unsigned long long t0 = __builtin_amdgcn_s_memrealtime(); unsigned polls = 0;
            while (!bad && !__all(ld4_sys(ln & 1 ? gate2 : gate) >= call)) {
                __builtin_amdgcn_s_sleep(2);
                bad = give_up(tmo, call, t0, polls);
            }
        }
        STAMP(6);
        if (bad) {
            hdr = RAWCNT << 16;
#pragma unroll
            for (int f = 0; f < 16; ++f) b[f] = (u32x4){0x7fc07fc0u, 0x7fc07fc0u, 0x7fc07fc0u, 0x7fc07fc0u};
        }
        if (__builtin_expect(hdr_cnt(hdr) == RAWCNT, 0)) { WAIT_V(0); st_raw(sl, vo, b); }
        else {
            {
                if (!open) { st16<-4096>(s0, vo, pk[0]); WAIT_V(0); st16<-3072>(s0, vo, pk[1]); WAIT_V(0); st16<0>(s1, vo, pk[8]); WAIT_V(0); st16<1024>(s1, vo, pk[9]); }
                WAIT_V(0); st_tail(s1, vo, pk[10], pk[11], tl, ln);
                WAIT_V(0); STAMP(7); if (staged & 1) st8_flag(fl, call, hdr | (1u << 28), ln); st16<-2048>(s0, vo, pk[2]);
                WAIT_V(0); st16<-1024>(s0, vo, pk[3]);
                WAIT_V(0); if (staged & 2) st8_flag(fl, call, hdr | (2u << 28), ln); st16<0>(s0, vo, pk[4]);
                WAIT_V(0); st16<1024>(s0, vo, pk[5]);
                WAIT_V(0); if (staged & 4) st8_flag(fl, call, hdr | (3u << 28), ln); st16<2048>(s0, vo, pk[6]);
                WAIT_V(0); st16<3072>(s0, vo, pk[7]);
            }
        }
        WAIT_V(0);
        st8_flag(fl, call, hdr | (4u << 28), ln);
    } else {
        WAIT_V(0);
        if (wr == 0) BAR;
        BAR;
        LAS unsigned char* dl = lds + wid * HTB;
#pragma unroll
        for (int i = 0; i < 8; ++i) *(LAS u32x4*)(dl + i * 1024 + ln * 16) = (u32x4){0u, 0u, 0u, 0u};
        const char* fl = ws + OFF_FP + (size_t)tile * FLS;
        const char* sl = ws + OFF_PL + (size_t)tile * UB;
        unsigned hdr = wait_unit(fl, call, 1u, tmo);
        const unsigned cnt = hdr_cnt(hdr); const bool raw = cnt == RAWCNT;
        const unsigned LL = (hdr & 0x7fffu) * 0x10001u;
        u32x4 r[10], G;
        unsigned tv = 0;
        const bool whole = WHOLE && (hdr >> 28) >= 4u;
        if (!raw) {
            unsigned F;
            if (hdr & (1u << 26)) ld_tail2(sl, vo, (unsigned)(size_t)(dl + DEC_TAIL));
            if (whole) { tv = ld4_sys(tmo); ld_packed(r, F, sl, vo, (unsigned)(size_t)(dl + DEC_TAIL)); }
            else ld_head(r, F, tv, sl, vo, (unsigned)(size_t)(dl + DEC_TAIL), tmo);
            unpack_head(F, hdr, dl, ln, G);
        }
        const char* ob = p.out + ((size_t)((rt & 1) * 256 + wc * 32) * NN + cn * 256 + wr * 128) * 2;
        const unsigned oo = (ln >> 4) * (4 * NN * 2) + (ln & 15) * 16;
#pragma unroll
        for (int s = 0; s < 4; ++s) {
            if ((hdr >> 28) <= (unsigned)s) hdr = wait_unit(fl, call, s + 1, tmo);
            u32x4 o[4];
            if (raw) { ld4x16(o[0], o[1], o[2], o[3], sl + s * 4096, vo); tv = ld4_sys(tmo); }
            else { if (s && !whole) ld2x16t(r[2 * s], r[2 * s + 1], tv, sl + s * 2048, vo, tmo); unpack_sub(r, G, s, dl, ln, LL, o); }
            const unsigned nanmask = (tv == call) ? 0x7fc07fc0u : 0u;
#pragma unroll
            for (int rr = 0; rr < 4; ++rr) {
                u32x4 y;
#pragma unroll
                for (int cc = 0; cc < 4; ++cc) {
                    const unsigned pv = o[rr][cc];
                    const f32x2 v = (f32x2){acc[cc >> 1][s >> 1][2 * (cc & 1)][s & 1][rr] + __uint_as_float(pv << 16),
                                            acc[cc >> 1][s >> 1][2 * (cc & 1) + 1][s & 1][rr] + __uint_as_float(pv & 0xffff0000u)};
                    const f32x2 g = gelu2(v);
                    y[cc] = cvt_pk_bf16(g.x, g.y) | nanmask;
                }
                st16<0>(ob + (size_t)((s >> 1) * 128 + (s & 1) * 16 + rr) * (NN * 2), oo, y);
            }
        }
    }
    STAMP(3);
}
}

extern "C" void kernel_launch(int rank, int nranks, ncclComm_t comm, hipStream_t stream,
                              void* const* d_in, const int* in_sizes, int n_in,
                              void* d_out, int out_size, const int* in_dtypes, int out_dtype,
                              void* const* d_out_all, void* const* d_ws_all, size_t ws_bytes) {
    (void)comm; (void)in_dtypes; (void)out_dtype; (void)d_out_all;
    if (nranks != NR || (unsigned)rank >= (unsigned)NR || n_in < 2 || in_sizes[0] != MM * KS || in_sizes[1] != KS * NN ||
        out_size != (MM / NR) * NN || ws_bytes < WS_NEED) {
        fprintf(stderr, "kernel_launch: unexpected problem geometry\n");
        return;
    }
    Params p;
    p.x = (const char*)d_in[0];
    p.w = (const char*)d_in[1];
    for (int i = 0; i < NR; ++i) p.ws[i] = (char*)d_ws_all[i];
    p.out = (char*)d_out;
    p.rank = rank; p.pad = 0;
    hipLaunchKernelGGL(fused_kernel, dim3(256), dim3(512), 0, stream, p);
    HIP_CHECK(hipGetLastError());
}
```

```cpp
#include <hip/hip_runtime.h>
#include <rccl/rccl.h>
#include <cstdio>
#include <cstdint>

#define HIP_CHECK(expr) do { hipError_t _e = (expr); if (_e != hipSuccess) { \
    fprintf(stderr, "HIP error %s at %s:%d\n", hipGetErrorString(_e), __FILE__, __LINE__); \
    return; } } while (0)

#define LAGXB 36
#define LAGWH 4
#define LAGXA 1
#define NW0 2
#define RAW0 2
#define FORCE_RAW 0
#define LWIN 6
#define NDEC 5
#define EARLYTK 1
#define PUSHP 0
#define MAINP 0
#define PWIN 3
#define OWND 30
#define PD0 0
#define PDT 40
#define GATE_EARLY 6

namespace {
#define LAS __attribute__((address_space(3)))
typedef short bf16x8 __attribute__((ext_vector_type(8)));
typedef float f32x4 __attribute__((ext_vector_type(4)));
typedef float f32x2 __attribute__((ext_vector_type(2)));
typedef unsigned u32x4 __attribute__((ext_vector_type(4)));
typedef unsigned u32x2 __attribute__((ext_vector_type(2)));
typedef unsigned short u16x2 __attribute__((ext_vector_type(2)));

constexpr int NR = 8;
constexpr int MM = 4096, NN = 16384, KS = 512;
constexpr int NQ = 128;
constexpr int NH = 16;
constexpr int NU = NH * 22;
constexpr int NTASK = NH * 70;
constexpr unsigned long long DEADLINE = 20000000ull;
constexpr unsigned RAWCNT = 0x3ffu, LOWCNT = 0x3feu;

constexpr size_t OFF_CNT = 0;
constexpr size_t OFF_TMO = 8192;
constexpr size_t OFF_TK = 8192 + 1024;
constexpr int FSTR = 160;
constexpr size_t OFF_FX = 16384;
constexpr size_t OFF_FW = 32768;
constexpr size_t OFF_FD = 81920;
constexpr size_t OFF_SD = 131072;
constexpr size_t OFF_STAMP = 262144;
constexpr size_t UB = 16384, FLS = 128;
constexpr size_t XQ = 256 * 1024, WQ = 128 * 1024;
constexpr size_t OFF_XI = 1u << 20;
constexpr size_t OFF_WI = OFF_XI + (size_t)NQ * XQ;
constexpr size_t OFF_WD = OFF_WI + (size_t)NQ * WQ;
constexpr size_t OFF_PS = OFF_WD + (size_t)7 * NH * WQ;
constexpr size_t OFF_ST = OFF_PS + (size_t)1024 * UB;
constexpr size_t ST_PAR = (size_t)1120 * UB;
constexpr size_t OFF_PK = OFF_ST + 2 * ST_PAR;
constexpr size_t OFF_LZ = OFF_PK + (size_t)512 * 32768;
constexpr size_t OFF_FA = OFF_LZ + (size_t)14 * 8 * RAW0 * UB;
constexpr size_t OFF_FP = OFF_FA + (size_t)14 * NU * FLS;
constexpr size_t WS_NEED = OFF_FP + 1024 * FLS;
static_assert(OFF_FW + 64 * FSTR * 4 <= OFF_FD && OFF_FD + 2048 <= OFF_SD && OFF_SD + 56 * 128 <= OFF_STAMP, "layout");
static_assert(WS_NEED <= (size_t)161 << 20, "workspace");

struct Params {
    const char* x;
    const char* w;
    char* ws[NR];
    char* out;
    int rank;
    int pad;
};

template <int OFF> __device__ __forceinline__ void st16(const char* sbase, unsigned voff, u32x4 v) {
    asm volatile("global_store_dwordx4 %0, %1, %2 offset:%3 sc0 sc1\n\ts_nop 1" :: "v"(voff), "v"(v), "s"(sbase), "i"(OFF) : "memory");
}
__device__ __forceinline__ unsigned ld4_sys(const void* p) {
    unsigned v;
    asm volatile("global_load_dword %0, %1, off sc0 sc1\n\ts_waitcnt vmcnt(0)" : "=&v"(v) : "v"(p) : "memory");
    return v;
}
__device__ __forceinline__ u32x2 ld8_sys(const void* p) {
    u32x2 v;
    asm volatile("global_load_dwordx2 %0, %1, off sc0 sc1\n\ts_waitcnt vmcnt(0)" : "=&v"(v) : "v"(p) : "memory");
    return v;
}
__device__ __forceinline__ void st8_flag(void* p, unsigned call, unsigned hdr, int lane) {
    if (lane == 0) __hip_atomic_store((unsigned long long*)p, ((unsigned long long)hdr << 32) | call, __ATOMIC_RELAXED, __HIP_MEMORY_SCOPE_SYSTEM);
}
#define WAIT_V_(n) asm volatile("s_waitcnt vmcnt(" #n ")" ::: "memory")
#define WAIT_V(n) WAIT_V_(n)
#define WAIT_L(n) asm volatile("s_waitcnt lgkmcnt(" #n ")" ::: "memory")
#define BAR __builtin_amdgcn_s_barrier()
#define SCHED __builtin_amdgcn_sched_barrier(0)

__device__ __forceinline__ unsigned cvt_pk_bf16(float lo, float hi) {
    unsigned r; asm("v_cvt_pk_bf16_f32 %0, %1, %2" : "=v"(r) : "v"(lo), "v"(hi)); return r;
}
__device__ __forceinline__ f32x2 gelu2(f32x2 y) {
    const float k1 = -2.0f * 0.7978845608028654f * 1.4426950408889634f, k2 = k1 * 0.044715f;
    f32x2 z = y * (y * y * k2 + k1);
    f32x2 e; e.x = __builtin_amdgcn_exp2f(z.x); e.y = __builtin_amdgcn_exp2f(z.y);
    e = e + 1.0f;
    f32x2 r; r.x = __builtin_amdgcn_rcpf(e.x); r.y = __builtin_amdgcn_rcpf(e.y);
    return y * r;
}
__device__ __forceinline__ unsigned pk_sub(unsigned a, unsigned b) { return __builtin_bit_cast(unsigned, __builtin_bit_cast(u16x2, a) - __builtin_bit_cast(u16x2, b)); }
__device__ __forceinline__ unsigned pk_add(unsigned a, unsigned b) { return __builtin_bit_cast(unsigned, __builtin_bit_cast(u16x2, a) + __builtin_bit_cast(u16x2, b)); }
__device__ __forceinline__ unsigned pk_max(unsigned a, unsigned b) { return __builtin_bit_cast(unsigned, __builtin_elementwise_max(__builtin_bit_cast(u16x2, a), __builtin_bit_cast(u16x2, b))); }
__device__ __forceinline__ unsigned bfi(unsigned m, unsigned a, unsigned b) { return __builtin_amdgcn_bitop3_b32(m, a, b, 0xca); }
__device__ __forceinline__ unsigned rotl(unsigned a, int n) { return __builtin_amdgcn_alignbit(a, a, 32 - n); }
__device__ __forceinline__ unsigned rotr(unsigned a, int n) { return __builtin_amdgcn_alignbit(a, a, n); }
__device__ __forceinline__ unsigned umax(unsigned a, unsigned b) { return a > b ? a : b; }

__device__ __forceinline__ void dma16(const char* sbase, unsigned voff, unsigned ldst) {
    unsigned keep;
    asm volatile("s_mov_b32 %0, m0\n\ts_mov_b32 m0, %3\n\ts_nop 0\n\tglobal_load_lds_dwordx4 %1, %2 sc0 sc1\n\ts_mov_b32 m0, %0"
                 : "=&s"(keep) : "v"(voff), "s"(sbase), "s"(ldst) : "memory");
}

#define STAMP(slot) do {} while (0)

__device__ __forceinline__ bool give_up(unsigned* tmo, unsigned call, unsigned long long t0, unsigned& polls) {
    if ((++polls & 15u) != 0u) return false;
    if (__builtin_amdgcn_readfirstlane(__hip_atomic_load(tmo, __ATOMIC_RELAXED, __HIP_MEMORY_SCOPE_AGENT)) == call ||
        __builtin_amdgcn_s_memrealtime() - t0 > DEADLINE) {
        __hip_atomic_store(tmo, call, __ATOMIC_RELAXED, __HIP_MEMORY_SCOPE_AGENT);
        return true;
    }
    return false;
}
__device__ __forceinline__ unsigned hdr_cnt(unsigned hdr) { return (hdr >> 15) & 0x3ffu; }
__device__ __forceinline__ unsigned wait_unit(const char* flag, unsigned call, bool full, unsigned* tmo) {
    const unsigned long long t0 = __builtin_amdgcn_s_memrealtime(); unsigned polls = 0;
    for (;;) {
        const u32x2 v = ld8_sys(flag);
        const unsigned hdr = __builtin_amdgcn_readfirstlane(v.y);
        if ((unsigned)__builtin_amdgcn_readfirstlane(v.x) == call && !(full && hdr_cnt(hdr) == LOWCNT)) return hdr;
        __builtin_amdgcn_s_sleep(1);
        if (give_up(tmo, call, t0, polls)) return RAWCNT << 15;
    }
}
__device__ __forceinline__ void wait_word(const unsigned* w, unsigned call, unsigned* tmo) {
    const unsigned long long t0 = __builtin_amdgcn_s_memrealtime(); unsigned polls = 0;
    while ((unsigned)__builtin_amdgcn_readfirstlane(ld4_sys(w)) < call) {
        __builtin_amdgcn_s_sleep(2);
        if (give_up(tmo, call, t0, polls)) break;
    }
}

constexpr int SCR_DENSE = 8192;
__device__ __forceinline__ unsigned umin(unsigned a, unsigned b) { return a < b ? a : b; }
__device__ __forceinline__ unsigned wave_scan(unsigned inc) {
    inc += __builtin_amdgcn_update_dpp(0, (int)inc, 0x111, 0xf, 0xf, true);
    inc += __builtin_amdgcn_update_dpp(0, (int)inc, 0x112, 0xf, 0xf, true);
    inc += __builtin_amdgcn_update_dpp(0, (int)inc, 0x114, 0xf, 0xf, true);
    inc += __builtin_amdgcn_update_dpp(0, (int)inc, 0x118, 0xf, 0xf, true);
    inc += __builtin_amdgcn_update_dpp(0, (int)inc, 0x142, 0xa, 0xf, false);
    inc += __builtin_amdgcn_update_dpp(0, (int)inc, 0x143, 0xc, 0xf, false);
    return inc;
}
__device__ __forceinline__ unsigned pack_base(const u32x4 (&b)[16]) {
    unsigned mx = 0;
#pragma unroll
    for (int f = 0; f < 16; ++f) mx = pk_max(mx, b[f][f & 3] & 0x7fff7fffu);
    unsigned m = umax(mx & 0xffffu, mx >> 16);
    m = umax(m, (unsigned)__builtin_amdgcn_update_dpp(0, (int)m, 0x128, 0xf, 0xf, false));
    m = umax(m, (unsigned)__builtin_amdgcn_update_dpp(0, (int)m, 0x124, 0xf, 0xf, false));
    m = umax(m, (unsigned)__builtin_amdgcn_update_dpp(0, (int)m, 0x122, 0xf, 0xf, false));
    m = umax(m, (unsigned)__builtin_amdgcn_update_dpp(0, (int)m, 0x121, 0xf, 0xf, false));
    const unsigned m0 = __builtin_amdgcn_readlane(m, 0), m1 = __builtin_amdgcn_readlane(m, 16), m2 = __builtin_amdgcn_readlane(m, 32), m3 = __builtin_amdgcn_readlane(m, 48);
    const unsigned ms = umin(umax(umin(m0, m1), umin(m2, m3)), umin(umax(m0, m1), umax(m2, m3)));
    return umax(ms, 1023u) - 1023u;
}
__device__ __forceinline__ u32x4 pack_low(const u32x4 (&b)[16], int i, unsigned L) {
    const unsigned LL = L | (L << 16);
    u32x4 r;
#pragma unroll
    for (int c = 0; c < 4; ++c) r[c] = __builtin_amdgcn_perm(pk_sub(b[2 * i + (c >> 1)][2 * (c & 1) + 1], LL), pk_sub(b[2 * i + (c >> 1)][2 * (c & 1)], LL), 0x06040200u);
    return r;
}
__device__ __forceinline__ unsigned pack_unit(const u32x4 (&b)[16], u32x4 (&pk)[12], unsigned L, LAS unsigned char* scr, int lane) {
    const unsigned LL = L | (L << 16);
    const unsigned wp0 = (unsigned)(size_t)scr + lane * 4;
    unsigned wp = wp0;
    unsigned z[4], seg[4], len[4], F = 0;
#pragma unroll
    for (int s = 0; s < 4; ++s) {
        unsigned hb[8];
#pragma unroll
        for (int jj = 0; jj < 8; ++jj) {
            const int j = 8 * s + jj;
            const unsigned d0 = pk_sub(b[j >> 1][2 * (j & 1)], LL), d1 = pk_sub(b[j >> 1][2 * (j & 1) + 1], LL);
            pk[j >> 2][j & 3] = __builtin_amdgcn_perm(d1, d0, 0x06040200u);
            hb[jj] = __builtin_amdgcn_perm(d1, d0, 0x07050301u);
            const unsigned t = hb[jj] & 0x7c7c7c7cu; unsigned tmp;
            *(LAS unsigned*)(size_t)wp = t | (unsigned)((j & 3) | ((j >> 2) << 7));
            asm("v_min_u32 %0, 1, %2\n\tv_lshl_add_u32 %1, %0, 8, %1" : "=&v"(tmp), "+v"(wp) : "v"(t));
            if (jj & 3) hb[jj] = rotl(hb[jj], 2 * (jj & 3));
        }
        pk[8][s] = bfi(0x81818181u, hb[0], bfi(0x06060606u, hb[1], bfi(0x18181818u, hb[2], hb[3])));
        pk[9][s] = bfi(0x81818181u, hb[4], bfi(0x06060606u, hb[5], bfi(0x18181818u, hb[6], hb[7])));
        const unsigned ga = bfi(0x02020202u, hb[0], bfi(0x08080808u, hb[1], bfi(0x20202020u, hb[2], hb[3])));
        const unsigned gb = bfi(0x02020202u, hb[4], bfi(0x08080808u, hb[5], bfi(0x20202020u, hb[6], hb[7])));
        z[s] = ~bfi(0xaaaaaaaau, ga, gb >> 1);
    }
    unsigned e8[8], nz4[4];
#pragma unroll
    for (int i = 0; i < 8; ++i) e8[i] = *(LAS unsigned*)(scr + i * 256 + lane * 4);
#pragma unroll
    for (int s = 0; s < 4; ++s) {
        unsigned t = z[s] | (z[s] >> 1); t |= t << 2;
        nz4[s] = t & 0x44444444u; F |= s < 2 ? nz4[s] >> (2 - s) : nz4[s] << (s - 2);
    }
    const unsigned nrow = (wp - wp0) >> 8, mine = (__builtin_popcount(F) << 14) | nrow;
    const unsigned inc = wave_scan(mine), tot = __builtin_amdgcn_readlane(inc, 63), ngrp = tot & 0xfffu;
    const unsigned lo = 256u + ((((tot >> 12) + 31u) >> 5) << 2);
    unsigned cnt = ngrp, bad = 0;
    if (ngrp <= 512u) {
        LAS unsigned* dense = (LAS unsigned*)(scr + SCR_DENSE);
        const unsigned lanebits = ((lane & 7) << 15) | ((lane >> 3) << 23), base = (inc - mine) & 0xfffu;
#pragma unroll
        for (int i = 0; i < 8; ++i) dense[(unsigned)i < nrow ? base + i : 512 + lane] = e8[i] | lanebits;
        for (unsigned r = 8; __builtin_expect(__any(nrow > r), 0); ++r)
            dense[r < nrow ? base + r : 512 + lane] = *(LAS unsigned*)(scr + r * 256 + lane * 4) | lanebits;
        unsigned e2[2] = {dense[lane], dense[64 + lane]};
        *(LAS u32x4*)(scr + 256 + lane * 16) = (u32x4){0u, 0u, 0u, 0u};
        *(LAS unsigned*)(scr + lane * 4) = F;
#pragma unroll
        for (int s = 0; s < 4; ++s) { seg[s] = z[s] & 15u; len[s] = nz4[s] & 7u; }
#pragma unroll
        for (int i = 1; i < 8; ++i)
#pragma unroll
            for (int s = 0; s < 4; ++s) { seg[s] |= ((z[s] >> (4 * i)) & 15u) << len[s]; len[s] += (nz4[s] >> (4 * i)) & 7u; }
        unsigned bit = (inc - mine) >> 12;
#pragma unroll
        for (int s = 0; s < 4; ++s) {
            const unsigned long long v = (unsigned long long)seg[s] << (bit & 31u);
            LAS unsigned* w = (LAS unsigned*)(scr + 256) + (bit >> 5);
            __hip_atomic_fetch_or(w, (unsigned)v, __ATOMIC_RELAXED, __HIP_MEMORY_SCOPE_WAVEFRONT);
            __hip_atomic_fetch_or(w + 1, (unsigned)(v >> 32), __ATOMIC_RELAXED, __HIP_MEMORY_SCOPE_WAVEFRONT);
            bit += len[s];
        }
        LAS unsigned short* lst = (LAS unsigned short*)(scr + lo);
        const unsigned hz = ((0u - L) >> 10) & 31u;
        for (unsigned i0 = 0;;) {
            unsigned t[2], pos[2], at[2];
#pragma unroll
            for (int c = 0; c < 2; ++c) {
                const unsigned e = e2[c], ix = i0 + c * 64 + lane;
                t[c] = ix < ngrp ? e & 0x7c7c7c7cu : 0u;
                pos[c] = ((e & 0x380u) << 3) | ((e >> 16) & 0x380u) | ((e >> 11) & 0x70u) | ((e & 3u) << 2);
                at[c] = ix < ngrp ? ix : 1024u + lane;
            }
            for (;;) {
#pragma unroll
                for (int c = 0; c < 2; ++c) {
                    const unsigned b8 = __builtin_ctz(t[c] | 0x80000000u) & 24u, hi5 = (t[c] >> (b8 + 2)) & 31u;
                    const unsigned code = hi5 == hz ? 5u : (hi5 + 5u) & 31u;
                    bad = umax(bad, code);
                    lst[at[c]] = (unsigned short)((code << 13) | pos[c] | (b8 >> 3));
                    t[c] &= ~(0xffu << b8);
                }
                const unsigned long long b0 = __ballot(t[0] != 0), b1 = __ballot(t[1] != 0);
                if (!(b0 | b1)) break;
                at[0] = t[0] != 0 ? __builtin_amdgcn_mbcnt_hi((unsigned)(b0 >> 32), __builtin_amdgcn_mbcnt_lo((unsigned)b0, umin(cnt, 512u))) : 1024u + lane;
                cnt += __builtin_popcountll(b0);
                at[1] = t[1] != 0 ? __builtin_amdgcn_mbcnt_hi((unsigned)(b1 >> 32), __builtin_amdgcn_mbcnt_lo((unsigned)b1, umin(cnt, 512u))) : 1024u + lane;
                cnt += __builtin_popcountll(b1);
            }
            i0 += 128;
            if (__builtin_expect(i0 >= ngrp, 1)) break;
            e2[0] = dense[i0 + lane]; e2[1] = dense[i0 + 64 + lane];
        }
    }
    pk[10] = *(LAS u32x4*)(scr + lane * 16);
    pk[11] = *(LAS u32x4*)(scr + 1024 + lane * 16);
    const unsigned tl = (lo + 2u * cnt + 31u) >> 5;
    if (cnt > 512u || tl > 64u || __any(bad > 7u) || FORCE_RAW) return RAWCNT << 15;
    return L | (cnt << 15) | (tl << 25);
}
__device__ __forceinline__ void st_tail(const char* sb8k  , unsigned voff, u32x4 t0, u32x4 t1, unsigned tl, int lane) {
    if (__builtin_expect(tl > 32u, 0)) { if ((unsigned)lane < (tl - 32u) * 2u) st16<3072>(sb8k, voff, t1); }
    if ((unsigned)lane < tl * 2u) st16<2048>(sb8k, voff, t0);
}
__device__ __forceinline__ void st_raw(const char* slot, unsigned voff, const u32x4 (&b)[16]) {
    const char* s0 = slot + 4096; const char* s1 = slot + 12288;
    st16<-4096>(s0, voff, b[0]); st16<-3072>(s0, voff, b[1]); st16<-2048>(s0, voff, b[2]); st16<-1024>(s0, voff, b[3]);
    st16<0>(s0, voff, b[4]); st16<1024>(s0, voff, b[5]); st16<2048>(s0, voff, b[6]); st16<3072>(s0, voff, b[7]);
    st16<-4096>(s1, voff, b[8]); st16<-3072>(s1, voff, b[9]); st16<-2048>(s1, voff, b[10]); st16<-1024>(s1, voff, b[11]);
    st16<0>(s1, voff, b[12]); st16<1024>(s1, voff, b[13]); st16<2048>(s1, voff, b[14]); st16<3072>(s1, voff, b[15]);
}
__device__ __forceinline__ void ld4x16(f32x4& a, f32x4& b, f32x4& c, f32x4& d, const char* sb, unsigned voff) {
    asm volatile("global_load_dwordx4 %0, %4, %5 sc0 sc1\n\tglobal_load_dwordx4 %1, %4, %5 offset:1024 sc0 sc1\n\t"
                 "global_load_dwordx4 %2, %4, %5 offset:2048 sc0 sc1\n\tglobal_load_dwordx4 %3, %4, %5 offset:3072 sc0 sc1\n\ts_waitcnt vmcnt(0)"
                 : "=&v"(a), "=&v"(b), "=&v"(c), "=&v"(d) : "v"(voff), "s"(sb) : "memory");
}
constexpr int DEC_SLOT = 8192, DEC_TAIL = DEC_SLOT + 10240;
__device__ __forceinline__ void patch_apply(LAS unsigned char* dl, unsigned lo, unsigned hdr, int lane, bool zero) {
    const unsigned cnt = hdr_cnt(hdr), lut_hi = 0x0804007cu | ((((0u - (hdr & 0x7fffu)) >> 10) & 31u) << 10);
    for (unsigned c = 0; c < cnt; c += 128) {
        const unsigned w = *(LAS unsigned*)(dl + DEC_TAIL + lo + c * 2 + lane * 4);
#pragma unroll
        for (int hf = 0; hf < 2; ++hf) {
            const unsigned e = hf ? w >> 16 : w & 0xffffu;
            const unsigned at = c + lane * 2 + hf < cnt ? e & 0x1fffu : (unsigned)DEC_TAIL + lane;
            dl[at] = zero ? 0 : (unsigned char)__builtin_amdgcn_perm(lut_hi, 0x7874706cu, e >> 13);
        }
    }
}
__device__ __forceinline__ unsigned unpack_head(unsigned F, unsigned hdr, LAS unsigned char* dl, int lane, u32x4& G) {
    const unsigned mine = __builtin_popcount(F) << 2, inc = wave_scan(mine);
    const unsigned lo = 256u + ((((unsigned)__builtin_amdgcn_readlane(inc, 63) + 31u) >> 5) << 2);
    unsigned bit = inc - mine, sg[4];
#pragma unroll
    for (int s = 0; s < 4; ++s) {
        const LAS unsigned* w = (const LAS unsigned*)(dl + DEC_TAIL + 256) + (bit >> 5);
        sg[s] = __builtin_amdgcn_alignbit(w[1], w[0], bit);
        bit += __builtin_popcount(F & (0x11111111u << s)) << 2;
    }
    patch_apply(dl, lo, hdr, lane, false);
#pragma unroll
    for (int s = 0; s < 4; ++s) {
        const unsigned nz = (F >> s) & 0x11111111u, pre = nz * 0x11111110u;
        const unsigned xe = sg[s] & 0x0f0f0f0fu, xo = (sg[s] >> 4) & 0x0f0f0f0fu;
        const unsigned b0 = __builtin_amdgcn_perm(xo, xe, 0x05010400u), b1 = __builtin_amdgcn_perm(xo, xe, 0x07030602u);
        const unsigned ze = __builtin_amdgcn_perm(b1, b0, pre & 0x07070707u), zo = __builtin_amdgcn_perm(b1, b0, (pre >> 4) & 0x07070707u);
        G[s] = ~((ze | (zo << 4)) & ((nz << 4) - nz));
    }
    return lo;
}
__device__ __forceinline__ void unpack_piece(u32x4 lo, const u32x4 (&P)[2], u32x4 G, int i, LAS unsigned char* dl, int lane, unsigned LL, u32x4 (&o)[2]) {
    const u32x4 pt = *(LAS u32x4*)(dl + i * 1024 + lane * 16);
    const unsigned A = P[i & 1][i >> 1], g = G[i >> 1] << (i & 1);
    unsigned hb[4];
    hb[0] = bfi(0x83838383u, bfi(0x02020202u, g, A), pt[0]); hb[1] = bfi(0x83838383u, rotr(bfi(0x08080808u, g, A), 2), pt[1]);
    hb[2] = bfi(0x83838383u, rotr(bfi(0x20202020u, g, A), 4), pt[2]); hb[3] = bfi(0x83838383u, rotr(bfi(0x80808080u, g, A), 6), pt[3]);
#pragma unroll
    for (int j = 0; j < 4; ++j) {
        o[j >> 1][2 * (j & 1)] = pk_add(__builtin_amdgcn_perm(hb[j], lo[j], 0x05010400u), LL);
        o[j >> 1][2 * (j & 1) + 1] = pk_add(__builtin_amdgcn_perm(hb[j], lo[j], 0x07030602u), LL);
    }
}

__device__ __forceinline__ void unit_fetch(const Params& p, int h, int u, int lane, unsigned ldsbuf) {
    const bool isx = u < 16;
    const char* base = isx ? p.x + ((size_t)u * 256 * KS + h * 32) * 2 : p.w + ((size_t)h * 32 * NN + (u - 16) * 256) * 2;
    unsigned voff = isx ? (lane & 15) * (KS * 2) + (lane >> 4) * 16 : (lane >> 4) * (8 * NN * 2) + (lane & 15) * 16;
    const unsigned s1 = isx ? 16 * KS * 2 : NN * 2, s2 = isx ? 16 * KS * 2 : 256 - 7 * NN * 2;
#pragma unroll
    for (int i = 0; i < 16; ++i) {
        unsigned keep;
        asm volatile("s_mov_b32 %0, m0\n\ts_mov_b32 m0, %3\n\ts_nop 0\n\tglobal_load_lds_dwordx4 %1, %2\n\ts_mov_b32 m0, %0"
                     : "=&s"(keep) : "v"(voff), "s"(base), "s"(ldsbuf + i * 1024) : "memory");
        voff += (i == 7) ? s2 : s1;
    }
}
__device__ __forceinline__ void unit_read(LAS unsigned char* buf, int lane, u32x4 (&r)[16]) {
#pragma unroll
    for (int i = 0; i < 16; ++i) r[i] = *(LAS u32x4*)(buf + i * 1024 + lane * 16);
    asm volatile("s_waitcnt lgkmcnt(0)" : "+v"(r[0]), "+v"(r[1]), "+v"(r[2]), "+v"(r[3]), "+v"(r[4]), "+v"(r[5]), "+v"(r[6]), "+v"(r[7]),
                 "+v"(r[8]), "+v"(r[9]), "+v"(r[10]), "+v"(r[11]), "+v"(r[12]), "+v"(r[13]), "+v"(r[14]), "+v"(r[15]) :: "memory");
}
__device__ __forceinline__ void unit_format(int u, u32x4 (&r)[16]) {
    if (u >= 16) {
#pragma unroll
        for (int g = 0; g < 2; ++g) {
            u32x4 o[8];
#pragma unroll
            for (int i = 0; i < 8; ++i)
#pragma unroll
                for (int a = 0; a < 4; ++a)
                    o[i][a] = __builtin_amdgcn_perm(r[g * 8 + 2 * a + 1][i >> 1], r[g * 8 + 2 * a][i >> 1], (i & 1) ? 0x07060302u : 0x05040100u);
#pragma unroll
            for (int i = 0; i < 8; ++i) r[g * 8 + i] = o[i];
        }
    }
}
struct Tabs { unsigned ord[NU]; unsigned task[NTASK]; };
constexpr Tabs make_tabs() {
    Tabs t{}; int n = 0;
    for (int v = 0; v < 8 * NW0; ++v) t.ord[n++] = (v >> 3) << 8 | (v & 7);
    for (int key = 0; key < 8 * NH + 512; ++key)
        for (int v = 0; v < 22; ++v) {
            const int d = key - (v < 8 ? ((v & 2) ? LAGWH : 0) : v < 15 ? LAGXA : LAGXB);
            if (d >= 0 && d % 8 == 0 && d / 8 < NH && !(v < 8 && d / 8 < NW0)) t.ord[n++] = (unsigned)((d / 8) << 8 | v);
        }
    n = 0;
    for (int k = 0; k < NU; ++k) {
        const unsigned h = t.ord[k] >> 8, v = t.ord[k] & 255, e = k | h << 9;
        for (unsigned l = 0; l < (v < 8 ? 7u : 1u); ++l) t.task[n++] = e | v << 13 | l << 18;
    }
    return t;
}
__constant__ const Tabs g_tabs = make_tabs();
__device__ __forceinline__ int x_tile(int v, int src) { return v >= 15 ? 2 * ((src + v - 14) & 7) + 1 : 2 * ((src + v - 7) & 7); }
__device__ __forceinline__ int st_slot(int h, int v, int link) { return v < 8 ? (link * NH + h) * 8 + v : 896 + h * 14 + v - 8; }

__device__ __forceinline__ void producer(const Params& p, LAS unsigned char* buf, int n0, int n1, int nlead, int lane, unsigned* cntp) {
    char* ws = p.ws[p.rank];
    const unsigned voff = lane * 16;
    const unsigned long long ts = __builtin_amdgcn_s_memrealtime();
    char* peer = p.ws[(p.rank + 1 + (lane & 7)) & 7];
    unsigned call = 0;
    for (int n = n0; n < NTASK; n = n == n0 ? n1 : n + 576) {
        const bool first = n < 56 * RAW0;
        const unsigned e = first ? (n / 7) | ((n / 7) >> 3) << 9 | ((n / 7) & 7) << 13 | (n % 7) << 18 : g_tabs.task[n];
        const int k = e & 511, h = (e >> 9) & 15, v = (e >> 13) & 31, link = e >> 18;
        const bool isx = v >= 8;
        const unsigned dest = (unsigned)(p.rank + 1 + link) & 7u;
        const int t = isx ? x_tile(v, p.rank) : dest * 8 + v;
        while (__builtin_amdgcn_s_memrealtime() - ts < (unsigned)(n < nlead ? 0 : PD0 + (n - nlead) * PDT / 16)) __builtin_amdgcn_s_sleep(2);
        unit_fetch(p, h, isx ? t : 16 + t, lane, (unsigned)(size_t)buf);
        if (n == n0) call = __builtin_amdgcn_readfirstlane(ld4_sys(cntp)) + 1u;
        WAIT_V(0);
        u32x4 r[16];
        unit_read(buf, lane, r);
        unit_format(isx ? 0 : 16, r);
        unsigned hdr = RAWCNT << 15;
        if (first) {
            const char* s0 = p.ws[dest] + OFF_LZ + (size_t)(((call & 1u) * 7 + 6 - link) * 8 * RAW0 + k) * UB + 4096; const char* s1 = s0 + 8192;
            st16<-4096>(s0, voff, r[0]); st16<-3072>(s0, voff, r[1]); st16<-2048>(s0, voff, r[2]); WAIT_V(PWIN); st16<-1024>(s0, voff, r[3]);
            WAIT_V(PWIN); st16<0>(s0, voff, r[4]); WAIT_V(PWIN); st16<1024>(s0, voff, r[5]); WAIT_V(PWIN); st16<2048>(s0, voff, r[6]); WAIT_V(PWIN); st16<3072>(s0, voff, r[7]);
            WAIT_V(PWIN); st16<-4096>(s1, voff, r[8]); WAIT_V(PWIN); st16<-3072>(s1, voff, r[9]); WAIT_V(PWIN); st16<-2048>(s1, voff, r[10]); WAIT_V(PWIN); st16<-1024>(s1, voff, r[11]);
            WAIT_V(PWIN); st16<0>(s1, voff, r[12]); WAIT_V(PWIN); st16<1024>(s1, voff, r[13]); WAIT_V(PWIN); st16<2048>(s1, voff, r[14]); WAIT_V(PWIN); st16<3072>(s1, voff, r[15]);
        } else {
            const char* sl = ws + OFF_ST + (call & 1u) * ST_PAR + (size_t)st_slot(h, v, link) * UB;
            u32x4 pk[12];
            hdr = pack_unit(r, pk, pack_base(r), buf, lane);
            if (__builtin_expect(hdr_cnt(hdr) == RAWCNT, 0)) st_raw(sl, voff, r);
            else {
                const char* s0 = sl + 4096; const char* s1 = sl + 8192;
                st16<-4096>(s0, voff, pk[0]); st16<-3072>(s0, voff, pk[1]); st16<-2048>(s0, voff, pk[2]); st16<-1024>(s0, voff, pk[3]);
                st16<0>(s0, voff, pk[4]); st16<1024>(s0, voff, pk[5]); st16<2048>(s0, voff, pk[6]); st16<3072>(s0, voff, pk[7]);
                st16<0>(s1, voff, pk[8]); st16<1024>(s1, voff, pk[9]); st_tail(s1, voff, pk[10], pk[11], hdr >> 25, lane);
            }
        }
        WAIT_V(0);
        if (isx ? lane < 7 : lane == link)
            __hip_atomic_store((unsigned long long*)(peer + OFF_FA + (size_t)(((call & 1u) * 7 + 6 - lane) * NU + k) * FLS), ((unsigned long long)hdr << 32) | call,
                               __ATOMIC_RELAXED, __HIP_MEMORY_SCOPE_SYSTEM);
        if (!isx) {
            st_raw(ws + OFF_WD + (size_t)(link * NH + h) * WQ + (size_t)v * UB, voff, r);
            WAIT_V(0);
            if (lane == 0) __hip_atomic_store((unsigned*)(ws + OFF_FW) + t * FSTR + h, call, __ATOMIC_RELAXED, __HIP_MEMORY_SCOPE_SYSTEM);
        }
        WAIT_L(0);
    }
}
__device__ __forceinline__ void own_image(const Params& p, LAS unsigned char* buf, int n, int lane, unsigned call) {
    char* ws = p.ws[p.rank];
    const int h = n / 24, v = n - h * 24, t = v < 16 ? v : p.rank * 8 + v - 16;
    unit_fetch(p, h, v < 16 ? t : 16 + t, lane, (unsigned)(size_t)buf);
    WAIT_V(0);
    u32x4 r[16];
    unit_read(buf, lane, r);
    unit_format(v, r);
    st_raw(ws + (v < 16 ? OFF_XI + (size_t)h * XQ : OFF_WI + (size_t)h * WQ) + (size_t)(v & 15) * UB, lane * 16, r);
    WAIT_V(0);
    if (lane == 0) __hip_atomic_store((unsigned*)(ws + (v < 16 ? OFF_FX : OFF_FW)) + t * FSTR + h, call, __ATOMIC_RELAXED, __HIP_MEMORY_SCOPE_SYSTEM);
}

constexpr int DEC_LDS = 24 * 1024;
struct PullCtl { u32x4 desc[16]; unsigned full[7]; unsigned tail; };
__device__ __forceinline__ void decoder(const Params& p, LAS unsigned char* dl, LAS PullCtl* ctl, int xcd, int li, int mem, int lane, unsigned call) {
    char* ws = p.ws[p.rank];
    unsigned* tmo = (unsigned*)(ws + OFF_TMO);
    const unsigned voff = lane * 16;
    const int src = (p.rank + 1 + li) & 7;
    const char* fa = ws + OFF_FA + (size_t)((call & 1u) * 7 + li) * NU * FLS;
    unsigned* tk = (unsigned*)(ws + OFF_TK) + li * 32;
    const unsigned tk0 = (call - 1u) * (NU - 8 * RAW0) - (RAW0 + NDEC) * 8;
#pragma unroll
    for (int i = 0; i < 8; ++i) *(LAS u32x4*)(dl + i * 1024 + lane * 16) = (u32x4){0u, 0u, 0u, 0u};
    unsigned k = (RAW0 + mem) * 8 + xcd, n = 0;
    while (k < (unsigned)NU) {
        const int e = g_tabs.ord[k], h = e >> 8, v = e & 255;
        const unsigned hdr = wait_unit(fa + (size_t)k * FLS, call, false, tmo);
        ++n;
        if (lane == 0) {
            const unsigned pos = __hip_atomic_fetch_add(&ctl->tail, 1u, __ATOMIC_RELAXED, __HIP_MEMORY_SCOPE_WORKGROUP);
            *(volatile LAS u32x4*)&ctl->desc[pos & 15] = (u32x4){pos + 1u, k | (unsigned)st_slot(h, v, 6 - li) << 16, hdr, (unsigned)mem | (n << 8)};
        }
        const bool isx = v >= 8, raw = hdr_cnt(hdr) == RAWCNT;
        const int q = NH + h * 7 + li, t = isx ? x_tile(v, src) : v;
        const char* dst = ws + (isx ? OFF_XI + (size_t)q * XQ : OFF_WI + (size_t)q * WQ) + (size_t)t * UB;
        unsigned* flag = (unsigned*)(ws + (isx ? OFF_FX : OFF_FW)) + (isx ? t : p.rank * 8 + t) * FSTR + q;
        unsigned tn = 1u;
        if (EARLYTK && lane == 0) asm volatile("global_atomic_add %0, %1, %0, off sc0\n\ts_waitcnt vmcnt(0)" : "+v"(tn) : "v"(tk) : "memory");
        while ((unsigned)__builtin_amdgcn_readfirstlane(*(volatile LAS unsigned*)&ctl->full[mem]) != n) __builtin_amdgcn_s_sleep(1);
        asm volatile("" ::: "memory");
        unsigned lo = 0;
        if (__builtin_expect(raw, 0)) {
            u32x4 r[16];
            unit_read(dl + DEC_SLOT, lane, r);
            st_raw(dst, voff, r);
        } else {
            const unsigned F = *(LAS unsigned*)(dl + DEC_TAIL + lane * 4), LL = (hdr & 0x7fffu) * 0x10001u;
            u32x4 P[2], G, lw[8];
            P[0] = *(LAS u32x4*)(dl + DEC_SLOT + 8192 + lane * 16); P[1] = *(LAS u32x4*)(dl + DEC_SLOT + 9216 + lane * 16);
#pragma unroll
            for (int i = 0; i < 8; ++i) lw[i] = *(LAS u32x4*)(dl + DEC_SLOT + i * 1024 + lane * 16);
            lo = unpack_head(F, hdr, dl, lane, G);
#pragma unroll
            for (int i = 0; i < 8; ++i) {
                u32x4 o[2]; unpack_piece(lw[i], P, G, i, dl, lane, LL, o);
                const char* sb = dst + i * 2048;
                st16<0>(sb, voff, o[0]); st16<1024>(sb, voff, o[1]);
            }
        }
        WAIT_V(0);
        if (lane == 0) __hip_atomic_store(flag, call, __ATOMIC_RELAXED, __HIP_MEMORY_SCOPE_SYSTEM);
        if (!EARLYTK && lane == 0) asm volatile("global_atomic_add %0, %1, %0, off sc0\n\ts_waitcnt vmcnt(0)" : "+v"(tn) : "v"(tk) : "memory");
        if (!raw) patch_apply(dl, lo, hdr, lane, true);
        WAIT_L(0);
        k = __builtin_amdgcn_readfirstlane(tn) - tk0;
    }
    if (lane == 0) {
        const unsigned pos = __hip_atomic_fetch_add(&ctl->tail, 1u, __ATOMIC_RELAXED, __HIP_MEMORY_SCOPE_WORKGROUP);
        *(volatile LAS u32x4*)&ctl->desc[pos & 15] = (u32x4){pos + 1u, 0u, 0u, 0x80u};
    }
}
__device__ __forceinline__ void opener(const Params& p, int li, int h, int ct, int lane, unsigned call) {
    char* ws = p.ws[p.rank];
    wait_unit(ws + OFF_FA + (size_t)(((call & 1u) * 7 + li) * NU + h * 8 + ct) * FLS, call, false, (unsigned*)(ws + OFF_TMO));
    const char* src = ws + OFF_LZ + (size_t)(((call & 1u) * 7 + li) * 8 * RAW0 + h * 8 + ct) * UB;
    const int q = NH + h * 7 + li;
    const char* dst = ws + OFF_WI + (size_t)q * WQ + (size_t)ct * UB;
    for (int i = 0; i < 4; ++i) {
        f32x4 t[4]; ld4x16(t[0], t[1], t[2], t[3], src + i * 4096, lane * 16);
        const char* d = dst + i * 4096;
        st16<0>(d, lane * 16, __builtin_bit_cast(u32x4, t[0])); st16<1024>(d, lane * 16, __builtin_bit_cast(u32x4, t[1]));
        st16<2048>(d, lane * 16, __builtin_bit_cast(u32x4, t[2])); st16<3072>(d, lane * 16, __builtin_bit_cast(u32x4, t[3]));
    }
    WAIT_V(0);
    if (lane == 0) __hip_atomic_store((unsigned*)(ws + OFF_FW) + (p.rank * 8 + ct) * FSTR + q, call, __ATOMIC_RELAXED, __HIP_MEMORY_SCOPE_SYSTEM);
}
__device__ __forceinline__ void loader(const Params& p, LAS unsigned char* lds, LAS PullCtl* ctl, int li, int lane, unsigned call) {
    const char* st = p.ws[(unsigned)(p.rank + 1 + li) & 7u] + OFF_ST + (call & 1u) * ST_PAR;
    const unsigned voff = lane * 16;
    unsigned pos = 0, done = 0, pend = 0, idle = 0;
#define LANDED() do { *(volatile LAS unsigned*)&ctl->full[pend & 7u] = pend >> 8; pend = 0; } while (0)
    while (done < NDEC || pend) {
        const u32x4 d = *(volatile LAS u32x4*)&ctl->desc[pos & 15];
        if ((unsigned)__builtin_amdgcn_readfirstlane(d.x) != pos + 1u) {
            if (!pend) __builtin_amdgcn_s_sleep(1);
            else if (idle == 0) WAIT_V(4);
            else if (idle == 1) WAIT_V(2);
            else { WAIT_V(0); LANDED(); }
            ++idle;
            continue;
        }
        idle = 0;
        const unsigned ks = __builtin_amdgcn_readfirstlane(d.y), hdr = __builtin_amdgcn_readfirstlane(d.z), w = __builtin_amdgcn_readfirstlane(d.w);
        ++pos;
        if (w & 0x80u) { ++done; continue; }
        if ((ks & 0xffffu) == NU - 1 - GATE_EARLY && lane < 8)
            __hip_atomic_store((unsigned*)(p.ws[p.rank] + OFF_SD) + (li * 8 + lane) * 32, call, __ATOMIC_RELAXED, __HIP_MEMORY_SCOPE_AGENT);
        const char* s = st + (size_t)(ks >> 16) * UB;
        const unsigned l = (unsigned)(size_t)lds + (w & 7u) * DEC_LDS + DEC_SLOT;
#pragma unroll
        for (int i = 0; i < 10; ++i) {
            WAIT_V(LWIN);
            if (i == LWIN && pend) LANDED();
            dma16(s + i * 1024, voff, l + i * 1024);
        }
        static_assert(LWIN < 10, "window");
        if (__builtin_expect(hdr_cnt(hdr) == RAWCNT, 0)) {
#pragma unroll
            for (int i = 10; i < 16; ++i) { WAIT_V(LWIN); dma16(s + i * 1024, voff, l + i * 1024); }
        } else {
            const unsigned tl = hdr >> 25;
            WAIT_V(LWIN);
            if ((unsigned)lane < tl * 2u) dma16(s + 10240, voff, l + 10240);
            if (__builtin_expect(tl > 32u, 0)) { if ((unsigned)lane < (tl - 32u) * 2u) dma16(s + 11264, voff, l + 11264); }
        }
        pend = w;
    }
}

__device__ __forceinline__ void post_ready(bool r0, bool r1, unsigned long long& td0, unsigned long long& td1, LAS unsigned* queue, int& nq, unsigned lane) {
    const unsigned long long a0 = __ballot(r0) & td0, a1 = __ballot(r1) & td1;
    if ((a0 >> lane) & 1) ((volatile LAS unsigned*)queue)[__builtin_amdgcn_mbcnt_hi((unsigned)(a0 >> 32), __builtin_amdgcn_mbcnt_lo((unsigned)a0, nq))] = lane | 0x10000u;
    nq += __builtin_popcountll(a0);
    if ((a1 >> lane) & 1) ((volatile LAS unsigned*)queue)[__builtin_amdgcn_mbcnt_hi((unsigned)(a1 >> 32), __builtin_amdgcn_mbcnt_lo((unsigned)a1, nq))] = lane | 0x10040u;
    nq += __builtin_popcountll(a1);
    td0 &= ~a0; td1 &= ~a1;
}
__device__ __forceinline__ void poll_ready(const unsigned* fx, const unsigned* fw, unsigned long long& td0, unsigned long long& td1,
                                           LAS unsigned* queue, int& nq, int need, unsigned call, unsigned* tmo) {
    const unsigned lane = __builtin_amdgcn_mbcnt_hi(~0u, __builtin_amdgcn_mbcnt_lo(~0u, 0u));
    {
        const volatile LAS unsigned* sn = queue + 256 + lane;
        const unsigned x0 = sn[0], x1 = sn[64], w0 = sn[128], w1 = sn[192];
        post_ready(x0 == call && w0 == call, x1 == call && w1 == call, td0, td1, queue, nq, lane);
    }
    const unsigned long long t0 = __builtin_amdgcn_s_memrealtime(); unsigned polls = 0;
    bool all = false;
    while (nq < need) {
        unsigned x0, x1, w0, w1;
        asm volatile("global_load_dword %0, %4, %5 sc0 sc1\n\tglobal_load_dword %1, %4, %5 offset:256 sc0 sc1\n\t"
                     "global_load_dword %2, %4, %6 sc0 sc1\n\tglobal_load_dword %3, %4, %6 offset:256 sc0 sc1\n\ts_waitcnt vmcnt(0)"
                     : "=&v"(x0), "=&v"(x1), "=&v"(w0), "=&v"(w1) : "v"(lane * 4), "s"(fx), "s"(fw) : "memory");
        post_ready(all || (x0 >= call && w0 >= call), all || (x1 >= call && w1 >= call), td0, td1, queue, nq, lane);
        if (nq >= need) break;
        all = give_up(tmo, call, t0, polls);
        __builtin_amdgcn_s_sleep(1);
    }
    WAIT_L(0);
}

constexpr int HTB = 16384;
constexpr int FIN_LDS = 20480;

__global__ __launch_bounds__(512, 2) void fused_kernel(const Params p) {
    __shared__ __attribute__((aligned(16))) unsigned char shm[8 * FIN_LDS];
    LAS unsigned char* lds = (LAS unsigned char*)shm;
    const int wid = __builtin_amdgcn_readfirstlane(threadIdx.x >> 6), lane = threadIdx.x & 63;
    const int xcd = blockIdx.x & 7, slot = blockIdx.x >> 3;
    char* ws = p.ws[p.rank];
    unsigned* tmo = (unsigned*)(ws + OFF_TMO);
    unsigned* cntp = (unsigned*)(ws + OFF_CNT) + blockIdx.x * 8 + wid;
    STAMP(0);
    const bool main_tile = slot >= 16;
    LAS PullCtl* ctl = (LAS PullCtl*)(lds + 5 * DEC_LDS + 2 * HTB);
    static_assert(NDEC <= 5 && RAW0 <= 2 && RAW0 <= NW0, "puller CU: load wave, 5 unpack waves, 2 waves for the opening units");
    if (slot < 7) { if (wid < 2) ((volatile LAS unsigned*)ctl)[wid * 64 + lane] = 0u; WAIT_L(0); BAR; }
    {
        constexpr int NOPEN = 56 * RAW0, NLEAD = NOPEN + MAINP * 128;
        int n0 = -1, n1 = NTASK; LAS unsigned char* buf = lds + wid * HTB;
        if (slot < 7) { if (PUSHP && wid >= 6 && wid < 6 + RAW0) { n0 = ((wid - 6) * 8 + xcd) * 7 + slot; buf = lds + 5 * DEC_LDS + (wid - 6) * HTB; } }
        else if (main_tile) { if (wid >= 4 && wid < 4 + MAINP) n0 = NOPEN + (wid - 4) * 128 + (blockIdx.x - 128); }
        else { const int i = wid * 72 + (blockIdx.x - 56);
               if (PUSHP) { n0 = NLEAD + i; n1 = n0 + 576; } else { n0 = i < NOPEN ? i : i + MAINP * 128; n1 = i + 576 + MAINP * 128; } }
        if (n0 >= 0) producer(p, buf, n0, n1, NLEAD, lane, cntp);
    }
    const unsigned call = __builtin_amdgcn_readfirstlane(ld4_sys(cntp)) + 1u;
    if (lane == 0) __hip_atomic_store(cntp, call, __ATOMIC_RELAXED, __HIP_MEMORY_SCOPE_AGENT);
    if (slot < 7) {
        if (wid == 0) { __builtin_amdgcn_s_setprio(3); loader(p, lds, ctl, slot, lane, call); __builtin_amdgcn_s_setprio(0); }
        else if (wid <= NDEC) decoder(p, lds + (wid - 1) * DEC_LDS, ctl, xcd, slot, wid - 1, lane, call);
        else if (wid >= 6 && wid < 6 + RAW0) opener(p, slot, wid - 6, xcd, lane, call);
    }
    if (main_tile) {
        if (wid >= 1 && wid <= 3) { __builtin_amdgcn_s_sleep(OWND); own_image(p, lds + wid * HTB, (wid - 1) * 128 + (blockIdx.x - 128), lane, call); }
    } else {
        STAMP(1);
        asm volatile("s_waitcnt vmcnt(0) lgkmcnt(0)" ::: "memory");
        BAR;
    }

    const int wr = wid >> 2, wc = wid & 3;
    f32x4 acc[2][2][4][2];
    int rt, cn;
    const unsigned* gate2 = cntp;
    if (slot < 8) {
        rt = 0; cn = xcd * 8 + slot;
        wait_word((const unsigned*)(ws + OFF_FD) + cn * 8 + wid, call, tmo);
        const char* pb = ws + OFF_PK + (size_t)(cn * 8 + wid) * 32768;
#pragma unroll
        for (int i = 0; i < 8; ++i)
            ld4x16(acc[i >> 2][(i >> 1) & 1][(i & 1) * 2][0], acc[i >> 2][(i >> 1) & 1][(i & 1) * 2][1], acc[i >> 2][(i >> 1) & 1][(i & 1) * 2 + 1][0], acc[i >> 2][(i >> 1) & 1][(i & 1) * 2 + 1][1], pb + i * 4096, lane * 16);
    } else
    for (int it = 0; it < 2; ++it) {
    int q0, nt, skip;
    const char* wimg;
    if (main_tile) {
        const int idx = slot - 16;
        rt = ((xcd >> 1) * 2 + ((idx >> 2) & 1)) * 2 + (idx >> 3);
        cn = p.rank * 8 + (xcd & 1) * 4 + (idx & 3);
        const int m = rt >> 1;
        q0 = (m == p.rank) ? NH : 0; skip = (m - p.rank - 1) & 7; nt = 56;
        wimg = ws + OFF_WI;
    } else {
        rt = p.rank * 2 + 1 - it;
        cn = xcd * 8 + (slot & 7);
        q0 = 0; skip = 7; nt = 8;
        wimg = (xcd == p.rank) ? ws + OFF_WI : ws + OFF_WD + (size_t)((xcd - p.rank - 1) & 7) * NH * WQ;
    }
    const unsigned* fx = (const unsigned*)(ws + OFF_FX) + rt * FSTR;
    const unsigned* fw = (const unsigned*)(ws + OFF_FW) + cn * FSTR;
    gate2 = (skip == 7) ? cntp : fw + NH + skip;
    const char* xb = ws + OFF_XI + (size_t)(rt * 16 + wid) * 1024;
    const char* wb = wimg + (size_t)((cn & 7) * 16 + wr * 8 + wc) * 1024;
    const unsigned voff = lane * 16;
    const unsigned ldsw = wid * 1024;
    const int aoff = wr * 4096 + lane * 16, boff = wc * 2048 + lane * 16;
#define SA(b, h) (((b) * 4 + (h)) * HTB)
#define SB(b, h) (((b) * 4 + 2 + (h)) * HTB)
#define GLDS(g, o, l) __builtin_amdgcn_global_load_lds((const unsigned*)((g) + (size_t)(unsigned)(o)), (LAS unsigned*)(lds + (l)), 16, 0, 17)
#define STAGE_A(b, h, qq) do { GLDS(wb + (h) * 4096, (qq.x << 17) + voff, SA(b, h) + ldsw); GLDS(wb + (h) * 4096, (qq.y << 17) + voff, SA(b, h) + ldsw + 8192); } while (0)
#define STAGE_B(b, h, qq) do { GLDS(xb + (h) * 8192, (qq.x << 18) + voff, SB(b, h) + ldsw); GLDS(xb + (h) * 8192, (qq.y << 18) + voff, SB(b, h) + ldsw + 8192); } while (0)
#define LDA(dst, b, h) do { _Pragma("unroll") for (int m = 0; m < 4; ++m) _Pragma("unroll") for (int k = 0; k < 2; ++k) dst[m][k] = *(const LAS bf16x8*)(lds + SA(b, h) + aoff + m * 1024 + k * 8192); } while (0)
#define LDB(dst, b, h) do { _Pragma("unroll") for (int n = 0; n < 2; ++n) _Pragma("unroll") for (int k = 0; k < 2; ++k) dst[n][k] = *(const LAS bf16x8*)(lds + SB(b, h) + boff + n * 1024 + k * 8192); } while (0)
#define MMA2(ai) do { __builtin_amdgcn_s_setprio(1); _Pragma("unroll") for (int k = 0; k < 2; ++k) _Pragma("unroll") for (int m = 0; m < 4; ++m) _Pragma("unroll") for (int n = 0; n < 2; ++n) { \
        acc[ai][0][m][n] = __builtin_amdgcn_mfma_f32_16x16x32_bf16(B0[n][k], At[m][k], acc[ai][0][m][n], 0, 0, 0); \
        acc[ai][1][m][n] = __builtin_amdgcn_mfma_f32_16x16x32_bf16(B1[n][k], At[m][k], acc[ai][1][m][n], 0, 0, 0); } __builtin_amdgcn_s_setprio(0); } while (0)
#define MUST_PICK(kt) do { if (__builtin_expect(nq < 2 * (kt) + 2, 0)) poll_ready(fx, fw, td0, td1, queue, nq, 2 * (kt) + 2, call, tmo); } while (0)
#define SNAP(kt) do { if (__builtin_expect(nq < 2 * (kt) + 2, 0)) \
        __builtin_amdgcn_global_load_lds((lane < 32 ? fx : fw - 128) + lane * 4, queue + 256, 16, 0, 17); } while (0)
#define GETQ(kt) (*(volatile LAS u32x2*)(queue + 2 * (kt)))
#define FIXQ(qq, kt) do { if (__builtin_expect(__builtin_amdgcn_readfirstlane(qq.x & qq.y) < 0x10000, 0)) qq = GETQ(kt); } while (0)
#pragma unroll
    for (int a = 0; a < 2; ++a)
#pragma unroll
        for (int b = 0; b < 2; ++b)
#pragma unroll
            for (int m = 0; m < 4; ++m)
#pragma unroll
                for (int n = 0; n < 2; ++n) acc[a][b][m][n] = (f32x4){0.f, 0.f, 0.f, 0.f};
    bf16x8 At[4][2], B0[2][2], B1[2][2];
    LAS unsigned* queue = (LAS unsigned*)(lds + 8 * HTB + 4096);
    unsigned long long td0 = __ballot(lane >= q0 && lane < 2 * nt + 64 * main_tile && !(lane >= NH && (lane - NH) % 7 == skip));
    unsigned long long td1 = __ballot(main_tile && (lane + 64 - NH) % 7 != skip);
    int nq = 1 << 20;
    if (wid == 0) { nq = 0; *(volatile LAS u32x2*)(queue + 2 * lane) = (u32x2){0u, 0u}; *(volatile LAS u32x4*)(queue + 256 + 4 * lane) = (u32x4){0u, 0u, 0u, 0u}; poll_ready(fx, fw, td0, td1, queue, nq, 4, call, tmo); }
    asm volatile("" ::: "memory"); BAR; asm volatile("" ::: "memory");
    u32x2 qe = GETQ(0), qo = GETQ(1);
    STAMP(4);
    STAGE_B(0, 0, qe); STAGE_A(0, 0, qe); STAGE_B(0, 1, qe); STAGE_A(0, 1, qe);
    STAGE_B(1, 0, qo); STAGE_A(1, 0, qo); STAGE_B(1, 1, qo);
    if (wr == 1) BAR;
    WAIT_V(8); BAR; BAR;
    for (int t = 0; t < nt; t += 2) {
        if (t == nt - 2) nq = 1 << 20;
        SNAP(t + 3);
        qe = GETQ(t + 2);
        LDB(B0, 0, 0); LDA(At, 0, 0); LDB(B1, 0, 1); STAGE_A(1, 1, qo);
        WAIT_V(8); WAIT_L(0); BAR; SCHED; MMA2(0); SCHED; MUST_PICK(t + 2); BAR; SCHED;
        FIXQ(qe, t + 2);
        LDA(At, 0, 1); STAGE_B(0, 0, qe); STAGE_A(0, 0, qe); STAGE_B(0, 1, qe);
        WAIT_V(8); WAIT_L(0); BAR; SCHED; MMA2(1); BAR; SCHED;
        if (t == nt - 2)
            __builtin_amdgcn_global_load_lds(lane == 1 ? tmo : gate2, (LAS unsigned*)(lds + 8 * HTB + wid * 256), 4, 0, 17);
        SNAP(t + 4);
        qo = GETQ(t + 3);
        LDB(B0, 1, 0); LDA(At, 1, 0); LDB(B1, 1, 1); STAGE_A(0, 1, qe);
        WAIT_V(8); WAIT_L(0); BAR; SCHED; MMA2(0); SCHED; MUST_PICK(t + 3); BAR; SCHED;
        FIXQ(qo, t + 3);
        LDA(At, 1, 1); STAGE_B(1, 0, qo); STAGE_A(1, 0, qo); STAGE_B(1, 1, qo);
        WAIT_V(8); WAIT_L(0); BAR; SCHED; MMA2(1); BAR; SCHED;
    }
    STAMP(2);
    if (main_tile) break;
    if (wr == 0) BAR;
    if (it == 0) {
        const char* pb = ws + OFF_PK + (size_t)(cn * 8 + wid) * 32768;
        asm volatile("s_nop 4" : "+s"(pb));
#pragma unroll
        for (int i = 0; i < 32; ++i) st16<0>(pb + i * 1024, voff, __builtin_bit_cast(u32x4, acc[i >> 4][(i >> 3) & 1][(i >> 1) & 3][i & 1]));
    }
    WAIT_V(0);
    if (it == 0 && lane == 0) __hip_atomic_store((unsigned*)(ws + OFF_FD) + cn * 8 + wid, call, __ATOMIC_RELAXED, __HIP_MEMORY_SCOPE_SYSTEM);
    asm volatile("" ::: "memory"); BAR; asm volatile("" ::: "memory");
    }

    int tz = threadIdx.x; asm volatile("" : "+v"(tz));
    const int ln = tz & 63;
    const unsigned vo = ln * 16;
    if (main_tile) {
        const u32x2 gv = *(volatile LAS u32x2*)(lds + 8 * HTB + wid * 256);
        u32x4 b[16], pk[12];
#pragma unroll
        for (int bj = 0; bj < 2; ++bj)
#pragma unroll
            for (int n = 0; n < 2; ++n)
#pragma unroll
                for (int r = 0; r < 4; ++r)
#pragma unroll
                    for (int ai = 0; ai < 2; ++ai)
#pragma unroll
                        for (int m2 = 0; m2 < 2; ++m2)
                            b[bj * 8 + n * 4 + r][ai * 2 + m2] = cvt_pk_bf16(acc[ai][bj][2 * m2][n][r], acc[ai][bj][2 * m2 + 1][n][r]);
        asm volatile("" : "+v"(b[0]), "+v"(b[1]), "+v"(b[2]), "+v"(b[3]), "+v"(b[4]), "+v"(b[5]), "+v"(b[6]), "+v"(b[7]), "+v"(b[8]), "+v"(b[9]), "+v"(b[10]), "+v"(b[11]), "+v"(b[12]), "+v"(b[13]), "+v"(b[14]), "+v"(b[15]) :: "memory");
        if (wr == 0) BAR;
        bool bad = (unsigned)__builtin_amdgcn_readfirstlane(gv.y) == call;
        if ((unsigned)__builtin_amdgcn_readfirstlane(gv.x) < call) { wait_word(gate2, call, tmo); bad = (unsigned)__builtin_amdgcn_readfirstlane(ld4_sys(tmo)) == call; }
        const char* sl = ws + OFF_PS + (size_t)((rt * 8 + (cn & 7)) * 8 + wid) * UB;
        char* fl = p.ws[(unsigned)(rt >> 1) & 7u] + OFF_FP + (size_t)(((rt & 1) * 64 + cn) * 8 + wid) * FLS;
        const char* s0 = sl + 4096; const char* s1 = sl + 8192;
        const unsigned L = pack_base(b);
        st16<-4096>(s0, vo, pack_low(b, 0, L)); st16<-3072>(s0, vo, pack_low(b, 1, L)); st16<-2048>(s0, vo, pack_low(b, 2, L));
        WAIT_V(0);
        st8_flag(fl, call, LOWCNT << 15, ln);
        STAMP(6);
        asm volatile("" ::: "memory"); BAR; asm volatile("" ::: "memory");
        unsigned hdr = pack_unit(b, pk, L, lds + wid * HTB, ln);
        if (bad) {
            hdr = RAWCNT << 15;
#pragma unroll
            for (int f = 0; f < 16; ++f) b[f] = (u32x4){0x7fc07fc0u, 0x7fc07fc0u, 0x7fc07fc0u, 0x7fc07fc0u};
        }
        if (__builtin_expect(hdr_cnt(hdr) == RAWCNT, 0)) st_raw(sl, vo, b);
        else {
            st16<0>(s1, vo, pk[8]); st16<1024>(s1, vo, pk[9]); st_tail(s1, vo, pk[10], pk[11], hdr >> 25, ln);
            st16<-1024>(s0, vo, pk[3]); st16<0>(s0, vo, pk[4]); st16<1024>(s0, vo, pk[5]); st16<2048>(s0, vo, pk[6]); st16<3072>(s0, vo, pk[7]);
        }
        WAIT_V(0);
        st8_flag(fl, call, hdr, ln);
    } else {
        const int rh = slot < 8 ? 1 : 0;
        LAS unsigned char* dl = lds + wid * FIN_LDS;
#pragma unroll
        for (int i = 0; i < 8; ++i) *(LAS u32x4*)(dl + i * 1024 + ln * 16) = (u32x4){0u, 0u, 0u, 0u};
        STAMP(24);
        const char* fl = ws + OFF_FP + (size_t)((rh * 64 + cn) * 8 + wid) * FLS;
        const char* sl = p.ws[xcd] + OFF_PS + (size_t)(((p.rank * 2 + rh) * 8 + (cn & 7)) * 8 + wid) * UB;
        const unsigned ls = (unsigned)(size_t)dl + DEC_SLOT;
        if (xcd != p.rank) wait_word((const unsigned*)(ws + OFF_SD) + (((xcd - p.rank - 1) & 7) * 8 + (slot & 7)) * 32, call, tmo);
        unsigned hdr = wait_unit(fl, call, false, tmo);
        STAMP(25);
        dma16(sl, vo, ls); WAIT_V(0); dma16(sl + 1024, vo, ls + 1024); WAIT_V(0); dma16(sl + 2048, vo, ls + 2048);
        if (hdr_cnt(hdr) == LOWCNT) hdr = wait_unit(fl, call, true, tmo);
        const bool raw = hdr_cnt(hdr) == RAWCNT;
        const unsigned LL = (hdr & 0x7fffu) * 0x10001u, tl = hdr >> 25;
        const unsigned nanmask = ((unsigned)__builtin_amdgcn_readfirstlane(ld4_sys(tmo)) == call) ? 0x7fc07fc0u : 0u;
        u32x4 P[2], G;
        if (!raw) {
            if ((unsigned)ln < tl * 2u) dma16(sl + 10240, vo, ls + 10240);
            if (__builtin_expect(tl > 32u, 0)) { if ((unsigned)ln < (tl - 32u) * 2u) dma16(sl + 11264, vo, ls + 11264); }
            WAIT_V(0); dma16(sl + 8192, vo, ls + 8192);
            unpack_head(*(LAS unsigned*)(dl + DEC_TAIL + ln * 4), hdr, dl, ln, G);
            WAIT_V(0); dma16(sl + 9216, vo, ls + 9216);
            WAIT_V(0);
            P[0] = *(LAS u32x4*)(dl + DEC_SLOT + 8192 + ln * 16); P[1] = *(LAS u32x4*)(dl + DEC_SLOT + 9216 + ln * 16);
        }
        STAMP(26);
        const char* ob = p.out + ((size_t)(rh * 256 + wc * 32) * NN + cn * 256 + wr * 128) * 2;
        const unsigned oo = (ln >> 4) * (4 * NN * 2) + (ln & 15) * 16;
#pragma unroll
        for (int i = 0; i < 8; ++i) {
            u32x4 o[2];
            if (raw) { dma16(sl + i * 2048, vo, ls); dma16(sl + i * 2048 + 1024, vo, ls + 1024); WAIT_V(0);
                       o[0] = *(LAS u32x4*)(dl + DEC_SLOT + ln * 16); o[1] = *(LAS u32x4*)(dl + DEC_SLOT + 1024 + ln * 16); asm volatile("s_waitcnt lgkmcnt(0)" : "+v"(o[0]), "+v"(o[1]) :: "memory"); }
            else {
                if (i == 0 || i == 2 || i >= 4) WAIT_V(0);
                if (i == 0) dma16(sl + 3072, vo, ls + 3072);
                if (i == 2) dma16(sl + 4096, vo, ls + 4096);
                if (i >= 4 && i < 7) dma16(sl + (i + 1) * 1024, vo, ls + (i + 1) * 1024);
                unpack_piece(*(LAS u32x4*)(dl + DEC_SLOT + i * 1024 + ln * 16), P, G, i, dl, ln, LL, o);
            }
#pragma unroll
            for (int j = 0; j < 2; ++j) {
                const int f = 2 * i + j;
                u32x4 y;
#pragma unroll
                for (int cc = 0; cc < 4; ++cc) {
                    const unsigned pv = o[j][cc];
                    const f32x2 v = (f32x2){acc[cc >> 1][f >> 3][2 * (cc & 1)][(f >> 2) & 1][f & 3] + __uint_as_float(pv << 16),
                                            acc[cc >> 1][f >> 3][2 * (cc & 1) + 1][(f >> 2) & 1][f & 3] + __uint_as_float(pv & 0xffff0000u)};
                    const f32x2 g = gelu2(v);
                    y[cc] = cvt_pk_bf16(g.x, g.y) | nanmask;
                }
                st16<0>(ob + (size_t)((f >> 3) * 128 + ((f >> 2) & 1) * 16 + (f & 3)) * (NN * 2), oo, y);
            }
        }
    }
    STAMP(3);
}
}

extern "C" void kernel_launch(int rank, int nranks, ncclComm_t comm, hipStream_t stream,
                              void* const* d_in, const int* in_sizes, int n_in,
                              void* d_out, int out_size, const int* in_dtypes, int out_dtype,
                              void* const* d_out_all, void* const* d_ws_all, size_t ws_bytes) {
    (void)comm; (void)in_dtypes; (void)out_dtype; (void)d_out_all;
    if (nranks != NR || (unsigned)rank >= (unsigned)NR || n_in < 2 || in_sizes[0] != MM * KS || in_sizes[1] != KS * NN ||
        out_size != (MM / NR) * NN || ws_bytes < WS_NEED) {
        fprintf(stderr, "kernel_launch: unexpected problem geometry\n");
        return;
    }
    Params p;
    p.x = (const char*)d_in[0];
    p.w = (const char*)d_in[1];
    for (int i = 0; i < NR; ++i) p.ws[i] = (char*)d_ws_all[i];
    p.out = (char*)d_out;
    p.rank = rank; p.pad = 0;
    hipLaunchKernelGGL(fused_kernel, dim3(256), dim3(512), 0, stream, p);
    HIP_CHECK(hipGetLastError());
}
```

```cpp
#include <hip/hip_runtime.h>
#include <rccl/rccl.h>
#include <cstdio>
#include <cstdint>

#define HIP_CHECK(expr) do { hipError_t _e = (expr); if (_e != hipSuccess) { \
    fprintf(stderr, "HIP error %s at %s:%d\n", hipGetErrorString(_e), __FILE__, __LINE__); \
    return; } } while (0)

#define LAGXB 36
#define LAGWH 4
#define LAGXA 1
#define NW0 2
#define RAW0 2
#define FORCE_RAW 0
#define LWIN 6
#define NDEC 5
#define EARLYTK 1
#define PUSHP 0
#define MAINP 0
#define OWNB 1
#define FMAP 1
#define NLOW 3
#define PWIN 3
#define OWND 30
#define PD0 0
#define PDT 40
#define GATE_EARLY 6

namespace {
#define LAS __attribute__((address_space(3)))
typedef short bf16x8 __attribute__((ext_vector_type(8)));
typedef float f32x4 __attribute__((ext_vector_type(4)));
typedef float f32x2 __attribute__((ext_vector_type(2)));
typedef unsigned u32x4 __attribute__((ext_vector_type(4)));
typedef unsigned u32x2 __attribute__((ext_vector_type(2)));
typedef unsigned short u16x2 __attribute__((ext_vector_type(2)));

constexpr int NR = 8;
constexpr int MM = 4096, NN = 16384, KS = 512;
constexpr int NQ = 128;
constexpr int NH = 16;
constexpr int NU = NH * 22;
constexpr int NTASK = NH * 70;
constexpr unsigned long long DEADLINE = 20000000ull;
constexpr unsigned RAWCNT = 0x3ffu, LOWCNT = 0x3feu;

constexpr size_t OFF_CNT = 0;
constexpr size_t OFF_TMO = 8192;
constexpr size_t OFF_TK = 8192 + 1024;
constexpr int FSTR = 160;
constexpr size_t OFF_FX = 16384;
constexpr size_t OFF_FW = 32768;
constexpr size_t OFF_FD = 81920;
constexpr size_t OFF_SD = 131072;
constexpr size_t OFF_STAMP = 262144;
constexpr size_t UB = 16384, FLS = 128;
constexpr size_t XQ = 256 * 1024, WQ = 128 * 1024;
constexpr size_t OFF_XI = 1u << 20;
constexpr size_t OFF_WI = OFF_XI + (size_t)NQ * XQ;
constexpr size_t OFF_WD = OFF_WI + (size_t)NQ * WQ;
constexpr size_t OFF_PS = OFF_WD + (size_t)7 * NH * WQ;
constexpr size_t OFF_ST = OFF_PS + (size_t)1024 * UB;
constexpr size_t ST_PAR = (size_t)1120 * UB;
constexpr size_t OFF_PK = OFF_ST + 2 * ST_PAR;
constexpr size_t OFF_LZ = OFF_PK + (size_t)512 * 32768;
constexpr size_t OFF_FA = OFF_LZ + (size_t)14 * 8 * RAW0 * UB;
constexpr size_t OFF_FP = OFF_FA + (size_t)14 * NU * FLS;
constexpr size_t WS_NEED = OFF_FP + 1024 * FLS;
static_assert(OFF_FW + 64 * FSTR * 4 <= OFF_FD && OFF_FD + 2048 <= OFF_SD && OFF_SD + 56 * 128 <= OFF_STAMP, "layout");
static_assert(WS_NEED <= (size_t)161 << 20, "workspace");

struct Params {
    const char* x;
    const char* w;
    char* ws[NR];
    char* out;
    int rank;
    int pad;
};

template <int OFF> __device__ __forceinline__ void st16(const char* sbase, unsigned voff, u32x4 v) {
    asm volatile("global_store_dwordx4 %0, %1, %2 offset:%3 sc0 sc1\n\ts_nop 1" :: "v"(voff), "v"(v), "s"(sbase), "i"(OFF) : "memory");
}
__device__ __forceinline__ unsigned ld4_sys(const void* p) {
    unsigned v;
    asm volatile("global_load_dword %0, %1, off sc0 sc1\n\ts_waitcnt vmcnt(0)" : "=&v"(v) : "v"(p) : "memory");
    return v;
}
__device__ __forceinline__ u32x2 ld8_sys(const void* p) {
    u32x2 v;
    asm volatile("global_load_dwordx2 %0, %1, off sc0 sc1\n\ts_waitcnt vmcnt(0)" : "=&v"(v) : "v"(p) : "memory");
    return v;
}
__device__ __forceinline__ void st8_flag(void* p, unsigned call, unsigned hdr, int lane) {
    if (lane == 0) __hip_atomic_store((unsigned long long*)p, ((unsigned long long)hdr << 32) | call, __ATOMIC_RELAXED, __HIP_MEMORY_SCOPE_SYSTEM);
}
#define WAIT_V_(n) asm volatile("s_waitcnt vmcnt(" #n ")" ::: "memory")
#define WAIT_V(n) WAIT_V_(n)
#define WAIT_L(n) asm volatile("s_waitcnt lgkmcnt(" #n ")" ::: "memory")
#define BAR __builtin_amdgcn_s_barrier()
#define SCHED __builtin_amdgcn_sched_barrier(0)

__device__ __forceinline__ unsigned cvt_pk_bf16(float lo, float hi) {
    unsigned r; asm("v_cvt_pk_bf16_f32 %0, %1, %2" : "=v"(r) : "v"(lo), "v"(hi)); return r;
}
__device__ __forceinline__ f32x2 gelu2(f32x2 y) {
    const float k1 = -2.0f * 0.7978845608028654f * 1.4426950408889634f, k2 = k1 * 0.044715f;
    f32x2 z = y * (y * y * k2 + k1);
    f32x2 e; e.x = __builtin_amdgcn_exp2f(z.x); e.y = __builtin_amdgcn_exp2f(z.y);
    e = e + 1.0f;
    f32x2 r; r.x = __builtin_amdgcn_rcpf(e.x); r.y = __builtin_amdgcn_rcpf(e.y);
    return y * r;
}
__device__ __forceinline__ unsigned pk_sub(unsigned a, unsigned b) { return __builtin_bit_cast(unsigned, __builtin_bit_cast(u16x2, a) - __builtin_bit_cast(u16x2, b)); }
__device__ __forceinline__ unsigned pk_add(unsigned a, unsigned b) { return __builtin_bit_cast(unsigned, __builtin_bit_cast(u16x2, a) + __builtin_bit_cast(u16x2, b)); }
__device__ __forceinline__ unsigned pk_max(unsigned a, unsigned b) { return __builtin_bit_cast(unsigned, __builtin_elementwise_max(__builtin_bit_cast(u16x2, a), __builtin_bit_cast(u16x2, b))); }
__device__ __forceinline__ unsigned bfi(unsigned m, unsigned a, unsigned b) { return __builtin_amdgcn_bitop3_b32(m, a, b, 0xca); }
__device__ __forceinline__ unsigned rotl(unsigned a, int n) { return __builtin_amdgcn_alignbit(a, a, 32 - n); }
__device__ __forceinline__ unsigned rotr(unsigned a, int n) { return __builtin_amdgcn_alignbit(a, a, n); }
__device__ __forceinline__ unsigned umax(unsigned a, unsigned b) { return a > b ? a : b; }

__device__ __forceinline__ void dma16(const char* sbase, unsigned voff, unsigned ldst) {
    unsigned keep;
    asm volatile("s_mov_b32 %0, m0\n\ts_mov_b32 m0, %3\n\ts_nop 0\n\tglobal_load_lds_dwordx4 %1, %2 sc0 sc1\n\ts_mov_b32 m0, %0"
                 : "=&s"(keep) : "v"(voff), "s"(sbase), "s"(ldst) : "memory");
}

#define STAMP(slot) do {} while (0)

__device__ __forceinline__ bool give_up(unsigned* tmo, unsigned call, unsigned long long t0, unsigned& polls) {
    if ((++polls & 15u) != 0u) return false;
    if (__builtin_amdgcn_readfirstlane(__hip_atomic_load(tmo, __ATOMIC_RELAXED, __HIP_MEMORY_SCOPE_AGENT)) == call ||
        __builtin_amdgcn_s_memrealtime() - t0 > DEADLINE) {
        __hip_atomic_store(tmo, call, __ATOMIC_RELAXED, __HIP_MEMORY_SCOPE_AGENT);
        return true;
    }
    return false;
}
__device__ __forceinline__ unsigned hdr_cnt(unsigned hdr) { return (hdr >> 15) & 0x3ffu; }
__device__ __forceinline__ unsigned wait_unit(const char* flag, unsigned call, bool full, unsigned* tmo) {
    const unsigned long long t0 = __builtin_amdgcn_s_memrealtime(); unsigned polls = 0;
    for (;;) {
        const u32x2 v = ld8_sys(flag);
        const unsigned hdr = __builtin_amdgcn_readfirstlane(v.y);
        if ((unsigned)__builtin_amdgcn_readfirstlane(v.x) == call && !(full && hdr_cnt(hdr) == LOWCNT)) return hdr;
        __builtin_amdgcn_s_sleep(1);
        if (give_up(tmo, call, t0, polls)) return RAWCNT << 15;
    }
}
__device__ __forceinline__ void wait_word(const unsigned* w, unsigned call, unsigned* tmo) {
    const unsigned long long t0 = __builtin_amdgcn_s_memrealtime(); unsigned polls = 0;
    while ((unsigned)__builtin_amdgcn_readfirstlane(ld4_sys(w)) < call) {
        __builtin_amdgcn_s_sleep(2);
        if (give_up(tmo, call, t0, polls)) break;
    }
}

constexpr int SCR_DENSE = 8192;
__device__ __forceinline__ unsigned umin(unsigned a, unsigned b) { return a < b ? a : b; }
__device__ __forceinline__ unsigned wave_scan(unsigned inc) {
    inc += __builtin_amdgcn_update_dpp(0, (int)inc, 0x111, 0xf, 0xf, true);
    inc += __builtin_amdgcn_update_dpp(0, (int)inc, 0x112, 0xf, 0xf, true);
    inc += __builtin_amdgcn_update_dpp(0, (int)inc, 0x114, 0xf, 0xf, true);
    inc += __builtin_amdgcn_update_dpp(0, (int)inc, 0x118, 0xf, 0xf, true);
    inc += __builtin_amdgcn_update_dpp(0, (int)inc, 0x142, 0xa, 0xf, false);
    inc += __builtin_amdgcn_update_dpp(0, (int)inc, 0x143, 0xc, 0xf, false);
    return inc;
}
__device__ __forceinline__ unsigned pack_base(const u32x4 (&b)[16]) {
    unsigned mx = 0;
#pragma unroll
    for (int f = 0; f < 16; ++f) mx = pk_max(mx, b[f][f & 3] & 0x7fff7fffu);
    unsigned m = umax(mx & 0xffffu, mx >> 16);
    m = umax(m, (unsigned)__builtin_amdgcn_update_dpp(0, (int)m, 0x128, 0xf, 0xf, false));
    m = umax(m, (unsigned)__builtin_amdgcn_update_dpp(0, (int)m, 0x124, 0xf, 0xf, false));
    m = umax(m, (unsigned)__builtin_amdgcn_update_dpp(0, (int)m, 0x122, 0xf, 0xf, false));
    m = umax(m, (unsigned)__builtin_amdgcn_update_dpp(0, (int)m, 0x121, 0xf, 0xf, false));
    const unsigned m0 = __builtin_amdgcn_readlane(m, 0), m1 = __builtin_amdgcn_readlane(m, 16), m2 = __builtin_amdgcn_readlane(m, 32), m3 = __builtin_amdgcn_readlane(m, 48);
    const unsigned ms = umin(umax(umin(m0, m1), umin(m2, m3)), umin(umax(m0, m1), umax(m2, m3)));
    return umax(ms, 1023u) - 1023u;
}
__device__ __forceinline__ u32x4 pack_low(const u32x4 (&b)[16], int i, unsigned L) {
    const unsigned LL = L | (L << 16);
    u32x4 r;
#pragma unroll
    for (int c = 0; c < 4; ++c) r[c] = __builtin_amdgcn_perm(pk_sub(b[2 * i + (c >> 1)][2 * (c & 1) + 1], LL), pk_sub(b[2 * i + (c >> 1)][2 * (c & 1)], LL), 0x06040200u);
    return r;
}
__device__ __forceinline__ unsigned pack_unit(const u32x4 (&b)[16], u32x4 (&pk)[12], unsigned L, LAS unsigned char* scr, int lane) {
    const unsigned LL = L | (L << 16);
    const unsigned wp0 = (unsigned)(size_t)scr + lane * 4;
    unsigned wp = wp0;
    unsigned z[4], seg[4], len[4], F = 0;
#pragma unroll
    for (int s = 0; s < 4; ++s) {
        unsigned hb[8];
#pragma unroll
        for (int jj = 0; jj < 8; ++jj) {
            const int j = 8 * s + jj;
            const unsigned d0 = pk_sub(b[j >> 1][2 * (j & 1)], LL), d1 = pk_sub(b[j >> 1][2 * (j & 1) + 1], LL);
            pk[j >> 2][j & 3] = __builtin_amdgcn_perm(d1, d0, 0x06040200u);
            hb[jj] = __builtin_amdgcn_perm(d1, d0, 0x07050301u);
            const unsigned t = hb[jj] & 0x7c7c7c7cu; unsigned tmp;
            *(LAS unsigned*)(size_t)wp = t | (unsigned)((j & 3) | ((j >> 2) << 7));
            asm("v_min_u32 %0, 1, %2\n\tv_lshl_add_u32 %1, %0, 8, %1" : "=&v"(tmp), "+v"(wp) : "v"(t));
            if (jj & 3) hb[jj] = rotl(hb[jj], 2 * (jj & 3));
        }
        pk[8][s] = bfi(0x81818181u, hb[0], bfi(0x06060606u, hb[1], bfi(0x18181818u, hb[2], hb[3])));
        pk[9][s] = bfi(0x81818181u, hb[4], bfi(0x06060606u, hb[5], bfi(0x18181818u, hb[6], hb[7])));
        const unsigned ga = bfi(0x02020202u, hb[0], bfi(0x08080808u, hb[1], bfi(0x20202020u, hb[2], hb[3])));
        const unsigned gb = bfi(0x02020202u, hb[4], bfi(0x08080808u, hb[5], bfi(0x20202020u, hb[6], hb[7])));
        z[s] = ~bfi(0xaaaaaaaau, ga, gb >> 1);
    }
    unsigned e8[8], nz4[4];
#pragma unroll
    for (int i = 0; i < 8; ++i) e8[i] = *(LAS unsigned*)(scr + i * 256 + lane * 4);
#pragma unroll
    for (int s = 0; s < 4; ++s) {
        unsigned t = z[s] | (z[s] >> 1); t |= t << 2;
        nz4[s] = t & 0x44444444u; F |= s < 2 ? nz4[s] >> (2 - s) : nz4[s] << (s - 2);
    }
    const unsigned nrow = (wp - wp0) >> 8, mine = (__builtin_popcount(F) << 14) | nrow;
    const unsigned inc = wave_scan(mine), tot = __builtin_amdgcn_readlane(inc, 63), ngrp = tot & 0xfffu;
    const unsigned lo = 256u + ((((tot >> 12) + 31u) >> 5) << 2);
    unsigned cnt = ngrp, bad = 0;
    if (ngrp <= 512u) {
        LAS unsigned* dense = (LAS unsigned*)(scr + SCR_DENSE);
        const unsigned lanebits = ((lane & 7) << 15) | ((lane >> 3) << 23), base = (inc - mine) & 0xfffu;
#pragma unroll
        for (int i = 0; i < 8; ++i) dense[(unsigned)i < nrow ? base + i : 512 + lane] = e8[i] | lanebits;
        for (unsigned r = 8; __builtin_expect(__any(nrow > r), 0); ++r)
            dense[r < nrow ? base + r : 512 + lane] = *(LAS unsigned*)(scr + r * 256 + lane * 4) | lanebits;
        unsigned e2[2] = {dense[lane], dense[64 + lane]};
        *(LAS u32x4*)(scr + 256 + lane * 16) = (u32x4){0u, 0u, 0u, 0u};
        *(LAS unsigned*)(scr + lane * 4) = F;
#pragma unroll
        for (int s = 0; s < 4; ++s) { seg[s] = z[s] & 15u; len[s] = nz4[s] & 7u; }
#pragma unroll
        for (int i = 1; i < 8; ++i)
#pragma unroll
            for (int s = 0; s < 4; ++s) { seg[s] |= ((z[s] >> (4 * i)) & 15u) << len[s]; len[s] += (nz4[s] >> (4 * i)) & 7u; }
        unsigned bit = (inc - mine) >> 12;
#pragma unroll
        for (int s = 0; s < 4; ++s) {
            const unsigned long long v = (unsigned long long)seg[s] << (bit & 31u);
            LAS unsigned* w = (LAS unsigned*)(scr + 256) + (bit >> 5);
            __hip_atomic_fetch_or(w, (unsigned)v, __ATOMIC_RELAXED, __HIP_MEMORY_SCOPE_WAVEFRONT);
            __hip_atomic_fetch_or(w + 1, (unsigned)(v >> 32), __ATOMIC_RELAXED, __HIP_MEMORY_SCOPE_WAVEFRONT);
            bit += len[s];
        }
        LAS unsigned short* lst = (LAS unsigned short*)(scr + lo);
        const unsigned hz = ((0u - L) >> 10) & 31u;
        for (unsigned i0 = 0;;) {
            unsigned t[2], pos[2], at[2];
#pragma unroll
            for (int c = 0; c < 2; ++c) {
                const unsigned e = e2[c], ix = i0 + c * 64 + lane;
                t[c] = ix < ngrp ? e & 0x7c7c7c7cu : 0u;
                pos[c] = ((e & 0x380u) << 3) | ((e >> 16) & 0x380u) | ((e >> 11) & 0x70u) | ((e & 3u) << 2);
                at[c] = ix < ngrp ? ix : 1024u + lane;
            }
            for (;;) {
#pragma unroll
                for (int c = 0; c < 2; ++c) {
                    const unsigned b8 = __builtin_ctz(t[c] | 0x80000000u) & 24u, hi5 = (t[c] >> (b8 + 2)) & 31u;
                    const unsigned code = hi5 == hz ? 5u : (hi5 + 5u) & 31u;
                    bad = umax(bad, code);
                    lst[at[c]] = (unsigned short)((code << 13) | pos[c] | (b8 >> 3));
                    t[c] &= ~(0xffu << b8);
                }
                const unsigned long long b0 = __ballot(t[0] != 0), b1 = __ballot(t[1] != 0);
                if (!(b0 | b1)) break;
                at[0] = t[0] != 0 ? __builtin_amdgcn_mbcnt_hi((unsigned)(b0 >> 32), __builtin_amdgcn_mbcnt_lo((unsigned)b0, umin(cnt, 512u))) : 1024u + lane;
                cnt += __builtin_popcountll(b0);
                at[1] = t[1] != 0 ? __builtin_amdgcn_mbcnt_hi((unsigned)(b1 >> 32), __builtin_amdgcn_mbcnt_lo((unsigned)b1, umin(cnt, 512u))) : 1024u + lane;
                cnt += __builtin_popcountll(b1);
            }
            i0 += 128;
            if (__builtin_expect(i0 >= ngrp, 1)) break;
            e2[0] = dense[i0 + lane]; e2[1] = dense[i0 + 64 + lane];
        }
    }
    pk[10] = *(LAS u32x4*)(scr + lane * 16);
    pk[11] = *(LAS u32x4*)(scr + 1024 + lane * 16);
    const unsigned tl = (lo + 2u * cnt + 31u) >> 5;
    if (cnt > 512u || tl > 64u || __any(bad > 7u) || FORCE_RAW) return RAWCNT << 15;
    return L | (cnt << 15) | (tl << 25);
}
__device__ __forceinline__ void st_tail(const char* sb8k  , unsigned voff, u32x4 t0, u32x4 t1, unsigned tl, int lane) {
    if (__builtin_expect(tl > 32u, 0)) { if ((unsigned)lane < (tl - 32u) * 2u) st16<3072>(sb8k, voff, t1); }
    if ((unsigned)lane < tl * 2u) st16<2048>(sb8k, voff, t0);
}
__device__ __forceinline__ void st_raw(const char* slot, unsigned voff, const u32x4 (&b)[16]) {
    const char* s0 = slot + 4096; const char* s1 = slot + 12288;
    st16<-4096>(s0, voff, b[0]); st16<-3072>(s0, voff, b[1]); st16<-2048>(s0, voff, b[2]); st16<-1024>(s0, voff, b[3]);
    st16<0>(s0, voff, b[4]); st16<1024>(s0, voff, b[5]); st16<2048>(s0, voff, b[6]); st16<3072>(s0, voff, b[7]);
    st16<-4096>(s1, voff, b[8]); st16<-3072>(s1, voff, b[9]); st16<-2048>(s1, voff, b[10]); st16<-1024>(s1, voff, b[11]);
    st16<0>(s1, voff, b[12]); st16<1024>(s1, voff, b[13]); st16<2048>(s1, voff, b[14]); st16<3072>(s1, voff, b[15]);
}
__device__ __forceinline__ void ld4x16(f32x4& a, f32x4& b, f32x4& c, f32x4& d, const char* sb, unsigned voff) {
    asm volatile("global_load_dwordx4 %0, %4, %5 sc0 sc1\n\tglobal_load_dwordx4 %1, %4, %5 offset:1024 sc0 sc1\n\t"
                 "global_load_dwordx4 %2, %4, %5 offset:2048 sc0 sc1\n\tglobal_load_dwordx4 %3, %4, %5 offset:3072 sc0 sc1\n\ts_waitcnt vmcnt(0)"
                 : "=&v"(a), "=&v"(b), "=&v"(c), "=&v"(d) : "v"(voff), "s"(sb) : "memory");
}
constexpr int DEC_SLOT = 8192, DEC_TAIL = DEC_SLOT + 10240;
__device__ __forceinline__ void patch_apply(LAS unsigned char* dl, unsigned lo, unsigned hdr, int lane, bool zero) {
    const unsigned cnt = hdr_cnt(hdr), lut_hi = 0x0804007cu | ((((0u - (hdr & 0x7fffu)) >> 10) & 31u) << 10);
    for (unsigned c = 0; c < cnt; c += 128) {
        const unsigned w = *(LAS unsigned*)(dl + DEC_TAIL + lo + c * 2 + lane * 4);
#pragma unroll
        for (int hf = 0; hf < 2; ++hf) {
            const unsigned e = hf ? w >> 16 : w & 0xffffu;
            const unsigned at = c + lane * 2 + hf < cnt ? e & 0x1fffu : (unsigned)DEC_TAIL + lane;
            dl[at] = zero ? 0 : (unsigned char)__builtin_amdgcn_perm(lut_hi, 0x7874706cu, e >> 13);
        }
    }
}
__device__ __forceinline__ unsigned unpack_head(unsigned F, unsigned hdr, LAS unsigned char* dl, int lane, u32x4& G) {
    const unsigned mine = __builtin_popcount(F) << 2, inc = wave_scan(mine);
    const unsigned lo = 256u + ((((unsigned)__builtin_amdgcn_readlane(inc, 63) + 31u) >> 5) << 2);
    unsigned bit = inc - mine, sg[4];
#pragma unroll
    for (int s = 0; s < 4; ++s) {
        const LAS unsigned* w = (const LAS unsigned*)(dl + DEC_TAIL + 256) + (bit >> 5);
        sg[s] = __builtin_amdgcn_alignbit(w[1], w[0], bit);
        bit += __builtin_popcount(F & (0x11111111u << s)) << 2;
    }
    patch_apply(dl, lo, hdr, lane, false);
#pragma unroll
    for (int s = 0; s < 4; ++s) {
        const unsigned nz = (F >> s) & 0x11111111u, pre = nz * 0x11111110u;
        const unsigned xe = sg[s] & 0x0f0f0f0fu, xo = (sg[s] >> 4) & 0x0f0f0f0fu;
        const unsigned b0 = __builtin_amdgcn_perm(xo, xe, 0x05010400u), b1 = __builtin_amdgcn_perm(xo, xe, 0x07030602u);
        const unsigned ze = __builtin_amdgcn_perm(b1, b0, pre & 0x07070707u), zo = __builtin_amdgcn_perm(b1, b0, (pre >> 4) & 0x07070707u);
        G[s] = ~((ze | (zo << 4)) & ((nz << 4) - nz));
    }
    return lo;
}
__device__ __forceinline__ void unpack_piece(u32x4 lo, const u32x4 (&P)[2], u32x4 G, int i, LAS unsigned char* dl, int lane, unsigned LL, u32x4 (&o)[2]) {
    const u32x4 pt = *(LAS u32x4*)(dl + i * 1024 + lane * 16);
    const unsigned A = P[i & 1][i >> 1], g = G[i >> 1] << (i & 1);
    unsigned hb[4];
    hb[0] = bfi(0x83838383u, bfi(0x02020202u, g, A), pt[0]); hb[1] = bfi(0x83838383u, rotr(bfi(0x08080808u, g, A), 2), pt[1]);
    hb[2] = bfi(0x83838383u, rotr(bfi(0x20202020u, g, A), 4), pt[2]); hb[3] = bfi(0x83838383u, rotr(bfi(0x80808080u, g, A), 6), pt[3]);
#pragma unroll
    for (int j = 0; j < 4; ++j) {
        o[j >> 1][2 * (j & 1)] = pk_add(__builtin_amdgcn_perm(hb[j], lo[j], 0x05010400u), LL);
        o[j >> 1][2 * (j & 1) + 1] = pk_add(__builtin_amdgcn_perm(hb[j], lo[j], 0x07030602u), LL);
    }
}

__device__ __forceinline__ void unit_fetch(const Params& p, int h, int u, int lane, unsigned ldsbuf) {
    const bool isx = u < 16;
    const char* base = isx ? p.x + ((size_t)u * 256 * KS + h * 32) * 2 : p.w + ((size_t)h * 32 * NN + (u - 16) * 256) * 2;
    unsigned voff = isx ? (lane & 15) * (KS * 2) + (lane >> 4) * 16 : (lane >> 4) * (8 * NN * 2) + (lane & 15) * 16;
    const unsigned s1 = isx ? 16 * KS * 2 : NN * 2, s2 = isx ? 16 * KS * 2 : 256 - 7 * NN * 2;
#pragma unroll
    for (int i = 0; i < 16; ++i) {
        unsigned keep;
        asm volatile("s_mov_b32 %0, m0\n\ts_mov_b32 m0, %3\n\ts_nop 0\n\tglobal_load_lds_dwordx4 %1, %2\n\ts_mov_b32 m0, %0"
                     : "=&s"(keep) : "v"(voff), "s"(base), "s"(ldsbuf + i * 1024) : "memory");
        voff += (i == 7) ? s2 : s1;
    }
}
__device__ __forceinline__ void unit_read(LAS unsigned char* buf, int lane, u32x4 (&r)[16]) {
#pragma unroll
    for (int i = 0; i < 16; ++i) r[i] = *(LAS u32x4*)(buf + i * 1024 + lane * 16);
    asm volatile("s_waitcnt lgkmcnt(0)" : "+v"(r[0]), "+v"(r[1]), "+v"(r[2]), "+v"(r[3]), "+v"(r[4]), "+v"(r[5]), "+v"(r[6]), "+v"(r[7]),
                 "+v"(r[8]), "+v"(r[9]), "+v"(r[10]), "+v"(r[11]), "+v"(r[12]), "+v"(r[13]), "+v"(r[14]), "+v"(r[15]) :: "memory");
}
__device__ __forceinline__ void unit_format(int u, u32x4 (&r)[16]) {
    if (u >= 16) {
#pragma unroll
        for (int g = 0; g < 2; ++g) {
            u32x4 o[8];
#pragma unroll
            for (int i = 0; i < 8; ++i)
#pragma unroll
                for (int a = 0; a < 4; ++a)
                    o[i][a] = __builtin_amdgcn_perm(r[g * 8 + 2 * a + 1][i >> 1], r[g * 8 + 2 * a][i >> 1], (i & 1) ? 0x07060302u : 0x05040100u);
#pragma unroll
            for (int i = 0; i < 8; ++i) r[g * 8 + i] = o[i];
        }
    }
}
struct Tabs { unsigned ord[NU]; unsigned task[NTASK]; };
constexpr Tabs make_tabs() {
    Tabs t{}; int n = 0;
    for (int v = 0; v < 8 * NW0; ++v) t.ord[n++] = (v >> 3) << 8 | (v & 7);
    for (int key = 0; key < 8 * NH + 512; ++key)
        for (int v = 0; v < 22; ++v) {
            const int d = key - (v < 8 ? ((v & 2) ? LAGWH : 0) : v < 15 ? LAGXA : LAGXB);
            if (d >= 0 && d % 8 == 0 && d / 8 < NH && !(v < 8 && d / 8 < NW0)) t.ord[n++] = (unsigned)((d / 8) << 8 | v);
        }
    n = 0;
    for (int k = 0; k < NU; ++k) {
        const unsigned h = t.ord[k] >> 8, v = t.ord[k] & 255, e = k | h << 9;
        for (unsigned l = 0; l < (v < 8 ? 7u : 1u); ++l) t.task[n++] = e | v << 13 | l << 18;
    }
    return t;
}
__constant__ const Tabs g_tabs = make_tabs();
__device__ __forceinline__ int x_tile(int v, int src) { return v >= 15 ? 2 * ((src + v - 14) & 7) + 1 : 2 * ((src + v - 7) & 7); }
__device__ __forceinline__ int st_slot(int h, int v, int link) { return v < 8 ? (link * NH + h) * 8 + v : 896 + h * 14 + v - 8; }

__device__ __forceinline__ void producer(const Params& p, LAS unsigned char* buf, int n0, int n1, int nlead, int lane, unsigned* cntp) {
    char* ws = p.ws[p.rank];
    const unsigned voff = lane * 16;
    const unsigned long long ts = __builtin_amdgcn_s_memrealtime();
    char* peer = p.ws[(p.rank + 1 + (lane & 7)) & 7];
    unsigned call = 0;
    for (int n = n0; n < NTASK; n = n == n0 ? n1 : n + 576) {
        const bool first = n < 56 * RAW0;
        const unsigned e = first ? (n / 7) | ((n / 7) >> 3) << 9 | ((n / 7) & 7) << 13 | (n % 7) << 18 : g_tabs.task[n];
        const int k = e & 511, h = (e >> 9) & 15, v = (e >> 13) & 31, link = e >> 18;
        const bool isx = v >= 8;
        const unsigned dest = (unsigned)(p.rank + 1 + link) & 7u;
        const int t = isx ? x_tile(v, p.rank) : dest * 8 + v;
        while (__builtin_amdgcn_s_memrealtime() - ts < (unsigned)(n < nlead ? 0 : PD0 + (n - nlead) * PDT / 16)) __builtin_amdgcn_s_sleep(2);
        unit_fetch(p, h, isx ? t : 16 + t, lane, (unsigned)(size_t)buf);
        if (n == n0) call = __builtin_amdgcn_readfirstlane(ld4_sys(cntp)) + 1u;
        WAIT_V(0);
        u32x4 r[16];
        unit_read(buf, lane, r);
        unit_format(isx ? 0 : 16, r);
        unsigned hdr = RAWCNT << 15;
        if (first) {
            const char* s0 = p.ws[dest] + OFF_LZ + (size_t)(((call & 1u) * 7 + 6 - link) * 8 * RAW0 + k) * UB + 4096; const char* s1 = s0 + 8192;
            st16<-4096>(s0, voff, r[0]); st16<-3072>(s0, voff, r[1]); st16<-2048>(s0, voff, r[2]); WAIT_V(PWIN); st16<-1024>(s0, voff, r[3]);
            WAIT_V(PWIN); st16<0>(s0, voff, r[4]); WAIT_V(PWIN); st16<1024>(s0, voff, r[5]); WAIT_V(PWIN); st16<2048>(s0, voff, r[6]); WAIT_V(PWIN); st16<3072>(s0, voff, r[7]);
            WAIT_V(PWIN); st16<-4096>(s1, voff, r[8]); WAIT_V(PWIN); st16<-3072>(s1, voff, r[9]); WAIT_V(PWIN); st16<-2048>(s1, voff, r[10]); WAIT_V(PWIN); st16<-1024>(s1, voff, r[11]);
            WAIT_V(PWIN); st16<0>(s1, voff, r[12]); WAIT_V(PWIN); st16<1024>(s1, voff, r[13]); WAIT_V(PWIN); st16<2048>(s1, voff, r[14]); WAIT_V(PWIN); st16<3072>(s1, voff, r[15]);
        } else {
            const char* sl = ws + OFF_ST + (call & 1u) * ST_PAR + (size_t)st_slot(h, v, link) * UB;
            u32x4 pk[12];
            hdr = pack_unit(r, pk, pack_base(r), buf, lane);
            if (__builtin_expect(hdr_cnt(hdr) == RAWCNT, 0)) st_raw(sl, voff, r);
            else {
                const char* s0 = sl + 4096; const char* s1 = sl + 8192;
                st16<-4096>(s0, voff, pk[0]); st16<-3072>(s0, voff, pk[1]); st16<-2048>(s0, voff, pk[2]); st16<-1024>(s0, voff, pk[3]);
                st16<0>(s0, voff, pk[4]); st16<1024>(s0, voff, pk[5]); st16<2048>(s0, voff, pk[6]); st16<3072>(s0, voff, pk[7]);
                st16<0>(s1, voff, pk[8]); st16<1024>(s1, voff, pk[9]); st_tail(s1, voff, pk[10], pk[11], hdr >> 25, lane);
            }
        }
        WAIT_V(0);
        if (isx ? lane < 7 : lane == link)
            __hip_atomic_store((unsigned long long*)(peer + OFF_FA + (size_t)(((call & 1u) * 7 + 6 - lane) * NU + k) * FLS), ((unsigned long long)hdr << 32) | call,
                               __ATOMIC_RELAXED, __HIP_MEMORY_SCOPE_SYSTEM);
        if (!isx) {
            st_raw(ws + OFF_WD + (size_t)(link * NH + h) * WQ + (size_t)v * UB, voff, r);
            WAIT_V(0);
            if (lane == 0) __hip_atomic_store((unsigned*)(ws + OFF_FW) + t * FSTR + h, call, __ATOMIC_RELAXED, __HIP_MEMORY_SCOPE_SYSTEM);
        }
        WAIT_L(0);
    }
}
__device__ __forceinline__ void own_image(const Params& p, LAS unsigned char* buf, int n, int lane, unsigned call) {
    char* ws = p.ws[p.rank];
    const int h = n / 24, v = n - h * 24, t = v < 16 ? v : p.rank * 8 + v - 16;
    unit_fetch(p, h, v < 16 ? t : 16 + t, lane, (unsigned)(size_t)buf);
    WAIT_V(0);
    u32x4 r[16];
    unit_read(buf, lane, r);
    unit_format(v, r);
    st_raw(ws + (v < 16 ? OFF_XI + (size_t)h * XQ : OFF_WI + (size_t)h * WQ) + (size_t)(v & 15) * UB, lane * 16, r);
    WAIT_V(0);
    if (lane == 0) __hip_atomic_store((unsigned*)(ws + (v < 16 ? OFF_FX : OFF_FW)) + t * FSTR + h, call, __ATOMIC_RELAXED, __HIP_MEMORY_SCOPE_SYSTEM);
}

constexpr int DEC_LDS = 24 * 1024;
struct PullCtl { u32x4 desc[16]; unsigned full[7]; unsigned tail; };
__device__ __forceinline__ void decoder(const Params& p, LAS unsigned char* dl, LAS PullCtl* ctl, int xcd, int li, int mem, int lane, unsigned call) {
    char* ws = p.ws[p.rank];
    unsigned* tmo = (unsigned*)(ws + OFF_TMO);
    const unsigned voff = lane * 16;
    const int src = (p.rank + 1 + li) & 7;
    const char* fa = ws + OFF_FA + (size_t)((call & 1u) * 7 + li) * NU * FLS;
    unsigned* tk = (unsigned*)(ws + OFF_TK) + li * 32;
    const unsigned tk0 = (call - 1u) * (NU - 8 * RAW0) - (RAW0 + NDEC) * 8;
#pragma unroll
    for (int i = 0; i < 8; ++i) *(LAS u32x4*)(dl + i * 1024 + lane * 16) = (u32x4){0u, 0u, 0u, 0u};
    unsigned k = (RAW0 + mem) * 8 + xcd, n = 0;
    while (k < (unsigned)NU) {
        const int e = g_tabs.ord[k], h = e >> 8, v = e & 255;
        const unsigned hdr = wait_unit(fa + (size_t)k * FLS, call, false, tmo);
        ++n;
        if (lane == 0) {
            const unsigned pos = __hip_atomic_fetch_add(&ctl->tail, 1u, __ATOMIC_RELAXED, __HIP_MEMORY_SCOPE_WORKGROUP);
            *(volatile LAS u32x4*)&ctl->desc[pos & 15] = (u32x4){pos + 1u, k | (unsigned)st_slot(h, v, 6 - li) << 16, hdr, (unsigned)mem | (n << 8)};
        }
        const bool isx = v >= 8, raw = hdr_cnt(hdr) == RAWCNT;
        const int q = NH + h * 7 + li, t = isx ? x_tile(v, src) : v;
        const char* dst = ws + (isx ? OFF_XI + (size_t)q * XQ : OFF_WI + (size_t)q * WQ) + (size_t)t * UB;
        unsigned* flag = (unsigned*)(ws + (isx ? OFF_FX : OFF_FW)) + (isx ? t : p.rank * 8 + t) * FSTR + q;
        unsigned tn = 1u;
        if (EARLYTK && lane == 0) asm volatile("global_atomic_add %0, %1, %0, off sc0\n\ts_waitcnt vmcnt(0)" : "+v"(tn) : "v"(tk) : "memory");
        while ((unsigned)__builtin_amdgcn_readfirstlane(*(volatile LAS unsigned*)&ctl->full[mem]) != n) __builtin_amdgcn_s_sleep(1);
        asm volatile("" ::: "memory");
        unsigned lo = 0;
        if (__builtin_expect(raw, 0)) {
            u32x4 r[16];
            unit_read(dl + DEC_SLOT, lane, r);
            st_raw(dst, voff, r);
        } else {
            const unsigned F = *(LAS unsigned*)(dl + DEC_TAIL + lane * 4), LL = (hdr & 0x7fffu) * 0x10001u;
            u32x4 P[2], G, lw[8];
            P[0] = *(LAS u32x4*)(dl + DEC_SLOT + 8192 + lane * 16); P[1] = *(LAS u32x4*)(dl + DEC_SLOT + 9216 + lane * 16);
#pragma unroll
            for (int i = 0; i < 8; ++i) lw[i] = *(LAS u32x4*)(dl + DEC_SLOT + i * 1024 + lane * 16);
            lo = unpack_head(F, hdr, dl, lane, G);
#pragma unroll
            for (int i = 0; i < 8; ++i) {
                u32x4 o[2]; unpack_piece(lw[i], P, G, i, dl, lane, LL, o);
                const char* sb = dst + i * 2048;
                st16<0>(sb, voff, o[0]); st16<1024>(sb, voff, o[1]);
            }
        }
        WAIT_V(0);
        if (lane == 0) __hip_atomic_store(flag, call, __ATOMIC_RELAXED, __HIP_MEMORY_SCOPE_SYSTEM);
        if (!EARLYTK && lane == 0) asm volatile("global_atomic_add %0, %1, %0, off sc0\n\ts_waitcnt vmcnt(0)" : "+v"(tn) : "v"(tk) : "memory");
        if (!raw) patch_apply(dl, lo, hdr, lane, true);
        WAIT_L(0);
        k = __builtin_amdgcn_readfirstlane(tn) - tk0;
    }
    if (lane == 0) {
        const unsigned pos = __hip_atomic_fetch_add(&ctl->tail, 1u, __ATOMIC_RELAXED, __HIP_MEMORY_SCOPE_WORKGROUP);
        *(volatile LAS u32x4*)&ctl->desc[pos & 15] = (u32x4){pos + 1u, 0u, 0u, 0x80u};
    }
}
__device__ __forceinline__ void opener(const Params& p, int li, int h, int ct, int lane, unsigned call) {
    char* ws = p.ws[p.rank];
    wait_unit(ws + OFF_FA + (size_t)(((call & 1u) * 7 + li) * NU + h * 8 + ct) * FLS, call, false, (unsigned*)(ws + OFF_TMO));
    const char* src = ws + OFF_LZ + (size_t)(((call & 1u) * 7 + li) * 8 * RAW0 + h * 8 + ct) * UB;
    const int q = NH + h * 7 + li;
    const char* dst = ws + OFF_WI + (size_t)q * WQ + (size_t)ct * UB;
    for (int i = 0; i < 4; ++i) {
        f32x4 t[4]; ld4x16(t[0], t[1], t[2], t[3], src + i * 4096, lane * 16);
        const char* d = dst + i * 4096;
        st16<0>(d, lane * 16, __builtin_bit_cast(u32x4, t[0])); st16<1024>(d, lane * 16, __builtin_bit_cast(u32x4, t[1]));
        st16<2048>(d, lane * 16, __builtin_bit_cast(u32x4, t[2])); st16<3072>(d, lane * 16, __builtin_bit_cast(u32x4, t[3]));
    }
    WAIT_V(0);
    if (lane == 0) __hip_atomic_store((unsigned*)(ws + OFF_FW) + (p.rank * 8 + ct) * FSTR + q, call, __ATOMIC_RELAXED, __HIP_MEMORY_SCOPE_SYSTEM);
}
__device__ __forceinline__ void loader(const Params& p, LAS unsigned char* lds, LAS PullCtl* ctl, int li, int lane, unsigned call) {
    const char* st = p.ws[(unsigned)(p.rank + 1 + li) & 7u] + OFF_ST + (call & 1u) * ST_PAR;
    const unsigned voff = lane * 16;
    unsigned pos = 0, done = 0, pend = 0, idle = 0;
#define LANDED() do { *(volatile LAS unsigned*)&ctl->full[pend & 7u] = pend >> 8; pend = 0; } while (0)
    while (done < NDEC || pend) {
        const u32x4 d = *(volatile LAS u32x4*)&ctl->desc[pos & 15];
        if ((unsigned)__builtin_amdgcn_readfirstlane(d.x) != pos + 1u) {
            if (!pend) __builtin_amdgcn_s_sleep(1);
            else if (idle == 0) WAIT_V(4);
            else if (idle == 1) WAIT_V(2);
            else { WAIT_V(0); LANDED(); }
            ++idle;
            continue;
        }
        idle = 0;
        const unsigned ks = __builtin_amdgcn_readfirstlane(d.y), hdr = __builtin_amdgcn_readfirstlane(d.z), w = __builtin_amdgcn_readfirstlane(d.w);
        ++pos;
        if (w & 0x80u) { ++done; continue; }
        if ((ks & 0xffffu) == NU - 1 - GATE_EARLY && lane < 8)
            __hip_atomic_store((unsigned*)(p.ws[p.rank] + OFF_SD) + (li * 8 + lane) * 32, call, __ATOMIC_RELAXED, __HIP_MEMORY_SCOPE_AGENT);
        const char* s = st + (size_t)(ks >> 16) * UB;
        const unsigned l = (unsigned)(size_t)lds + (w & 7u) * DEC_LDS + DEC_SLOT;
#pragma unroll
        for (int i = 0; i < 10; ++i) {
            WAIT_V(LWIN);
            if (i == LWIN && pend) LANDED();
            dma16(s + i * 1024, voff, l + i * 1024);
        }
        static_assert(LWIN < 10, "window");
        if (__builtin_expect(hdr_cnt(hdr) == RAWCNT, 0)) {
#pragma unroll
            for (int i = 10; i < 16; ++i) { WAIT_V(LWIN); dma16(s + i * 1024, voff, l + i * 1024); }
        } else {
            const unsigned tl = hdr >> 25;
            WAIT_V(LWIN);
            if ((unsigned)lane < tl * 2u) dma16(s + 10240, voff, l + 10240);
            if (__builtin_expect(tl > 32u, 0)) { if ((unsigned)lane < (tl - 32u) * 2u) dma16(s + 11264, voff, l + 11264); }
        }
        pend = w;
    }
}

__device__ __forceinline__ void post_ready(bool r0, bool r1, unsigned long long& td0, unsigned long long& td1, LAS unsigned* queue, int& nq, unsigned lane) {
    const unsigned long long a0 = __ballot(r0) & td0, a1 = __ballot(r1) & td1;
    if ((a0 >> lane) & 1) ((volatile LAS unsigned*)queue)[__builtin_amdgcn_mbcnt_hi((unsigned)(a0 >> 32), __builtin_amdgcn_mbcnt_lo((unsigned)a0, nq))] = lane | 0x10000u;
    nq += __builtin_popcountll(a0);
    if ((a1 >> lane) & 1) ((volatile LAS unsigned*)queue)[__builtin_amdgcn_mbcnt_hi((unsigned)(a1 >> 32), __builtin_amdgcn_mbcnt_lo((unsigned)a1, nq))] = lane | 0x10040u;
    nq += __builtin_popcountll(a1);
    td0 &= ~a0; td1 &= ~a1;
}
__device__ __forceinline__ void poll_ready(const unsigned* fx, const unsigned* fw, unsigned long long& td0, unsigned long long& td1,
                                           LAS unsigned* queue, int& nq, int need, unsigned call, unsigned* tmo) {
    const unsigned lane = __builtin_amdgcn_mbcnt_hi(~0u, __builtin_amdgcn_mbcnt_lo(~0u, 0u));
    {
        const volatile LAS unsigned* sn = queue + 256 + lane;
        const unsigned x0 = sn[0], x1 = sn[64], w0 = sn[128], w1 = sn[192];
        post_ready(x0 == call && w0 == call, x1 == call && w1 == call, td0, td1, queue, nq, lane);
    }
    const unsigned long long t0 = __builtin_amdgcn_s_memrealtime(); unsigned polls = 0;
    bool all = false;
    while (nq < need) {
        unsigned x0, x1, w0, w1;
        asm volatile("global_load_dword %0, %4, %5 sc0 sc1\n\tglobal_load_dword %1, %4, %5 offset:256 sc0 sc1\n\t"
                     "global_load_dword %2, %4, %6 sc0 sc1\n\tglobal_load_dword %3, %4, %6 offset:256 sc0 sc1\n\ts_waitcnt vmcnt(0)"
                     : "=&v"(x0), "=&v"(x1), "=&v"(w0), "=&v"(w1) : "v"(lane * 4), "s"(fx), "s"(fw) : "memory");
        post_ready(all || (x0 >= call && w0 >= call), all || (x1 >= call && w1 >= call), td0, td1, queue, nq, lane);
        if (nq >= need) break;
        all = give_up(tmo, call, t0, polls);
        __builtin_amdgcn_s_sleep(1);
    }
    WAIT_L(0);
}

constexpr int HTB = 16384;
constexpr int FIN_LDS = 20480;

__global__ __launch_bounds__(512, 2) void fused_kernel(const Params p) {
    __shared__ __attribute__((aligned(16))) unsigned char shm[8 * FIN_LDS];
    LAS unsigned char* lds = (LAS unsigned char*)shm;
    const int wid = __builtin_amdgcn_readfirstlane(threadIdx.x >> 6), lane = threadIdx.x & 63;
    const int xcd = blockIdx.x & 7, slot = blockIdx.x >> 3;
    char* ws = p.ws[p.rank];
    unsigned* tmo = (unsigned*)(ws + OFF_TMO);
    unsigned* cntp = (unsigned*)(ws + OFF_CNT) + blockIdx.x * 8 + wid;
    STAMP(0);
    const bool main_tile = slot >= 16;
    const int fsrc = FMAP ? slot & 7 : xcd, fct = FMAP ? xcd : slot & 7;
    LAS PullCtl* ctl = (LAS PullCtl*)(lds + 5 * DEC_LDS + 2 * HTB);
    static_assert(NDEC <= 5 && RAW0 <= 2 && RAW0 <= NW0, "puller CU: load wave, 5 unpack waves, 2 waves for the opening units");
    if (slot < 7) { if (wid < 2) ((volatile LAS unsigned*)ctl)[wid * 64 + lane] = 0u; WAIT_L(0); BAR; }
    {
        constexpr int NOPEN = 56 * RAW0, NLEAD = NOPEN + MAINP * 128;
        int n0 = -1, n1 = NTASK; LAS unsigned char* buf = lds + wid * HTB;
        if (slot < 7) { if (PUSHP && wid >= 6 && wid < 6 + RAW0) { n0 = ((wid - 6) * 8 + xcd) * 7 + slot; buf = lds + 5 * DEC_LDS + (wid - 6) * HTB; } }
        else if (main_tile) { if (wid >= 4 && wid < 4 + MAINP) n0 = NOPEN + (wid - 4) * 128 + (blockIdx.x - 128); }
        else { const int i = wid * 72 + (blockIdx.x - 56);
               if (PUSHP) { n0 = NLEAD + i; n1 = n0 + 576; } else { n0 = i < NOPEN ? i : i + MAINP * 128; n1 = i + 576 + MAINP * 128; } }
        if (n0 >= 0) producer(p, buf, n0, n1, NLEAD, lane, cntp);
    }
    const unsigned call = __builtin_amdgcn_readfirstlane(ld4_sys(cntp)) + 1u;
    if (lane == 0) __hip_atomic_store(cntp, call, __ATOMIC_RELAXED, __HIP_MEMORY_SCOPE_AGENT);
    if (slot < 7) {
        if (wid == 0) { __builtin_amdgcn_s_setprio(3); loader(p, lds, ctl, slot, lane, call); __builtin_amdgcn_s_setprio(0); }
        else if (wid <= NDEC) decoder(p, lds + (wid - 1) * DEC_LDS, ctl, xcd, slot, wid - 1, lane, call);
        else if (wid >= 6 && wid < 6 + RAW0) opener(p, slot, wid - 6, xcd, lane, call);
    }
    if (main_tile) {
        if (slot >= 24 && wid >= 1 && wid <= 6) { for (int i = 1; i < wid; ++i) __builtin_amdgcn_s_sleep(OWND); own_image(p, lds + wid * HTB, (wid - 1) * 64 + (blockIdx.x - 192), lane, call); }
    } else {
        STAMP(1);
        if (fsrc == p.rank) return;
        asm volatile("s_waitcnt vmcnt(0) lgkmcnt(0)" ::: "memory");
        BAR;
    }

    const int wr = wid >> 2, wc = wid & 3;
    f32x4 acc[2][2][4][2];
    int rt, cn;
    const unsigned* gate2 = cntp;
    if (slot < 8) {
        rt = 0; cn = fsrc * 8 + fct;
        wait_word((const unsigned*)(ws + OFF_FD) + cn * 8 + wid, call, tmo);
        const char* pb = ws + OFF_PK + (size_t)(cn * 8 + wid) * 32768;
#pragma unroll
        for (int i = 0; i < 8; ++i)
            ld4x16(acc[i >> 2][(i >> 1) & 1][(i & 1) * 2][0], acc[i >> 2][(i >> 1) & 1][(i & 1) * 2][1], acc[i >> 2][(i >> 1) & 1][(i & 1) * 2 + 1][0], acc[i >> 2][(i >> 1) & 1][(i & 1) * 2 + 1][1], pb + i * 4096, lane * 16);
    } else
    for (int it = 0; it < 2; ++it) {
    int nt, skip;
    const char* wimg;
    if (main_tile) {
        const int idx = slot - 16;
        rt = ((xcd >> 1) * 2 + ((idx >> 2) & 1)) * 2 + (idx >> 3);
        cn = p.rank * 8 + (xcd & 1) * 4 + (idx & 3);
        const int m = rt >> 1;
        skip = (m - p.rank - 1) & 7; nt = (m == p.rank) ? 64 : 56;
        wimg = ws + OFF_WI;
    } else {
        rt = p.rank * 2 + 1 - it;
        cn = fsrc * 8 + fct;
        skip = 7; nt = 8;
        wimg = ws + OFF_WD + (size_t)((fsrc - p.rank - 1) & 7) * NH * WQ;
    }
    const unsigned* fx = (const unsigned*)(ws + OFF_FX) + rt * FSTR;
    const unsigned* fw = (const unsigned*)(ws + OFF_FW) + cn * FSTR;
    gate2 = (skip == 7) ? cntp : fw + NH + skip;
    const char* xb = ws + OFF_XI + (size_t)(rt * 16 + wid) * 1024;
    const char* wb = wimg + (size_t)((cn & 7) * 16 + wr * 8 + wc) * 1024;
    const unsigned voff = lane * 16;
    const unsigned ldsw = wid * 1024;
    const int aoff = wr * 4096 + lane * 16, boff = wc * 2048 + lane * 16;
#define SA(b, h) (((b) * 4 + (h)) * HTB)
#define SB(b, h) (((b) * 4 + 2 + (h)) * HTB)
#define GLDS(g, o, l) __builtin_amdgcn_global_load_lds((const unsigned*)((g) + (size_t)(unsigned)(o)), (LAS unsigned*)(lds + (l)), 16, 0, 17)
#define STAGE_A(b, h, qq) do { GLDS(wb + (h) * 4096, (qq.x << 17) + voff, SA(b, h) + ldsw); GLDS(wb + (h) * 4096, (qq.y << 17) + voff, SA(b, h) + ldsw + 8192); } while (0)
#define STAGE_B(b, h, qq) do { GLDS(xb + (h) * 8192, (qq.x << 18) + voff, SB(b, h) + ldsw); GLDS(xb + (h) * 8192, (qq.y << 18) + voff, SB(b, h) + ldsw + 8192); } while (0)
#define LDA(dst, b, h) do { _Pragma("unroll") for (int m = 0; m < 4; ++m) _Pragma("unroll") for (int k = 0; k < 2; ++k) dst[m][k] = *(const LAS bf16x8*)(lds + SA(b, h) + aoff + m * 1024 + k * 8192); } while (0)
#define LDB(dst, b, h) do { _Pragma("unroll") for (int n = 0; n < 2; ++n) _Pragma("unroll") for (int k = 0; k < 2; ++k) dst[n][k] = *(const LAS bf16x8*)(lds + SB(b, h) + boff + n * 1024 + k * 8192); } while (0)
#define MMA2(ai) do { __builtin_amdgcn_s_setprio(1); _Pragma("unroll") for (int k = 0; k < 2; ++k) _Pragma("unroll") for (int m = 0; m < 4; ++m) _Pragma("unroll") for (int n = 0; n < 2; ++n) { \
        acc[ai][0][m][n] = __builtin_amdgcn_mfma_f32_16x16x32_bf16(B0[n][k], At[m][k], acc[ai][0][m][n], 0, 0, 0); \
        acc[ai][1][m][n] = __builtin_amdgcn_mfma_f32_16x16x32_bf16(B1[n][k], At[m][k], acc[ai][1][m][n], 0, 0, 0); } __builtin_amdgcn_s_setprio(0); } while (0)
#define MUST_PICK(kt) do { if (__builtin_expect(nq < 2 * (kt) + 2, 0)) poll_ready(fx, fw, td0, td1, queue, nq, 2 * (kt) + 2, call, tmo); } while (0)
#define SNAP(kt) do { if (__builtin_expect(nq < 2 * (kt) + 2, 0)) \
        __builtin_amdgcn_global_load_lds((lane < 32 ? fx : fw - 128) + lane * 4, queue + 256, 16, 0, 17); } while (0)
#define GETQ(kt) (*(volatile LAS u32x2*)(queue + 2 * (kt)))
#define FIXQ(qq, kt) do { if (__builtin_expect(__builtin_amdgcn_readfirstlane(qq.x & qq.y) < 0x10000, 0)) qq = GETQ(kt); } while (0)
#pragma unroll
    for (int a = 0; a < 2; ++a)
#pragma unroll
        for (int b = 0; b < 2; ++b)
#pragma unroll
            for (int m = 0; m < 4; ++m)
#pragma unroll
                for (int n = 0; n < 2; ++n) acc[a][b][m][n] = (f32x4){0.f, 0.f, 0.f, 0.f};
    bf16x8 At[4][2], B0[2][2], B1[2][2];
    LAS unsigned* queue = (LAS unsigned*)(lds + 8 * HTB + 4096);
    unsigned long long td0 = __ballot(lane < 2 * nt && !(lane >= NH && (lane - NH) % 7 == skip));
    unsigned long long td1 = __ballot(main_tile && (lane + 64 - NH) % 7 != skip);
    int nq = 1 << 20;
    if (wid == 0) { nq = 0; *(volatile LAS u32x4*)(queue + 4 * lane) = (u32x4){0u, 0u, 0u, 0u}; *(volatile LAS u32x4*)(queue + 256 + 4 * lane) = (u32x4){0u, 0u, 0u, 0u}; poll_ready(fx, fw, td0, td1, queue, nq, 4, call, tmo); }
    asm volatile("" ::: "memory"); BAR; asm volatile("" ::: "memory");
    u32x2 qe = GETQ(0), qo = GETQ(1);
    STAMP(4);
    STAGE_B(0, 0, qe); STAGE_A(0, 0, qe); STAGE_B(0, 1, qe); STAGE_A(0, 1, qe);
    STAGE_B(1, 0, qo); STAGE_A(1, 0, qo); STAGE_B(1, 1, qo);
    if (wr == 1) BAR;
    WAIT_V(8); BAR; BAR;
    for (int t = 0; t < nt; t += 2) {
        if (t == nt - 2) nq = 1 << 20;
        SNAP(t + 3);
        qe = GETQ(t + 2);
        LDB(B0, 0, 0); LDA(At, 0, 0); LDB(B1, 0, 1); STAGE_A(1, 1, qo);
        WAIT_V(8); WAIT_L(0); BAR; SCHED; MMA2(0); SCHED; MUST_PICK(t + 2); BAR; SCHED;
        FIXQ(qe, t + 2);
        LDA(At, 0, 1); STAGE_B(0, 0, qe); STAGE_A(0, 0, qe); STAGE_B(0, 1, qe);
        WAIT_V(8); WAIT_L(0); BAR; SCHED; MMA2(1); BAR; SCHED;
        if (t == nt - 2)
            __builtin_amdgcn_global_load_lds(lane == 1 ? tmo : gate2, (LAS unsigned*)(lds + 8 * HTB + wid * 256), 4, 0, 17);
        SNAP(t + 4);
        qo = GETQ(t + 3);
        LDB(B0, 1, 0); LDA(At, 1, 0); LDB(B1, 1, 1); STAGE_A(0, 1, qe);
        WAIT_V(8); WAIT_L(0); BAR; SCHED; MMA2(0); SCHED; MUST_PICK(t + 3); BAR; SCHED;
        FIXQ(qo, t + 3);
        LDA(At, 1, 1); STAGE_B(1, 0, qo); STAGE_A(1, 0, qo); STAGE_B(1, 1, qo);
        WAIT_V(8); WAIT_L(0); BAR; SCHED; MMA2(1); BAR; SCHED;
    }
    STAMP(2);
    if (main_tile) break;
    if (wr == 0) BAR;
    if (it == 0) {
        const char* pb = ws + OFF_PK + (size_t)(cn * 8 + wid) * 32768;
        asm volatile("s_nop 4" : "+s"(pb));
#pragma unroll
        for (int i = 0; i < 32; ++i) st16<0>(pb + i * 1024, voff, __builtin_bit_cast(u32x4, acc[i >> 4][(i >> 3) & 1][(i >> 1) & 3][i & 1]));
    }
    WAIT_V(0);
    if (it == 0 && lane == 0) __hip_atomic_store((unsigned*)(ws + OFF_FD) + cn * 8 + wid, call, __ATOMIC_RELAXED, __HIP_MEMORY_SCOPE_SYSTEM);
    asm volatile("" ::: "memory"); BAR; asm volatile("" ::: "memory");
    }

    int tz = threadIdx.x; asm volatile("" : "+v"(tz));
    const int ln = tz & 63;
    const unsigned vo = ln * 16;
    const bool own = main_tile && (rt >> 1) == p.rank;
    int computing = __builtin_amdgcn_readfirstlane(main_tile && !own); asm volatile("" : "+s"(computing));
    if (computing != 1) {
        const int rh = own ? rt & 1 : slot < 8 ? 1 : 0;
        LAS unsigned char* dl = lds + wid * FIN_LDS;
        const char* sl = p.ws[fsrc] + OFF_PS + (size_t)(((p.rank * 2 + rh) * 8 + (cn & 7)) * 8 + wid) * UB;
        const unsigned ls = (unsigned)(size_t)dl + DEC_SLOT;
        unsigned hdr = 0;
        if (own) { if (wr == 0) BAR; }
        else {
#pragma unroll
            for (int i = 0; i < 8; ++i) *(LAS u32x4*)(dl + i * 1024 + ln * 16) = (u32x4){0u, 0u, 0u, 0u};
            STAMP(24);
            const char* fl = ws + OFF_FP + (size_t)((rh * 64 + cn) * 8 + wid) * FLS;
            wait_word((const unsigned*)(ws + OFF_SD) + (((fsrc - p.rank - 1) & 7) * 8 + fct) * 32, call, tmo);
            hdr = wait_unit(fl, call, false, tmo);
            STAMP(25);
            dma16(sl, vo, ls); WAIT_V(0); dma16(sl + 1024, vo, ls + 1024); WAIT_V(0); dma16(sl + 2048, vo, ls + 2048);
            if (NLOW > 3) { WAIT_V(0); dma16(sl + 3072, vo, ls + 3072); }
            if (NLOW > 4) { WAIT_V(0); dma16(sl + 4096, vo, ls + 4096); }
            if (hdr_cnt(hdr) == LOWCNT) hdr = wait_unit(fl, call, true, tmo);
        }
        const bool raw = hdr_cnt(hdr) == RAWCNT;
        const unsigned LL = (hdr & 0x7fffu) * 0x10001u, tl = hdr >> 25;
        if (!raw && !own) {
            WAIT_V(0);
            if ((unsigned)ln < tl * 2u) dma16(sl + 10240, vo, ls + 10240);
            if (__builtin_expect(tl > 32u, 0)) { if ((unsigned)ln < (tl - 32u) * 2u) dma16(sl + 11264, vo, ls + 11264); }
        }
        const unsigned nanmask = ((unsigned)__builtin_amdgcn_readfirstlane(ld4_sys(tmo)) == call) ? 0x7fc07fc0u : 0u;
        u32x4 P[2], G;
        if (!raw && !own) {
            dma16(sl + 8192, vo, ls + 8192);
            unpack_head(*(LAS unsigned*)(dl + DEC_TAIL + ln * 4), hdr, dl, ln, G);
            WAIT_V(0); dma16(sl + 9216, vo, ls + 9216);
            P[0] = *(LAS u32x4*)(dl + DEC_SLOT + 8192 + ln * 16);
        }
        STAMP(26);
        const char* ob = p.out + ((size_t)(rh * 256 + wc * 32) * NN + cn * 256 + wr * 128) * 2;
        const unsigned oo = (ln >> 4) * (4 * NN * 2) + (ln & 15) * 16;
#pragma unroll
        for (int st = 0; st < 8; ++st) {
            const int i = st == 1 ? 2 : st == 2 ? 1 : st;
            u32x4 o[2];
            if (own) o[0] = o[1] = (u32x4){0u, 0u, 0u, 0u};
            else if (raw) { dma16(sl + i * 2048, vo, ls); dma16(sl + i * 2048 + 1024, vo, ls + 1024); WAIT_V(0);
                       o[0] = *(LAS u32x4*)(dl + DEC_SLOT + ln * 16); o[1] = *(LAS u32x4*)(dl + DEC_SLOT + 1024 + ln * 16); asm volatile("s_waitcnt lgkmcnt(0)" : "+v"(o[0]), "+v"(o[1]) :: "memory"); }
            else {
#define NEXTP(j) dma16(sl + (j) * 1024, vo, ls + (j) * 1024)
                if (st == 2) { WAIT_V(4); NEXTP(NLOW); P[1] = *(LAS u32x4*)(dl + DEC_SLOT + 9216 + ln * 16); }
                if (NLOW == 3 && st >= 3) { WAIT_V(2); if (st < 7) NEXTP(st + 1); }
                if (NLOW == 4 && st == 4) { WAIT_V(4); NEXTP(5); }
                if (NLOW == 4 && st > 4) { WAIT_V(2); if (st < 7) NEXTP(st + 1); }
                if (NLOW == 5 && st == 4) { WAIT_V(4); NEXTP(6); }
                if (NLOW == 5 && st == 6) { WAIT_V(4); NEXTP(7); }
                if (NLOW == 5 && st == 7) WAIT_V(2);
                unpack_piece(*(LAS u32x4*)(dl + DEC_SLOT + i * 1024 + ln * 16), P, G, i, dl, ln, LL, o);
            }
#pragma unroll
            for (int j = 0; j < 2; ++j) {
                const int f = 2 * i + j;
                u32x4 y;
#pragma unroll
                for (int cc = 0; cc < 4; ++cc) {
                    const unsigned pv = o[j][cc];
                    const f32x2 v = (f32x2){acc[cc >> 1][f >> 3][2 * (cc & 1)][(f >> 2) & 1][f & 3] + __uint_as_float(pv << 16),
                                            acc[cc >> 1][f >> 3][2 * (cc & 1) + 1][(f >> 2) & 1][f & 3] + __uint_as_float(pv & 0xffff0000u)};
                    const f32x2 g = gelu2(v);
                    y[cc] = cvt_pk_bf16(g.x, g.y) | nanmask;
                }
                st16<0>(ob + (size_t)((f >> 3) * 128 + ((f >> 2) & 1) * 16 + (f & 3)) * (NN * 2), oo, y);
            }
        }
    } else {
        const u32x2 gv = *(volatile LAS u32x2*)(lds + 8 * HTB + wid * 256);
        u32x4 b[16], pk[12];
#pragma unroll
        for (int bj = 0; bj < 2; ++bj)
#pragma unroll
            for (int n = 0; n < 2; ++n)
#pragma unroll
                for (int r = 0; r < 4; ++r)
#pragma unroll
                    for (int ai = 0; ai < 2; ++ai)
#pragma unroll
                        for (int m2 = 0; m2 < 2; ++m2)
                            b[bj * 8 + n * 4 + r][ai * 2 + m2] = cvt_pk_bf16(acc[ai][bj][2 * m2][n][r], acc[ai][bj][2 * m2 + 1][n][r]);
        asm volatile("" : "+v"(b[0]), "+v"(b[1]), "+v"(b[2]), "+v"(b[3]), "+v"(b[4]), "+v"(b[5]), "+v"(b[6]), "+v"(b[7]), "+v"(b[8]), "+v"(b[9]), "+v"(b[10]), "+v"(b[11]), "+v"(b[12]), "+v"(b[13]), "+v"(b[14]), "+v"(b[15]) :: "memory");
        if (wr == 0) BAR;
        bool bad = (unsigned)__builtin_amdgcn_readfirstlane(gv.y) == call;
        if ((unsigned)__builtin_amdgcn_readfirstlane(gv.x) < call) { wait_word(gate2, call, tmo); bad = (unsigned)__builtin_amdgcn_readfirstlane(ld4_sys(tmo)) == call; }
        const char* sl = ws + OFF_PS + (size_t)((rt * 8 + (cn & 7)) * 8 + wid) * UB;
        char* fl = p.ws[(unsigned)(rt >> 1) & 7u] + OFF_FP + (size_t)(((rt & 1) * 64 + cn) * 8 + wid) * FLS;
        const char* s0 = sl + 4096; const char* s1 = sl + 8192;
        const unsigned L = pack_base(b);
        st16<-4096>(s0, vo, pack_low(b, 0, L)); st16<-3072>(s0, vo, pack_low(b, 1, L)); st16<-2048>(s0, vo, pack_low(b, 2, L));
        if (NLOW > 3) st16<-1024>(s0, vo, pack_low(b, 3, L));
        if (NLOW > 4) st16<0>(s0, vo, pack_low(b, 4, L));
        WAIT_V(0);
        st8_flag(fl, call, LOWCNT << 15, ln);
        STAMP(6);
        asm volatile("" ::: "memory"); BAR; asm volatile("" ::: "memory");
        unsigned hdr = pack_unit(b, pk, L, lds + wid * HTB, ln);
        if (bad) {
            hdr = RAWCNT << 15;
#pragma unroll
            for (int f = 0; f < 16; ++f) b[f] = (u32x4){0x7fc07fc0u, 0x7fc07fc0u, 0x7fc07fc0u, 0x7fc07fc0u};
        }
        if (__builtin_expect(hdr_cnt(hdr) == RAWCNT, 0)) st_raw(sl, vo, b);
        else {
            st16<0>(s1, vo, pk[8]); st16<1024>(s1, vo, pk[9]); st_tail(s1, vo, pk[10], pk[11], hdr >> 25, ln);
            if (NLOW <= 3) st16<-1024>(s0, vo, pk[3]);
            if (NLOW <= 4) st16<0>(s0, vo, pk[4]);
            st16<1024>(s0, vo, pk[5]); st16<2048>(s0, vo, pk[6]); st16<3072>(s0, vo, pk[7]);
        }
        WAIT_V(0);
        st8_flag(fl, call, hdr, ln);
    }
    STAMP(3);
}
}

extern "C" void kernel_launch(int rank, int nranks, ncclComm_t comm, hipStream_t stream,
                              void* const* d_in, const int* in_sizes, int n_in,
                              void* d_out, int out_size, const int* in_dtypes, int out_dtype,
                              void* const* d_out_all, void* const* d_ws_all, size_t ws_bytes) {
    (void)comm; (void)in_dtypes; (void)out_dtype; (void)d_out_all;
    if (nranks != NR || (unsigned)rank >= (unsigned)NR || n_in < 2 || in_sizes[0] != MM * KS || in_sizes[1] != KS * NN ||
        out_size != (MM / NR) * NN || ws_bytes < WS_NEED) {
        fprintf(stderr, "kernel_launch: unexpected problem geometry\n");
        return;
    }
    Params p;
    p.x = (const char*)d_in[0];
    p.w = (const char*)d_in[1];
    for (int i = 0; i < NR; ++i) p.ws[i] = (char*)d_ws_all[i];
    p.out = (char*)d_out;
    p.rank = rank; p.pad = 0;
    hipLaunchKernelGGL(fused_kernel, dim3(256), dim3(512), 0, stream, p);
    HIP_CHECK(hipGetLastError());
}
```

```cpp
#include <hip/hip_runtime.h>
#include <rccl/rccl.h>
#include <cstdio>
#include <cstdint>

#define HIP_CHECK(expr) do { hipError_t _e = (expr); if (_e != hipSuccess) { \
    fprintf(stderr, "HIP error %s at %s:%d\n", hipGetErrorString(_e), __FILE__, __LINE__); \
    return; } } while (0)

#define LAGXB 36
#define LAGOWN 69
#define LAGWH 4
#define LAGXA 1
#define NW0 2
#define RAW0 2
#define FORCE_RAW 0
#define LWIN 6
#define NDEC 5
#define EARLYTK 1
#define PUSHP 0
#define MAINP 0
#define OWNB 1
#define NLOW 3
#define OWNM 0x0f
#define PWIN 3
#define OWND 30
#define PD0 0
#define PDT 40
#define GATE_EARLY 6

namespace {
#define LAS __attribute__((address_space(3)))
typedef short bf16x8 __attribute__((ext_vector_type(8)));
typedef float f32x4 __attribute__((ext_vector_type(4)));
typedef float f32x2 __attribute__((ext_vector_type(2)));
typedef unsigned u32x4 __attribute__((ext_vector_type(4)));
typedef unsigned u32x2 __attribute__((ext_vector_type(2)));
typedef unsigned short u16x2 __attribute__((ext_vector_type(2)));

constexpr int NR = 8;
constexpr int MM = 4096, NN = 16384, KS = 512;
constexpr int NQ = 128;
constexpr int NH = 16;
constexpr int NU = NH * 22;
constexpr int NTASK = NH * 70;
constexpr unsigned long long DEADLINE = 20000000ull;
constexpr unsigned RAWCNT = 0x3ffu, LOWCNT = 0x3feu;

constexpr size_t OFF_CNT = 0;
constexpr size_t OFF_TMO = 8192;
constexpr size_t OFF_TK = 8192 + 1024;
constexpr int FSTR = 160;
constexpr size_t OFF_FX = 16384;
constexpr size_t OFF_FW = 32768;
constexpr size_t OFF_FD = 81920;
constexpr size_t OFF_SD = 131072;
constexpr size_t OFF_STAMP = 262144;
constexpr size_t UB = 16384, FLS = 128;
constexpr size_t XQ = 256 * 1024, WQ = 128 * 1024;
constexpr size_t OFF_XI = 1u << 20;
constexpr size_t OFF_WI = OFF_XI + (size_t)NQ * XQ;
constexpr size_t OFF_WD = OFF_WI + (size_t)NQ * WQ;
constexpr size_t OFF_PS = OFF_WD + (size_t)7 * NH * WQ;
constexpr size_t OFF_ST = OFF_PS + (size_t)1024 * UB;
constexpr size_t ST_PAR = (size_t)1120 * UB;
constexpr size_t OFF_PK = OFF_ST + 2 * ST_PAR;
constexpr size_t OFF_LZ = OFF_PK + (size_t)640 * 32768;
constexpr size_t OFF_FA = OFF_LZ + (size_t)14 * 8 * RAW0 * UB;
constexpr size_t OFF_FP = OFF_FA + (size_t)14 * NU * FLS;
constexpr size_t WS_NEED = OFF_FP + 1024 * FLS;
static_assert(OFF_FW + 64 * FSTR * 4 <= OFF_FD && OFF_FD + 2560 <= OFF_SD && OFF_SD + 56 * 128 <= OFF_STAMP, "layout");
static_assert(WS_NEED <= (size_t)161 << 20, "workspace");

struct Params {
    const char* x;
    const char* w;
    char* ws[NR];
    char* out;
    int rank;
    int pad;
};

template <int OFF> __device__ __forceinline__ void st16(const char* sbase, unsigned voff, u32x4 v) {
    asm volatile("global_store_dwordx4 %0, %1, %2 offset:%3 sc0 sc1\n\ts_nop 1" :: "v"(voff), "v"(v), "s"(sbase), "i"(OFF) : "memory");
}
__device__ __forceinline__ unsigned ld4_sys(const void* p) {
    unsigned v;
    asm volatile("global_load_dword %0, %1, off sc0 sc1\n\ts_waitcnt vmcnt(0)" : "=&v"(v) : "v"(p) : "memory");
    return v;
}
__device__ __forceinline__ u32x2 ld8_sys(const void* p) {
    u32x2 v;
    asm volatile("global_load_dwordx2 %0, %1, off sc0 sc1\n\ts_waitcnt vmcnt(0)" : "=&v"(v) : "v"(p) : "memory");
    return v;
}
__device__ __forceinline__ void st8_flag(void* p, unsigned call, unsigned hdr, int lane) {
    if (lane == 0) __hip_atomic_store((unsigned long long*)p, ((unsigned long long)hdr << 32) | call, __ATOMIC_RELAXED, __HIP_MEMORY_SCOPE_SYSTEM);
}
#define WAIT_V_(n) asm volatile("s_waitcnt vmcnt(" #n ")" ::: "memory")
#define WAIT_V(n) WAIT_V_(n)
#define WAIT_L(n) asm volatile("s_waitcnt lgkmcnt(" #n ")" ::: "memory")
#define BAR __builtin_amdgcn_s_barrier()
#define SCHED __builtin_amdgcn_sched_barrier(0)

__device__ __forceinline__ unsigned cvt_pk_bf16(float lo, float hi) {
    unsigned r; asm("v_cvt_pk_bf16_f32 %0, %1, %2" : "=v"(r) : "v"(lo), "v"(hi)); return r;
}
__device__ __forceinline__ f32x2 gelu2(f32x2 y) {
    const float k1 = -2.0f * 0.7978845608028654f * 1.4426950408889634f, k2 = k1 * 0.044715f;
    f32x2 z = y * (y * y * k2 + k1);
    f32x2 e; e.x = __builtin_amdgcn_exp2f(z.x); e.y = __builtin_amdgcn_exp2f(z.y);
    e = e + 1.0f;
    f32x2 r; r.x = __builtin_amdgcn_rcpf(e.x); r.y = __builtin_amdgcn_rcpf(e.y);
    return y * r;
}
__device__ __forceinline__ unsigned pk_sub(unsigned a, unsigned b) { return __builtin_bit_cast(unsigned, __builtin_bit_cast(u16x2, a) - __builtin_bit_cast(u16x2, b)); }
__device__ __forceinline__ unsigned pk_add(unsigned a, unsigned b) { return __builtin_bit_cast(unsigned, __builtin_bit_cast(u16x2, a) + __builtin_bit_cast(u16x2, b)); }
__device__ __forceinline__ unsigned pk_max(unsigned a, unsigned b) { return __builtin_bit_cast(unsigned, __builtin_elementwise_max(__builtin_bit_cast(u16x2, a), __builtin_bit_cast(u16x2, b))); }
__device__ __forceinline__ unsigned bfi(unsigned m, unsigned a, unsigned b) { return __builtin_amdgcn_bitop3_b32(m, a, b, 0xca); }
__device__ __forceinline__ unsigned rotl(unsigned a, int n) { return __builtin_amdgcn_alignbit(a, a, 32 - n); }
__device__ __forceinline__ unsigned rotr(unsigned a, int n) { return __builtin_amdgcn_alignbit(a, a, n); }
__device__ __forceinline__ unsigned umax(unsigned a, unsigned b) { return a > b ? a : b; }

__device__ __forceinline__ void dma16(const char* sbase, unsigned voff, unsigned ldst) {
    unsigned keep;
    asm volatile("s_mov_b32 %0, m0\n\ts_mov_b32 m0, %3\n\ts_nop 0\n\tglobal_load_lds_dwordx4 %1, %2 sc0 sc1\n\ts_mov_b32 m0, %0"
                 : "=&s"(keep) : "v"(voff), "s"(sbase), "s"(ldst) : "memory");
}

#define STAMP(slot) do {} while (0)

__device__ __forceinline__ bool give_up(unsigned* tmo, unsigned call, unsigned long long t0, unsigned& polls) {
    if ((++polls & 15u) != 0u) return false;
    if (__builtin_amdgcn_readfirstlane(__hip_atomic_load(tmo, __ATOMIC_RELAXED, __HIP_MEMORY_SCOPE_AGENT)) == call ||
        __builtin_amdgcn_s_memrealtime() - t0 > DEADLINE) {
        __hip_atomic_store(tmo, call, __ATOMIC_RELAXED, __HIP_MEMORY_SCOPE_AGENT);
        return true;
    }
    return false;
}
__device__ __forceinline__ unsigned hdr_cnt(unsigned hdr) { return (hdr >> 15) & 0x3ffu; }
__device__ __forceinline__ unsigned wait_unit(const char* flag, unsigned call, bool full, unsigned* tmo) {
    const unsigned long long t0 = __builtin_amdgcn_s_memrealtime(); unsigned polls = 0;
    for (;;) {
        const u32x2 v = ld8_sys(flag);
        const unsigned hdr = __builtin_amdgcn_readfirstlane(v.y);
        if ((unsigned)__builtin_amdgcn_readfirstlane(v.x) == call && !(full && hdr_cnt(hdr) == LOWCNT)) return hdr;
        __builtin_amdgcn_s_sleep(1);
        if (give_up(tmo, call, t0, polls)) return RAWCNT << 15;
    }
}
__device__ __forceinline__ void wait_word(const unsigned* w, unsigned call, unsigned* tmo) {
    const unsigned long long t0 = __builtin_amdgcn_s_memrealtime(); unsigned polls = 0;
    while ((unsigned)__builtin_amdgcn_readfirstlane(ld4_sys(w)) < call) {
        __builtin_amdgcn_s_sleep(2);
        if (give_up(tmo, call, t0, polls)) break;
    }
}

constexpr int SCR_DENSE = 8192;
__device__ __forceinline__ unsigned umin(unsigned a, unsigned b) { return a < b ? a : b; }
__device__ __forceinline__ unsigned wave_scan(unsigned inc) {
    inc += __builtin_amdgcn_update_dpp(0, (int)inc, 0x111, 0xf, 0xf, true);
    inc += __builtin_amdgcn_update_dpp(0, (int)inc, 0x112, 0xf, 0xf, true);
    inc += __builtin_amdgcn_update_dpp(0, (int)inc, 0x114, 0xf, 0xf, true);
    inc += __builtin_amdgcn_update_dpp(0, (int)inc, 0x118, 0xf, 0xf, true);
    inc += __builtin_amdgcn_update_dpp(0, (int)inc, 0x142, 0xa, 0xf, false);
    inc += __builtin_amdgcn_update_dpp(0, (int)inc, 0x143, 0xc, 0xf, false);
    return inc;
}
__device__ __forceinline__ unsigned pack_base(const u32x4 (&b)[16]) {
    unsigned mx = 0;
#pragma unroll
    for (int f = 0; f < 16; ++f) mx = pk_max(mx, b[f][f & 3] & 0x7fff7fffu);
    unsigned m = umax(mx & 0xffffu, mx >> 16);
    m = umax(m, (unsigned)__builtin_amdgcn_update_dpp(0, (int)m, 0x128, 0xf, 0xf, false));
    m = umax(m, (unsigned)__builtin_amdgcn_update_dpp(0, (int)m, 0x124, 0xf, 0xf, false));
    m = umax(m, (unsigned)__builtin_amdgcn_update_dpp(0, (int)m, 0x122, 0xf, 0xf, false));
    m = umax(m, (unsigned)__builtin_amdgcn_update_dpp(0, (int)m, 0x121, 0xf, 0xf, false));
    const unsigned m0 = __builtin_amdgcn_readlane(m, 0), m1 = __builtin_amdgcn_readlane(m, 16), m2 = __builtin_amdgcn_readlane(m, 32), m3 = __builtin_amdgcn_readlane(m, 48);
    const unsigned ms = umin(umax(umin(m0, m1), umin(m2, m3)), umin(umax(m0, m1), umax(m2, m3)));
    return umax(ms, 1023u) - 1023u;
}
__device__ __forceinline__ u32x4 pack_low(const u32x4 (&b)[16], int i, unsigned L) {
    const unsigned LL = L | (L << 16);
    u32x4 r;
#pragma unroll
    for (int c = 0; c < 4; ++c) r[c] = __builtin_amdgcn_perm(pk_sub(b[2 * i + (c >> 1)][2 * (c & 1) + 1], LL), pk_sub(b[2 * i + (c >> 1)][2 * (c & 1)], LL), 0x06040200u);
    return r;
}
__device__ __forceinline__ unsigned pack_unit(const u32x4 (&b)[16], u32x4 (&pk)[12], unsigned L, LAS unsigned char* scr, int lane) {
    const unsigned LL = L | (L << 16);
    const unsigned wp0 = (unsigned)(size_t)scr + lane * 4;
    unsigned wp = wp0;
    unsigned z[4], seg[4], len[4], F = 0;
#pragma unroll
    for (int s = 0; s < 4; ++s) {
        unsigned hb[8];
#pragma unroll
        for (int jj = 0; jj < 8; ++jj) {
            const int j = 8 * s + jj;
            const unsigned d0 = pk_sub(b[j >> 1][2 * (j & 1)], LL), d1 = pk_sub(b[j >> 1][2 * (j & 1) + 1], LL);
            pk[j >> 2][j & 3] = __builtin_amdgcn_perm(d1, d0, 0x06040200u);
            hb[jj] = __builtin_amdgcn_perm(d1, d0, 0x07050301u);
            const unsigned t = hb[jj] & 0x7c7c7c7cu; unsigned tmp;
            *(LAS unsigned*)(size_t)wp = t | (unsigned)((j & 3) | ((j >> 2) << 7));
            asm("v_min_u32 %0, 1, %2\n\tv_lshl_add_u32 %1, %0, 8, %1" : "=&v"(tmp), "+v"(wp) : "v"(t));
            if (jj & 3) hb[jj] = rotl(hb[jj], 2 * (jj & 3));
        }
        pk[8][s] = bfi(0x81818181u, hb[0], bfi(0x06060606u, hb[1], bfi(0x18181818u, hb[2], hb[3])));
        pk[9][s] = bfi(0x81818181u, hb[4], bfi(0x06060606u, hb[5], bfi(0x18181818u, hb[6], hb[7])));
        const unsigned ga = bfi(0x02020202u, hb[0], bfi(0x08080808u, hb[1], bfi(0x20202020u, hb[2], hb[3])));
        const unsigned gb = bfi(0x02020202u, hb[4], bfi(0x08080808u, hb[5], bfi(0x20202020u, hb[6], hb[7])));
        z[s] = ~bfi(0xaaaaaaaau, ga, gb >> 1);
    }
    unsigned e8[8], nz4[4];
#pragma unroll
    for (int i = 0; i < 8; ++i) e8[i] = *(LAS unsigned*)(scr + i * 256 + lane * 4);
#pragma unroll
    for (int s = 0; s < 4; ++s) {
        unsigned t = z[s] | (z[s] >> 1); t |= t << 2;
        nz4[s] = t & 0x44444444u; F |= s < 2 ? nz4[s] >> (2 - s) : nz4[s] << (s - 2);
    }
    const unsigned nrow = (wp - wp0) >> 8, mine = (__builtin_popcount(F) << 14) | nrow;
    const unsigned inc = wave_scan(mine), tot = __builtin_amdgcn_readlane(inc, 63), ngrp = tot & 0xfffu;
    const unsigned lo = 256u + ((((tot >> 12) + 31u) >> 5) << 2);
    unsigned cnt = ngrp, bad = 0;
    if (ngrp <= 512u) {
        LAS unsigned* dense = (LAS unsigned*)(scr + SCR_DENSE);
        const unsigned lanebits = ((lane & 7) << 15) | ((lane >> 3) << 23), base = (inc - mine) & 0xfffu;
#pragma unroll
        for (int i = 0; i < 8; ++i) dense[(unsigned)i < nrow ? base + i : 512 + lane] = e8[i] | lanebits;
        for (unsigned r = 8; __builtin_expect(__any(nrow > r), 0); ++r)
            dense[r < nrow ? base + r : 512 + lane] = *(LAS unsigned*)(scr + r * 256 + lane * 4) | lanebits;
        unsigned e2[2] = {dense[lane], dense[64 + lane]};
        *(LAS u32x4*)(scr + 256 + lane * 16) = (u32x4){0u, 0u, 0u, 0u};
        *(LAS unsigned*)(scr + lane * 4) = F;
#pragma unroll
        for (int s = 0; s < 4; ++s) { seg[s] = z[s] & 15u; len[s] = nz4[s] & 7u; }
#pragma unroll
        for (int i = 1; i < 8; ++i)
#pragma unroll
            for (int s = 0; s < 4; ++s) { seg[s] |= ((z[s] >> (4 * i)) & 15u) << len[s]; len[s] += (nz4[s] >> (4 * i)) & 7u; }
        unsigned bit = (inc - mine) >> 12;
#pragma unroll
        for (int s = 0; s < 4; ++s) {
            const unsigned long long v = (unsigned long long)seg[s] << (bit & 31u);
            LAS unsigned* w = (LAS unsigned*)(scr + 256) + (bit >> 5);
            __hip_atomic_fetch_or(w, (unsigned)v, __ATOMIC_RELAXED, __HIP_MEMORY_SCOPE_WAVEFRONT);
            __hip_atomic_fetch_or(w + 1, (unsigned)(v >> 32), __ATOMIC_RELAXED, __HIP_MEMORY_SCOPE_WAVEFRONT);
            bit += len[s];
        }
        LAS unsigned short* lst = (LAS unsigned short*)(scr + lo);
        const unsigned hz = ((0u - L) >> 10) & 31u;
        for (unsigned i0 = 0;;) {
            unsigned t[2], pos[2], at[2];
#pragma unroll
            for (int c = 0; c < 2; ++c) {
                const unsigned e = e2[c], ix = i0 + c * 64 + lane;
                t[c] = ix < ngrp ? e & 0x7c7c7c7cu : 0u;
                pos[c] = ((e & 0x380u) << 3) | ((e >> 16) & 0x380u) | ((e >> 11) & 0x70u) | ((e & 3u) << 2);
                at[c] = ix < ngrp ? ix : 1024u + lane;
            }
            for (;;) {
#pragma unroll
                for (int c = 0; c < 2; ++c) {
                    const unsigned b8 = __builtin_ctz(t[c] | 0x80000000u) & 24u, hi5 = (t[c] >> (b8 + 2)) & 31u;
                    const unsigned code = hi5 == hz ? 5u : (hi5 + 5u) & 31u;
                    bad = umax(bad, code);
                    lst[at[c]] = (unsigned short)((code << 13) | pos[c] | (b8 >> 3));
                    t[c] &= ~(0xffu << b8);
                }
                const unsigned long long b0 = __ballot(t[0] != 0), b1 = __ballot(t[1] != 0);
                if (!(b0 | b1)) break;
                at[0] = t[0] != 0 ? __builtin_amdgcn_mbcnt_hi((unsigned)(b0 >> 32), __builtin_amdgcn_mbcnt_lo((unsigned)b0, umin(cnt, 512u))) : 1024u + lane;
                cnt += __builtin_popcountll(b0);
                at[1] = t[1] != 0 ? __builtin_amdgcn_mbcnt_hi((unsigned)(b1 >> 32), __builtin_amdgcn_mbcnt_lo((unsigned)b1, umin(cnt, 512u))) : 1024u + lane;
                cnt += __builtin_popcountll(b1);
            }
            i0 += 128;
            if (__builtin_expect(i0 >= ngrp, 1)) break;
            e2[0] = dense[i0 + lane]; e2[1] = dense[i0 + 64 + lane];
        }
    }
    pk[10] = *(LAS u32x4*)(scr + lane * 16);
    pk[11] = *(LAS u32x4*)(scr + 1024 + lane * 16);
    const unsigned tl = (lo + 2u * cnt + 31u) >> 5;
    if (cnt > 512u || tl > 64u || __any(bad > 7u) || FORCE_RAW) return RAWCNT << 15;
    return L | (cnt << 15) | (tl << 25);
}
__device__ __forceinline__ void st_tail(const char* sb8k  , unsigned voff, u32x4 t0, u32x4 t1, unsigned tl, int lane) {
    if (__builtin_expect(tl > 32u, 0)) { if ((unsigned)lane < (tl - 32u) * 2u) st16<3072>(sb8k, voff, t1); }
    if ((unsigned)lane < tl * 2u) st16<2048>(sb8k, voff, t0);
}
__device__ __forceinline__ void st_raw(const char* slot, unsigned voff, const u32x4 (&b)[16]) {
    const char* s0 = slot + 4096; const char* s1 = slot + 12288;
    st16<-4096>(s0, voff, b[0]); st16<-3072>(s0, voff, b[1]); st16<-2048>(s0, voff, b[2]); st16<-1024>(s0, voff, b[3]);
    st16<0>(s0, voff, b[4]); st16<1024>(s0, voff, b[5]); st16<2048>(s0, voff, b[6]); st16<3072>(s0, voff, b[7]);
    st16<-4096>(s1, voff, b[8]); st16<-3072>(s1, voff, b[9]); st16<-2048>(s1, voff, b[10]); st16<-1024>(s1, voff, b[11]);
    st16<0>(s1, voff, b[12]); st16<1024>(s1, voff, b[13]); st16<2048>(s1, voff, b[14]); st16<3072>(s1, voff, b[15]);
}
__device__ __forceinline__ void ld4x16(f32x4& a, f32x4& b, f32x4& c, f32x4& d, const char* sb, unsigned voff) {
    asm volatile("global_load_dwordx4 %0, %4, %5 sc0 sc1\n\tglobal_load_dwordx4 %1, %4, %5 offset:1024 sc0 sc1\n\t"
                 "global_load_dwordx4 %2, %4, %5 offset:2048 sc0 sc1\n\tglobal_load_dwordx4 %3, %4, %5 offset:3072 sc0 sc1\n\ts_waitcnt vmcnt(0)"
                 : "=&v"(a), "=&v"(b), "=&v"(c), "=&v"(d) : "v"(voff), "s"(sb) : "memory");
}
__device__ __forceinline__ void ld8x16s(f32x4 (&t)[8], const char* sb  , unsigned voff) {
    asm volatile("global_load_dwordx4 %0, %8, %9 offset:-4096 sc0 sc1\n\tglobal_load_dwordx4 %1, %8, %9 offset:-2048 sc0 sc1\n\t"
                 "global_load_dwordx4 %2, %8, %9 sc0 sc1\n\tglobal_load_dwordx4 %3, %8, %9 offset:2048 sc0 sc1\n\t"
                 "global_load_dwordx4 %4, %8, %10 offset:-4096 sc0 sc1\n\tglobal_load_dwordx4 %5, %8, %10 offset:-2048 sc0 sc1\n\t"
                 "global_load_dwordx4 %6, %8, %10 sc0 sc1\n\tglobal_load_dwordx4 %7, %8, %10 offset:2048 sc0 sc1\n\ts_waitcnt vmcnt(0)"
                 : "=&v"(t[0]), "=&v"(t[1]), "=&v"(t[2]), "=&v"(t[3]), "=&v"(t[4]), "=&v"(t[5]), "=&v"(t[6]), "=&v"(t[7])
                 : "v"(voff), "s"(sb), "s"(sb + 16384) : "memory");
}
constexpr int DEC_SLOT = 8192, DEC_TAIL = DEC_SLOT + 10240;
__device__ __forceinline__ void patch_apply(LAS unsigned char* dl, unsigned lo, unsigned hdr, int lane, bool zero) {
    const unsigned cnt = hdr_cnt(hdr), lut_hi = 0x0804007cu | ((((0u - (hdr & 0x7fffu)) >> 10) & 31u) << 10);
    for (unsigned c = 0; c < cnt; c += 128) {
        const unsigned w = *(LAS unsigned*)(dl + DEC_TAIL + lo + c * 2 + lane * 4);
#pragma unroll
        for (int hf = 0; hf < 2; ++hf) {
            const unsigned e = hf ? w >> 16 : w & 0xffffu;
            const unsigned at = c + lane * 2 + hf < cnt ? e & 0x1fffu : (unsigned)DEC_TAIL + lane;
            dl[at] = zero ? 0 : (unsigned char)__builtin_amdgcn_perm(lut_hi, 0x7874706cu, e >> 13);
        }
    }
}
__device__ __forceinline__ unsigned unpack_head(unsigned F, unsigned hdr, LAS unsigned char* dl, int lane, u32x4& G) {
    const unsigned mine = __builtin_popcount(F) << 2, inc = wave_scan(mine);
    const unsigned lo = 256u + ((((unsigned)__builtin_amdgcn_readlane(inc, 63) + 31u) >> 5) << 2);
    unsigned bit = inc - mine, sg[4];
#pragma unroll
    for (int s = 0; s < 4; ++s) {
        const LAS unsigned* w = (const LAS unsigned*)(dl + DEC_TAIL + 256) + (bit >> 5);
        sg[s] = __builtin_amdgcn_alignbit(w[1], w[0], bit);
        bit += __builtin_popcount(F & (0x11111111u << s)) << 2;
    }
    patch_apply(dl, lo, hdr, lane, false);
#pragma unroll
    for (int s = 0; s < 4; ++s) {
        const unsigned nz = (F >> s) & 0x11111111u, pre = nz * 0x11111110u;
        const unsigned xe = sg[s] & 0x0f0f0f0fu, xo = (sg[s] >> 4) & 0x0f0f0f0fu;
        const unsigned b0 = __builtin_amdgcn_perm(xo, xe, 0x05010400u), b1 = __builtin_amdgcn_perm(xo, xe, 0x07030602u);
        const unsigned ze = __builtin_amdgcn_perm(b1, b0, pre & 0x07070707u), zo = __builtin_amdgcn_perm(b1, b0, (pre >> 4) & 0x07070707u);
        G[s] = ~((ze | (zo << 4)) & ((nz << 4) - nz));
    }
    return lo;
}
__device__ __forceinline__ void unpack_piece(u32x4 lo, const u32x4 (&P)[2], u32x4 G, int i, LAS unsigned char* dl, int lane, unsigned LL, u32x4 (&o)[2]) {
    const u32x4 pt = *(LAS u32x4*)(dl + i * 1024 + lane * 16);
    const unsigned A = P[i & 1][i >> 1], g = G[i >> 1] << (i & 1);
    unsigned hb[4];
    hb[0] = bfi(0x83838383u, bfi(0x02020202u, g, A), pt[0]); hb[1] = bfi(0x83838383u, rotr(bfi(0x08080808u, g, A), 2), pt[1]);
    hb[2] = bfi(0x83838383u, rotr(bfi(0x20202020u, g, A), 4), pt[2]); hb[3] = bfi(0x83838383u, rotr(bfi(0x80808080u, g, A), 6), pt[3]);
#pragma unroll
    for (int j = 0; j < 4; ++j) {
        o[j >> 1][2 * (j & 1)] = pk_add(__builtin_amdgcn_perm(hb[j], lo[j], 0x05010400u), LL);
        o[j >> 1][2 * (j & 1) + 1] = pk_add(__builtin_amdgcn_perm(hb[j], lo[j], 0x07030602u), LL);
    }
}

__device__ __forceinline__ void unit_fetch(const Params& p, int h, int u, int lane, unsigned ldsbuf) {
    const bool isx = u < 16;
    const char* base = isx ? p.x + ((size_t)u * 256 * KS + h * 32) * 2 : p.w + ((size_t)h * 32 * NN + (u - 16) * 256) * 2;
    unsigned voff = isx ? (lane & 15) * (KS * 2) + (lane >> 4) * 16 : (lane >> 4) * (8 * NN * 2) + (lane & 15) * 16;
    const unsigned s1 = isx ? 16 * KS * 2 : NN * 2, s2 = isx ? 16 * KS * 2 : 256 - 7 * NN * 2;
#pragma unroll
    for (int i = 0; i < 16; ++i) {
        unsigned keep;
        asm volatile("s_mov_b32 %0, m0\n\ts_mov_b32 m0, %3\n\ts_nop 0\n\tglobal_load_lds_dwordx4 %1, %2\n\ts_mov_b32 m0, %0"
                     : "=&s"(keep) : "v"(voff), "s"(base), "s"(ldsbuf + i * 1024) : "memory");
        voff += (i == 7) ? s2 : s1;
    }
}
__device__ __forceinline__ void unit_read(LAS unsigned char* buf, int lane, u32x4 (&r)[16]) {
#pragma unroll
    for (int i = 0; i < 16; ++i) r[i] = *(LAS u32x4*)(buf + i * 1024 + lane * 16);
    asm volatile("s_waitcnt lgkmcnt(0)" : "+v"(r[0]), "+v"(r[1]), "+v"(r[2]), "+v"(r[3]), "+v"(r[4]), "+v"(r[5]), "+v"(r[6]), "+v"(r[7]),
                 "+v"(r[8]), "+v"(r[9]), "+v"(r[10]), "+v"(r[11]), "+v"(r[12]), "+v"(r[13]), "+v"(r[14]), "+v"(r[15]) :: "memory");
}
__device__ __forceinline__ void unit_format(int u, u32x4 (&r)[16]) {
    if (u >= 16) {
#pragma unroll
        for (int g = 0; g < 2; ++g) {
            u32x4 o[8];
#pragma unroll
            for (int i = 0; i < 8; ++i)
#pragma unroll
                for (int a = 0; a < 4; ++a)
                    o[i][a] = __builtin_amdgcn_perm(r[g * 8 + 2 * a + 1][i >> 1], r[g * 8 + 2 * a][i >> 1], (i & 1) ? 0x07060302u : 0x05040100u);
#pragma unroll
            for (int i = 0; i < 8; ++i) r[g * 8 + i] = o[i];
        }
    }
}
struct Tabs { unsigned ord[NU]; unsigned task[NTASK]; unsigned pull[7][NU]; };
constexpr Tabs make_tabs() {
    Tabs t{}; int n = 0;
    for (int v = 0; v < 8 * NW0; ++v) t.ord[n++] = (v >> 3) << 8 | (v & 7);
    for (int key = 0; key < 8 * NH + 128; ++key)
        for (int v = 0; v < 22; ++v) {
            const int d = key - (v < 8 ? ((v & 2) ? LAGWH : 0) : v < 15 ? LAGXA : LAGXB);
            if (d >= 0 && d % 8 == 0 && d / 8 < NH && !(v < 8 && d / 8 < NW0)) t.ord[n++] = (unsigned)((d / 8) << 8 | v);
        }
    int inv[NU] = {};
    for (int k = 0; k < NU; ++k) inv[(t.ord[k] >> 8) * 22 + (t.ord[k] & 255)] = k;
    for (int li = 0; li < 7; ++li) {
        n = 0;
        for (int k = 0; k < 8 * NW0; ++k) t.pull[li][n++] = k << 16 | t.ord[k];
        for (int key = 0; key < 8 * NH + 128; ++key)
            for (int v = 0; v < 22; ++v) {
                const int d = key - (v == 14 - li || v == 21 - li ? LAGOWN : v < 8 ? ((v & 2) ? LAGWH : 0) : v < 15 ? LAGXA : LAGXB);
                if (d >= 0 && d % 8 == 0 && d / 8 < NH && !(v < 8 && d / 8 < NW0)) t.pull[li][n++] = (unsigned)(inv[d / 8 * 22 + v] << 16 | (d / 8) << 8 | v);
            }
    }
    n = 0;
    for (int k = 0; k < NU; ++k) {
        const unsigned h = t.ord[k] >> 8, v = t.ord[k] & 255, e = k | h << 9;
        for (unsigned l = 0; l < (v < 8 ? 7u : 1u); ++l) t.task[n++] = e | v << 13 | l << 18;
    }
    return t;
}
__constant__ const Tabs g_tabs = make_tabs();
__device__ __forceinline__ int x_tile(int v, int src) { return v >= 15 ? 2 * ((src + v - 14) & 7) + 1 : 2 * ((src + v - 7) & 7); }
__device__ __forceinline__ int st_slot(int h, int v, int link) { return v < 8 ? (link * NH + h) * 8 + v : 896 + h * 14 + v - 8; }

__device__ __forceinline__ void producer(const Params& p, LAS unsigned char* buf, int n0, int n1, int nlead, int lane, unsigned* cntp) {
    char* ws = p.ws[p.rank];
    const unsigned voff = lane * 16;
    const unsigned long long ts = __builtin_amdgcn_s_memrealtime();
    char* peer = p.ws[(p.rank + 1 + (lane & 7)) & 7];
    unsigned call = 0;
    for (int n = n0; n < NTASK; n = n == n0 ? n1 : n + 576) {
        const bool first = n < 56 * RAW0;
        const unsigned e = first ? (n / 7) | ((n / 7) >> 3) << 9 | ((n / 7) & 7) << 13 | (n % 7) << 18 : g_tabs.task[n];
        const int k = e & 511, h = (e >> 9) & 15, v = (e >> 13) & 31, link = e >> 18;
        const bool isx = v >= 8;
        const unsigned dest = (unsigned)(p.rank + 1 + link) & 7u;
        const int t = isx ? x_tile(v, p.rank) : dest * 8 + v;
        while (__builtin_amdgcn_s_memrealtime() - ts < (unsigned)(n < nlead ? 0 : PD0 + (n - nlead) * PDT / 16)) __builtin_amdgcn_s_sleep(2);
        unit_fetch(p, h, isx ? t : 16 + t, lane, (unsigned)(size_t)buf);
        if (n == n0) call = __builtin_amdgcn_readfirstlane(ld4_sys(cntp)) + 1u;
        WAIT_V(0);
        u32x4 r[16];
        unit_read(buf, lane, r);
        unit_format(isx ? 0 : 16, r);
        unsigned hdr = RAWCNT << 15;
        if (first) {
            const char* s0 = p.ws[dest] + OFF_LZ + (size_t)(((call & 1u) * 7 + 6 - link) * 8 * RAW0 + k) * UB + 4096; const char* s1 = s0 + 8192;
            st16<-4096>(s0, voff, r[0]); st16<-3072>(s0, voff, r[1]); st16<-2048>(s0, voff, r[2]); WAIT_V(PWIN); st16<-1024>(s0, voff, r[3]);
            WAIT_V(PWIN); st16<0>(s0, voff, r[4]); WAIT_V(PWIN); st16<1024>(s0, voff, r[5]); WAIT_V(PWIN); st16<2048>(s0, voff, r[6]); WAIT_V(PWIN); st16<3072>(s0, voff, r[7]);
            WAIT_V(PWIN); st16<-4096>(s1, voff, r[8]); WAIT_V(PWIN); st16<-3072>(s1, voff, r[9]); WAIT_V(PWIN); st16<-2048>(s1, voff, r[10]); WAIT_V(PWIN); st16<-1024>(s1, voff, r[11]);
            WAIT_V(PWIN); st16<0>(s1, voff, r[12]); WAIT_V(PWIN); st16<1024>(s1, voff, r[13]); WAIT_V(PWIN); st16<2048>(s1, voff, r[14]); WAIT_V(PWIN); st16<3072>(s1, voff, r[15]);
        } else {
            const char* sl = ws + OFF_ST + (call & 1u) * ST_PAR + (size_t)st_slot(h, v, link) * UB;
            u32x4 pk[12];
            hdr = pack_unit(r, pk, pack_base(r), buf, lane);
            if (__builtin_expect(hdr_cnt(hdr) == RAWCNT, 0)) st_raw(sl, voff, r);
            else {
                const char* s0 = sl + 4096; const char* s1 = sl + 8192;
                st16<-4096>(s0, voff, pk[0]); st16<-3072>(s0, voff, pk[1]); st16<-2048>(s0, voff, pk[2]); st16<-1024>(s0, voff, pk[3]);
                st16<0>(s0, voff, pk[4]); st16<1024>(s0, voff, pk[5]); st16<2048>(s0, voff, pk[6]); st16<3072>(s0, voff, pk[7]);
                st16<0>(s1, voff, pk[8]); st16<1024>(s1, voff, pk[9]); st_tail(s1, voff, pk[10], pk[11], hdr >> 25, lane);
            }
        }
        WAIT_V(0);
        if (isx ? lane < 7 : lane == link)
            __hip_atomic_store((unsigned long long*)(peer + OFF_FA + (size_t)(((call & 1u) * 7 + 6 - lane) * NU + k) * FLS), ((unsigned long long)hdr << 32) | call,
                               __ATOMIC_RELAXED, __HIP_MEMORY_SCOPE_SYSTEM);
        if (!isx) {
            st_raw(ws + OFF_WD + (size_t)(link * NH + h) * WQ + (size_t)v * UB, voff, r);
            WAIT_V(0);
            if (lane == 0) __hip_atomic_store((unsigned*)(ws + OFF_FW) + t * FSTR + h, call, __ATOMIC_RELAXED, __HIP_MEMORY_SCOPE_SYSTEM);
        }
        WAIT_L(0);
    }
}
__device__ __forceinline__ void own_image(const Params& p, LAS unsigned char* buf, int n, int lane, unsigned call) {
    char* ws = p.ws[p.rank];
    const int h = n / 24, v = n - h * 24, t = v < 16 ? v : p.rank * 8 + v - 16;
    unit_fetch(p, h, v < 16 ? t : 16 + t, lane, (unsigned)(size_t)buf);
    WAIT_V(0);
    u32x4 r[16];
    unit_read(buf, lane, r);
    unit_format(v, r);
    st_raw(ws + (v < 16 ? OFF_XI + (size_t)h * XQ : OFF_WI + (size_t)h * WQ) + (size_t)(v & 15) * UB, lane * 16, r);
    WAIT_V(0);
    if (lane == 0) __hip_atomic_store((unsigned*)(ws + (v < 16 ? OFF_FX : OFF_FW)) + t * FSTR + h, call, __ATOMIC_RELAXED, __HIP_MEMORY_SCOPE_SYSTEM);
}

constexpr int DEC_LDS = 24 * 1024;
struct PullCtl { u32x4 desc[16]; unsigned full[7]; unsigned tail; };
__device__ __forceinline__ void decoder(const Params& p, LAS unsigned char* dl, LAS PullCtl* ctl, int xcd, int li, int mem, int lane, unsigned call) {
    char* ws = p.ws[p.rank];
    unsigned* tmo = (unsigned*)(ws + OFF_TMO);
    const unsigned voff = lane * 16;
    const int src = (p.rank + 1 + li) & 7;
    const char* fa = ws + OFF_FA + (size_t)((call & 1u) * 7 + li) * NU * FLS;
    unsigned* tk = (unsigned*)(ws + OFF_TK) + li * 32;
    const unsigned tk0 = (call - 1u) * (NU - 8 * RAW0) - (RAW0 + NDEC) * 8;
#pragma unroll
    for (int i = 0; i < 8; ++i) *(LAS u32x4*)(dl + i * 1024 + lane * 16) = (u32x4){0u, 0u, 0u, 0u};
    unsigned k = (RAW0 + mem) * 8 + xcd, n = 0;
    while (k < (unsigned)NU) {
        const int e = g_tabs.pull[li][k], h = (e >> 8) & 255, v = e & 255;
        const unsigned hdr = wait_unit(fa + (size_t)(e >> 16) * FLS, call, false, tmo);
        ++n;
        if (lane == 0) {
            const unsigned pos = __hip_atomic_fetch_add(&ctl->tail, 1u, __ATOMIC_RELAXED, __HIP_MEMORY_SCOPE_WORKGROUP);
            *(volatile LAS u32x4*)&ctl->desc[pos & 15] = (u32x4){pos + 1u, k | (unsigned)st_slot(h, v, 6 - li) << 16, hdr, (unsigned)mem | (n << 8)};
        }
        const bool isx = v >= 8, raw = hdr_cnt(hdr) == RAWCNT;
        const int q = NH + h * 7 + li, t = isx ? x_tile(v, src) : v;
        const char* dst = ws + (isx ? OFF_XI + (size_t)q * XQ : OFF_WI + (size_t)q * WQ) + (size_t)t * UB;
        unsigned* flag = (unsigned*)(ws + (isx ? OFF_FX : OFF_FW)) + (isx ? t : p.rank * 8 + t) * FSTR + q;
        unsigned tn = 1u;
        if (EARLYTK && lane == 0) asm volatile("global_atomic_add %0, %1, %0, off sc0\n\ts_waitcnt vmcnt(0)" : "+v"(tn) : "v"(tk) : "memory");
        while ((unsigned)__builtin_amdgcn_readfirstlane(*(volatile LAS unsigned*)&ctl->full[mem]) != n) __builtin_amdgcn_s_sleep(1);
        asm volatile("" ::: "memory");
        unsigned lo = 0;
        if (__builtin_expect(raw, 0)) {
            u32x4 r[16];
            unit_read(dl + DEC_SLOT, lane, r);
            st_raw(dst, voff, r);
        } else {
            const unsigned F = *(LAS unsigned*)(dl + DEC_TAIL + lane * 4), LL = (hdr & 0x7fffu) * 0x10001u;
            u32x4 P[2], G, lw[8];
            P[0] = *(LAS u32x4*)(dl + DEC_SLOT + 8192 + lane * 16); P[1] = *(LAS u32x4*)(dl + DEC_SLOT + 9216 + lane * 16);
#pragma unroll
            for (int i = 0; i < 8; ++i) lw[i] = *(LAS u32x4*)(dl + DEC_SLOT + i * 1024 + lane * 16);
            lo = unpack_head(F, hdr, dl, lane, G);
#pragma unroll
            for (int i = 0; i < 8; ++i) {
                u32x4 o[2]; unpack_piece(lw[i], P, G, i, dl, lane, LL, o);
                const char* sb = dst + i * 2048;
                st16<0>(sb, voff, o[0]); st16<1024>(sb, voff, o[1]);
            }
        }
        WAIT_V(0);
        if (lane == 0) __hip_atomic_store(flag, call, __ATOMIC_RELAXED, __HIP_MEMORY_SCOPE_SYSTEM);
        if (!EARLYTK && lane == 0) asm volatile("global_atomic_add %0, %1, %0, off sc0\n\ts_waitcnt vmcnt(0)" : "+v"(tn) : "v"(tk) : "memory");
        if (!raw) patch_apply(dl, lo, hdr, lane, true);
        WAIT_L(0);
        k = __builtin_amdgcn_readfirstlane(tn) - tk0;
    }
    if (lane == 0) {
        const unsigned pos = __hip_atomic_fetch_add(&ctl->tail, 1u, __ATOMIC_RELAXED, __HIP_MEMORY_SCOPE_WORKGROUP);
        *(volatile LAS u32x4*)&ctl->desc[pos & 15] = (u32x4){pos + 1u, 0u, 0u, 0x80u};
    }
}
__device__ __forceinline__ void opener(const Params& p, int li, int h, int ct, int lane, unsigned call) {
    char* ws = p.ws[p.rank];
    wait_unit(ws + OFF_FA + (size_t)(((call & 1u) * 7 + li) * NU + h * 8 + ct) * FLS, call, false, (unsigned*)(ws + OFF_TMO));
    const char* src = ws + OFF_LZ + (size_t)(((call & 1u) * 7 + li) * 8 * RAW0 + h * 8 + ct) * UB;
    const int q = NH + h * 7 + li;
    const char* dst = ws + OFF_WI + (size_t)q * WQ + (size_t)ct * UB;
    for (int i = 0; i < 4; ++i) {
        f32x4 t[4]; ld4x16(t[0], t[1], t[2], t[3], src + i * 4096, lane * 16);
        const char* d = dst + i * 4096;
        st16<0>(d, lane * 16, __builtin_bit_cast(u32x4, t[0])); st16<1024>(d, lane * 16, __builtin_bit_cast(u32x4, t[1]));
        st16<2048>(d, lane * 16, __builtin_bit_cast(u32x4, t[2])); st16<3072>(d, lane * 16, __builtin_bit_cast(u32x4, t[3]));
    }
    WAIT_V(0);
    if (lane == 0) __hip_atomic_store((unsigned*)(ws + OFF_FW) + (p.rank * 8 + ct) * FSTR + q, call, __ATOMIC_RELAXED, __HIP_MEMORY_SCOPE_SYSTEM);
}
__device__ __forceinline__ void loader(const Params& p, LAS unsigned char* lds, LAS PullCtl* ctl, int li, int lane, unsigned call) {
    const char* st = p.ws[(unsigned)(p.rank + 1 + li) & 7u] + OFF_ST + (call & 1u) * ST_PAR;
    const unsigned voff = lane * 16;
    unsigned pos = 0, done = 0, pend = 0, idle = 0;
#define LANDED() do { *(volatile LAS unsigned*)&ctl->full[pend & 7u] = pend >> 8; pend = 0; } while (0)
    while (done < NDEC || pend) {
        const u32x4 d = *(volatile LAS u32x4*)&ctl->desc[pos & 15];
        if ((unsigned)__builtin_amdgcn_readfirstlane(d.x) != pos + 1u) {
            if (!pend) __builtin_amdgcn_s_sleep(1);
            else if (idle == 0) WAIT_V(4);
            else if (idle == 1) WAIT_V(2);
            else { WAIT_V(0); LANDED(); }
            ++idle;
            continue;
        }
        idle = 0;
        const unsigned ks = __builtin_amdgcn_readfirstlane(d.y), hdr = __builtin_amdgcn_readfirstlane(d.z), w = __builtin_amdgcn_readfirstlane(d.w);
        ++pos;
        if (w & 0x80u) { ++done; continue; }
        if ((ks & 0xffffu) == NU - 1 - GATE_EARLY && lane < 8)
            __hip_atomic_store((unsigned*)(p.ws[p.rank] + OFF_SD) + (li * 8 + lane) * 32, call, __ATOMIC_RELAXED, __HIP_MEMORY_SCOPE_AGENT);
        const char* s = st + (size_t)(ks >> 16) * UB;
        const unsigned l = (unsigned)(size_t)lds + (w & 7u) * DEC_LDS + DEC_SLOT;
#pragma unroll
        for (int i = 0; i < 10; ++i) {
            WAIT_V(LWIN);
            if (i == LWIN && pend) LANDED();
            dma16(s + i * 1024, voff, l + i * 1024);
        }
        static_assert(LWIN < 10, "window");
        if (__builtin_expect(hdr_cnt(hdr) == RAWCNT, 0)) {
#pragma unroll
            for (int i = 10; i < 16; ++i) { WAIT_V(LWIN); dma16(s + i * 1024, voff, l + i * 1024); }
        } else {
            const unsigned tl = hdr >> 25;
            WAIT_V(LWIN);
            if ((unsigned)lane < tl * 2u) dma16(s + 10240, voff, l + 10240);
            if (__builtin_expect(tl > 32u, 0)) { if ((unsigned)lane < (tl - 32u) * 2u) dma16(s + 11264, voff, l + 11264); }
        }
        pend = w;
    }
}

__device__ __forceinline__ void post_ready(bool r0, bool r1, unsigned long long& td0, unsigned long long& td1, LAS unsigned* queue, int& nq, unsigned lane) {
    const unsigned long long a0 = __ballot(r0) & td0, a1 = __ballot(r1) & td1;
    if ((a0 >> lane) & 1) ((volatile LAS unsigned*)queue)[__builtin_amdgcn_mbcnt_hi((unsigned)(a0 >> 32), __builtin_amdgcn_mbcnt_lo((unsigned)a0, nq))] = lane | 0x10000u;
    nq += __builtin_popcountll(a0);
    if ((a1 >> lane) & 1) ((volatile LAS unsigned*)queue)[__builtin_amdgcn_mbcnt_hi((unsigned)(a1 >> 32), __builtin_amdgcn_mbcnt_lo((unsigned)a1, nq))] = lane | 0x10040u;
    nq += __builtin_popcountll(a1);
    td0 &= ~a0; td1 &= ~a1;
}
__device__ __forceinline__ void poll_ready(const unsigned* fx, const unsigned* fw, unsigned long long& td0, unsigned long long& td1,
                                           LAS unsigned* queue, int& nq, int need, unsigned call, unsigned* tmo) {
    const unsigned lane = __builtin_amdgcn_mbcnt_hi(~0u, __builtin_amdgcn_mbcnt_lo(~0u, 0u));
    {
        const volatile LAS unsigned* sn = queue + 256 + lane;
        const unsigned x0 = sn[0], x1 = sn[64], w0 = sn[128], w1 = sn[192];
        post_ready(x0 == call && w0 == call, x1 == call && w1 == call, td0, td1, queue, nq, lane);
    }
    const unsigned long long t0 = __builtin_amdgcn_s_memrealtime(); unsigned polls = 0;
    bool all = false;
    while (nq < need) {
        unsigned x0, x1, w0, w1;
        asm volatile("global_load_dword %0, %4, %5 sc0 sc1\n\tglobal_load_dword %1, %4, %5 offset:256 sc0 sc1\n\t"
                     "global_load_dword %2, %4, %6 sc0 sc1\n\tglobal_load_dword %3, %4, %6 offset:256 sc0 sc1\n\ts_waitcnt vmcnt(0)"
                     : "=&v"(x0), "=&v"(x1), "=&v"(w0), "=&v"(w1) : "v"(lane * 4), "s"(fx), "s"(fw) : "memory");
        post_ready(all || (x0 >= call && w0 >= call), all || (x1 >= call && w1 >= call), td0, td1, queue, nq, lane);
        if (nq >= need) break;
        all = give_up(tmo, call, t0, polls);
        __builtin_amdgcn_s_sleep(1);
    }
    WAIT_L(0);
}

constexpr int HTB = 16384;
constexpr int FIN_LDS = 20480;

__global__ __launch_bounds__(512, 2) void fused_kernel(const Params p) {
    __shared__ __attribute__((aligned(16))) unsigned char shm[8 * FIN_LDS];
    LAS unsigned char* lds = (LAS unsigned char*)shm;
    const int wid = __builtin_amdgcn_readfirstlane(threadIdx.x >> 6), lane = threadIdx.x & 63;
    const int xcd = blockIdx.x & 7, slot = blockIdx.x >> 3;
    char* ws = p.ws[p.rank];
    unsigned* tmo = (unsigned*)(ws + OFF_TMO);
    unsigned* cntp = (unsigned*)(ws + OFF_CNT) + blockIdx.x * 8 + wid;
    STAMP(0);
    const bool main_tile = slot >= 16;
    const int fsrc = (p.rank + 1 + slot) & 7, fct = xcd;
    const bool helper = !main_tile && (slot & 7) == 7;
    LAS PullCtl* ctl = (LAS PullCtl*)(lds + 5 * DEC_LDS + 2 * HTB);
    static_assert(NDEC <= 5 && RAW0 <= 2 && RAW0 <= NW0, "puller CU: load wave, 5 unpack waves, 2 waves for the opening units");
    if (slot < 7) { if (wid < 2) ((volatile LAS unsigned*)ctl)[wid * 64 + lane] = 0u; WAIT_L(0); BAR; }
    {
        constexpr int NOPEN = 56 * RAW0, NLEAD = NOPEN + MAINP * 128;
        int n0 = -1, n1 = NTASK; LAS unsigned char* buf = lds + wid * HTB;
        if (slot < 7) { if (PUSHP && wid >= 6 && wid < 6 + RAW0) { n0 = ((wid - 6) * 8 + xcd) * 7 + slot; buf = lds + 5 * DEC_LDS + (wid - 6) * HTB; } }
        else if (main_tile) { if (wid >= 4 && wid < 4 + MAINP) n0 = NOPEN + (wid - 4) * 128 + (blockIdx.x - 128); }
        else { const int i = wid * 72 + (blockIdx.x - 56);
               if (PUSHP) { n0 = NLEAD + i; n1 = n0 + 576; } else { n0 = i < NOPEN ? i : i + MAINP * 128; n1 = i + 576 + MAINP * 128; } }
        if (n0 >= 0) producer(p, buf, n0, n1, NLEAD, lane, cntp);
    }
    const unsigned call = __builtin_amdgcn_readfirstlane(ld4_sys(cntp)) + 1u;
    if (lane == 0) __hip_atomic_store(cntp, call, __ATOMIC_RELAXED, __HIP_MEMORY_SCOPE_AGENT);
    if (slot < 7) {
        if (wid == 0) { __builtin_amdgcn_s_setprio(3); loader(p, lds, ctl, slot, lane, call); __builtin_amdgcn_s_setprio(0); }
        else if (wid <= NDEC) decoder(p, lds + (wid - 1) * DEC_LDS, ctl, xcd, slot, wid - 1, lane, call);
        else if (wid >= 6 && wid < 6 + RAW0) opener(p, slot, wid - 6, xcd, lane, call);
    }
    if (main_tile) {
        if (slot >= 24 && wid >= 1 && wid <= 6) { for (int i = 1; i < wid; ++i) __builtin_amdgcn_s_sleep(OWND); own_image(p, lds + wid * HTB, (wid - 1) * 64 + (blockIdx.x - 192), lane, call); }
    } else {
        STAMP(1);
        asm volatile("s_waitcnt vmcnt(0) lgkmcnt(0)" ::: "memory");
        BAR;
    }

    const int wr = wid >> 2, wc = wid & 3;
    f32x4 acc[2][2][4][2];
    int rt, cn;
    const unsigned* gate2 = cntp;
    if (slot < 7) {
        rt = 0; cn = fsrc * 8 + fct;
        wait_word((const unsigned*)(ws + OFF_FD) + cn * 8 + wid, call, tmo);
        const char* pb = ws + OFF_PK + (size_t)(cn * 8 + wid) * 32768;
#pragma unroll
        for (int i = 0; i < 8; ++i)
            ld4x16(acc[i >> 2][(i >> 1) & 1][(i & 1) * 2][0], acc[i >> 2][(i >> 1) & 1][(i & 1) * 2][1], acc[i >> 2][(i >> 1) & 1][(i & 1) * 2 + 1][0], acc[i >> 2][(i >> 1) & 1][(i & 1) * 2 + 1][1], pb + i * 4096, lane * 16);
    } else
    for (int it = 0; it < 2; ++it) {
    int sm, skip = 7;
    const char* wimg;
    if (main_tile) {
        const int idx = slot - 16;
        rt = ((xcd >> 1) * 2 + ((idx >> 2) & 1)) * 2 + (idx >> 3);
        cn = p.rank * 8 + (xcd & 1) * 4 + (idx & 3);
        const int m = rt >> 1;
        skip = (m - p.rank - 1) & 7; sm = (m == p.rank) ? OWNM : 0xff & ~(1 << skip);
        wimg = ws + OFF_WI;
    } else {
        rt = p.rank * 2 + (helper ? slot >> 3 : 1 - it);
        cn = fsrc * 8 + fct;
        sm = helper ? 0xff ^ OWNM : 0x80;
        wimg = helper ? ws + OFF_WI : ws + OFF_WD + (size_t)(slot & 7) * NH * WQ;
    }
    const int nt = 8 * __builtin_popcount(sm);
    const unsigned* fx = (const unsigned*)(ws + OFF_FX) + rt * FSTR;
    const unsigned* fw = (const unsigned*)(ws + OFF_FW) + cn * FSTR;
    gate2 = (skip == 7) ? cntp : fw + NH + skip;
    const char* xb = ws + OFF_XI + (size_t)(rt * 16 + wid) * 1024;
    const char* wb = wimg + (size_t)((cn & 7) * 16 + wr * 8 + wc) * 1024;
    const unsigned voff = lane * 16;
    const unsigned ldsw = wid * 1024;
    const int aoff = wr * 4096 + lane * 16, boff = wc * 2048 + lane * 16;
#define SA(b, h) (((b) * 4 + (h)) * HTB)
#define SB(b, h) (((b) * 4 + 2 + (h)) * HTB)
#define GLDS(g, o, l) __builtin_amdgcn_global_load_lds((const unsigned*)((g) + (size_t)(unsigned)(o)), (LAS unsigned*)(lds + (l)), 16, 0, 17)
#define STAGE_A(b, h, qq) do { GLDS(wb + (h) * 4096, (qq.x << 17) + voff, SA(b, h) + ldsw); GLDS(wb + (h) * 4096, (qq.y << 17) + voff, SA(b, h) + ldsw + 8192); } while (0)
#define STAGE_B(b, h, qq) do { GLDS(xb + (h) * 8192, (qq.x << 18) + voff, SB(b, h) + ldsw); GLDS(xb + (h) * 8192, (qq.y << 18) + voff, SB(b, h) + ldsw + 8192); } while (0)
#define LDA(dst, b, h) do { _Pragma("unroll") for (int m = 0; m < 4; ++m) _Pragma("unroll") for (int k = 0; k < 2; ++k) dst[m][k] = *(const LAS bf16x8*)(lds + SA(b, h) + aoff + m * 1024 + k * 8192); } while (0)
#define LDB(dst, b, h) do { _Pragma("unroll") for (int n = 0; n < 2; ++n) _Pragma("unroll") for (int k = 0; k < 2; ++k) dst[n][k] = *(const LAS bf16x8*)(lds + SB(b, h) + boff + n * 1024 + k * 8192); } while (0)
#define MMA2(ai) do { __builtin_amdgcn_s_setprio(1); _Pragma("unroll") for (int k = 0; k < 2; ++k) _Pragma("unroll") for (int m = 0; m < 4; ++m) _Pragma("unroll") for (int n = 0; n < 2; ++n) { \
        acc[ai][0][m][n] = __builtin_amdgcn_mfma_f32_16x16x32_bf16(B0[n][k], At[m][k], acc[ai][0][m][n], 0, 0, 0); \
        acc[ai][1][m][n] = __builtin_amdgcn_mfma_f32_16x16x32_bf16(B1[n][k], At[m][k], acc[ai][1][m][n], 0, 0, 0); } __builtin_amdgcn_s_setprio(0); } while (0)
#define MUST_PICK(kt) do { if (__builtin_expect(nq < 2 * (kt) + 2, 0)) poll_ready(fx, fw, td0, td1, queue, nq, 2 * (kt) + 2, call, tmo); } while (0)
#define SNAP(kt) do { if (__builtin_expect(nq < 2 * (kt) + 2, 0)) \
        __builtin_amdgcn_global_load_lds((lane < 32 ? fx : fw - 128) + lane * 4, queue + 256, 16, 0, 17); } while (0)
#define GETQ(kt) (*(volatile LAS u32x2*)(queue + 2 * (kt)))
#define FIXQ(qq, kt) do { if (__builtin_expect(__builtin_amdgcn_readfirstlane(qq.x & qq.y) < 0x10000, 0)) qq = GETQ(kt); } while (0)
#pragma unroll
    for (int a = 0; a < 2; ++a)
#pragma unroll
        for (int b = 0; b < 2; ++b)
#pragma unroll
            for (int m = 0; m < 4; ++m)
#pragma unroll
                for (int n = 0; n < 2; ++n) acc[a][b][m][n] = (f32x4){0.f, 0.f, 0.f, 0.f};
    bf16x8 At[4][2], B0[2][2], B1[2][2];
    LAS unsigned* queue = (LAS unsigned*)(lds + 8 * HTB + 4096);
    unsigned long long td0 = __ballot((sm >> (lane < NH ? 7 : (lane - NH) % 7)) & 1), td1 = __ballot((sm >> ((lane + 64 - NH) % 7)) & 1);
    int nq = 1 << 20;
    if (wid == 0) { nq = 0; *(volatile LAS u32x4*)(queue + 4 * lane) = (u32x4){0u, 0u, 0u, 0u}; *(volatile LAS u32x4*)(queue + 256 + 4 * lane) = (u32x4){0u, 0u, 0u, 0u}; poll_ready(fx, fw, td0, td1, queue, nq, 4, call, tmo); }
    asm volatile("" ::: "memory"); BAR; asm volatile("" ::: "memory");
    u32x2 qe = GETQ(0), qo = GETQ(1);
    STAMP(4);
    STAGE_B(0, 0, qe); STAGE_A(0, 0, qe); STAGE_B(0, 1, qe); STAGE_A(0, 1, qe);
    STAGE_B(1, 0, qo); STAGE_A(1, 0, qo); STAGE_B(1, 1, qo);
    if (wr == 1) BAR;
    WAIT_V(8); BAR; BAR;
    for (int t = 0; t < nt; t += 2) {
        if (t == nt - 2) nq = 1 << 20;
        SNAP(t + 3);
        qe = GETQ(t + 2);
        LDB(B0, 0, 0); LDA(At, 0, 0); LDB(B1, 0, 1); STAGE_A(1, 1, qo);
        WAIT_V(8); WAIT_L(0); BAR; SCHED; MMA2(0); SCHED; MUST_PICK(t + 2); BAR; SCHED;
        FIXQ(qe, t + 2);
        LDA(At, 0, 1); STAGE_B(0, 0, qe); STAGE_A(0, 0, qe); STAGE_B(0, 1, qe);
        WAIT_V(8); WAIT_L(0); BAR; SCHED; MMA2(1); BAR; SCHED;
        if (t == nt - 2)
            __builtin_amdgcn_global_load_lds(lane == 1 ? tmo : gate2, (LAS unsigned*)(lds + 8 * HTB + wid * 256), 4, 0, 17);
        SNAP(t + 4);
        qo = GETQ(t + 3);
        LDB(B0, 1, 0); LDA(At, 1, 0); LDB(B1, 1, 1); STAGE_A(0, 1, qe);
        WAIT_V(8); WAIT_L(0); BAR; SCHED; MMA2(0); SCHED; MUST_PICK(t + 3); BAR; SCHED;
        FIXQ(qo, t + 3);
        LDA(At, 1, 1); STAGE_B(1, 0, qo); STAGE_A(1, 0, qo); STAGE_B(1, 1, qo);
        WAIT_V(8); WAIT_L(0); BAR; SCHED; MMA2(1); BAR; SCHED;
    }
    STAMP(2);
    if (main_tile) break;
    if (wr == 0) BAR;
    const int pki = (helper ? 64 + (rt & 1) * 8 + fct : cn) * 8 + wid;
    if (it == 0) {
        const char* pb = ws + OFF_PK + (size_t)pki * 32768;
        asm volatile("s_nop 4" : "+s"(pb));
#pragma unroll
        for (int i = 0; i < 32; ++i) st16<0>(pb + i * 1024, voff, __builtin_bit_cast(u32x4, acc[i >> 4][(i >> 3) & 1][(i >> 1) & 3][i & 1]));
    }
    WAIT_V(0);
    if (it == 0 && lane == 0) __hip_atomic_store((unsigned*)(ws + OFF_FD) + pki, call, __ATOMIC_RELAXED, __HIP_MEMORY_SCOPE_SYSTEM);
    if (helper) return;
    asm volatile("" ::: "memory"); BAR; asm volatile("" ::: "memory");
    }

    int tz = threadIdx.x; asm volatile("" : "+v"(tz));
    const int ln = tz & 63;
    const unsigned vo = ln * 16;
    const bool own = main_tile && (rt >> 1) == p.rank;
    int computing = __builtin_amdgcn_readfirstlane(main_tile && !own); asm volatile("" : "+s"(computing));
    if (computing != 1) {
        const int rh = own ? rt & 1 : slot < 8 ? 1 : 0;
        LAS unsigned char* dl = lds + wid * FIN_LDS;
        const char* sl = p.ws[fsrc] + OFF_PS + (size_t)(((p.rank * 2 + rh) * 8 + (cn & 7)) * 8 + wid) * UB;
        const unsigned ls = (unsigned)(size_t)dl + DEC_SLOT;
        unsigned hdr = 0;
        if (own) {
            if (wr == 0) BAR;
            const int pki = (64 + rh * 8 + (cn & 7)) * 8 + wid;
            wait_word((const unsigned*)(ws + OFF_FD) + pki, call, tmo);
            sl = ws + OFF_PK + (size_t)pki * 32768;
        } else {
#pragma unroll
            for (int i = 0; i < 8; ++i) *(LAS u32x4*)(dl + i * 1024 + ln * 16) = (u32x4){0u, 0u, 0u, 0u};
            STAMP(24);
            const char* fl = ws + OFF_FP + (size_t)((rh * 64 + cn) * 8 + wid) * FLS;
            wait_word((const unsigned*)(ws + OFF_SD) + ((slot & 7) * 8 + fct) * 32, call, tmo);
            hdr = wait_unit(fl, call, false, tmo);
            STAMP(25);
            dma16(sl, vo, ls); WAIT_V(0); dma16(sl + 1024, vo, ls + 1024); WAIT_V(0); dma16(sl + 2048, vo, ls + 2048);
            if (NLOW > 3) { WAIT_V(0); dma16(sl + 3072, vo, ls + 3072); }
            if (NLOW > 4) { WAIT_V(0); dma16(sl + 4096, vo, ls + 4096); }
            if (hdr_cnt(hdr) == LOWCNT) hdr = wait_unit(fl, call, true, tmo);
        }
        const bool raw = hdr_cnt(hdr) == RAWCNT;
        const unsigned LL = (hdr & 0x7fffu) * 0x10001u, tl = hdr >> 25;
        if (!raw && !own) {
            WAIT_V(0);
            if ((unsigned)ln < tl * 2u) dma16(sl + 10240, vo, ls + 10240);
            if (__builtin_expect(tl > 32u, 0)) { if ((unsigned)ln < (tl - 32u) * 2u) dma16(sl + 11264, vo, ls + 11264); }
        }
        const unsigned nanmask = ((unsigned)__builtin_amdgcn_readfirstlane(ld4_sys(tmo)) == call) ? 0x7fc07fc0u : 0u;
        u32x4 P[2], G;
        if (!raw && !own) {
            dma16(sl + 8192, vo, ls + 8192);
            unpack_head(*(LAS unsigned*)(dl + DEC_TAIL + ln * 4), hdr, dl, ln, G);
            WAIT_V(0); dma16(sl + 9216, vo, ls + 9216);
            P[0] = *(LAS u32x4*)(dl + DEC_SLOT + 8192 + ln * 16);
        }
        STAMP(26);
        const char* ob = p.out + ((size_t)(rh * 256 + wc * 32) * NN + cn * 256 + wr * 128) * 2;
        const unsigned oo = (ln >> 4) * (4 * NN * 2) + (ln & 15) * 16;
#pragma unroll
        for (int st = 0; st < 8; ++st) {
            const int i = st == 1 ? 2 : st == 2 ? 1 : st;
            u32x4 o[2];
            if (own) {
                o[0] = o[1] = (u32x4){0u, 0u, 0u, 0u};
                if (st == 0 || st == 1 || st == 4 || st == 6) {
                    const int bj = i >> 2, n = (i >> 1) & 1;
                    f32x4 t[8]; ld8x16s(t, sl + (bj * 8 + n) * 1024 + 4096, vo);
#pragma unroll
                    for (int j = 0; j < 8; ++j) acc[j >> 2][bj][j & 3][n] += t[j];
                }
            } else if (raw) { dma16(sl + i * 2048, vo, ls); dma16(sl + i * 2048 + 1024, vo, ls + 1024); WAIT_V(0);
                       o[0] = *(LAS u32x4*)(dl + DEC_SLOT + ln * 16); o[1] = *(LAS u32x4*)(dl + DEC_SLOT + 1024 + ln * 16); asm volatile("s_waitcnt lgkmcnt(0)" : "+v"(o[0]), "+v"(o[1]) :: "memory"); }
            else {
#define NEXTP(j) dma16(sl + (j) * 1024, vo, ls + (j) * 1024)
                if (st == 2) { WAIT_V(4); NEXTP(NLOW); P[1] = *(LAS u32x4*)(dl + DEC_SLOT + 9216 + ln * 16); }
                if (NLOW == 3 && st >= 3) { WAIT_V(2); if (st < 7) NEXTP(st + 1); }
                if (NLOW == 4 && st == 4) { WAIT_V(4); NEXTP(5); }
                if (NLOW == 4 && st > 4) { WAIT_V(2); if (st < 7) NEXTP(st + 1); }
                if (NLOW == 5 && st == 4) { WAIT_V(4); NEXTP(6); }
                if (NLOW == 5 && st == 6) { WAIT_V(4); NEXTP(7); }
                if (NLOW == 5 && st == 7) WAIT_V(2);
                unpack_piece(*(LAS u32x4*)(dl + DEC_SLOT + i * 1024 + ln * 16), P, G, i, dl, ln, LL, o);
            }
#pragma unroll
            for (int j = 0; j < 2; ++j) {
                const int f = 2 * i + j;
                u32x4 y;
#pragma unroll
                for (int cc = 0; cc < 4; ++cc) {
                    const unsigned pv = o[j][cc];
                    const f32x2 v = (f32x2){acc[cc >> 1][f >> 3][2 * (cc & 1)][(f >> 2) & 1][f & 3] + __uint_as_float(pv << 16),
                                            acc[cc >> 1][f >> 3][2 * (cc & 1) + 1][(f >> 2) & 1][f & 3] + __uint_as_float(pv & 0xffff0000u)};
                    const f32x2 g = gelu2(v);
                    y[cc] = cvt_pk_bf16(g.x, g.y) | nanmask;
                }
                st16<0>(ob + (size_t)((f >> 3) * 128 + ((f >> 2) & 1) * 16 + (f & 3)) * (NN * 2), oo, y);
            }
        }
    } else {
        const u32x2 gv = *(volatile LAS u32x2*)(lds + 8 * HTB + wid * 256);
        u32x4 b[16], pk[12];
#pragma unroll
        for (int bj = 0; bj < 2; ++bj)
#pragma unroll
            for (int n = 0; n < 2; ++n)
#pragma unroll
                for (int r = 0; r < 4; ++r)
#pragma unroll
                    for (int ai = 0; ai < 2; ++ai)
#pragma unroll
                        for (int m2 = 0; m2 < 2; ++m2)
                            b[bj * 8 + n * 4 + r][ai * 2 + m2] = cvt_pk_bf16(acc[ai][bj][2 * m2][n][r], acc[ai][bj][2 * m2 + 1][n][r]);
        asm volatile("" : "+v"(b[0]), "+v"(b[1]), "+v"(b[2]), "+v"(b[3]), "+v"(b[4]), "+v"(b[5]), "+v"(b[6]), "+v"(b[7]), "+v"(b[8]), "+v"(b[9]), "+v"(b[10]), "+v"(b[11]), "+v"(b[12]), "+v"(b[13]), "+v"(b[14]), "+v"(b[15]) :: "memory");
        if (wr == 0) BAR;
        bool bad = (unsigned)__builtin_amdgcn_readfirstlane(gv.y) == call;
        if ((unsigned)__builtin_amdgcn_readfirstlane(gv.x) < call) { wait_word(gate2, call, tmo); bad = (unsigned)__builtin_amdgcn_readfirstlane(ld4_sys(tmo)) == call; }
        const char* sl = ws + OFF_PS + (size_t)((rt * 8 + (cn & 7)) * 8 + wid) * UB;
        char* fl = p.ws[(unsigned)(rt >> 1) & 7u] + OFF_FP + (size_t)(((rt & 1) * 64 + cn) * 8 + wid) * FLS;
        const char* s0 = sl + 4096; const char* s1 = sl + 8192;
        const unsigned L = pack_base(b);
        st16<-4096>(s0, vo, pack_low(b, 0, L)); st16<-3072>(s0, vo, pack_low(b, 1, L)); st16<-2048>(s0, vo, pack_low(b, 2, L));
        if (NLOW > 3) st16<-1024>(s0, vo, pack_low(b, 3, L));
        if (NLOW > 4) st16<0>(s0, vo, pack_low(b, 4, L));
        WAIT_V(0);
        st8_flag(fl, call, LOWCNT << 15, ln);
        STAMP(6);
        asm volatile("" ::: "memory"); BAR; asm volatile("" ::: "memory");
        unsigned hdr = pack_unit(b, pk, L, lds + wid * HTB, ln);
        if (bad) {
            hdr = RAWCNT << 15;
#pragma unroll
            for (int f = 0; f < 16; ++f) b[f] = (u32x4){0x7fc07fc0u, 0x7fc07fc0u, 0x7fc07fc0u, 0x7fc07fc0u};
        }
        if (__builtin_expect(hdr_cnt(hdr) == RAWCNT, 0)) st_raw(sl, vo, b);
        else {
            st16<0>(s1, vo, pk[8]); st16<1024>(s1, vo, pk[9]); st_tail(s1, vo, pk[10], pk[11], hdr >> 25, ln);
            if (NLOW <= 3) st16<-1024>(s0, vo, pk[3]);
            if (NLOW <= 4) st16<0>(s0, vo, pk[4]);
            st16<1024>(s0, vo, pk[5]); st16<2048>(s0, vo, pk[6]); st16<3072>(s0, vo, pk[7]);
        }
        WAIT_V(0);
        st8_flag(fl, call, hdr, ln);
    }
    STAMP(3);
}
}

extern "C" void kernel_launch(int rank, int nranks, ncclComm_t comm, hipStream_t stream,
                              void* const* d_in, const int* in_sizes, int n_in,
                              void* d_out, int out_size, const int* in_dtypes, int out_dtype,
                              void* const* d_out_all, void* const* d_ws_all, size_t ws_bytes) {
    (void)comm; (void)in_dtypes; (void)out_dtype; (void)d_out_all;
    if (nranks != NR || (unsigned)rank >= (unsigned)NR || n_in < 2 || in_sizes[0] != MM * KS || in_sizes[1] != KS * NN ||
        out_size != (MM / NR) * NN || ws_bytes < WS_NEED) {
        fprintf(stderr, "kernel_launch: unexpected problem geometry\n");
        return;
    }
    Params p;
    p.x = (const char*)d_in[0];
    p.w = (const char*)d_in[1];
    for (int i = 0; i < NR; ++i) p.ws[i] = (char*)d_ws_all[i];
    p.out = (char*)d_out;
    p.rank = rank; p.pad = 0;
    hipLaunchKernelGGL(fused_kernel, dim3(256), dim3(512), 0, stream, p);
    HIP_CHECK(hipGetLastError());
}
```

```cpp
#include <hip/hip_runtime.h>
#include <rccl/rccl.h>
#include <cstdio>
#include <cstdint>

#define HIP_CHECK(expr) do { hipError_t _e = (expr); if (_e != hipSuccess) { \
    fprintf(stderr, "HIP error %s at %s:%d\n", hipGetErrorString(_e), __FILE__, __LINE__); \
    return; } } while (0)

#define LAGXA 1
#define LAGXB 36
#define LAGOWN 69
#define LAGWH 4
#define RAW0 2
#define LWIN 6
#define PWIN 3
#define NDEC 5
#define PDT 40
#define GATE_EARLY 6
#define FORCE_RAW 0

namespace {
#define LAS __attribute__((address_space(3)))
typedef short bf16x8 __attribute__((ext_vector_type(8)));
typedef float f32x4 __attribute__((ext_vector_type(4)));
typedef float f32x2 __attribute__((ext_vector_type(2)));
typedef unsigned u32x4 __attribute__((ext_vector_type(4)));
typedef unsigned u32x2 __attribute__((ext_vector_type(2)));
typedef unsigned short u16x2 __attribute__((ext_vector_type(2)));

constexpr int NR = 8;
constexpr int MM = 4096, NN = 16384, KS = 512;
constexpr int NQ = 128;
constexpr int NH = 16;
constexpr int NU = NH * 22;
constexpr int NTASK = NH * 70;
constexpr unsigned long long DEADLINE = 100000000ull;
constexpr unsigned RAWCNT = 0x3ffu, LOWCNT = 0x3feu;

constexpr size_t OFF_CNT = 0;
constexpr size_t OFF_TMO = 8192;
constexpr size_t OFF_TK = 8192 + 1024;
constexpr int FSTR = 160;
constexpr size_t OFF_FX = 16384;
constexpr size_t OFF_FW = 32768;
constexpr size_t OFF_FD = 81920;
constexpr size_t OFF_SD = 131072;
constexpr size_t UB = 16384, FLS = 128;
constexpr size_t XQ = 256 * 1024, WQ = 128 * 1024;
constexpr size_t OFF_XI = 1u << 20;
constexpr size_t OFF_WI = OFF_XI + (size_t)NQ * XQ;
constexpr size_t OFF_WD = OFF_WI + (size_t)NQ * WQ;
constexpr size_t OFF_PS = OFF_WD + (size_t)7 * NH * WQ;
constexpr size_t OFF_ST = OFF_PS + (size_t)1024 * UB;
constexpr size_t ST_PAR = (size_t)1120 * UB;
constexpr size_t OFF_PK = OFF_ST + 2 * ST_PAR;
constexpr size_t OFF_LZ = OFF_PK + (size_t)640 * 32768;
constexpr size_t OFF_FA = OFF_LZ + (size_t)14 * 8 * RAW0 * UB;
constexpr size_t OFF_FP = OFF_FA + (size_t)14 * NU * FLS;
constexpr size_t WS_NEED = OFF_FP + 1024 * FLS;
static_assert(OFF_FW + 64 * FSTR * 4 <= OFF_FD && OFF_FD + 2560 <= OFF_SD && OFF_SD + 56 * 128 <= OFF_XI, "layout");
static_assert(WS_NEED <= (size_t)161 << 20, "workspace");

struct Params {
    const char* x;
    const char* w;
    char* ws[NR];
    char* out;
    int rank;
    int pad;
};

template <int OFF> __device__ __forceinline__ void st16(const char* sbase, unsigned voff, u32x4 v) {
    asm volatile("global_store_dwordx4 %0, %1, %2 offset:%3 sc0 sc1\n\ts_nop 1" :: "v"(voff), "v"(v), "s"(sbase), "i"(OFF) : "memory");
}
__device__ __forceinline__ unsigned ld4_sys(const void* p) {
    unsigned v;
    asm volatile("global_load_dword %0, %1, off sc0 sc1\n\ts_waitcnt vmcnt(0)" : "=&v"(v) : "v"(p) : "memory");
    return v;
}
__device__ __forceinline__ u32x2 ld8_sys(const void* p) {
    u32x2 v;
    asm volatile("global_load_dwordx2 %0, %1, off sc0 sc1\n\ts_waitcnt vmcnt(0)" : "=&v"(v) : "v"(p) : "memory");
    return v;
}
__device__ __forceinline__ void st8_flag(void* p, unsigned call, unsigned hdr, int lane) {
    if (lane == 0) __hip_atomic_store((unsigned long long*)p, ((unsigned long long)hdr << 32) | call, __ATOMIC_RELAXED, __HIP_MEMORY_SCOPE_SYSTEM);
}
#define WAIT_V_(n) asm volatile("s_waitcnt vmcnt(" #n ")" ::: "memory")
#define WAIT_V(n) WAIT_V_(n)
#define WAIT_L(n) asm volatile("s_waitcnt lgkmcnt(" #n ")" ::: "memory")
#define BAR __builtin_amdgcn_s_barrier()
#define SCHED __builtin_amdgcn_sched_barrier(0)

typedef __bf16 bf16x2 __attribute__((ext_vector_type(2)));
__device__ __forceinline__ unsigned cvt_pk_bf16(float lo, float hi) {
    return __builtin_bit_cast(unsigned, __builtin_convertvector((f32x2){lo, hi}, bf16x2));
}
#define CFENCE asm volatile("" ::: "memory")
__device__ __forceinline__ f32x2 gelu2(f32x2 y) {
    const float k1 = -2.0f * 0.7978845608028654f * 1.4426950408889634f, k2 = k1 * 0.044715f;
    f32x2 z = y * (y * y * k2 + k1);
    f32x2 e; e.x = __builtin_amdgcn_exp2f(z.x); e.y = __builtin_amdgcn_exp2f(z.y);
    e = e + 1.0f;
    f32x2 r; r.x = __builtin_amdgcn_rcpf(e.x); r.y = __builtin_amdgcn_rcpf(e.y);
    return y * r;
}
__device__ __forceinline__ unsigned pk_sub(unsigned a, unsigned b) { return __builtin_bit_cast(unsigned, __builtin_bit_cast(u16x2, a) - __builtin_bit_cast(u16x2, b)); }
__device__ __forceinline__ unsigned pk_add(unsigned a, unsigned b) { return __builtin_bit_cast(unsigned, __builtin_bit_cast(u16x2, a) + __builtin_bit_cast(u16x2, b)); }
__device__ __forceinline__ unsigned pk_max(unsigned a, unsigned b) { return __builtin_bit_cast(unsigned, __builtin_elementwise_max(__builtin_bit_cast(u16x2, a), __builtin_bit_cast(u16x2, b))); }
__device__ __forceinline__ unsigned bfi(unsigned m, unsigned a, unsigned b) { return __builtin_amdgcn_bitop3_b32(m, a, b, 0xca); }
__device__ __forceinline__ unsigned rotl(unsigned a, int n) { return __builtin_amdgcn_alignbit(a, a, 32 - n); }
__device__ __forceinline__ unsigned rotr(unsigned a, int n) { return __builtin_amdgcn_alignbit(a, a, n); }
__device__ __forceinline__ unsigned umax(unsigned a, unsigned b) { return a > b ? a : b; }

__device__ __forceinline__ void dma16(const char* sbase, unsigned voff, unsigned ldst) {
    unsigned keep;
    asm volatile("s_mov_b32 %0, m0\n\ts_mov_b32 m0, %3\n\ts_nop 0\n\tglobal_load_lds_dwordx4 %1, %2 sc0 sc1\n\ts_mov_b32 m0, %0"
                 : "=&s"(keep) : "v"(voff), "s"(sbase), "s"(ldst) : "memory");
}

__device__ __forceinline__ bool give_up(unsigned* tmo, unsigned call, unsigned long long t0, unsigned& polls) {
    if ((++polls & 15u) != 0u) return false;
    if (__builtin_amdgcn_readfirstlane(__hip_atomic_load(tmo, __ATOMIC_RELAXED, __HIP_MEMORY_SCOPE_AGENT)) == call ||
        __builtin_amdgcn_s_memrealtime() - t0 > DEADLINE) {
        __hip_atomic_store(tmo, call, __ATOMIC_RELAXED, __HIP_MEMORY_SCOPE_AGENT);
        return true;
    }
    return false;
}
__device__ __forceinline__ unsigned hdr_cnt(unsigned hdr) { return (hdr >> 15) & 0x3ffu; }
__device__ __forceinline__ unsigned wait_unit(const char* flag, unsigned call, bool full, unsigned* tmo) {
    const unsigned long long t0 = __builtin_amdgcn_s_memrealtime(); unsigned polls = 0;
    for (;;) {
        const u32x2 v = ld8_sys(flag);
        const unsigned hdr = __builtin_amdgcn_readfirstlane(v.y);
        if ((unsigned)__builtin_amdgcn_readfirstlane(v.x) == call && !(full && hdr_cnt(hdr) == LOWCNT)) return hdr;
        __builtin_amdgcn_s_sleep(1);
        if (give_up(tmo, call, t0, polls)) return RAWCNT << 15;
    }
}
__device__ __forceinline__ void wait_word(const unsigned* w, unsigned call, unsigned* tmo) {
    const unsigned long long t0 = __builtin_amdgcn_s_memrealtime(); unsigned polls = 0;
    while ((unsigned)__builtin_amdgcn_readfirstlane(ld4_sys(w)) < call) {
        __builtin_amdgcn_s_sleep(2);
        if (give_up(tmo, call, t0, polls)) break;
    }
}

constexpr int SCR_DENSE = 8192;
__device__ __forceinline__ unsigned umin(unsigned a, unsigned b) { return a < b ? a : b; }
__device__ __forceinline__ unsigned wave_scan(unsigned inc) {
    inc += __builtin_amdgcn_update_dpp(0, (int)inc, 0x111, 0xf, 0xf, true);
    inc += __builtin_amdgcn_update_dpp(0, (int)inc, 0x112, 0xf, 0xf, true);
    inc += __builtin_amdgcn_update_dpp(0, (int)inc, 0x114, 0xf, 0xf, true);
    inc += __builtin_amdgcn_update_dpp(0, (int)inc, 0x118, 0xf, 0xf, true);
    inc += __builtin_amdgcn_update_dpp(0, (int)inc, 0x142, 0xa, 0xf, false);
    inc += __builtin_amdgcn_update_dpp(0, (int)inc, 0x143, 0xc, 0xf, false);
    return inc;
}
__device__ __forceinline__ unsigned pack_base(const u32x4 (&b)[16]) {
    unsigned mx = 0;
#pragma unroll
    for (int f = 0; f < 16; ++f) mx = pk_max(mx, b[f][f & 3] & 0x7fff7fffu);
    unsigned m = umax(mx & 0xffffu, mx >> 16);
    m = umax(m, (unsigned)__builtin_amdgcn_update_dpp(0, (int)m, 0x128, 0xf, 0xf, false));
    m = umax(m, (unsigned)__builtin_amdgcn_update_dpp(0, (int)m, 0x124, 0xf, 0xf, false));
    m = umax(m, (unsigned)__builtin_amdgcn_update_dpp(0, (int)m, 0x122, 0xf, 0xf, false));
    m = umax(m, (unsigned)__builtin_amdgcn_update_dpp(0, (int)m, 0x121, 0xf, 0xf, false));
    const unsigned m0 = __builtin_amdgcn_readlane(m, 0), m1 = __builtin_amdgcn_readlane(m, 16), m2 = __builtin_amdgcn_readlane(m, 32), m3 = __builtin_amdgcn_readlane(m, 48);
    const unsigned ms = umin(umax(umin(m0, m1), umin(m2, m3)), umin(umax(m0, m1), umax(m2, m3)));
    return umax(ms, 1023u) - 1023u;
}
__device__ __forceinline__ u32x4 pack_low(const u32x4 (&b)[16], int i, unsigned L) {
    const unsigned LL = L | (L << 16);
    u32x4 r;
#pragma unroll
    for (int c = 0; c < 4; ++c) r[c] = __builtin_amdgcn_perm(pk_sub(b[2 * i + (c >> 1)][2 * (c & 1) + 1], LL), pk_sub(b[2 * i + (c >> 1)][2 * (c & 1)], LL), 0x06040200u);
    return r;
}
__device__ __forceinline__ unsigned pack_unit(const u32x4 (&b)[16], u32x4 (&pk)[12], unsigned L, LAS unsigned char* scr, int lane) {
    const unsigned LL = L | (L << 16);
    const unsigned wp0 = (unsigned)(size_t)scr + lane * 4;
    unsigned wp = wp0;
    unsigned z[4], seg[4], len[4], F = 0;
#pragma unroll
    for (int s = 0; s < 4; ++s) {
        unsigned hb[8];
#pragma unroll
        for (int jj = 0; jj < 8; ++jj) {
            const int j = 8 * s + jj;
            const unsigned d0 = pk_sub(b[j >> 1][2 * (j & 1)], LL), d1 = pk_sub(b[j >> 1][2 * (j & 1) + 1], LL);
            pk[j >> 2][j & 3] = __builtin_amdgcn_perm(d1, d0, 0x06040200u);
            hb[jj] = __builtin_amdgcn_perm(d1, d0, 0x07050301u);
            const unsigned t = hb[jj] & 0x7c7c7c7cu; unsigned tmp;
            *(LAS unsigned*)(size_t)wp = t | (unsigned)((j & 3) | ((j >> 2) << 7));
            asm("v_min_u32 %0, 1, %2\n\tv_lshl_add_u32 %1, %0, 8, %1" : "=&v"(tmp), "+v"(wp) : "v"(t));
            if (jj & 3) hb[jj] = rotl(hb[jj], 2 * (jj & 3));
        }
        pk[8][s] = bfi(0x81818181u, hb[0], bfi(0x06060606u, hb[1], bfi(0x18181818u, hb[2], hb[3])));
        pk[9][s] = bfi(0x81818181u, hb[4], bfi(0x06060606u, hb[5], bfi(0x18181818u, hb[6], hb[7])));
        const unsigned ga = bfi(0x02020202u, hb[0], bfi(0x08080808u, hb[1], bfi(0x20202020u, hb[2], hb[3])));
        const unsigned gb = bfi(0x02020202u, hb[4], bfi(0x08080808u, hb[5], bfi(0x20202020u, hb[6], hb[7])));
        z[s] = ~bfi(0xaaaaaaaau, ga, gb >> 1);
    }
    unsigned e8[8], nz4[4];
#pragma unroll
    for (int i = 0; i < 8; ++i) e8[i] = *(LAS unsigned*)(scr + i * 256 + lane * 4);
#pragma unroll
    for (int s = 0; s < 4; ++s) {
        unsigned t = z[s] | (z[s] >> 1); t |= t << 2;
        nz4[s] = t & 0x44444444u; F |= s < 2 ? nz4[s] >> (2 - s) : nz4[s] << (s - 2);
    }
    const unsigned nrow = (wp - wp0) >> 8, mine = (__builtin_popcount(F) << 14) | nrow;
    const unsigned inc = wave_scan(mine), tot = __builtin_amdgcn_readlane(inc, 63), ngrp = tot & 0xfffu;
    const unsigned lo = 256u + ((((tot >> 12) + 31u) >> 5) << 2);
    unsigned cnt = ngrp, bad = 0;
    if (ngrp <= 512u) {
        LAS unsigned* dense = (LAS unsigned*)(scr + SCR_DENSE);
        const unsigned lanebits = ((lane & 7) << 15) | ((lane >> 3) << 23), base = (inc - mine) & 0xfffu;
#pragma unroll
        for (int i = 0; i < 8; ++i) dense[(unsigned)i < nrow ? base + i : 512 + lane] = e8[i] | lanebits;
        for (unsigned r = 8; __builtin_expect(__any(nrow > r), 0); ++r)
            dense[r < nrow ? base + r : 512 + lane] = *(LAS unsigned*)(scr + r * 256 + lane * 4) | lanebits;
        unsigned e2[2] = {dense[lane], dense[64 + lane]};
        CFENCE;
        *(LAS u32x4*)(scr + 256 + lane * 16) = (u32x4){0u, 0u, 0u, 0u};
        *(LAS unsigned*)(scr + lane * 4) = F;
        CFENCE;
#pragma unroll
        for (int s = 0; s < 4; ++s) { seg[s] = z[s] & 15u; len[s] = nz4[s] & 7u; }
#pragma unroll
        for (int i = 1; i < 8; ++i)
#pragma unroll
            for (int s = 0; s < 4; ++s) { seg[s] |= ((z[s] >> (4 * i)) & 15u) << len[s]; len[s] += (nz4[s] >> (4 * i)) & 7u; }
        unsigned bit = (inc - mine) >> 12;
#pragma unroll
        for (int s = 0; s < 4; ++s) {
            const unsigned long long v = (unsigned long long)seg[s] << (bit & 31u);
            LAS unsigned* w = (LAS unsigned*)(scr + 256) + (bit >> 5);
            __hip_atomic_fetch_or(w, (unsigned)v, __ATOMIC_RELAXED, __HIP_MEMORY_SCOPE_WAVEFRONT);
            __hip_atomic_fetch_or(w + 1, (unsigned)(v >> 32), __ATOMIC_RELAXED, __HIP_MEMORY_SCOPE_WAVEFRONT);
            bit += len[s];
        }
        CFENCE;
        LAS unsigned short* lst = (LAS unsigned short*)(scr + lo);
        const unsigned hz = ((0u - L) >> 10) & 31u;
        for (unsigned i0 = 0;;) {
            unsigned t[2], pos[2], at[2];
#pragma unroll
            for (int c = 0; c < 2; ++c) {
                const unsigned e = e2[c], ix = i0 + c * 64 + lane;
                t[c] = ix < ngrp ? e & 0x7c7c7c7cu : 0u;
                pos[c] = ((e & 0x380u) << 3) | ((e >> 16) & 0x380u) | ((e >> 11) & 0x70u) | ((e & 3u) << 2);
                at[c] = ix < ngrp ? ix : 1024u + lane;
            }
            for (;;) {
#pragma unroll
                for (int c = 0; c < 2; ++c) {
                    const unsigned b8 = __builtin_ctz(t[c] | 0x80000000u) & 24u, hi5 = (t[c] >> (b8 + 2)) & 31u;
                    const unsigned code = hi5 == hz ? 5u : (hi5 + 5u) & 31u;
                    bad = umax(bad, code);
                    lst[at[c]] = (unsigned short)((code << 13) | pos[c] | (b8 >> 3));
                    t[c] &= ~(0xffu << b8);
                }
                const unsigned long long b0 = __ballot(t[0] != 0), b1 = __ballot(t[1] != 0);
                if (!(b0 | b1)) break;
                at[0] = t[0] != 0 ? __builtin_amdgcn_mbcnt_hi((unsigned)(b0 >> 32), __builtin_amdgcn_mbcnt_lo((unsigned)b0, umin(cnt, 512u))) : 1024u + lane;
                cnt += __builtin_popcountll(b0);
                at[1] = t[1] != 0 ? __builtin_amdgcn_mbcnt_hi((unsigned)(b1 >> 32), __builtin_amdgcn_mbcnt_lo((unsigned)b1, umin(cnt, 512u))) : 1024u + lane;
                cnt += __builtin_popcountll(b1);
            }
            i0 += 128;
            if (__builtin_expect(i0 >= ngrp, 1)) break;
            e2[0] = dense[i0 + lane]; e2[1] = dense[i0 + 64 + lane];
        }
    }
    CFENCE;
    pk[10] = *(LAS u32x4*)(scr + lane * 16);
    pk[11] = *(LAS u32x4*)(scr + 1024 + lane * 16);
    const unsigned tl = (lo + 2u * cnt + 31u) >> 5;
    if (cnt > 512u || tl > 64u || __any(bad > 7u) || FORCE_RAW) return RAWCNT << 15;
    return L | (cnt << 15) | (tl << 25);
}
__device__ __forceinline__ void st_tail(const char* sb8k  , unsigned voff, u32x4 t0, u32x4 t1, unsigned tl, int lane) {
    if (__builtin_expect(tl > 32u, 0)) { if ((unsigned)lane < (tl - 32u) * 2u) st16<3072>(sb8k, voff, t1); }
    if ((unsigned)lane < tl * 2u) st16<2048>(sb8k, voff, t0);
}
__device__ __forceinline__ void st_raw(const char* slot, unsigned voff, const u32x4 (&b)[16]) {
    const char* s0 = slot + 4096; const char* s1 = slot + 12288;
    st16<-4096>(s0, voff, b[0]); st16<-3072>(s0, voff, b[1]); st16<-2048>(s0, voff, b[2]); st16<-1024>(s0, voff, b[3]);
    st16<0>(s0, voff, b[4]); st16<1024>(s0, voff, b[5]); st16<2048>(s0, voff, b[6]); st16<3072>(s0, voff, b[7]);
    st16<-4096>(s1, voff, b[8]); st16<-3072>(s1, voff, b[9]); st16<-2048>(s1, voff, b[10]); st16<-1024>(s1, voff, b[11]);
    st16<0>(s1, voff, b[12]); st16<1024>(s1, voff, b[13]); st16<2048>(s1, voff, b[14]); st16<3072>(s1, voff, b[15]);
}
__device__ __forceinline__ void ld4x16(f32x4& a, f32x4& b, f32x4& c, f32x4& d, const char* sb, unsigned voff) {
    asm volatile("global_load_dwordx4 %0, %4, %5 sc0 sc1\n\tglobal_load_dwordx4 %1, %4, %5 offset:1024 sc0 sc1\n\t"
                 "global_load_dwordx4 %2, %4, %5 offset:2048 sc0 sc1\n\tglobal_load_dwordx4 %3, %4, %5 offset:3072 sc0 sc1\n\ts_waitcnt vmcnt(0)"
                 : "=&v"(a), "=&v"(b), "=&v"(c), "=&v"(d) : "v"(voff), "s"(sb) : "memory");
}
__device__ __forceinline__ void ld8x16s(f32x4 (&t)[8], const char* sb  , unsigned voff) {
    asm volatile("global_load_dwordx4 %0, %8, %9 offset:-4096 sc0 sc1\n\tglobal_load_dwordx4 %1, %8, %9 offset:-2048 sc0 sc1\n\t"
                 "global_load_dwordx4 %2, %8, %9 sc0 sc1\n\tglobal_load_dwordx4 %3, %8, %9 offset:2048 sc0 sc1\n\t"
                 "global_load_dwordx4 %4, %8, %10 offset:-4096 sc0 sc1\n\tglobal_load_dwordx4 %5, %8, %10 offset:-2048 sc0 sc1\n\t"
                 "global_load_dwordx4 %6, %8, %10 sc0 sc1\n\tglobal_load_dwordx4 %7, %8, %10 offset:2048 sc0 sc1\n\ts_waitcnt vmcnt(0)"
                 : "=&v"(t[0]), "=&v"(t[1]), "=&v"(t[2]), "=&v"(t[3]), "=&v"(t[4]), "=&v"(t[5]), "=&v"(t[6]), "=&v"(t[7])
                 : "v"(voff), "s"(sb), "s"(sb + 16384) : "memory");
}
constexpr int DEC_SLOT = 8192, DEC_TAIL = DEC_SLOT + 10240;
__device__ __forceinline__ void patch_apply(LAS unsigned char* dl, unsigned lo, unsigned hdr, int lane, bool zero) {
    const unsigned cnt = hdr_cnt(hdr), lut_hi = 0x0804007cu | ((((0u - (hdr & 0x7fffu)) >> 10) & 31u) << 10);
    for (unsigned c = 0; c < cnt; c += 128) {
        const unsigned w = *(LAS unsigned*)(dl + DEC_TAIL + lo + c * 2 + lane * 4);
#pragma unroll
        for (int hf = 0; hf < 2; ++hf) {
            const unsigned e = hf ? w >> 16 : w & 0xffffu;
            const unsigned at = c + lane * 2 + hf < cnt ? e & 0x1fffu : (unsigned)DEC_TAIL + lane;
            dl[at] = zero ? 0 : (unsigned char)__builtin_amdgcn_perm(lut_hi, 0x7874706cu, e >> 13);
        }
    }
}
__device__ __forceinline__ unsigned unpack_head(unsigned F, unsigned hdr, LAS unsigned char* dl, int lane, u32x4& G) {
    const unsigned mine = __builtin_popcount(F) << 2, inc = wave_scan(mine);
    const unsigned lo = 256u + ((((unsigned)__builtin_amdgcn_readlane(inc, 63) + 31u) >> 5) << 2);
    unsigned bit = inc - mine, sg[4];
#pragma unroll
    for (int s = 0; s < 4; ++s) {
        const LAS unsigned* w = (const LAS unsigned*)(dl + DEC_TAIL + 256) + (bit >> 5);
        sg[s] = __builtin_amdgcn_alignbit(w[1], w[0], bit);
        bit += __builtin_popcount(F & (0x11111111u << s)) << 2;
    }
    patch_apply(dl, lo, hdr, lane, false);
#pragma unroll
    for (int s = 0; s < 4; ++s) {
        const unsigned nz = (F >> s) & 0x11111111u, pre = nz * 0x11111110u;
        const unsigned xe = sg[s] & 0x0f0f0f0fu, xo = (sg[s] >> 4) & 0x0f0f0f0fu;
        const unsigned b0 = __builtin_amdgcn_perm(xo, xe, 0x05010400u), b1 = __builtin_amdgcn_perm(xo, xe, 0x07030602u);
        const unsigned ze = __builtin_amdgcn_perm(b1, b0, pre & 0x07070707u), zo = __builtin_amdgcn_perm(b1, b0, (pre >> 4) & 0x07070707u);
        G[s] = ~((ze | (zo << 4)) & ((nz << 4) - nz));
    }
    return lo;
}
__device__ __forceinline__ void unpack_piece(u32x4 lo, const u32x4 (&P)[2], u32x4 G, int i, LAS unsigned char* dl, int lane, unsigned LL, u32x4 (&o)[2]) {
    const u32x4 pt = *(LAS u32x4*)(dl + i * 1024 + lane * 16);
    const unsigned A = P[i & 1][i >> 1], g = G[i >> 1] << (i & 1);
    unsigned hb[4];
    hb[0] = bfi(0x83838383u, bfi(0x02020202u, g, A), pt[0]); hb[1] = bfi(0x83838383u, rotr(bfi(0x08080808u, g, A), 2), pt[1]);
    hb[2] = bfi(0x83838383u, rotr(bfi(0x20202020u, g, A), 4), pt[2]); hb[3] = bfi(0x83838383u, rotr(bfi(0x80808080u, g, A), 6), pt[3]);
#pragma unroll
    for (int j = 0; j < 4; ++j) {
        o[j >> 1][2 * (j & 1)] = pk_add(__builtin_amdgcn_perm(hb[j], lo[j], 0x05010400u), LL);
        o[j >> 1][2 * (j & 1) + 1] = pk_add(__builtin_amdgcn_perm(hb[j], lo[j], 0x07030602u), LL);
    }
}

__device__ __forceinline__ void unit_fetch(const Params& p, int h, int u, int lane, unsigned ldsbuf) {
    const bool isx = u < 16;
    const char* base = isx ? p.x + ((size_t)u * 256 * KS + h * 32) * 2 : p.w + ((size_t)h * 32 * NN + (u - 16) * 256) * 2;
    unsigned voff = isx ? (lane & 15) * (KS * 2) + (lane >> 4) * 16 : (lane >> 4) * (8 * NN * 2) + (lane & 15) * 16;
    const unsigned s1 = isx ? 16 * KS * 2 : NN * 2, s2 = isx ? 16 * KS * 2 : 256 - 7 * NN * 2;
#pragma unroll
    for (int i = 0; i < 16; ++i) {
        unsigned keep;
        asm volatile("s_mov_b32 %0, m0\n\ts_mov_b32 m0, %3\n\ts_nop 0\n\tglobal_load_lds_dwordx4 %1, %2\n\ts_mov_b32 m0, %0"
                     : "=&s"(keep) : "v"(voff), "s"(base), "s"(ldsbuf + i * 1024) : "memory");
        voff += (i == 7) ? s2 : s1;
    }
}
__device__ __forceinline__ void unit_read(LAS unsigned char* buf, int lane, u32x4 (&r)[16]) {
#pragma unroll
    for (int i = 0; i < 16; ++i) r[i] = *(LAS u32x4*)(buf + i * 1024 + lane * 16);
    asm volatile("s_waitcnt lgkmcnt(0)" : "+v"(r[0]), "+v"(r[1]), "+v"(r[2]), "+v"(r[3]), "+v"(r[4]), "+v"(r[5]), "+v"(r[6]), "+v"(r[7]),
                 "+v"(r[8]), "+v"(r[9]), "+v"(r[10]), "+v"(r[11]), "+v"(r[12]), "+v"(r[13]), "+v"(r[14]), "+v"(r[15]) :: "memory");
}
__device__ __forceinline__ void unit_format(int u, u32x4 (&r)[16]) {
    if (u >= 16) {
#pragma unroll
        for (int g = 0; g < 2; ++g) {
            u32x4 o[8];
#pragma unroll
            for (int i = 0; i < 8; ++i)
#pragma unroll
                for (int a = 0; a < 4; ++a)
                    o[i][a] = __builtin_amdgcn_perm(r[g * 8 + 2 * a + 1][i >> 1], r[g * 8 + 2 * a][i >> 1], (i & 1) ? 0x07060302u : 0x05040100u);
#pragma unroll
            for (int i = 0; i < 8; ++i) r[g * 8 + i] = o[i];
        }
    }
}
struct Tabs { unsigned ord[NU]; unsigned task[NTASK]; unsigned pull[7][NU]; };
constexpr Tabs make_tabs() {
    Tabs t{}; int n = 0;
    for (int v = 0; v < 8 * RAW0; ++v) t.ord[n++] = (v >> 3) << 8 | (v & 7);
    for (int key = 0; key < 8 * NH + 128; ++key)
        for (int v = 0; v < 22; ++v) {
            const int d = key - (v < 8 ? ((v & 2) ? LAGWH : 0) : v < 15 ? LAGXA : LAGXB);
            if (d >= 0 && d % 8 == 0 && d / 8 < NH && !(v < 8 && d / 8 < RAW0)) t.ord[n++] = (unsigned)((d / 8) << 8 | v);
        }
    int inv[NU] = {};
    for (int k = 0; k < NU; ++k) inv[(t.ord[k] >> 8) * 22 + (t.ord[k] & 255)] = k;
    for (int li = 0; li < 7; ++li) {
        n = 0;
        for (int k = 0; k < 8 * RAW0; ++k) t.pull[li][n++] = k << 16 | t.ord[k];
        for (int key = 0; key < 8 * NH + 128; ++key)
            for (int v = 0; v < 22; ++v) {
                const int d = key - (v == 14 - li || v == 21 - li ? LAGOWN : v < 8 ? ((v & 2) ? LAGWH : 0) : v < 15 ? LAGXA : LAGXB);
                if (d >= 0 && d % 8 == 0 && d / 8 < NH && !(v < 8 && d / 8 < RAW0)) t.pull[li][n++] = (unsigned)(inv[d / 8 * 22 + v] << 16 | (d / 8) << 8 | v);
            }
    }
    n = 0;
    for (int k = 0; k < NU; ++k) {
        const unsigned h = t.ord[k] >> 8, v = t.ord[k] & 255, e = k | h << 9;
        for (unsigned l = 0; l < (v < 8 ? 7u : 1u); ++l) t.task[n++] = e | v << 13 | l << 18;
    }
    return t;
}
__constant__ const Tabs g_tabs = make_tabs();
__device__ __forceinline__ int x_tile(int v, int src) { return v >= 15 ? 2 * ((src + v - 14) & 7) + 1 : 2 * ((src + v - 7) & 7); }
__device__ __forceinline__ int st_slot(int h, int v, int link) { return v < 8 ? (link * NH + h) * 8 + v : 896 + h * 14 + v - 8; }

__device__ __forceinline__ void producer(const Params& p, LAS unsigned char* buf, int n0, int lane, unsigned* cntp) {
    char* ws = p.ws[p.rank];
    const unsigned voff = lane * 16;
    const unsigned long long ts = __builtin_amdgcn_s_memrealtime();
    char* peer = p.ws[(p.rank + 1 + (lane & 7)) & 7];
    unsigned call = 0;
    for (int n = n0; n < NTASK; n += 576) {
        const bool first = n < 56 * RAW0;
        const unsigned e = first ? (n / 7) | ((n / 7) >> 3) << 9 | ((n / 7) & 7) << 13 | (n % 7) << 18 : g_tabs.task[n];
        const int k = e & 511, h = (e >> 9) & 15, v = (e >> 13) & 31, link = e >> 18;
        const bool isx = v >= 8;
        const unsigned dest = (unsigned)(p.rank + 1 + link) & 7u;
        const int t = isx ? x_tile(v, p.rank) : dest * 8 + v;
        while (__builtin_amdgcn_s_memrealtime() - ts < (unsigned)(first ? 0 : (n - 56 * RAW0) * PDT / 16)) __builtin_amdgcn_s_sleep(2);
        unit_fetch(p, h, isx ? t : 16 + t, lane, (unsigned)(size_t)buf);
        if (n == n0) call = __builtin_amdgcn_readfirstlane(ld4_sys(cntp)) + 1u;
        WAIT_V(0);
        u32x4 r[16];
        unit_read(buf, lane, r);
        unit_format(isx ? 0 : 16, r);
        unsigned hdr = RAWCNT << 15;
        if (first) {
            const char* s0 = p.ws[dest] + OFF_LZ + (size_t)(((call & 1u) * 7 + 6 - link) * 8 * RAW0 + k) * UB + 4096; const char* s1 = s0 + 8192;
            st16<-4096>(s0, voff, r[0]); st16<-3072>(s0, voff, r[1]); st16<-2048>(s0, voff, r[2]); WAIT_V(PWIN); st16<-1024>(s0, voff, r[3]);
            WAIT_V(PWIN); st16<0>(s0, voff, r[4]); WAIT_V(PWIN); st16<1024>(s0, voff, r[5]); WAIT_V(PWIN); st16<2048>(s0, voff, r[6]); WAIT_V(PWIN); st16<3072>(s0, voff, r[7]);
            WAIT_V(PWIN); st16<-4096>(s1, voff, r[8]); WAIT_V(PWIN); st16<-3072>(s1, voff, r[9]); WAIT_V(PWIN); st16<-2048>(s1, voff, r[10]); WAIT_V(PWIN); st16<-1024>(s1, voff, r[11]);
            WAIT_V(PWIN); st16<0>(s1, voff, r[12]); WAIT_V(PWIN); st16<1024>(s1, voff, r[13]); WAIT_V(PWIN); st16<2048>(s1, voff, r[14]); WAIT_V(PWIN); st16<3072>(s1, voff, r[15]);
        } else {
            const char* sl = ws + OFF_ST + (call & 1u) * ST_PAR + (size_t)st_slot(h, v, link) * UB;
            u32x4 pk[12];
            hdr = pack_unit(r, pk, pack_base(r), buf, lane);
            if (__builtin_expect(hdr_cnt(hdr) == RAWCNT, 0)) st_raw(sl, voff, r);
            else {
                const char* s0 = sl + 4096; const char* s1 = sl + 8192;
                st16<-4096>(s0, voff, pk[0]); st16<-3072>(s0, voff, pk[1]); st16<-2048>(s0, voff, pk[2]); st16<-1024>(s0, voff, pk[3]);
                st16<0>(s0, voff, pk[4]); st16<1024>(s0, voff, pk[5]); st16<2048>(s0, voff, pk[6]); st16<3072>(s0, voff, pk[7]);
                st16<0>(s1, voff, pk[8]); st16<1024>(s1, voff, pk[9]); st_tail(s1, voff, pk[10], pk[11], hdr >> 25, lane);
            }
        }
        WAIT_V(0);
        if (isx ? lane < 7 : lane == link)
            __hip_atomic_store((unsigned long long*)(peer + OFF_FA + (size_t)(((call & 1u) * 7 + 6 - lane) * NU + k) * FLS), ((unsigned long long)hdr << 32) | call,
                               __ATOMIC_RELAXED, __HIP_MEMORY_SCOPE_SYSTEM);
        if (!isx) {
            st_raw(ws + OFF_WD + (size_t)(link * NH + h) * WQ + (size_t)v * UB, voff, r);
            WAIT_V(0);
            if (lane == 0) __hip_atomic_store((unsigned*)(ws + OFF_FW) + t * FSTR + h, call, __ATOMIC_RELAXED, __HIP_MEMORY_SCOPE_SYSTEM);
        }
        WAIT_L(0);
    }
}
__device__ __forceinline__ void own_image(const Params& p, LAS unsigned char* buf, int n, int lane, unsigned call) {
    char* ws = p.ws[p.rank];
    const int h = n / 24, v = n - h * 24, t = v < 16 ? v : p.rank * 8 + v - 16;
    unit_fetch(p, h, v < 16 ? t : 16 + t, lane, (unsigned)(size_t)buf);
    WAIT_V(0);
    u32x4 r[16];
    unit_read(buf, lane, r);
    unit_format(v, r);
    st_raw(ws + (v < 16 ? OFF_XI + (size_t)h * XQ : OFF_WI + (size_t)h * WQ) + (size_t)(v & 15) * UB, lane * 16, r);
    WAIT_V(0);
    if (lane == 0) __hip_atomic_store((unsigned*)(ws + (v < 16 ? OFF_FX : OFF_FW)) + t * FSTR + h, call, __ATOMIC_RELAXED, __HIP_MEMORY_SCOPE_SYSTEM);
}

constexpr int DEC_LDS = 24 * 1024;
struct PullCtl { u32x4 desc[16]; unsigned full[7]; unsigned tail; };
__device__ __forceinline__ void decoder(const Params& p, LAS unsigned char* dl, LAS PullCtl* ctl, int xcd, int li, int mem, int lane, unsigned call) {
    char* ws = p.ws[p.rank];
    unsigned* tmo = (unsigned*)(ws + OFF_TMO);
    const unsigned voff = lane * 16;
    const int src = (p.rank + 1 + li) & 7;
    const char* fa = ws + OFF_FA + (size_t)((call & 1u) * 7 + li) * NU * FLS;
    unsigned* tk = (unsigned*)(ws + OFF_TK) + li * 32;
    const unsigned tk0 = (call - 1u) * (NU - 8 * RAW0) - (RAW0 + NDEC) * 8;
#pragma unroll
    for (int i = 0; i < 8; ++i) *(LAS u32x4*)(dl + i * 1024 + lane * 16) = (u32x4){0u, 0u, 0u, 0u};
    unsigned k = (RAW0 + mem) * 8 + xcd, n = 0;
    while (k < (unsigned)NU) {
        const int e = g_tabs.pull[li][k], h = (e >> 8) & 255, v = e & 255;
        const unsigned hdr = wait_unit(fa + (size_t)(e >> 16) * FLS, call, false, tmo);
        ++n;
        if (lane == 0) {
            const unsigned pos = __hip_atomic_fetch_add(&ctl->tail, 1u, __ATOMIC_RELAXED, __HIP_MEMORY_SCOPE_WORKGROUP);
            *(volatile LAS u32x4*)&ctl->desc[pos & 15] = (u32x4){pos + 1u, k | (unsigned)st_slot(h, v, 6 - li) << 16, hdr, (unsigned)mem | (n << 8)};
        }
        const bool isx = v >= 8, raw = hdr_cnt(hdr) == RAWCNT;
        const int q = NH + h * 7 + li, t = isx ? x_tile(v, src) : v;
        const char* dst = ws + (isx ? OFF_XI + (size_t)q * XQ : OFF_WI + (size_t)q * WQ) + (size_t)t * UB;
        unsigned* flag = (unsigned*)(ws + (isx ? OFF_FX : OFF_FW)) + (isx ? t : p.rank * 8 + t) * FSTR + q;
        unsigned tn = 1u;
        if (lane == 0) asm volatile("global_atomic_add %0, %1, %0, off sc0\n\ts_waitcnt vmcnt(0)" : "+v"(tn) : "v"(tk) : "memory");
        while ((unsigned)__builtin_amdgcn_readfirstlane(*(volatile LAS unsigned*)&ctl->full[mem]) != n) __builtin_amdgcn_s_sleep(1);
        asm volatile("" ::: "memory");
        unsigned lo = 0;
        if (__builtin_expect(raw, 0)) {
            u32x4 r[16];
            unit_read(dl + DEC_SLOT, lane, r);
            st_raw(dst, voff, r);
        } else {
            const unsigned F = *(LAS unsigned*)(dl + DEC_TAIL + lane * 4), LL = (hdr & 0x7fffu) * 0x10001u;
            u32x4 P[2], G, lw[8];
            P[0] = *(LAS u32x4*)(dl + DEC_SLOT + 8192 + lane * 16); P[1] = *(LAS u32x4*)(dl + DEC_SLOT + 9216 + lane * 16);
#pragma unroll
            for (int i = 0; i < 8; ++i) lw[i] = *(LAS u32x4*)(dl + DEC_SLOT + i * 1024 + lane * 16);
            lo = unpack_head(F, hdr, dl, lane, G);
#pragma unroll
            for (int i = 0; i < 8; ++i) {
                u32x4 o[2]; unpack_piece(lw[i], P, G, i, dl, lane, LL, o);
                const char* sb = dst + i * 2048;
                st16<0>(sb, voff, o[0]); st16<1024>(sb, voff, o[1]);
            }
        }
        WAIT_V(0);
        if (lane == 0) __hip_atomic_store(flag, call, __ATOMIC_RELAXED, __HIP_MEMORY_SCOPE_SYSTEM);
        if (!raw) patch_apply(dl, lo, hdr, lane, true);
        WAIT_L(0);
        k = __builtin_amdgcn_readfirstlane(tn) - tk0;
    }
    if (lane == 0) {
        const unsigned pos = __hip_atomic_fetch_add(&ctl->tail, 1u, __ATOMIC_RELAXED, __HIP_MEMORY_SCOPE_WORKGROUP);
        *(volatile LAS u32x4*)&ctl->desc[pos & 15] = (u32x4){pos + 1u, 0u, 0u, 0x80u};
    }
}
__device__ __forceinline__ void opener(const Params& p, int li, int h, int ct, int lane, unsigned call) {
    char* ws = p.ws[p.rank];
    wait_unit(ws + OFF_FA + (size_t)(((call & 1u) * 7 + li) * NU + h * 8 + ct) * FLS, call, false, (unsigned*)(ws + OFF_TMO));
    const char* src = ws + OFF_LZ + (size_t)(((call & 1u) * 7 + li) * 8 * RAW0 + h * 8 + ct) * UB;
    const int q = NH + h * 7 + li;
    const char* dst = ws + OFF_WI + (size_t)q * WQ + (size_t)ct * UB;
    for (int i = 0; i < 4; ++i) {
        f32x4 t[4]; ld4x16(t[0], t[1], t[2], t[3], src + i * 4096, lane * 16);
        const char* d = dst + i * 4096;
        st16<0>(d, lane * 16, __builtin_bit_cast(u32x4, t[0])); st16<1024>(d, lane * 16, __builtin_bit_cast(u32x4, t[1]));
        st16<2048>(d, lane * 16, __builtin_bit_cast(u32x4, t[2])); st16<3072>(d, lane * 16, __builtin_bit_cast(u32x4, t[3]));
    }
    WAIT_V(0);
    if (lane == 0) __hip_atomic_store((unsigned*)(ws + OFF_FW) + (p.rank * 8 + ct) * FSTR + q, call, __ATOMIC_RELAXED, __HIP_MEMORY_SCOPE_SYSTEM);
}
__device__ __forceinline__ void loader(const Params& p, LAS unsigned char* lds, LAS PullCtl* ctl, int li, int lane, unsigned call) {
    const char* st = p.ws[(unsigned)(p.rank + 1 + li) & 7u] + OFF_ST + (call & 1u) * ST_PAR;
    const unsigned voff = lane * 16;
    unsigned pos = 0, done = 0, pend = 0, idle = 0;
#define LANDED() do { *(volatile LAS unsigned*)&ctl->full[pend & 7u] = pend >> 8; pend = 0; } while (0)
    while (done < NDEC || pend) {
        const u32x4 d = *(volatile LAS u32x4*)&ctl->desc[pos & 15];
        if ((unsigned)__builtin_amdgcn_readfirstlane(d.x) != pos + 1u) {
            if (!pend) __builtin_amdgcn_s_sleep(1);
            else if (idle == 0) WAIT_V(4);
            else if (idle == 1) WAIT_V(2);
            else { WAIT_V(0); LANDED(); }
            ++idle;
            continue;
        }
        idle = 0;
        const unsigned ks = __builtin_amdgcn_readfirstlane(d.y), hdr = __builtin_amdgcn_readfirstlane(d.z), w = __builtin_amdgcn_readfirstlane(d.w);
        ++pos;
        if (w & 0x80u) { ++done; continue; }
        if ((ks & 0xffffu) == NU - 1 - GATE_EARLY && lane < 8)
            __hip_atomic_store((unsigned*)(p.ws[p.rank] + OFF_SD) + (li * 8 + lane) * 32, call, __ATOMIC_RELAXED, __HIP_MEMORY_SCOPE_AGENT);
        const char* s = st + (size_t)(ks >> 16) * UB;
        const unsigned l = (unsigned)(size_t)lds + (w & 7u) * DEC_LDS + DEC_SLOT;
#pragma unroll
        for (int i = 0; i < 10; ++i) {
            WAIT_V(LWIN);
            if (i == LWIN && pend) LANDED();
            dma16(s + i * 1024, voff, l + i * 1024);
        }
        static_assert(LWIN < 10, "window");
        if (__builtin_expect(hdr_cnt(hdr) == RAWCNT, 0)) {
#pragma unroll
            for (int i = 10; i < 16; ++i) { WAIT_V(LWIN); dma16(s + i * 1024, voff, l + i * 1024); }
        } else {
            const unsigned tl = hdr >> 25;
            WAIT_V(LWIN);
            if ((unsigned)lane < tl * 2u) dma16(s + 10240, voff, l + 10240);
            if (__builtin_expect(tl > 32u, 0)) { if ((unsigned)lane < (tl - 32u) * 2u) dma16(s + 11264, voff, l + 11264); }
        }
        pend = w;
    }
}

__device__ __forceinline__ void post_ready(bool r0, bool r1, unsigned long long& td0, unsigned long long& td1, LAS unsigned* queue, int& nq, unsigned lane) {
    const unsigned long long a0 = __ballot(r0) & td0, a1 = __ballot(r1) & td1;
    if ((a0 >> lane) & 1) ((volatile LAS unsigned*)queue)[__builtin_amdgcn_mbcnt_hi((unsigned)(a0 >> 32), __builtin_amdgcn_mbcnt_lo((unsigned)a0, nq))] = lane | 0x10000u;
    nq += __builtin_popcountll(a0);
    if ((a1 >> lane) & 1) ((volatile LAS unsigned*)queue)[__builtin_amdgcn_mbcnt_hi((unsigned)(a1 >> 32), __builtin_amdgcn_mbcnt_lo((unsigned)a1, nq))] = lane | 0x10040u;
    nq += __builtin_popcountll(a1);
    td0 &= ~a0; td1 &= ~a1;
}
__device__ __forceinline__ void poll_ready(const unsigned* fx, const unsigned* fw, unsigned long long& td0, unsigned long long& td1,
                                           LAS unsigned* queue, int& nq, int need, unsigned call, unsigned* tmo) {
    const unsigned lane = __builtin_amdgcn_mbcnt_hi(~0u, __builtin_amdgcn_mbcnt_lo(~0u, 0u));
    {
        const volatile LAS unsigned* sn = queue + 256 + lane;
        const unsigned x0 = sn[0], x1 = sn[64], w0 = sn[128], w1 = sn[192];
        post_ready(x0 == call && w0 == call, x1 == call && w1 == call, td0, td1, queue, nq, lane);
    }
    const unsigned long long t0 = __builtin_amdgcn_s_memrealtime(); unsigned polls = 0;
    bool all = false;
    while (nq < need) {
        unsigned x0, x1, w0, w1;
        asm volatile("global_load_dword %0, %4, %5 sc0 sc1\n\tglobal_load_dword %1, %4, %5 offset:256 sc0 sc1\n\t"
                     "global_load_dword %2, %4, %6 sc0 sc1\n\tglobal_load_dword %3, %4, %6 offset:256 sc0 sc1\n\ts_waitcnt vmcnt(0)"
                     : "=&v"(x0), "=&v"(x1), "=&v"(w0), "=&v"(w1) : "v"(lane * 4), "s"(fx), "s"(fw) : "memory");
        post_ready(all || (x0 >= call && w0 >= call), all || (x1 >= call && w1 >= call), td0, td1, queue, nq, lane);
        if (nq >= need) break;
        all = give_up(tmo, call, t0, polls);
        __builtin_amdgcn_s_sleep(1);
    }
    WAIT_L(0);
}

constexpr int HTB = 16384;
constexpr int FIN_LDS = 20480;

__global__ __launch_bounds__(512, 2) void fused_kernel(const Params p) {
    __shared__ __attribute__((aligned(16))) unsigned char shm[8 * FIN_LDS];
    LAS unsigned char* lds = (LAS unsigned char*)shm;
    const int wid = __builtin_amdgcn_readfirstlane(threadIdx.x >> 6), lane = threadIdx.x & 63;
    const int xcd = blockIdx.x & 7, slot = blockIdx.x >> 3;
    char* ws = p.ws[p.rank];
    unsigned* tmo = (unsigned*)(ws + OFF_TMO);
    unsigned* cntp = (unsigned*)(ws + OFF_CNT) + blockIdx.x * 8 + wid;
    const bool main_tile = slot >= 16;
    const int fsrc = (p.rank + 1 + slot) & 7, fct = xcd;
    const bool helper = !main_tile && (slot & 7) == 7;
    LAS PullCtl* ctl = (LAS PullCtl*)(lds + NDEC * DEC_LDS);
    static_assert(NDEC <= 5 && RAW0 <= 2, "puller CU: load wave, 5 unpack waves, 2 waves for the opening units");
    if (slot < 7) { if (wid < 2) ((volatile LAS unsigned*)ctl)[wid * 64 + lane] = 0u; WAIT_L(0); BAR; }
    if (slot >= 7 && !main_tile) producer(p, lds + wid * HTB, wid * 72 + (blockIdx.x - 56), lane, cntp);
    const unsigned call = __builtin_amdgcn_readfirstlane(ld4_sys(cntp)) + 1u;
    if (lane == 0) __hip_atomic_store(cntp, call, __ATOMIC_RELAXED, __HIP_MEMORY_SCOPE_AGENT);
    if (slot < 7) {
        if (wid == 0) { __builtin_amdgcn_s_setprio(3); loader(p, lds, ctl, slot, lane, call); __builtin_amdgcn_s_setprio(0); }
        else if (wid <= NDEC) decoder(p, lds + (wid - 1) * DEC_LDS, ctl, xcd, slot, wid - 1, lane, call);
        else if (wid >= 6 && wid < 6 + RAW0) opener(p, slot, wid - 6, xcd, lane, call);
    }
    if (main_tile) {
        if (slot >= 24 && wid >= 1 && wid <= 6) { for (int i = 1; i < wid; ++i) __builtin_amdgcn_s_sleep(30); own_image(p, lds + wid * HTB, (wid - 1) * 64 + (blockIdx.x - 192), lane, call); }
    } else {
        asm volatile("s_waitcnt vmcnt(0) lgkmcnt(0)" ::: "memory");
        BAR;
    }

    const int wr = wid >> 2, wc = wid & 3;
    f32x4 acc[2][2][4][2];
    int rt, cn;
    const unsigned* gate2 = cntp;
    if (slot < 7) {
        rt = 0; cn = fsrc * 8 + fct;
        wait_word((const unsigned*)(ws + OFF_FD) + cn * 8 + wid, call, tmo);
        const char* pb = ws + OFF_PK + (size_t)(cn * 8 + wid) * 32768;
#pragma unroll
        for (int i = 0; i < 8; ++i)
            ld4x16(acc[i >> 2][(i >> 1) & 1][(i & 1) * 2][0], acc[i >> 2][(i >> 1) & 1][(i & 1) * 2][1], acc[i >> 2][(i >> 1) & 1][(i & 1) * 2 + 1][0], acc[i >> 2][(i >> 1) & 1][(i & 1) * 2 + 1][1], pb + i * 4096, lane * 16);
    } else
    for (int it = 0; it < 2; ++it) {
    int sm, skip = 7;
    const char* wimg;
    if (main_tile) {
        const int idx = slot - 16;
        rt = ((xcd >> 1) * 2 + ((idx >> 2) & 1)) * 2 + (idx >> 3);
        cn = p.rank * 8 + (xcd & 1) * 4 + (idx & 3);
        const int m = rt >> 1;
        skip = (m - p.rank - 1) & 7; sm = (m == p.rank) ? 0x0f : 0xff & ~(1 << skip);
        wimg = ws + OFF_WI;
    } else {
        rt = p.rank * 2 + (helper ? slot >> 3 : 1 - it);
        cn = fsrc * 8 + fct;
        sm = helper ? 0xf0 : 0x80;
        wimg = helper ? ws + OFF_WI : ws + OFF_WD + (size_t)(slot & 7) * NH * WQ;
    }
    const int nt = 8 * __builtin_popcount(sm);
    const unsigned* fx = (const unsigned*)(ws + OFF_FX) + rt * FSTR;
    const unsigned* fw = (const unsigned*)(ws + OFF_FW) + cn * FSTR;
    gate2 = (skip == 7) ? cntp : fw + NH + skip;
    const char* xb = ws + OFF_XI + (size_t)(rt * 16 + wid) * 1024;
    const char* wb = wimg + (size_t)((cn & 7) * 16 + wr * 8 + wc) * 1024;
    const unsigned voff = lane * 16;
    const unsigned ldsw = wid * 1024;
    const int aoff = wr * 4096 + lane * 16, boff = wc * 2048 + lane * 16;
#define SA(b, h) (((b) * 4 + (h)) * HTB)
#define SB(b, h) (((b) * 4 + 2 + (h)) * HTB)
#define GLDS(g, o, l) __builtin_amdgcn_global_load_lds((const unsigned*)((g) + (size_t)(unsigned)(o)), (LAS unsigned*)(lds + (l)), 16, 0, 17)
#define STAGE_A(b, h, qq) do { GLDS(wb + (h) * 4096, (qq.x << 17) + voff, SA(b, h) + ldsw); GLDS(wb + (h) * 4096, (qq.y << 17) + voff, SA(b, h) + ldsw + 8192); } while (0)
#define STAGE_B(b, h, qq) do { GLDS(xb + (h) * 8192, (qq.x << 18) + voff, SB(b, h) + ldsw); GLDS(xb + (h) * 8192, (qq.y << 18) + voff, SB(b, h) + ldsw + 8192); } while (0)
#define LDA(dst, b, h) do { _Pragma("unroll") for (int m = 0; m < 4; ++m) _Pragma("unroll") for (int k = 0; k < 2; ++k) dst[m][k] = *(const LAS bf16x8*)(lds + SA(b, h) + aoff + m * 1024 + k * 8192); } while (0)
#define LDB(dst, b, h) do { _Pragma("unroll") for (int n = 0; n < 2; ++n) _Pragma("unroll") for (int k = 0; k < 2; ++k) dst[n][k] = *(const LAS bf16x8*)(lds + SB(b, h) + boff + n * 1024 + k * 8192); } while (0)
#define MMA2(ai) do { __builtin_amdgcn_s_setprio(1); _Pragma("unroll") for (int k = 0; k < 2; ++k) _Pragma("unroll") for (int m = 0; m < 4; ++m) _Pragma("unroll") for (int n = 0; n < 2; ++n) { \
        acc[ai][0][m][n] = __builtin_amdgcn_mfma_f32_16x16x32_bf16(B0[n][k], At[m][k], acc[ai][0][m][n], 0, 0, 0); \
        acc[ai][1][m][n] = __builtin_amdgcn_mfma_f32_16x16x32_bf16(B1[n][k], At[m][k], acc[ai][1][m][n], 0, 0, 0); } __builtin_amdgcn_s_setprio(0); } while (0)
#define MUST_PICK(kt) do { if (__builtin_expect(nq < 2 * (kt) + 2, 0)) poll_ready(fx, fw, td0, td1, queue, nq, 2 * (kt) + 2, call, tmo); } while (0)
#define SNAP(kt) do { if (__builtin_expect(nq < 2 * (kt) + 2, 0)) \
        __builtin_amdgcn_global_load_lds((lane < 32 ? fx : fw - 128) + lane * 4, queue + 256, 16, 0, 17); } while (0)
#define GETQ(kt) (*(volatile LAS u32x2*)(queue + 2 * (kt)))
#define FIXQ(qq, kt) do { if (__builtin_expect(__builtin_amdgcn_readfirstlane(qq.x & qq.y) < 0x10000, 0)) qq = GETQ(kt); } while (0)
#pragma unroll
    for (int a = 0; a < 2; ++a)
#pragma unroll
        for (int b = 0; b < 2; ++b)
#pragma unroll
            for (int m = 0; m < 4; ++m)
#pragma unroll
                for (int n = 0; n < 2; ++n) acc[a][b][m][n] = (f32x4){0.f, 0.f, 0.f, 0.f};
    bf16x8 At[4][2], B0[2][2], B1[2][2];
    LAS unsigned* queue = (LAS unsigned*)(lds + 8 * HTB + 4096);
    unsigned long long td0 = __ballot((sm >> (lane < NH ? 7 : (lane - NH) % 7)) & 1), td1 = __ballot((sm >> ((lane + 64 - NH) % 7)) & 1);
    int nq = 1 << 20;
    if (wid == 0) { nq = 0; *(volatile LAS u32x4*)(queue + 4 * lane) = (u32x4){0u, 0u, 0u, 0u}; *(volatile LAS u32x4*)(queue + 256 + 4 * lane) = (u32x4){0u, 0u, 0u, 0u}; poll_ready(fx, fw, td0, td1, queue, nq, 4, call, tmo); }
    asm volatile("" ::: "memory"); BAR; asm volatile("" ::: "memory");
    u32x2 qe = GETQ(0), qo = GETQ(1);
    STAGE_B(0, 0, qe); STAGE_A(0, 0, qe); STAGE_B(0, 1, qe); STAGE_A(0, 1, qe);
    STAGE_B(1, 0, qo); STAGE_A(1, 0, qo); STAGE_B(1, 1, qo);
    if (wr == 1) BAR;
    WAIT_V(8); BAR; BAR;
    for (int t = 0; t < nt; t += 2) {
        if (t == nt - 2) nq = 1 << 20;
        SNAP(t + 3);
        qe = GETQ(t + 2);
        LDB(B0, 0, 0); LDA(At, 0, 0); LDB(B1, 0, 1); STAGE_A(1, 1, qo);
        WAIT_V(8); WAIT_L(0); BAR; SCHED; MMA2(0); SCHED; MUST_PICK(t + 2); BAR; SCHED;
        FIXQ(qe, t + 2);
        LDA(At, 0, 1); STAGE_B(0, 0, qe); STAGE_A(0, 0, qe); STAGE_B(0, 1, qe);
        WAIT_V(8); WAIT_L(0); BAR; SCHED; MMA2(1); BAR; SCHED;
        if (t == nt - 2)
            __builtin_amdgcn_global_load_lds(lane == 1 ? tmo : gate2, (LAS unsigned*)(lds + 8 * HTB + wid * 256), 4, 0, 17);
        SNAP(t + 4);
        qo = GETQ(t + 3);
        LDB(B0, 1, 0); LDA(At, 1, 0); LDB(B1, 1, 1); STAGE_A(0, 1, qe);
        WAIT_V(8); WAIT_L(0); BAR; SCHED; MMA2(0); SCHED; MUST_PICK(t + 3); BAR; SCHED;
        FIXQ(qo, t + 3);
        LDA(At, 1, 1); STAGE_B(1, 0, qo); STAGE_A(1, 0, qo); STAGE_B(1, 1, qo);
        WAIT_V(8); WAIT_L(0); BAR; SCHED; MMA2(1); BAR; SCHED;
    }
    if (main_tile) break;
    if (wr == 0) BAR;
    const int pki = (helper ? 64 + (rt & 1) * 8 + fct : cn) * 8 + wid;
    if (it == 0) {
        const char* pb = ws + OFF_PK + (size_t)pki * 32768;
        asm volatile("s_nop 4" : "+s"(pb));
#pragma unroll
        for (int i = 0; i < 32; ++i) st16<0>(pb + i * 1024, voff, __builtin_bit_cast(u32x4, acc[i >> 4][(i >> 3) & 1][(i >> 1) & 3][i & 1]));
    }
    WAIT_V(0);
    if (it == 0 && lane == 0) __hip_atomic_store((unsigned*)(ws + OFF_FD) + pki, call, __ATOMIC_RELAXED, __HIP_MEMORY_SCOPE_SYSTEM);
    if (helper) return;
    asm volatile("" ::: "memory"); BAR; asm volatile("" ::: "memory");
    }

    int tz = threadIdx.x; asm volatile("" : "+v"(tz));
    const int ln = tz & 63;
    const unsigned vo = ln * 16;
    const bool own = main_tile && (rt >> 1) == p.rank;
    int computing = __builtin_amdgcn_readfirstlane(main_tile && !own); asm volatile("" : "+s"(computing));
    if (computing != 1) {
        const int rh = own ? rt & 1 : slot < 8 ? 1 : 0;
        LAS unsigned char* dl = lds + wid * FIN_LDS;
        const char* sl = p.ws[fsrc] + OFF_PS + (size_t)(((p.rank * 2 + rh) * 8 + (cn & 7)) * 8 + wid) * UB;
        const unsigned ls = (unsigned)(size_t)dl + DEC_SLOT;
        unsigned hdr = 0;
        if (own) {
            if (wr == 0) BAR;
            const int pki = (64 + rh * 8 + (cn & 7)) * 8 + wid;
            wait_word((const unsigned*)(ws + OFF_FD) + pki, call, tmo);
            sl = ws + OFF_PK + (size_t)pki * 32768;
        } else {
#pragma unroll
            for (int i = 0; i < 8; ++i) *(LAS u32x4*)(dl + i * 1024 + ln * 16) = (u32x4){0u, 0u, 0u, 0u};
            const char* fl = ws + OFF_FP + (size_t)((rh * 64 + cn) * 8 + wid) * FLS;
            wait_word((const unsigned*)(ws + OFF_SD) + ((slot & 7) * 8 + fct) * 32, call, tmo);
            hdr = wait_unit(fl, call, false, tmo);
            dma16(sl, vo, ls); WAIT_V(0); dma16(sl + 1024, vo, ls + 1024); WAIT_V(0); dma16(sl + 2048, vo, ls + 2048);
            if (hdr_cnt(hdr) == LOWCNT) hdr = wait_unit(fl, call, true, tmo);
        }
        const bool raw = hdr_cnt(hdr) == RAWCNT;
        const unsigned LL = (hdr & 0x7fffu) * 0x10001u, tl = hdr >> 25;
        if (!raw && !own) {
            WAIT_V(0);
            if ((unsigned)ln < tl * 2u) dma16(sl + 10240, vo, ls + 10240);
            if (__builtin_expect(tl > 32u, 0)) { if ((unsigned)ln < (tl - 32u) * 2u) dma16(sl + 11264, vo, ls + 11264); }
        }
        const unsigned nanmask = ((unsigned)__builtin_amdgcn_readfirstlane(ld4_sys(tmo)) == call) ? 0x7fc07fc0u : 0u;
        u32x4 P[2], G;
        if (!raw && !own) {
            dma16(sl + 8192, vo, ls + 8192);
            unpack_head(*(LAS unsigned*)(dl + DEC_TAIL + ln * 4), hdr, dl, ln, G);
            WAIT_V(0); dma16(sl + 9216, vo, ls + 9216);
            P[0] = *(LAS u32x4*)(dl + DEC_SLOT + 8192 + ln * 16);
        }
        const char* ob = p.out + ((size_t)(rh * 256 + wc * 32) * NN + cn * 256 + wr * 128) * 2;
        const unsigned oo = (ln >> 4) * (4 * NN * 2) + (ln & 15) * 16;
#pragma unroll
        for (int st = 0; st < 8; ++st) {
            const int i = st == 1 ? 2 : st == 2 ? 1 : st;
            u32x4 o[2];
            if (own) {
                o[0] = o[1] = (u32x4){0u, 0u, 0u, 0u};
                if (st == 0 || st == 1 || st == 4 || st == 6) {
                    const int bj = i >> 2, n = (i >> 1) & 1;
                    f32x4 t[8]; ld8x16s(t, sl + (bj * 8 + n) * 1024 + 4096, vo);
#pragma unroll
                    for (int j = 0; j < 8; ++j) acc[j >> 2][bj][j & 3][n] += t[j];
                }
            } else if (raw) { dma16(sl + i * 2048, vo, ls); dma16(sl + i * 2048 + 1024, vo, ls + 1024); WAIT_V(0);
                       o[0] = *(LAS u32x4*)(dl + DEC_SLOT + ln * 16); o[1] = *(LAS u32x4*)(dl + DEC_SLOT + 1024 + ln * 16); asm volatile("s_waitcnt lgkmcnt(0)" : "+v"(o[0]), "+v"(o[1]) :: "memory"); }
            else {
#define NEXTP(j) dma16(sl + (j) * 1024, vo, ls + (j) * 1024)
                if (st == 2) { WAIT_V(4); NEXTP(3); P[1] = *(LAS u32x4*)(dl + DEC_SLOT + 9216 + ln * 16); }
                if (st >= 3) { WAIT_V(2); if (st < 7) NEXTP(st + 1); }
                unpack_piece(*(LAS u32x4*)(dl + DEC_SLOT + i * 1024 + ln * 16), P, G, i, dl, ln, LL, o);
            }
#pragma unroll
            for (int j = 0; j < 2; ++j) {
                const int f = 2 * i + j;
                u32x4 y;
#pragma unroll
                for (int cc = 0; cc < 4; ++cc) {
                    const unsigned pv = o[j][cc];
                    const f32x2 v = (f32x2){acc[cc >> 1][f >> 3][2 * (cc & 1)][(f >> 2) & 1][f & 3] + __uint_as_float(pv << 16),
                                            acc[cc >> 1][f >> 3][2 * (cc & 1) + 1][(f >> 2) & 1][f & 3] + __uint_as_float(pv & 0xffff0000u)};
                    const f32x2 g = gelu2(v);
                    y[cc] = cvt_pk_bf16(g.x, g.y) | nanmask;
                }
                st16<0>(ob + (size_t)((f >> 3) * 128 + ((f >> 2) & 1) * 16 + (f & 3)) * (NN * 2), oo, y);
            }
        }
    } else {
        const u32x2 gv = *(volatile LAS u32x2*)(lds + 8 * HTB + wid * 256);
        u32x4 b[16], pk[12];
#pragma unroll
        for (int bj = 0; bj < 2; ++bj)
#pragma unroll
            for (int n = 0; n < 2; ++n)
#pragma unroll
                for (int r = 0; r < 4; ++r)
#pragma unroll
                    for (int ai = 0; ai < 2; ++ai)
#pragma unroll
                        for (int m2 = 0; m2 < 2; ++m2)
                            b[bj * 8 + n * 4 + r][ai * 2 + m2] = cvt_pk_bf16(acc[ai][bj][2 * m2][n][r], acc[ai][bj][2 * m2 + 1][n][r]);
        asm volatile("" : "+v"(b[0]), "+v"(b[1]), "+v"(b[2]), "+v"(b[3]), "+v"(b[4]), "+v"(b[5]), "+v"(b[6]), "+v"(b[7]), "+v"(b[8]), "+v"(b[9]), "+v"(b[10]), "+v"(b[11]), "+v"(b[12]), "+v"(b[13]), "+v"(b[14]), "+v"(b[15]) :: "memory");
        if (wr == 0) BAR;
        bool bad = (unsigned)__builtin_amdgcn_readfirstlane(gv.y) == call;
        if ((unsigned)__builtin_amdgcn_readfirstlane(gv.x) < call) { wait_word(gate2, call, tmo); bad = (unsigned)__builtin_amdgcn_readfirstlane(ld4_sys(tmo)) == call; }
        const char* sl = ws + OFF_PS + (size_t)((rt * 8 + (cn & 7)) * 8 + wid) * UB;
        char* fl = p.ws[(unsigned)(rt >> 1) & 7u] + OFF_FP + (size_t)(((rt & 1) * 64 + cn) * 8 + wid) * FLS;
        const char* s0 = sl + 4096; const char* s1 = sl + 8192;
        const unsigned L = pack_base(b);
        st16<-4096>(s0, vo, pack_low(b, 0, L)); st16<-3072>(s0, vo, pack_low(b, 1, L)); st16<-2048>(s0, vo, pack_low(b, 2, L));
        WAIT_V(0);
        st8_flag(fl, call, LOWCNT << 15, ln);
        asm volatile("" ::: "memory"); BAR; asm volatile("" ::: "memory");
        unsigned hdr = pack_unit(b, pk, L, lds + wid * HTB, ln);
        if (bad) {
            hdr = RAWCNT << 15;
#pragma unroll
            for (int f = 0; f < 16; ++f) b[f] = (u32x4){0x7fc07fc0u, 0x7fc07fc0u, 0x7fc07fc0u, 0x7fc07fc0u};
        }
        if (__builtin_expect(hdr_cnt(hdr) == RAWCNT, 0)) st_raw(sl, vo, b);
        else {
            st16<0>(s1, vo, pk[8]); st16<1024>(s1, vo, pk[9]); st_tail(s1, vo, pk[10], pk[11], hdr >> 25, ln);
            st16<-1024>(s0, vo, pk[3]); st16<0>(s0, vo, pk[4]); st16<1024>(s0, vo, pk[5]); st16<2048>(s0, vo, pk[6]); st16<3072>(s0, vo, pk[7]);
        }
        WAIT_V(0);
        st8_flag(fl, call, hdr, ln);
    }
}
}

extern "C" void kernel_launch(int rank, int nranks, ncclComm_t comm, hipStream_t stream,
                              void* const* d_in, const int* in_sizes, int n_in,
                              void* d_out, int out_size, const int* in_dtypes, int out_dtype,
                              void* const* d_out_all, void* const* d_ws_all, size_t ws_bytes) {
    (void)comm; (void)in_dtypes; (void)out_dtype; (void)d_out_all;
    if (nranks != NR || (unsigned)rank >= (unsigned)NR || n_in < 2 || in_sizes[0] != MM * KS || in_sizes[1] != KS * NN ||
        out_size != (MM / NR) * NN || ws_bytes < WS_NEED) {
        fprintf(stderr, "kernel_launch: unexpected problem geometry\n");
        return;
    }
    Params p;
    p.x = (const char*)d_in[0];
    p.w = (const char*)d_in[1];
    for (int i = 0; i < NR; ++i) p.ws[i] = (char*)d_ws_all[i];
    p.out = (char*)d_out;
    p.rank = rank; p.pad = 0;
    hipLaunchKernelGGL(fused_kernel, dim3(256), dim3(512), 0, stream, p);
    HIP_CHECK(hipGetLastError());
}
```

```cpp
#include <hip/hip_runtime.h>
#include <rccl/rccl.h>
#include <cstdio>
#include <cstdint>

#define HIP_CHECK(expr) do { hipError_t _e = (expr); if (_e != hipSuccess) { \
    fprintf(stderr, "HIP error %s at %s:%d\n", hipGetErrorString(_e), __FILE__, __LINE__); \
    return; } } while (0)

#define LAGXA 1
#define LAGXB 36
#define LAGOWN 69
#define LAGWH 4
#define RAW0 2
#define LWIN 5
#define PWIN 3
#define NDEC 5
#define PDT 24
#define LD0 500
#define GATE_EARLY 6
#define FORCE_RAW 0

namespace {
#define LAS __attribute__((address_space(3)))
typedef short bf16x8 __attribute__((ext_vector_type(8)));
typedef float f32x4 __attribute__((ext_vector_type(4)));
typedef float f32x2 __attribute__((ext_vector_type(2)));
typedef unsigned u32x4 __attribute__((ext_vector_type(4)));
typedef unsigned u32x2 __attribute__((ext_vector_type(2)));
typedef unsigned short u16x2 __attribute__((ext_vector_type(2)));

constexpr int NR = 8;
constexpr int MM = 4096, NN = 16384, KS = 512;
constexpr int NQ = 128;
constexpr int NH = 16;
constexpr int NU = NH * 22;
constexpr int NTASK = NH * 70;
constexpr unsigned long long DEADLINE = 100000000ull;
constexpr unsigned RAWCNT = 0x3ffu, LOWCNT = 0x3feu;

constexpr size_t OFF_CNT = 0;
constexpr size_t OFF_TMO = 8192;
constexpr size_t OFF_TK = 8192 + 1024;
constexpr int FSTR = 160;
constexpr size_t OFF_FX = 16384;
constexpr size_t OFF_FW = 32768;
constexpr size_t OFF_FD = 81920;
constexpr size_t OFF_SD = 131072;
constexpr size_t UB = 16384, FLS = 128;
constexpr size_t XQ = 256 * 1024, WQ = 128 * 1024;
constexpr size_t OFF_XI = 1u << 20;
constexpr size_t OFF_WI = OFF_XI + (size_t)NQ * XQ;
constexpr size_t OFF_WD = OFF_WI + (size_t)NQ * WQ;
constexpr size_t OFF_PS = OFF_WD + (size_t)7 * NH * WQ;
constexpr size_t OFF_ST = OFF_PS + (size_t)1024 * UB;
constexpr size_t ST_PAR = (size_t)1120 * UB;
constexpr size_t OFF_PK = OFF_ST + 2 * ST_PAR;
constexpr size_t OFF_LZ = OFF_PK + (size_t)640 * 32768;
constexpr size_t OFF_FA = OFF_LZ + (size_t)14 * 8 * RAW0 * UB;
constexpr size_t OFF_FP = OFF_FA + (size_t)14 * NU * FLS;
constexpr size_t WS_NEED = OFF_FP + 1024 * FLS;
static_assert(OFF_FW + 64 * FSTR * 4 <= OFF_FD && OFF_FD + 2560 <= OFF_SD && OFF_SD + 56 * 128 <= OFF_XI, "layout");
static_assert(WS_NEED <= (size_t)161 << 20, "workspace");

struct Params {
    const char* x;
    const char* w;
    char* ws[NR];
    char* out;
    int rank;
    int pad;
};

template <int OFF> __device__ __forceinline__ void st16(const char* sbase, unsigned voff, u32x4 v) {
    asm volatile("global_store_dwordx4 %0, %1, %2 offset:%3 sc0 sc1\n\ts_nop 1" :: "v"(voff), "v"(v), "s"(sbase), "i"(OFF) : "memory");
}
__device__ __forceinline__ unsigned ld4_sys(const void* p) {
    unsigned v;
    asm volatile("global_load_dword %0, %1, off sc0 sc1\n\ts_waitcnt vmcnt(0)" : "=&v"(v) : "v"(p) : "memory");
    return v;
}
__device__ __forceinline__ u32x2 ld8_sys(const void* p) {
    u32x2 v;
    asm volatile("global_load_dwordx2 %0, %1, off sc0 sc1\n\ts_waitcnt vmcnt(0)" : "=&v"(v) : "v"(p) : "memory");
    return v;
}
__device__ __forceinline__ void st8_flag(void* p, unsigned call, unsigned hdr, int lane) {
    if (lane == 0) __hip_atomic_store((unsigned long long*)p, ((unsigned long long)hdr << 32) | call, __ATOMIC_RELAXED, __HIP_MEMORY_SCOPE_SYSTEM);
}
#define WAIT_V_(n) asm volatile("s_waitcnt vmcnt(" #n ")" ::: "memory")
#define WAIT_V(n) WAIT_V_(n)
#define WAIT_L(n) asm volatile("s_waitcnt lgkmcnt(" #n ")" ::: "memory")
#define BAR __builtin_amdgcn_s_barrier()
#define SCHED __builtin_amdgcn_sched_barrier(0)

typedef __bf16 bf16x2 __attribute__((ext_vector_type(2)));
__device__ __forceinline__ unsigned cvt_pk_bf16(float lo, float hi) {
    return __builtin_bit_cast(unsigned, __builtin_convertvector((f32x2){lo, hi}, bf16x2));
}
#define CFENCE asm volatile("" ::: "memory")
__device__ __forceinline__ f32x2 gelu2(f32x2 y) {
    const float k1 = -2.0f * 0.7978845608028654f * 1.4426950408889634f, k2 = k1 * 0.044715f;
    f32x2 z = y * (y * y * k2 + k1);
    f32x2 e; e.x = __builtin_amdgcn_exp2f(z.x); e.y = __builtin_amdgcn_exp2f(z.y);
    e = e + 1.0f;
    f32x2 r; r.x = __builtin_amdgcn_rcpf(e.x); r.y = __builtin_amdgcn_rcpf(e.y);
    return y * r;
}
__device__ __forceinline__ unsigned pk_sub(unsigned a, unsigned b) { return __builtin_bit_cast(unsigned, __builtin_bit_cast(u16x2, a) - __builtin_bit_cast(u16x2, b)); }
__device__ __forceinline__ unsigned pk_add(unsigned a, unsigned b) { return __builtin_bit_cast(unsigned, __builtin_bit_cast(u16x2, a) + __builtin_bit_cast(u16x2, b)); }
__device__ __forceinline__ unsigned pk_max(unsigned a, unsigned b) { return __builtin_bit_cast(unsigned, __builtin_elementwise_max(__builtin_bit_cast(u16x2, a), __builtin_bit_cast(u16x2, b))); }
__device__ __forceinline__ unsigned bfi(unsigned m, unsigned a, unsigned b) { return __builtin_amdgcn_bitop3_b32(m, a, b, 0xca); }
__device__ __forceinline__ unsigned rotl(unsigned a, int n) { return __builtin_amdgcn_alignbit(a, a, 32 - n); }
__device__ __forceinline__ unsigned rotr(unsigned a, int n) { return __builtin_amdgcn_alignbit(a, a, n); }
__device__ __forceinline__ unsigned umax(unsigned a, unsigned b) { return a > b ? a : b; }

__device__ __forceinline__ void dma16(const char* sbase, unsigned voff, unsigned ldst) {
    unsigned keep;
    asm volatile("s_mov_b32 %0, m0\n\ts_mov_b32 m0, %3\n\ts_nop 0\n\tglobal_load_lds_dwordx4 %1, %2 sc0 sc1\n\ts_mov_b32 m0, %0"
                 : "=&s"(keep) : "v"(voff), "s"(sbase), "s"(ldst) : "memory");
}

__device__ __forceinline__ bool give_up(unsigned* tmo, unsigned call, unsigned long long t0, unsigned& polls) {
    if ((++polls & 15u) != 0u) return false;
    if (__builtin_amdgcn_readfirstlane(__hip_atomic_load(tmo, __ATOMIC_RELAXED, __HIP_MEMORY_SCOPE_AGENT)) == call ||
        __builtin_amdgcn_s_memrealtime() - t0 > DEADLINE) {
        __hip_atomic_store(tmo, call, __ATOMIC_RELAXED, __HIP_MEMORY_SCOPE_AGENT);
        return true;
    }
    return false;
}
__device__ __forceinline__ unsigned hdr_cnt(unsigned hdr) { return (hdr >> 15) & 0x3ffu; }
__device__ __forceinline__ unsigned wait_unit(const char* flag, unsigned call, bool full, unsigned* tmo) {
    const unsigned long long t0 = __builtin_amdgcn_s_memrealtime(); unsigned polls = 0;
    for (;;) {
        const u32x2 v = ld8_sys(flag);
        const unsigned hdr = __builtin_amdgcn_readfirstlane(v.y);
        if ((unsigned)__builtin_amdgcn_readfirstlane(v.x) == call && !(full && hdr_cnt(hdr) == LOWCNT)) return hdr;
        __builtin_amdgcn_s_sleep(1);
        if (give_up(tmo, call, t0, polls)) return RAWCNT << 15;
    }
}
__device__ __forceinline__ void wait_word(const unsigned* w, unsigned call, unsigned* tmo) {
    const unsigned long long t0 = __builtin_amdgcn_s_memrealtime(); unsigned polls = 0;
    while ((unsigned)__builtin_amdgcn_readfirstlane(ld4_sys(w)) < call) {
        __builtin_amdgcn_s_sleep(2);
        if (give_up(tmo, call, t0, polls)) break;
    }
}

constexpr int SCR_DENSE = 8192;
__device__ __forceinline__ unsigned umin(unsigned a, unsigned b) { return a < b ? a : b; }
__device__ __forceinline__ unsigned wave_scan(unsigned inc) {
    inc += __builtin_amdgcn_update_dpp(0, (int)inc, 0x111, 0xf, 0xf, true);
    inc += __builtin_amdgcn_update_dpp(0, (int)inc, 0x112, 0xf, 0xf, true);
    inc += __builtin_amdgcn_update_dpp(0, (int)inc, 0x114, 0xf, 0xf, true);
    inc += __builtin_amdgcn_update_dpp(0, (int)inc, 0x118, 0xf, 0xf, true);
    inc += __builtin_amdgcn_update_dpp(0, (int)inc, 0x142, 0xa, 0xf, false);
    inc += __builtin_amdgcn_update_dpp(0, (int)inc, 0x143, 0xc, 0xf, false);
    return inc;
}
__device__ __forceinline__ unsigned pack_base(const u32x4 (&b)[16]) {
    unsigned mx = 0;
#pragma unroll
    for (int f = 0; f < 16; ++f) mx = pk_max(mx, b[f][f & 3] & 0x7fff7fffu);
    unsigned m = umax(mx & 0xffffu, mx >> 16);
    m = umax(m, (unsigned)__builtin_amdgcn_update_dpp(0, (int)m, 0x128, 0xf, 0xf, false));
    m = umax(m, (unsigned)__builtin_amdgcn_update_dpp(0, (int)m, 0x124, 0xf, 0xf, false));
    m = umax(m, (unsigned)__builtin_amdgcn_update_dpp(0, (int)m, 0x122, 0xf, 0xf, false));
    m = umax(m, (unsigned)__builtin_amdgcn_update_dpp(0, (int)m, 0x121, 0xf, 0xf, false));
    const unsigned m0 = __builtin_amdgcn_readlane(m, 0), m1 = __builtin_amdgcn_readlane(m, 16), m2 = __builtin_amdgcn_readlane(m, 32), m3 = __builtin_amdgcn_readlane(m, 48);
    const unsigned ms = umin(umax(umin(m0, m1), umin(m2, m3)), umin(umax(m0, m1), umax(m2, m3)));
    return umax(ms, 1023u) - 1023u;
}
__device__ __forceinline__ u32x4 pack_low(const u32x4 (&b)[16], int i, unsigned L) {
    const unsigned LL = L | (L << 16);
    u32x4 r;
#pragma unroll
    for (int c = 0; c < 4; ++c) r[c] = __builtin_amdgcn_perm(pk_sub(b[2 * i + (c >> 1)][2 * (c & 1) + 1], LL), pk_sub(b[2 * i + (c >> 1)][2 * (c & 1)], LL), 0x06040200u);
    return r;
}
__device__ __forceinline__ unsigned pack_unit(const u32x4 (&b)[16], u32x4 (&pk)[12], unsigned L, LAS unsigned char* scr, int lane) {
    const unsigned LL = L | (L << 16);
    const unsigned wp0 = (unsigned)(size_t)scr + lane * 4;
    unsigned wp = wp0;
    unsigned z[4], seg[4], len[4], F = 0;
#pragma unroll
    for (int s = 0; s < 4; ++s) {
        unsigned hb[8];
#pragma unroll
        for (int jj = 0; jj < 8; ++jj) {
            const int j = 8 * s + jj;
            const unsigned d0 = pk_sub(b[j >> 1][2 * (j & 1)], LL), d1 = pk_sub(b[j >> 1][2 * (j & 1) + 1], LL);
            pk[j >> 2][j & 3] = __builtin_amdgcn_perm(d1, d0, 0x06040200u);
            hb[jj] = __builtin_amdgcn_perm(d1, d0, 0x07050301u);
            const unsigned t = hb[jj] & 0x7c7c7c7cu; unsigned tmp;
            *(LAS unsigned*)(size_t)wp = t | (unsigned)((j & 3) | ((j >> 2) << 7));
            asm("v_min_u32 %0, 1, %2\n\tv_lshl_add_u32 %1, %0, 8, %1" : "=&v"(tmp), "+v"(wp) : "v"(t));
            if (jj & 3) hb[jj] = rotl(hb[jj], 2 * (jj & 3));
        }
        pk[8][s] = bfi(0x81818181u, hb[0], bfi(0x06060606u, hb[1], bfi(0x18181818u, hb[2], hb[3])));
        pk[9][s] = bfi(0x81818181u, hb[4], bfi(0x06060606u, hb[5], bfi(0x18181818u, hb[6], hb[7])));
        const unsigned ga = bfi(0x02020202u, hb[0], bfi(0x08080808u, hb[1], bfi(0x20202020u, hb[2], hb[3])));
        const unsigned gb = bfi(0x02020202u, hb[4], bfi(0x08080808u, hb[5], bfi(0x20202020u, hb[6], hb[7])));
        z[s] = ~bfi(0xaaaaaaaau, ga, gb >> 1);
    }
    unsigned e8[8], nz4[4];
#pragma unroll
    for (int i = 0; i < 8; ++i) e8[i] = *(LAS unsigned*)(scr + i * 256 + lane * 4);
#pragma unroll
    for (int s = 0; s < 4; ++s) {
        unsigned t = z[s] | (z[s] >> 1); t |= t << 2;
        nz4[s] = t & 0x44444444u; F |= s < 2 ? nz4[s] >> (2 - s) : nz4[s] << (s - 2);
    }
    const unsigned nrow = (wp - wp0) >> 8, mine = (__builtin_popcount(F) << 14) | nrow;
    const unsigned inc = wave_scan(mine), tot = __builtin_amdgcn_readlane(inc, 63), ngrp = tot & 0xfffu;
    const unsigned lo = 256u + ((((tot >> 12) + 31u) >> 5) << 2);
    unsigned cnt = ngrp, bad = 0;
    if (ngrp <= 512u) {
        LAS unsigned* dense = (LAS unsigned*)(scr + SCR_DENSE);
        const unsigned lanebits = ((lane & 7) << 15) | ((lane >> 3) << 23), base = (inc - mine) & 0xfffu;
#pragma unroll
        for (int i = 0; i < 8; ++i) dense[(unsigned)i < nrow ? base + i : 512 + lane] = e8[i] | lanebits;
        for (unsigned r = 8; __builtin_expect(__any(nrow > r), 0); ++r)
            dense[r < nrow ? base + r : 512 + lane] = *(LAS unsigned*)(scr + r * 256 + lane * 4) | lanebits;
        unsigned e2[2] = {dense[lane], dense[64 + lane]};
        CFENCE;
        *(LAS u32x4*)(scr + 256 + lane * 16) = (u32x4){0u, 0u, 0u, 0u};
        *(LAS unsigned*)(scr + lane * 4) = F;
        CFENCE;
#pragma unroll
        for (int s = 0; s < 4; ++s) { seg[s] = z[s] & 15u; len[s] = nz4[s] & 7u; }
#pragma unroll
        for (int i = 1; i < 8; ++i)
#pragma unroll
            for (int s = 0; s < 4; ++s) { seg[s] |= ((z[s] >> (4 * i)) & 15u) << len[s]; len[s] += (nz4[s] >> (4 * i)) & 7u; }
        unsigned bit = (inc - mine) >> 12;
#pragma unroll
        for (int s = 0; s < 4; ++s) {
            const unsigned long long v = (unsigned long long)seg[s] << (bit & 31u);
            LAS unsigned* w = (LAS unsigned*)(scr + 256) + (bit >> 5);
            __hip_atomic_fetch_or(w, (unsigned)v, __ATOMIC_RELAXED, __HIP_MEMORY_SCOPE_WAVEFRONT);
            __hip_atomic_fetch_or(w + 1, (unsigned)(v >> 32), __ATOMIC_RELAXED, __HIP_MEMORY_SCOPE_WAVEFRONT);
            bit += len[s];
        }
        CFENCE;
        LAS unsigned short* lst = (LAS unsigned short*)(scr + lo);
        const unsigned hz = ((0u - L) >> 10) & 31u;
        for (unsigned i0 = 0;;) {
            unsigned t[2], pos[2], at[2];
#pragma unroll
            for (int c = 0; c < 2; ++c) {
                const unsigned e = e2[c], ix = i0 + c * 64 + lane;
                t[c] = ix < ngrp ? e & 0x7c7c7c7cu : 0u;
                pos[c] = ((e & 0x380u) << 3) | ((e >> 16) & 0x380u) | ((e >> 11) & 0x70u) | ((e & 3u) << 2);
                at[c] = ix < ngrp ? ix : 1024u + lane;
            }
            for (;;) {
#pragma unroll
                for (int c = 0; c < 2; ++c) {
                    const unsigned b8 = __builtin_ctz(t[c] | 0x80000000u) & 24u, hi5 = (t[c] >> (b8 + 2)) & 31u;
                    const unsigned code = hi5 == hz ? 5u : (hi5 + 5u) & 31u;
                    bad = umax(bad, code);
                    lst[at[c]] = (unsigned short)((code << 13) | pos[c] | (b8 >> 3));
                    t[c] &= ~(0xffu << b8);
                }
                const unsigned long long b0 = __ballot(t[0] != 0), b1 = __ballot(t[1] != 0);
                if (!(b0 | b1)) break;
                at[0] = t[0] != 0 ? __builtin_amdgcn_mbcnt_hi((unsigned)(b0 >> 32), __builtin_amdgcn_mbcnt_lo((unsigned)b0, umin(cnt, 512u))) : 1024u + lane;
                cnt += __builtin_popcountll(b0);
                at[1] = t[1] != 0 ? __builtin_amdgcn_mbcnt_hi((unsigned)(b1 >> 32), __builtin_amdgcn_mbcnt_lo((unsigned)b1, umin(cnt, 512u))) : 1024u + lane;
                cnt += __builtin_popcountll(b1);
            }
            i0 += 128;
            if (__builtin_expect(i0 >= ngrp, 1)) break;
            e2[0] = dense[i0 + lane]; e2[1] = dense[i0 + 64 + lane];
        }
    }
    CFENCE;
    pk[10] = *(LAS u32x4*)(scr + lane * 16);
    pk[11] = *(LAS u32x4*)(scr + 1024 + lane * 16);
    const unsigned tl = (lo + 2u * cnt + 31u) >> 5;
    if (cnt > 512u || tl > 64u || __any(bad > 7u) || FORCE_RAW) return RAWCNT << 15;
    return L | (cnt << 15) | (tl << 25);
}
__device__ __forceinline__ void st_tail(const char* sb8k  , unsigned voff, u32x4 t0, u32x4 t1, unsigned tl, int lane) {
    if (__builtin_expect(tl > 32u, 0)) { if ((unsigned)lane < (tl - 32u) * 2u) st16<3072>(sb8k, voff, t1); }
    if ((unsigned)lane < tl * 2u) st16<2048>(sb8k, voff, t0);
}
__device__ __forceinline__ void st_raw(const char* slot, unsigned voff, const u32x4 (&b)[16]) {
    const char* s0 = slot + 4096; const char* s1 = slot + 12288;
    st16<-4096>(s0, voff, b[0]); st16<-3072>(s0, voff, b[1]); st16<-2048>(s0, voff, b[2]); st16<-1024>(s0, voff, b[3]);
    st16<0>(s0, voff, b[4]); st16<1024>(s0, voff, b[5]); st16<2048>(s0, voff, b[6]); st16<3072>(s0, voff, b[7]);
    st16<-4096>(s1, voff, b[8]); st16<-3072>(s1, voff, b[9]); st16<-2048>(s1, voff, b[10]); st16<-1024>(s1, voff, b[11]);
    st16<0>(s1, voff, b[12]); st16<1024>(s1, voff, b[13]); st16<2048>(s1, voff, b[14]); st16<3072>(s1, voff, b[15]);
}
__device__ __forceinline__ void ld4x16(f32x4& a, f32x4& b, f32x4& c, f32x4& d, const char* sb, unsigned voff) {
    asm volatile("global_load_dwordx4 %0, %4, %5 sc0 sc1\n\tglobal_load_dwordx4 %1, %4, %5 offset:1024 sc0 sc1\n\t"
                 "global_load_dwordx4 %2, %4, %5 offset:2048 sc0 sc1\n\tglobal_load_dwordx4 %3, %4, %5 offset:3072 sc0 sc1\n\ts_waitcnt vmcnt(0)"
                 : "=&v"(a), "=&v"(b), "=&v"(c), "=&v"(d) : "v"(voff), "s"(sb) : "memory");
}
__device__ __forceinline__ void ld8x16s(f32x4 (&t)[8], const char* sb  , unsigned voff) {
    asm volatile("global_load_dwordx4 %0, %8, %9 offset:-4096 sc0 sc1\n\tglobal_load_dwordx4 %1, %8, %9 offset:-2048 sc0 sc1\n\t"
                 "global_load_dwordx4 %2, %8, %9 sc0 sc1\n\tglobal_load_dwordx4 %3, %8, %9 offset:2048 sc0 sc1\n\t"
                 "global_load_dwordx4 %4, %8, %10 offset:-4096 sc0 sc1\n\tglobal_load_dwordx4 %5, %8, %10 offset:-2048 sc0 sc1\n\t"
                 "global_load_dwordx4 %6, %8, %10 sc0 sc1\n\tglobal_load_dwordx4 %7, %8, %10 offset:2048 sc0 sc1\n\ts_waitcnt vmcnt(0)"
                 : "=&v"(t[0]), "=&v"(t[1]), "=&v"(t[2]), "=&v"(t[3]), "=&v"(t[4]), "=&v"(t[5]), "=&v"(t[6]), "=&v"(t[7])
                 : "v"(voff), "s"(sb), "s"(sb + 16384) : "memory");
}
constexpr int DEC_SLOT = 8192, DEC_TAIL = DEC_SLOT + 10240;
__device__ __forceinline__ void patch_apply(LAS unsigned char* dl, unsigned lo, unsigned hdr, int lane, bool zero) {
    const unsigned cnt = hdr_cnt(hdr), lut_hi = 0x0804007cu | ((((0u - (hdr & 0x7fffu)) >> 10) & 31u) << 10);
    for (unsigned c = 0; c < cnt; c += 128) {
        const unsigned w = *(LAS unsigned*)(dl + DEC_TAIL + lo + c * 2 + lane * 4);
#pragma unroll
        for (int hf = 0; hf < 2; ++hf) {
            const unsigned e = hf ? w >> 16 : w & 0xffffu;
            const unsigned at = c + lane * 2 + hf < cnt ? e & 0x1fffu : (unsigned)DEC_TAIL + lane;
            dl[at] = zero ? 0 : (unsigned char)__builtin_amdgcn_perm(lut_hi, 0x7874706cu, e >> 13);
        }
    }
}
__device__ __forceinline__ unsigned unpack_head(unsigned F, unsigned hdr, LAS unsigned char* dl, int lane, u32x4& G) {
    const unsigned mine = __builtin_popcount(F) << 2, inc = wave_scan(mine);
    const unsigned lo = 256u + ((((unsigned)__builtin_amdgcn_readlane(inc, 63) + 31u) >> 5) << 2);
    unsigned bit = inc - mine, sg[4];
#pragma unroll
    for (int s = 0; s < 4; ++s) {
        const LAS unsigned* w = (const LAS unsigned*)(dl + DEC_TAIL + 256) + (bit >> 5);
        sg[s] = __builtin_amdgcn_alignbit(w[1], w[0], bit);
        bit += __builtin_popcount(F & (0x11111111u << s)) << 2;
    }
    patch_apply(dl, lo, hdr, lane, false);
#pragma unroll
    for (int s = 0; s < 4; ++s) {
        const unsigned nz = (F >> s) & 0x11111111u, pre = nz * 0x11111110u;
        const unsigned xe = sg[s] & 0x0f0f0f0fu, xo = (sg[s] >> 4) & 0x0f0f0f0fu;
        const unsigned b0 = __builtin_amdgcn_perm(xo, xe, 0x05010400u), b1 = __builtin_amdgcn_perm(xo, xe, 0x07030602u);
        const unsigned ze = __builtin_amdgcn_perm(b1, b0, pre & 0x07070707u), zo = __builtin_amdgcn_perm(b1, b0, (pre >> 4) & 0x07070707u);
        G[s] = ~((ze | (zo << 4)) & ((nz << 4) - nz));
    }
    return lo;
}
__device__ __forceinline__ void unpack_piece(u32x4 lo, const u32x4 (&P)[2], u32x4 G, int i, LAS unsigned char* dl, int lane, unsigned LL, u32x4 (&o)[2]) {
    const u32x4 pt = *(LAS u32x4*)(dl + i * 1024 + lane * 16);
    const unsigned A = P[i & 1][i >> 1], g = G[i >> 1] << (i & 1);
    unsigned hb[4];
    hb[0] = bfi(0x83838383u, bfi(0x02020202u, g, A), pt[0]); hb[1] = bfi(0x83838383u, rotr(bfi(0x08080808u, g, A), 2), pt[1]);
    hb[2] = bfi(0x83838383u, rotr(bfi(0x20202020u, g, A), 4), pt[2]); hb[3] = bfi(0x83838383u, rotr(bfi(0x80808080u, g, A), 6), pt[3]);
#pragma unroll
    for (int j = 0; j < 4; ++j) {
        o[j >> 1][2 * (j & 1)] = pk_add(__builtin_amdgcn_perm(hb[j], lo[j], 0x05010400u), LL);
        o[j >> 1][2 * (j & 1) + 1] = pk_add(__builtin_amdgcn_perm(hb[j], lo[j], 0x07030602u), LL);
    }
}

__device__ __forceinline__ void unit_fetch(const Params& p, int h, int u, int lane, unsigned ldsbuf) {
    const bool isx = u < 16;
    const char* base = isx ? p.x + ((size_t)u * 256 * KS + h * 32) * 2 : p.w + ((size_t)h * 32 * NN + (u - 16) * 256) * 2;
    unsigned voff = isx ? (lane & 15) * (KS * 2) + (lane >> 4) * 16 : (lane >> 4) * (8 * NN * 2) + (lane & 15) * 16;
    const unsigned s1 = isx ? 16 * KS * 2 : NN * 2, s2 = isx ? 16 * KS * 2 : 256 - 7 * NN * 2;
#pragma unroll
    for (int i = 0; i < 16; ++i) {
        unsigned keep;
        asm volatile("s_mov_b32 %0, m0\n\ts_mov_b32 m0, %3\n\ts_nop 0\n\tglobal_load_lds_dwordx4 %1, %2\n\ts_mov_b32 m0, %0"
                     : "=&s"(keep) : "v"(voff), "s"(base), "s"(ldsbuf + i * 1024) : "memory");
        voff += (i == 7) ? s2 : s1;
    }
}
__device__ __forceinline__ void unit_read(LAS unsigned char* buf, int lane, u32x4 (&r)[16]) {
#pragma unroll
    for (int i = 0; i < 16; ++i) r[i] = *(LAS u32x4*)(buf + i * 1024 + lane * 16);
    asm volatile("s_waitcnt lgkmcnt(0)" : "+v"(r[0]), "+v"(r[1]), "+v"(r[2]), "+v"(r[3]), "+v"(r[4]), "+v"(r[5]), "+v"(r[6]), "+v"(r[7]),
                 "+v"(r[8]), "+v"(r[9]), "+v"(r[10]), "+v"(r[11]), "+v"(r[12]), "+v"(r[13]), "+v"(r[14]), "+v"(r[15]) :: "memory");
}
__device__ __forceinline__ void unit_format(int u, u32x4 (&r)[16]) {
    if (u >= 16) {
#pragma unroll
        for (int g = 0; g < 2; ++g) {
            u32x4 o[8];
#pragma unroll
            for (int i = 0; i < 8; ++i)
#pragma unroll
                for (int a = 0; a < 4; ++a)
                    o[i][a] = __builtin_amdgcn_perm(r[g * 8 + 2 * a + 1][i >> 1], r[g * 8 + 2 * a][i >> 1], (i & 1) ? 0x07060302u : 0x05040100u);
#pragma unroll
            for (int i = 0; i < 8; ++i) r[g * 8 + i] = o[i];
        }
    }
}
struct Tabs { unsigned ord[NU]; unsigned task[NTASK]; unsigned pull[7][NU]; };
constexpr Tabs make_tabs() {
    Tabs t{}; int n = 0;
    for (int v = 0; v < 8 * RAW0; ++v) t.ord[n++] = (v >> 3) << 8 | (v & 7);
    for (int key = 0; key < 8 * NH + 128; ++key)
        for (int v = 0; v < 22; ++v) {
            const int d = key - (v < 8 ? ((v & 2) ? LAGWH : 0) : v < 15 ? LAGXA : LAGXB);
            if (d >= 0 && d % 8 == 0 && d / 8 < NH && !(v < 8 && d / 8 < RAW0)) t.ord[n++] = (unsigned)((d / 8) << 8 | v);
        }
    int inv[NU] = {};
    for (int k = 0; k < NU; ++k) inv[(t.ord[k] >> 8) * 22 + (t.ord[k] & 255)] = k;
    for (int li = 0; li < 7; ++li) {
        n = 0;
        for (int k = 0; k < 8 * RAW0; ++k) t.pull[li][n++] = k << 16 | t.ord[k];
        for (int key = 0; key < 8 * NH + 128; ++key)
            for (int v = 0; v < 22; ++v) {
                const int d = key - (v == 14 - li || v == 21 - li ? LAGOWN : v < 8 ? ((v & 2) ? LAGWH : 0) : v < 15 ? LAGXA : LAGXB);
                if (d >= 0 && d % 8 == 0 && d / 8 < NH && !(v < 8 && d / 8 < RAW0)) t.pull[li][n++] = (unsigned)(inv[d / 8 * 22 + v] << 16 | (d / 8) << 8 | v);
            }
    }
    n = 0;
    for (int k = 0; k < NU; ++k) {
        const unsigned h = t.ord[k] >> 8, v = t.ord[k] & 255, e = k | h << 9;
        for (unsigned l = 0; l < (v < 8 ? 7u : 1u); ++l) t.task[n++] = e | v << 13 | l << 18;
    }
    return t;
}
__constant__ const Tabs g_tabs = make_tabs();
__device__ __forceinline__ int x_tile(int v, int src) { return v >= 15 ? 2 * ((src + v - 14) & 7) + 1 : 2 * ((src + v - 7) & 7); }
__device__ __forceinline__ int st_slot(int h, int v, int link) { return v < 8 ? (link * NH + h) * 8 + v : 896 + h * 14 + v - 8; }

__device__ __forceinline__ void producer(const Params& p, LAS unsigned char* buf, int n0, int lane, unsigned* cntp) {
    char* ws = p.ws[p.rank];
    const unsigned voff = lane * 16;
    const unsigned long long ts = __builtin_amdgcn_s_memrealtime();
    char* peer = p.ws[(p.rank + 1 + (lane & 7)) & 7];
    unsigned call = 0;
    for (int n = n0; n < NTASK; n += 576) {
        const bool first = n < 56 * RAW0;
        const unsigned e = first ? (n / 7) | ((n / 7) >> 3) << 9 | ((n / 7) & 7) << 13 | (n % 7) << 18 : g_tabs.task[n];
        const int k = e & 511, h = (e >> 9) & 15, v = (e >> 13) & 31, link = e >> 18;
        const bool isx = v >= 8;
        const unsigned dest = (unsigned)(p.rank + 1 + link) & 7u;
        const int t = isx ? x_tile(v, p.rank) : dest * 8 + v;
        while (__builtin_amdgcn_s_memrealtime() - ts < (unsigned)(first ? 0 : (n - 56 * RAW0) * PDT / 16)) __builtin_amdgcn_s_sleep(2);
        unit_fetch(p, h, isx ? t : 16 + t, lane, (unsigned)(size_t)buf);
        if (n == n0) call = __builtin_amdgcn_readfirstlane(ld4_sys(cntp)) + 1u;
        WAIT_V(0);
        u32x4 r[16];
        unit_read(buf, lane, r);
        unit_format(isx ? 0 : 16, r);
        unsigned hdr = RAWCNT << 15;
        if (first) {
            const char* s0 = p.ws[dest] + OFF_LZ + (size_t)(((call & 1u) * 7 + 6 - link) * 8 * RAW0 + k) * UB + 4096; const char* s1 = s0 + 8192;
            st16<-4096>(s0, voff, r[0]); st16<-3072>(s0, voff, r[1]); st16<-2048>(s0, voff, r[2]); WAIT_V(PWIN); st16<-1024>(s0, voff, r[3]);
            WAIT_V(PWIN); st16<0>(s0, voff, r[4]); WAIT_V(PWIN); st16<1024>(s0, voff, r[5]); WAIT_V(PWIN); st16<2048>(s0, voff, r[6]); WAIT_V(PWIN); st16<3072>(s0, voff, r[7]);
            WAIT_V(PWIN); st16<-4096>(s1, voff, r[8]); WAIT_V(PWIN); st16<-3072>(s1, voff, r[9]); WAIT_V(PWIN); st16<-2048>(s1, voff, r[10]); WAIT_V(PWIN); st16<-1024>(s1, voff, r[11]);
            WAIT_V(PWIN); st16<0>(s1, voff, r[12]); WAIT_V(PWIN); st16<1024>(s1, voff, r[13]); WAIT_V(PWIN); st16<2048>(s1, voff, r[14]); WAIT_V(PWIN); st16<3072>(s1, voff, r[15]);
        } else {
            const char* sl = ws + OFF_ST + (call & 1u) * ST_PAR + (size_t)st_slot(h, v, link) * UB;
            u32x4 pk[12];
            hdr = pack_unit(r, pk, pack_base(r), buf, lane);
            if (__builtin_expect(hdr_cnt(hdr) == RAWCNT, 0)) st_raw(sl, voff, r);
            else {
                const char* s0 = sl + 4096; const char* s1 = sl + 8192;
                st16<-4096>(s0, voff, pk[0]); st16<-3072>(s0, voff, pk[1]); st16<-2048>(s0, voff, pk[2]); st16<-1024>(s0, voff, pk[3]);
                st16<0>(s0, voff, pk[4]); st16<1024>(s0, voff, pk[5]); st16<2048>(s0, voff, pk[6]); st16<3072>(s0, voff, pk[7]);
                st16<0>(s1, voff, pk[8]); st16<1024>(s1, voff, pk[9]); st_tail(s1, voff, pk[10], pk[11], hdr >> 25, lane);
            }
        }
        WAIT_V(0);
        if (isx ? lane < 7 : lane == link)
            __hip_atomic_store((unsigned long long*)(peer + OFF_FA + (size_t)(((call & 1u) * 7 + 6 - lane) * NU + k) * FLS), ((unsigned long long)hdr << 32) | call,
                               __ATOMIC_RELAXED, __HIP_MEMORY_SCOPE_SYSTEM);
        if (!isx) {
            st_raw(ws + OFF_WD + (size_t)(link * NH + h) * WQ + (size_t)v * UB, voff, r);
            WAIT_V(0);
            if (lane == 0) __hip_atomic_store((unsigned*)(ws + OFF_FW) + t * FSTR + h, call, __ATOMIC_RELAXED, __HIP_MEMORY_SCOPE_SYSTEM);
        }
        WAIT_L(0);
    }
}
__device__ __forceinline__ void own_image(const Params& p, LAS unsigned char* buf, int n, int lane, unsigned call) {
    char* ws = p.ws[p.rank];
    const int h = n < 256 ? n >> 4 : (n - 256) >> 3, v = n >= 256 ? 2 * (n & 7) + 1 : (n & 1) ? 16 + ((n >> 1) & 7) : (n & 14), t = v < 16 ? v : p.rank * 8 + v - 16;
    unit_fetch(p, h, v < 16 ? t : 16 + t, lane, (unsigned)(size_t)buf);
    WAIT_V(0);
    u32x4 r[16];
    unit_read(buf, lane, r);
    unit_format(v, r);
    st_raw(ws + (v < 16 ? OFF_XI + (size_t)h * XQ : OFF_WI + (size_t)h * WQ) + (size_t)(v & 15) * UB, lane * 16, r);
    WAIT_V(0);
    if (lane == 0) __hip_atomic_store((unsigned*)(ws + (v < 16 ? OFF_FX : OFF_FW)) + t * FSTR + h, call, __ATOMIC_RELAXED, __HIP_MEMORY_SCOPE_SYSTEM);
}

constexpr int DEC_LDS = 24 * 1024;
struct PullCtl { u32x4 desc[16]; unsigned full[7]; unsigned tail; };
__device__ __forceinline__ void decoder(const Params& p, LAS unsigned char* dl, LAS PullCtl* ctl, int xcd, int li, int mem, int lane, unsigned call) {
    char* ws = p.ws[p.rank];
    unsigned* tmo = (unsigned*)(ws + OFF_TMO);
    const unsigned voff = lane * 16;
    const int src = (p.rank + 1 + li) & 7;
    const char* fa = ws + OFF_FA + (size_t)((call & 1u) * 7 + li) * NU * FLS;
    unsigned* tk = (unsigned*)(ws + OFF_TK) + li * 32;
    const unsigned tk0 = (call - 1u) * (NU - 8 * RAW0) - (RAW0 + NDEC) * 8;
#pragma unroll
    for (int i = 0; i < 8; ++i) *(LAS u32x4*)(dl + i * 1024 + lane * 16) = (u32x4){0u, 0u, 0u, 0u};
    unsigned k = (RAW0 + mem) * 8 + xcd, n = 0;
    while (k < (unsigned)NU) {
        const int e = g_tabs.pull[li][k], h = (e >> 8) & 255, v = e & 255;
        const unsigned hdr = wait_unit(fa + (size_t)(e >> 16) * FLS, call, false, tmo);
        ++n;
        if (lane == 0) {
            const unsigned pos = __hip_atomic_fetch_add(&ctl->tail, 1u, __ATOMIC_RELAXED, __HIP_MEMORY_SCOPE_WORKGROUP);
            *(volatile LAS u32x4*)&ctl->desc[pos & 15] = (u32x4){pos + 1u, k | (unsigned)st_slot(h, v, 6 - li) << 16, hdr, (unsigned)mem | (n << 8)};
        }
        const bool isx = v >= 8, raw = hdr_cnt(hdr) == RAWCNT;
        const int q = NH + h * 7 + li, t = isx ? x_tile(v, src) : v;
        const char* dst = ws + (isx ? OFF_XI + (size_t)q * XQ : OFF_WI + (size_t)q * WQ) + (size_t)t * UB;
        unsigned* flag = (unsigned*)(ws + (isx ? OFF_FX : OFF_FW)) + (isx ? t : p.rank * 8 + t) * FSTR + q;
        unsigned tn = 1u;
        if (lane == 0) asm volatile("global_atomic_add %0, %1, %0, off sc0\n\ts_waitcnt vmcnt(0)" : "+v"(tn) : "v"(tk) : "memory");
        while ((unsigned)__builtin_amdgcn_readfirstlane(*(volatile LAS unsigned*)&ctl->full[mem]) != n) __builtin_amdgcn_s_sleep(1);
        asm volatile("" ::: "memory");
        unsigned lo = 0;
        if (__builtin_expect(raw, 0)) {
            u32x4 r[16];
            unit_read(dl + DEC_SLOT, lane, r);
            st_raw(dst, voff, r);
        } else {
            const unsigned F = *(LAS unsigned*)(dl + DEC_TAIL + lane * 4), LL = (hdr & 0x7fffu) * 0x10001u;
            u32x4 P[2], G, lw[8];
            P[0] = *(LAS u32x4*)(dl + DEC_SLOT + 8192 + lane * 16); P[1] = *(LAS u32x4*)(dl + DEC_SLOT + 9216 + lane * 16);
#pragma unroll
            for (int i = 0; i < 8; ++i) lw[i] = *(LAS u32x4*)(dl + DEC_SLOT + i * 1024 + lane * 16);
            lo = unpack_head(F, hdr, dl, lane, G);
#pragma unroll
            for (int i = 0; i < 8; ++i) {
                u32x4 o[2]; unpack_piece(lw[i], P, G, i, dl, lane, LL, o);
                const char* sb = dst + i * 2048;
                st16<0>(sb, voff, o[0]); st16<1024>(sb, voff, o[1]);
            }
        }
        WAIT_V(0);
        if (lane == 0) __hip_atomic_store(flag, call, __ATOMIC_RELAXED, __HIP_MEMORY_SCOPE_SYSTEM);
        if (!raw) patch_apply(dl, lo, hdr, lane, true);
        WAIT_L(0);
        k = __builtin_amdgcn_readfirstlane(tn) - tk0;
    }
    if (lane == 0) {
        const unsigned pos = __hip_atomic_fetch_add(&ctl->tail, 1u, __ATOMIC_RELAXED, __HIP_MEMORY_SCOPE_WORKGROUP);
        *(volatile LAS u32x4*)&ctl->desc[pos & 15] = (u32x4){pos + 1u, 0u, 0u, 0x80u};
    }
}
__device__ __forceinline__ void opener(const Params& p, int li, int h, int ct, int lane, unsigned call) {
    char* ws = p.ws[p.rank];
    wait_unit(ws + OFF_FA + (size_t)(((call & 1u) * 7 + li) * NU + h * 8 + ct) * FLS, call, false, (unsigned*)(ws + OFF_TMO));
    const char* src = ws + OFF_LZ + (size_t)(((call & 1u) * 7 + li) * 8 * RAW0 + h * 8 + ct) * UB;
    const int q = NH + h * 7 + li;
    const char* dst = ws + OFF_WI + (size_t)q * WQ + (size_t)ct * UB;
    for (int i = 0; i < 4; ++i) {
        f32x4 t[4]; ld4x16(t[0], t[1], t[2], t[3], src + i * 4096, lane * 16);
        const char* d = dst + i * 4096;
        st16<0>(d, lane * 16, __builtin_bit_cast(u32x4, t[0])); st16<1024>(d, lane * 16, __builtin_bit_cast(u32x4, t[1]));
        st16<2048>(d, lane * 16, __builtin_bit_cast(u32x4, t[2])); st16<3072>(d, lane * 16, __builtin_bit_cast(u32x4, t[3]));
    }
    WAIT_V(0);
    if (lane == 0) __hip_atomic_store((unsigned*)(ws + OFF_FW) + (p.rank * 8 + ct) * FSTR + q, call, __ATOMIC_RELAXED, __HIP_MEMORY_SCOPE_SYSTEM);
}
__device__ __forceinline__ void loader(const Params& p, LAS unsigned char* lds, LAS PullCtl* ctl, int li, int lane, unsigned call) {
    const char* st = p.ws[(unsigned)(p.rank + 1 + li) & 7u] + OFF_ST + (call & 1u) * ST_PAR;
    const unsigned voff = lane * 16;
    unsigned pos = 0, done = 0, pend = 0, idle = 0;
    { const unsigned long long ts = __builtin_amdgcn_s_memrealtime();
      while (__builtin_amdgcn_s_memrealtime() - ts < LD0) __builtin_amdgcn_s_sleep(4); }
#define LANDED() do { *(volatile LAS unsigned*)&ctl->full[pend & 7u] = pend >> 8; pend = 0; } while (0)
    while (done < NDEC || pend) {
        const u32x4 d = *(volatile LAS u32x4*)&ctl->desc[pos & 15];
        if ((unsigned)__builtin_amdgcn_readfirstlane(d.x) != pos + 1u) {
            if (!pend) __builtin_amdgcn_s_sleep(1);
            else if (idle == 0) WAIT_V(4);
            else if (idle == 1) WAIT_V(2);
            else { WAIT_V(0); LANDED(); }
            ++idle;
            continue;
        }
        idle = 0;
        const unsigned ks = __builtin_amdgcn_readfirstlane(d.y), hdr = __builtin_amdgcn_readfirstlane(d.z), w = __builtin_amdgcn_readfirstlane(d.w);
        ++pos;
        if (w & 0x80u) { ++done; continue; }
        if ((ks & 0xffffu) == NU - 1 - GATE_EARLY && lane < 8)
            __hip_atomic_store((unsigned*)(p.ws[p.rank] + OFF_SD) + (li * 8 + lane) * 32, call, __ATOMIC_RELAXED, __HIP_MEMORY_SCOPE_AGENT);
        const char* s = st + (size_t)(ks >> 16) * UB;
        const unsigned l = (unsigned)(size_t)lds + (w & 7u) * DEC_LDS + DEC_SLOT;
#pragma unroll
        for (int i = 0; i < 10; ++i) {
            WAIT_V(LWIN);
            if (i == LWIN && pend) LANDED();
            dma16(s + i * 1024, voff, l + i * 1024);
        }
        static_assert(LWIN < 10, "window");
        if (__builtin_expect(hdr_cnt(hdr) == RAWCNT, 0)) {
#pragma unroll
            for (int i = 10; i < 16; ++i) { WAIT_V(LWIN); dma16(s + i * 1024, voff, l + i * 1024); }
        } else {
            const unsigned tl = hdr >> 25;
            WAIT_V(LWIN);
            if ((unsigned)lane < tl * 2u) dma16(s + 10240, voff, l + 10240);
            if (__builtin_expect(tl > 32u, 0)) { if ((unsigned)lane < (tl - 32u) * 2u) dma16(s + 11264, voff, l + 11264); }
        }
        pend = w;
    }
}

__device__ __forceinline__ void post_ready(bool r0, bool r1, unsigned long long& td0, unsigned long long& td1, LAS unsigned* queue, int& nq, unsigned lane) {
    const unsigned long long a0 = __ballot(r0) & td0, a1 = __ballot(r1) & td1;
    if ((a0 >> lane) & 1) ((volatile LAS unsigned*)queue)[__builtin_amdgcn_mbcnt_hi((unsigned)(a0 >> 32), __builtin_amdgcn_mbcnt_lo((unsigned)a0, nq))] = lane | 0x10000u;
    nq += __builtin_popcountll(a0);
    if ((a1 >> lane) & 1) ((volatile LAS unsigned*)queue)[__builtin_amdgcn_mbcnt_hi((unsigned)(a1 >> 32), __builtin_amdgcn_mbcnt_lo((unsigned)a1, nq))] = lane | 0x10040u;
    nq += __builtin_popcountll(a1);
    td0 &= ~a0; td1 &= ~a1;
}
__device__ __forceinline__ void poll_ready(const unsigned* fx, const unsigned* fw, unsigned long long& td0, unsigned long long& td1,
                                           LAS unsigned* queue, int& nq, int need, unsigned call, unsigned* tmo) {
    const unsigned lane = __builtin_amdgcn_mbcnt_hi(~0u, __builtin_amdgcn_mbcnt_lo(~0u, 0u));
    {
        const volatile LAS unsigned* sn = queue + 256 + lane;
        const unsigned x0 = sn[0], x1 = sn[64], w0 = sn[128], w1 = sn[192];
        post_ready(x0 == call && w0 == call, x1 == call && w1 == call, td0, td1, queue, nq, lane);
    }
    const unsigned long long t0 = __builtin_amdgcn_s_memrealtime(); unsigned polls = 0;
    bool all = false;
    while (nq < need) {
        unsigned x0, x1, w0, w1;
        asm volatile("global_load_dword %0, %4, %5 sc0 sc1\n\tglobal_load_dword %1, %4, %5 offset:256 sc0 sc1\n\t"
                     "global_load_dword %2, %4, %6 sc0 sc1\n\tglobal_load_dword %3, %4, %6 offset:256 sc0 sc1\n\ts_waitcnt vmcnt(0)"
                     : "=&v"(x0), "=&v"(x1), "=&v"(w0), "=&v"(w1) : "v"(lane * 4), "s"(fx), "s"(fw) : "memory");
        post_ready(all || (x0 >= call && w0 >= call), all || (x1 >= call && w1 >= call), td0, td1, queue, nq, lane);
        if (nq >= need) break;
        all = give_up(tmo, call, t0, polls);
        __builtin_amdgcn_s_sleep(1);
    }
    WAIT_L(0);
}

constexpr int HTB = 16384;
constexpr int FIN_LDS = 20480;

__global__ __launch_bounds__(512, 2) void fused_kernel(const Params p) {
    __shared__ __attribute__((aligned(16))) unsigned char shm[8 * FIN_LDS];
    LAS unsigned char* lds = (LAS unsigned char*)shm;
    const int wid = __builtin_amdgcn_readfirstlane(threadIdx.x >> 6), lane = threadIdx.x & 63;
    const int xcd = blockIdx.x & 7, slot = blockIdx.x >> 3;
    char* ws = p.ws[p.rank];
    unsigned* tmo = (unsigned*)(ws + OFF_TMO);
    unsigned* cntp = (unsigned*)(ws + OFF_CNT) + blockIdx.x * 8 + wid;
    const bool main_tile = slot >= 16;
    const int fsrc = (p.rank + 1 + slot) & 7, fct = xcd;
    const bool helper = !main_tile && (slot & 7) == 7;
    LAS PullCtl* ctl = (LAS PullCtl*)(lds + NDEC * DEC_LDS);
    static_assert(NDEC <= 5 && RAW0 <= 2, "puller CU: load wave, 5 unpack waves, 2 waves for the opening units");
    if (slot < 7) { if (wid < 2) ((volatile LAS unsigned*)ctl)[wid * 64 + lane] = 0u; WAIT_L(0); BAR; }
    if (slot >= 7 && !main_tile) producer(p, lds + wid * HTB, wid * 72 + (blockIdx.x - 56), lane, cntp);
    const unsigned call = __builtin_amdgcn_readfirstlane(ld4_sys(cntp)) + 1u;
    if (lane == 0) __hip_atomic_store(cntp, call, __ATOMIC_RELAXED, __HIP_MEMORY_SCOPE_AGENT);
    if (slot < 7) {
        if (wid == 0) { __builtin_amdgcn_s_setprio(3); loader(p, lds, ctl, slot, lane, call); __builtin_amdgcn_s_setprio(0); }
        else if (wid <= NDEC) decoder(p, lds + (wid - 1) * DEC_LDS, ctl, xcd, slot, wid - 1, lane, call);
        else if (wid >= 6 && wid < 6 + RAW0) opener(p, slot, wid - 6, xcd, lane, call);
    }
    if (main_tile) {
        if ((xcd & 1) && wid >= 1 && wid <= 6) { for (int i = 1; i < wid; ++i) __builtin_amdgcn_s_sleep(40); own_image(p, lds + wid * HTB, (wid - 1) * 64 + ((blockIdx.x - 128) >> 1), lane, call); }
    } else {
        asm volatile("s_waitcnt vmcnt(0) lgkmcnt(0)" ::: "memory");
        BAR;
    }

    const int wr = wid >> 2, wc = wid & 3;
    f32x4 acc[2][2][4][2];
    int rt, cn;
    const unsigned* gate2 = cntp;
    if (slot < 7) {
        rt = 0; cn = fsrc * 8 + fct;
        wait_word((const unsigned*)(ws + OFF_FD) + cn * 8 + wid, call, tmo);
        const char* pb = ws + OFF_PK + (size_t)(cn * 8 + wid) * 32768;
#pragma unroll
        for (int i = 0; i < 8; ++i)
            ld4x16(acc[i >> 2][(i >> 1) & 1][(i & 1) * 2][0], acc[i >> 2][(i >> 1) & 1][(i & 1) * 2][1], acc[i >> 2][(i >> 1) & 1][(i & 1) * 2 + 1][0], acc[i >> 2][(i >> 1) & 1][(i & 1) * 2 + 1][1], pb + i * 4096, lane * 16);
    } else
    for (int it = (main_tile || helper) ? 0 : (xcd & 1) ? -1 : 1; it < 2; ++it) {
    int sm, skip = 7;
    const char* wimg;
    if (main_tile) {
        const int idx = slot - 16;
        rt = (((xcd >> 1) & 1) * 4 + (idx >> 2)) * 2 + (xcd & 1);
        cn = p.rank * 8 + (xcd >> 2) * 4 + (idx & 3);
        const int m = rt >> 1;
        skip = (m - p.rank - 1) & 7; sm = (m == p.rank) ? 0x0f : 0xff & ~(1 << skip);
        wimg = ws + OFF_WI;
    } else {
        rt = p.rank * 2 + (helper ? slot >> 3 : it < 1);
        cn = fsrc * 8 + (fct ^ (it < 0));
        sm = helper ? 0xf0 : 0x80;
        wimg = helper ? ws + OFF_WI : ws + OFF_WD + (size_t)(slot & 7) * NH * WQ;
    }
    const int nt = 8 * __builtin_popcount(sm);
    const unsigned* fx = (const unsigned*)(ws + OFF_FX) + rt * FSTR;
    const unsigned* fw = (const unsigned*)(ws + OFF_FW) + cn * FSTR;
    gate2 = (skip == 7) ? cntp : fw + NH + skip;
    const char* xb = ws + OFF_XI + (size_t)(rt * 16 + wid) * 1024;
    const char* wb = wimg + (size_t)((cn & 7) * 16 + wr * 8 + wc) * 1024;
    const unsigned voff = lane * 16;
    const unsigned ldsw = wid * 1024;
    const int aoff = wr * 4096 + lane * 16, boff = wc * 2048 + lane * 16;
#define SA(b, h) (((b) * 4 + (h)) * HTB)
#define SB(b, h) (((b) * 4 + 2 + (h)) * HTB)
#define GLDS(g, o, l) __builtin_amdgcn_global_load_lds((const unsigned*)((g) + (size_t)(unsigned)(o)), (LAS unsigned*)(lds + (l)), 16, 0, 17)
#define STAGE_A(b, h, qq) do { GLDS(wb + (h) * 4096, (qq.x << 17) + voff, SA(b, h) + ldsw); GLDS(wb + (h) * 4096, (qq.y << 17) + voff, SA(b, h) + ldsw + 8192); } while (0)
#define STAGE_B(b, h, qq) do { GLDS(xb + (h) * 8192, (qq.x << 18) + voff, SB(b, h) + ldsw); GLDS(xb + (h) * 8192, (qq.y << 18) + voff, SB(b, h) + ldsw + 8192); } while (0)
#define LDA(dst, b, h) do { _Pragma("unroll") for (int m = 0; m < 4; ++m) _Pragma("unroll") for (int k = 0; k < 2; ++k) dst[m][k] = *(const LAS bf16x8*)(lds + SA(b, h) + aoff + m * 1024 + k * 8192); } while (0)
#define LDB(dst, b, h) do { _Pragma("unroll") for (int n = 0; n < 2; ++n) _Pragma("unroll") for (int k = 0; k < 2; ++k) dst[n][k] = *(const LAS bf16x8*)(lds + SB(b, h) + boff + n * 1024 + k * 8192); } while (0)
#define MMA2(ai) do { __builtin_amdgcn_s_setprio(1); _Pragma("unroll") for (int k = 0; k < 2; ++k) _Pragma("unroll") for (int m = 0; m < 4; ++m) _Pragma("unroll") for (int n = 0; n < 2; ++n) { \
        acc[ai][0][m][n] = __builtin_amdgcn_mfma_f32_16x16x32_bf16(B0[n][k], At[m][k], acc[ai][0][m][n], 0, 0, 0); \
        acc[ai][1][m][n] = __builtin_amdgcn_mfma_f32_16x16x32_bf16(B1[n][k], At[m][k], acc[ai][1][m][n], 0, 0, 0); } __builtin_amdgcn_s_setprio(0); } while (0)
#define MUST_PICK(kt) do { if (__builtin_expect(nq < 2 * (kt) + 2, 0)) poll_ready(fx, fw, td0, td1, queue, nq, 2 * (kt) + 2, call, tmo); } while (0)
#define SNAP(kt) do { if (__builtin_expect(nq < 2 * (kt) + 2, 0)) \
        __builtin_amdgcn_global_load_lds((lane < 32 ? fx : fw - 128) + lane * 4, queue + 256, 16, 0, 17); } while (0)
#define GETQ(kt) (*(volatile LAS u32x2*)(queue + 2 * (kt)))
#define FIXQ(qq, kt) do { if (__builtin_expect(__builtin_amdgcn_readfirstlane(qq.x & qq.y) < 0x10000, 0)) qq = GETQ(kt); } while (0)
#pragma unroll
    for (int a = 0; a < 2; ++a)
#pragma unroll
        for (int b = 0; b < 2; ++b)
#pragma unroll
            for (int m = 0; m < 4; ++m)
#pragma unroll
                for (int n = 0; n < 2; ++n) acc[a][b][m][n] = (f32x4){0.f, 0.f, 0.f, 0.f};
    bf16x8 At[4][2], B0[2][2], B1[2][2];
    LAS unsigned* queue = (LAS unsigned*)(lds + 8 * HTB + 4096);
    unsigned long long td0 = __ballot((sm >> (lane < NH ? 7 : (lane - NH) % 7)) & 1), td1 = __ballot((sm >> ((lane + 64 - NH) % 7)) & 1);
    int nq = 1 << 20;
    if (wid == 0) { nq = 0; *(volatile LAS u32x4*)(queue + 4 * lane) = (u32x4){0u, 0u, 0u, 0u}; *(volatile LAS u32x4*)(queue + 256 + 4 * lane) = (u32x4){0u, 0u, 0u, 0u}; poll_ready(fx, fw, td0, td1, queue, nq, 4, call, tmo); }
    asm volatile("" ::: "memory"); BAR; asm volatile("" ::: "memory");
    u32x2 qe = GETQ(0), qo = GETQ(1);
    STAGE_B(0, 0, qe); STAGE_A(0, 0, qe); STAGE_B(0, 1, qe); STAGE_A(0, 1, qe);
    STAGE_B(1, 0, qo); STAGE_A(1, 0, qo); STAGE_B(1, 1, qo);
    if (wr == 1) BAR;
    WAIT_V(8); BAR; BAR;
    for (int t = 0; t < nt; t += 2) {
        asm volatile(".p2align 6");
        if (t == nt - 2) nq = 1 << 20;
        SNAP(t + 3);
        qe = GETQ(t + 2);
        LDB(B0, 0, 0); LDA(At, 0, 0); LDB(B1, 0, 1); STAGE_A(1, 1, qo);
        WAIT_V(8); WAIT_L(0); BAR; SCHED; MMA2(0); SCHED; MUST_PICK(t + 2); BAR; SCHED;
        FIXQ(qe, t + 2);
        LDA(At, 0, 1); STAGE_B(0, 0, qe); STAGE_A(0, 0, qe); STAGE_B(0, 1, qe);
        WAIT_V(8); WAIT_L(0); BAR; SCHED; MMA2(1); BAR; SCHED;
        if (t == nt - 2)
            __builtin_amdgcn_global_load_lds(lane == 1 ? tmo : gate2, (LAS unsigned*)(lds + 8 * HTB + wid * 256), 4, 0, 17);
        SNAP(t + 4);
        qo = GETQ(t + 3);
        LDB(B0, 1, 0); LDA(At, 1, 0); LDB(B1, 1, 1); STAGE_A(0, 1, qe);
        WAIT_V(8); WAIT_L(0); BAR; SCHED; MMA2(0); SCHED; MUST_PICK(t + 3); BAR; SCHED;
        FIXQ(qo, t + 3);
        LDA(At, 1, 1); STAGE_B(1, 0, qo); STAGE_A(1, 0, qo); STAGE_B(1, 1, qo);
        WAIT_V(8); WAIT_L(0); BAR; SCHED; MMA2(1); BAR; SCHED;
    }
    if (main_tile) break;
    if (wr == 0) BAR;
    const int pki = (helper ? 64 + (rt & 1) * 8 + fct : cn) * 8 + wid;
    if (it <= 0) {
        const char* pb = ws + OFF_PK + (size_t)pki * 32768;
        asm volatile("s_nop 4" : "+s"(pb));
#pragma unroll
        for (int i = 0; i < 32; ++i) st16<0>(pb + i * 1024, voff, __builtin_bit_cast(u32x4, acc[i >> 4][(i >> 3) & 1][(i >> 1) & 3][i & 1]));
    }
    WAIT_V(0);
    if (it <= 0 && lane == 0) __hip_atomic_store((unsigned*)(ws + OFF_FD) + pki, call, __ATOMIC_RELAXED, __HIP_MEMORY_SCOPE_SYSTEM);
    if (helper) return;
    asm volatile("" ::: "memory"); BAR; asm volatile("" ::: "memory");
    }

    int tz = threadIdx.x; asm volatile("" : "+v"(tz));
    const int ln = tz & 63;
    const unsigned vo = ln * 16;
    const bool own = main_tile && (rt >> 1) == p.rank;
    int computing = __builtin_amdgcn_readfirstlane(main_tile && !own); asm volatile("" : "+s"(computing));
    if (computing != 1) {
        const int rh = own ? rt & 1 : slot < 8 ? 1 : 0;
        LAS unsigned char* dl = lds + wid * FIN_LDS;
        const char* sl = p.ws[fsrc] + OFF_PS + (size_t)(((p.rank * 2 + rh) * 8 + (cn & 7)) * 8 + wid) * UB;
        const unsigned ls = (unsigned)(size_t)dl + DEC_SLOT;
        unsigned hdr = 0;
        if (own) {
            if (wr == 0) BAR;
            const int pki = (64 + rh * 8 + (cn & 7)) * 8 + wid;
            wait_word((const unsigned*)(ws + OFF_FD) + pki, call, tmo);
            sl = ws + OFF_PK + (size_t)pki * 32768;
        } else {
#pragma unroll
            for (int i = 0; i < 8; ++i) *(LAS u32x4*)(dl + i * 1024 + ln * 16) = (u32x4){0u, 0u, 0u, 0u};
            const char* fl = ws + OFF_FP + (size_t)((rh * 64 + cn) * 8 + wid) * FLS;
            wait_word((const unsigned*)(ws + OFF_SD) + ((slot & 7) * 8 + fct) * 32, call, tmo);
            hdr = wait_unit(fl, call, false, tmo);
            dma16(sl, vo, ls); WAIT_V(0); dma16(sl + 1024, vo, ls + 1024); WAIT_V(0); dma16(sl + 2048, vo, ls + 2048);
            if (hdr_cnt(hdr) == LOWCNT) hdr = wait_unit(fl, call, true, tmo);
        }
        const bool raw = hdr_cnt(hdr) == RAWCNT;
        const unsigned LL = (hdr & 0x7fffu) * 0x10001u, tl = hdr >> 25;
        if (!raw && !own) {
            WAIT_V(0);
            if ((unsigned)ln < tl * 2u) dma16(sl + 10240, vo, ls + 10240);
            if (__builtin_expect(tl > 32u, 0)) { if ((unsigned)ln < (tl - 32u) * 2u) dma16(sl + 11264, vo, ls + 11264); }
        }
        const unsigned nanmask = ((unsigned)__builtin_amdgcn_readfirstlane(ld4_sys(tmo)) == call) ? 0x7fc07fc0u : 0u;
        u32x4 P[2], G;
        if (!raw && !own) {
            dma16(sl + 8192, vo, ls + 8192);
            unpack_head(*(LAS unsigned*)(dl + DEC_TAIL + ln * 4), hdr, dl, ln, G);
            WAIT_V(0); dma16(sl + 9216, vo, ls + 9216);
            P[0] = *(LAS u32x4*)(dl + DEC_SLOT + 8192 + ln * 16);
        }
        const char* ob = p.out + ((size_t)(rh * 256 + wc * 32) * NN + cn * 256 + wr * 128) * 2;
        const unsigned oo = (ln >> 4) * (4 * NN * 2) + (ln & 15) * 16;
#pragma unroll
        for (int st = 0; st < 8; ++st) {
            const int i = st == 1 ? 2 : st == 2 ? 1 : st;
            u32x4 o[2];
            if (own) {
                o[0] = o[1] = (u32x4){0u, 0u, 0u, 0u};
                if (st == 0 || st == 1 || st == 4 || st == 6) {
                    const int bj = i >> 2, n = (i >> 1) & 1;
                    f32x4 t[8]; ld8x16s(t, sl + (bj * 8 + n) * 1024 + 4096, vo);
#pragma unroll
                    for (int j = 0; j < 8; ++j) acc[j >> 2][bj][j & 3][n] += t[j];
                }
            } else if (raw) { dma16(sl + i * 2048, vo, ls); dma16(sl + i * 2048 + 1024, vo, ls + 1024); WAIT_V(0);
                       o[0] = *(LAS u32x4*)(dl + DEC_SLOT + ln * 16); o[1] = *(LAS u32x4*)(dl + DEC_SLOT + 1024 + ln * 16); asm volatile("s_waitcnt lgkmcnt(0)" : "+v"(o[0]), "+v"(o[1]) :: "memory"); }
            else {
#define NEXTP(j) dma16(sl + (j) * 1024, vo, ls + (j) * 1024)
                if (st == 2) { WAIT_V(4); NEXTP(3); P[1] = *(LAS u32x4*)(dl + DEC_SLOT + 9216 + ln * 16); }
                if (st >= 3) { WAIT_V(2); if (st < 7) NEXTP(st + 1); }
                unpack_piece(*(LAS u32x4*)(dl + DEC_SLOT + i * 1024 + ln * 16), P, G, i, dl, ln, LL, o);
            }
#pragma unroll
            for (int j = 0; j < 2; ++j) {
                const int f = 2 * i + j;
                u32x4 y;
#pragma unroll
                for (int cc = 0; cc < 4; ++cc) {
                    const unsigned pv = o[j][cc];
                    const f32x2 v = (f32x2){acc[cc >> 1][f >> 3][2 * (cc & 1)][(f >> 2) & 1][f & 3] + __uint_as_float(pv << 16),
                                            acc[cc >> 1][f >> 3][2 * (cc & 1) + 1][(f >> 2) & 1][f & 3] + __uint_as_float(pv & 0xffff0000u)};
                    const f32x2 g = gelu2(v);
                    y[cc] = cvt_pk_bf16(g.x, g.y) | nanmask;
                }
                st16<0>(ob + (size_t)((f >> 3) * 128 + ((f >> 2) & 1) * 16 + (f & 3)) * (NN * 2), oo, y);
            }
        }
    } else {
        const u32x2 gv = *(volatile LAS u32x2*)(lds + 8 * HTB + wid * 256);
        u32x4 b[16], pk[12];
#pragma unroll
        for (int bj = 0; bj < 2; ++bj)
#pragma unroll
            for (int n = 0; n < 2; ++n)
#pragma unroll
                for (int r = 0; r < 4; ++r)
#pragma unroll
                    for (int ai = 0; ai < 2; ++ai)
#pragma unroll
                        for (int m2 = 0; m2 < 2; ++m2)
                            b[bj * 8 + n * 4 + r][ai * 2 + m2] = cvt_pk_bf16(acc[ai][bj][2 * m2][n][r], acc[ai][bj][2 * m2 + 1][n][r]);
        asm volatile("" : "+v"(b[0]), "+v"(b[1]), "+v"(b[2]), "+v"(b[3]), "+v"(b[4]), "+v"(b[5]), "+v"(b[6]), "+v"(b[7]), "+v"(b[8]), "+v"(b[9]), "+v"(b[10]), "+v"(b[11]), "+v"(b[12]), "+v"(b[13]), "+v"(b[14]), "+v"(b[15]) :: "memory");
        if (wr == 0) BAR;
        bool bad = (unsigned)__builtin_amdgcn_readfirstlane(gv.y) == call;
        if ((unsigned)__builtin_amdgcn_readfirstlane(gv.x) < call) { wait_word(gate2, call, tmo); bad = (unsigned)__builtin_amdgcn_readfirstlane(ld4_sys(tmo)) == call; }
        const char* sl = ws + OFF_PS + (size_t)((rt * 8 + (cn & 7)) * 8 + wid) * UB;
        char* fl = p.ws[(unsigned)(rt >> 1) & 7u] + OFF_FP + (size_t)(((rt & 1) * 64 + cn) * 8 + wid) * FLS;
        const char* s0 = sl + 4096; const char* s1 = sl + 8192;
        const unsigned L = pack_base(b);
        st16<-4096>(s0, vo, pack_low(b, 0, L)); st16<-3072>(s0, vo, pack_low(b, 1, L)); st16<-2048>(s0, vo, pack_low(b, 2, L));
        WAIT_V(0);
        st8_flag(fl, call, LOWCNT << 15, ln);
        asm volatile("" ::: "memory"); BAR; asm volatile("" ::: "memory");
        unsigned hdr = pack_unit(b, pk, L, lds + wid * HTB, ln);
        if (bad) {
            hdr = RAWCNT << 15;
#pragma unroll
            for (int f = 0; f < 16; ++f) b[f] = (u32x4){0x7fc07fc0u, 0x7fc07fc0u, 0x7fc07fc0u, 0x7fc07fc0u};
        }
        if (__builtin_expect(hdr_cnt(hdr) == RAWCNT, 0)) st_raw(sl, vo, b);
        else {
            st16<0>(s1, vo, pk[8]); st16<1024>(s1, vo, pk[9]); st_tail(s1, vo, pk[10], pk[11], hdr >> 25, ln);
            st16<-1024>(s0, vo, pk[3]); st16<0>(s0, vo, pk[4]); st16<1024>(s0, vo, pk[5]); st16<2048>(s0, vo, pk[6]); st16<3072>(s0, vo, pk[7]);
        }
        WAIT_V(0);
        st8_flag(fl, call, hdr, ln);
    }
}
}

extern "C" void kernel_launch(int rank, int nranks, ncclComm_t comm, hipStream_t stream,
                              void* const* d_in, const int* in_sizes, int n_in,
                              void* d_out, int out_size, const int* in_dtypes, int out_dtype,
                              void* const* d_out_all, void* const* d_ws_all, size_t ws_bytes) {
    (void)comm; (void)in_dtypes; (void)out_dtype; (void)d_out_all;
    if (nranks != NR || (unsigned)rank >= (unsigned)NR || n_in < 2 || in_sizes[0] != MM * KS || in_sizes[1] != KS * NN ||
        out_size != (MM / NR) * NN || ws_bytes < WS_NEED) {
        fprintf(stderr, "kernel_launch: unexpected problem geometry\n");
        return;
    }
    Params p;
    p.x = (const char*)d_in[0];
    p.w = (const char*)d_in[1];
    for (int i = 0; i < NR; ++i) p.ws[i] = (char*)d_ws_all[i];
    p.out = (char*)d_out;
    p.rank = rank; p.pad = 0;
    hipLaunchKernelGGL(fused_kernel, dim3(256), dim3(512), 0, stream, p);
    HIP_CHECK(hipGetLastError());
}
```
